```python
import numpy as np

kernel_name = "gpu_ag_gemm_m4096_k4096_n4096_e4m3e5m2_dqsilu"


ABSMAX_RTOL = 0.05

COMPUTE_DTYPE = "float32"
EPILOGUE = "dequant_silu"
IN_QUANT = None
OUT_QUANT = None
IN_FP8 = "e4m3e5m2"
IN_DTYPES = {"x": "e4m3", "w_mat": "e5m2",
             "scale_x": "float32", "scale_w": "float32"}
WORLD_SIZE = 8

def _snap_fp8(x, mant_bits, max_abs, min_normal):
    a = np.abs(x.astype(np.float64))
    _m, _e = np.frexp(a)
    q = np.where(a >= min_normal, np.ldexp(1.0, _e - 1 - mant_bits),
                 min_normal * 2.0 ** -mant_bits)
    snapped = np.minimum(np.round(a / q) * q, max_abs)
    return (np.sign(x) * snapped).astype(np.float32)

def setup_inputs():

    x = _snap_fp8(np.random.randn(4096, 4096).astype(np.float32), 3, 448.0, 2.0 ** -6)
    w_mat = _snap_fp8(np.random.randn(4096, 4096).astype(np.float32), 2, 57344.0, 2.0 ** -14)
    scale_x = (0.02 * (1.0 + np.random.rand(1))).astype(np.float32)
    scale_w = (0.02 * (1.0 + np.random.rand(1))).astype(np.float32)
    return {"x": x, "w_mat": w_mat, "scale_x": scale_x, "scale_w": scale_w}

def reference(x, w_mat, scale_x, scale_w):
    acc = (x @ w_mat).astype(np.float32)
    y = (acc * (scale_x * scale_w)).astype(np.float32)
    return (y / (1.0 + np.exp(-np.clip(y, -60.0, 60.0)))).astype(np.float32)
```

```cpp
#include <hip/hip_runtime.h>
#include <rccl/rccl.h>
#include <cstdio>

typedef unsigned int u32;
typedef unsigned long long u64;
typedef u32 u32x4 __attribute__((ext_vector_type(4)));
typedef int i32x8 __attribute__((ext_vector_type(8)));
typedef float f32x4 __attribute__((ext_vector_type(4)));
typedef float f32x2 __attribute__((ext_vector_type(2)));

namespace {

constexpr int NR = 8;
constexpr int KD = 4096;
constexpr int MS = 512;
constexpr int NS = 512;
constexpr u32 SHARD = (u32)MS * KD;
constexpr u32 TILE_B = 16 * KD;
constexpr int NT = MS / 16;
constexpr size_t BUF_B = (size_t)NR * SHARD;
constexpr int NBUF = 2;
constexpr size_t FLAG_OFF = NBUF * BUF_B;
constexpr size_t FLAG_B = (size_t)NR * NT * 128;
constexpr size_t CNT_OFF = FLAG_OFF + FLAG_B;
constexpr size_t CNT_B = 256 * 8 * 4;
constexpr size_t TMO_OFF = CNT_OFF + CNT_B;
constexpr size_t DBG_OFF = TMO_OFF + 4096;
constexpr size_t DBG_B = 256 * 64 * 8;
constexpr size_t WS_NEED = DBG_OFF + DBG_B;
constexpr u64 DEADLINE = 100000000ull;
constexpr int NSLOT = 24;
#define PUSH_WIN 2

struct Params {
  const unsigned char* x;
  const unsigned char* w;
  const float* sx;
  const float* sw;
  float* out;
  char* ws[NR];
  int rank;
};

#define AUX_SYS 17

#define STAMP(i) do {} while (0)
#define STAMPW(i) do {} while (0)

__global__ void __launch_bounds__(512, 2) ag_gemm_kernel(Params p) {
  __shared__ __attribute__((aligned(16))) float red[2][8][16 * 64];
  const int lane = threadIdx.x & 63;
  const int wv = __builtin_amdgcn_readfirstlane(threadIdx.x >> 6);
  const int xg = blockIdx.x & 7;
  const int ci = blockIdx.x >> 3;
  const int rank = p.rank;
  char* myws = p.ws[rank & 7];
  if (ci == 7) return;
  STAMP(0);
  const int m = lane & 15, q = lane >> 4;

  u32* cntp = (u32*)(myws + CNT_OFF) + blockIdx.x * 8 + wv;

  if (ci < 7) {
    u32* acked = (u32*)&red[0][0][0];
    if (threadIdx.x == 0) __hip_atomic_store(acked, 0u, __ATOMIC_RELAXED, __HIP_MEMORY_SCOPE_WORKGROUP);
    auto rx = __builtin_amdgcn_make_buffer_rsrc((void*)p.x, 0, (int)SHARD, 0x00020000);
    u32x4 d[4][8];
    const u32 voff = (u32)(wv * 16 + m) * KD + xg * 512 + q * 16;
#pragma unroll
    for (int i = 0; i < 8; ++i) d[0][i] = __builtin_amdgcn_raw_buffer_load_b128(rx, voff + i * 64, 0, 0);
    u32 call = __hip_atomic_load(cntp, __ATOMIC_RELAXED, __HIP_MEMORY_SCOPE_AGENT) + 1;
#pragma unroll
    for (int u = 1; u < 4; ++u)
#pragma unroll
      for (int i = 0; i < 8; ++i)
        d[u][i] = __builtin_amdgcn_raw_buffer_load_b128(rx, voff + i * 64, u * 8 * TILE_B, 0);
    call = __builtin_amdgcn_readfirstlane(call);
    if (lane == 0) __hip_atomic_store(cntp, call, __ATOMIC_RELAXED, __HIP_MEMORY_SCOPE_AGENT);
    const u32 peer = (u32)(rank + 1 + ci) & 7u;
    char* pws = p.ws[peer];
    char* dst = pws + (size_t)(call & 1) * BUF_B + (size_t)rank * SHARD;
    auto rd = __builtin_amdgcn_make_buffer_rsrc((void*)dst, 0, (int)SHARD, 0x00020000);
    u32* flg = (u32*)(pws + FLAG_OFF) + (rank * NT) * 32 + xg;
    asm volatile("s_waitcnt lgkmcnt(0)" ::: "memory");
    __builtin_amdgcn_s_barrier();
    const u32 soff = (u32)wv * TILE_B + xg * 8192 + lane * 16;
#pragma unroll
    for (int u = 0; u < 4; ++u) {
      const int t = wv + 8 * u;
      while ((int)__hip_atomic_load(acked, __ATOMIC_RELAXED, __HIP_MEMORY_SCOPE_WORKGROUP) < t - PUSH_WIN)
        __builtin_amdgcn_s_sleep(1);
      if (u == 0) STAMPW(8 + wv);
#pragma unroll
      for (int i = 0; i < 8; ++i)
        __builtin_amdgcn_raw_buffer_store_b128(d[u][i], rd, soff + i * 1024, u * 8 * TILE_B, AUX_SYS);
      asm volatile("s_waitcnt vmcnt(0)" ::: "memory");
      if (lane == 0) {
        __hip_atomic_store(flg + t * 32, call, __ATOMIC_RELAXED, __HIP_MEMORY_SCOPE_SYSTEM);
        __hip_atomic_fetch_add(acked, 1u, __ATOMIC_RELAXED, __HIP_MEMORY_SCOPE_WORKGROUP);
      }
      STAMPW(16 + t);
    }
    return;
  }

  const int g = (ci - 8) & 7;
  const int slot = ((ci - 8) >> 3) * 8 + xg;
  u32 call = __hip_atomic_load(cntp, __ATOMIC_RELAXED, __HIP_MEMORY_SCOPE_AGENT) + 1;

  i32x8 bf[4][4];
  {
    auto rw = __builtin_amdgcn_make_buffer_rsrc((void*)p.w, 0, (int)SHARD, 0x00020000);
    const u32 voff = (u32)(wv * 512 + q * 16) * NS + g * 64 + m * 4;
#pragma unroll
    for (int ks = 0; ks < 4; ++ks)
#pragma unroll
      for (int h = 0; h < 2; ++h)
#pragma unroll
        for (int r = 0; r < 4; ++r) {
          u32 d0 = __builtin_amdgcn_raw_buffer_load_b32(rw, voff + (r * 4 + 0) * NS, (ks * 128 + h * 64) * NS, 0);
          u32 d1 = __builtin_amdgcn_raw_buffer_load_b32(rw, voff + (r * 4 + 1) * NS, (ks * 128 + h * 64) * NS, 0);
          u32 d2 = __builtin_amdgcn_raw_buffer_load_b32(rw, voff + (r * 4 + 2) * NS, (ks * 128 + h * 64) * NS, 0);
          u32 d3 = __builtin_amdgcn_raw_buffer_load_b32(rw, voff + (r * 4 + 3) * NS, (ks * 128 + h * 64) * NS, 0);
          u32 t0 = __builtin_amdgcn_perm(d1, d0, 0x05010400u), t1 = __builtin_amdgcn_perm(d1, d0, 0x07030602u);
          u32 t2 = __builtin_amdgcn_perm(d3, d2, 0x05010400u), t3 = __builtin_amdgcn_perm(d3, d2, 0x07030602u);
          bf[0][ks][h * 4 + r] = (int)__builtin_amdgcn_perm(t2, t0, 0x05040100u);
          bf[1][ks][h * 4 + r] = (int)__builtin_amdgcn_perm(t2, t0, 0x07060302u);
          bf[2][ks][h * 4 + r] = (int)__builtin_amdgcn_perm(t3, t1, 0x05040100u);
          bf[3][ks][h * 4 + r] = (int)__builtin_amdgcn_perm(t3, t1, 0x07060302u);
        }
  }
  call = __builtin_amdgcn_readfirstlane(call);
  if (lane == 0) __hip_atomic_store(cntp, call, __ATOMIC_RELAXED, __HIP_MEMORY_SCOPE_AGENT);
  const float scale = p.sx[0] * p.sw[0];
  STAMP(1);

  auto rxl = __builtin_amdgcn_make_buffer_rsrc((void*)p.x, 0, (int)SHARD, 0x00020000);
  auto rxg = __builtin_amdgcn_make_buffer_rsrc((void*)(myws + (size_t)(call & 1) * BUF_B), 0, (int)BUF_B, 0x00020000);
  u32* flags = (u32*)(myws + FLAG_OFF);
  u32* tmo = (u32*)(myws + TMO_OFF);
  const u32 voff_l = (u32)m * KD + wv * 512 + q * 16;
  const u32 voff_g = (u32)wv * 8192 + lane * 16;

  int it = 0;
  for (int s = slot; s < NT * NR; s += NSLOT, ++it) {
    int src, t;
    u32x4 a[8];
    bool bad = false;
    if (s < NT) {
      src = rank; t = s;
#pragma unroll
      for (int i = 0; i < 8; ++i) a[i] = __builtin_amdgcn_raw_buffer_load_b128(rxl, voff_l + i * 64, t * TILE_B, 0);
    } else {
      t = (s - NT) / 7;
      src = (rank + 7 - ((s - NT) - t * 7)) & 7;
      u32* f = flags + (src * NT + t) * 32 + wv;
      {
        const u64 t0 = __builtin_amdgcn_s_memrealtime();
        u32 n = 0;
        while (__hip_atomic_load(f, __ATOMIC_RELAXED, __HIP_MEMORY_SCOPE_SYSTEM) < call) {
          if ((++n & 63u) == 0) {
            if (__hip_atomic_load(tmo, __ATOMIC_RELAXED, __HIP_MEMORY_SCOPE_AGENT) == call ||
                __builtin_amdgcn_s_memrealtime() - t0 > DEADLINE) {
              __hip_atomic_store(tmo, call, __ATOMIC_RELAXED, __HIP_MEMORY_SCOPE_AGENT);
              bad = true;
              break;
            }
          }
          __builtin_amdgcn_s_sleep(1);
        }
      }
      __builtin_amdgcn_fence(__ATOMIC_ACQUIRE, "wavefront");
      STAMP(2 + 3 * it);
#pragma unroll
      for (int i = 0; i < 8; ++i)
        a[i] = __builtin_amdgcn_raw_buffer_load_b128(rxg, voff_g + i * 1024, src * SHARD + t * TILE_B, AUX_SYS);
    }
    f32x4 acc[4];
    const float init = bad ? __builtin_nanf("") : 0.f;
#pragma unroll
    for (int c = 0; c < 4; ++c) acc[c] = f32x4{init, init, init, init};
#pragma unroll
    for (int ks = 0; ks < 4; ++ks) {
      i32x8 af;
#pragma unroll
      for (int e = 0; e < 4; ++e) { af[e] = (int)a[2 * ks][e]; af[4 + e] = (int)a[2 * ks + 1][e]; }
#pragma unroll
      for (int c = 0; c < 4; ++c)
        acc[c] = __builtin_amdgcn_mfma_scale_f32_16x16x128_f8f6f4(af, bf[c][ks], acc[c], 0, 1, 0, 0, 0, 0);
    }
    STAMP(3 + 3 * it);
    float* rb = &red[it & 1][0][0];
#pragma unroll
    for (int j = 0; j < 4; ++j)
      *(f32x4*)(rb + wv * 1024 + (q * 4 + j) * 64 + m * 4) = f32x4{acc[0][j], acc[1][j], acc[2][j], acc[3][j]};
    asm volatile("s_waitcnt lgkmcnt(0)" ::: "memory");
    __builtin_amdgcn_s_barrier();
    f32x2 sum = *(const f32x2*)(rb + threadIdx.x * 2);
#pragma unroll
    for (int v = 1; v < 8; ++v) sum += *(const f32x2*)(rb + v * 1024 + threadIdx.x * 2);
    f32x2 o;
#pragma unroll
    for (int e = 0; e < 2; ++e) {
      const float y = sum[e] * scale;
      const float yc = __builtin_fminf(__builtin_fmaxf(y, -60.f), 60.f);
      o[e] = y * __builtin_amdgcn_rcpf(1.f + __expf(-yc));
    }
    const int row = src * MS + t * 16 + (threadIdx.x >> 5);
    *(f32x2*)(p.out + (size_t)row * NS + g * 64 + (threadIdx.x & 31) * 2) = o;
    STAMP(4 + 3 * it);
  }
}

}

extern "C" void kernel_launch(int rank, int nranks, ncclComm_t comm, hipStream_t stream,
                              void* const* d_in, const int* in_sizes, int n_in,
                              void* d_out, int out_size, const int* in_dtypes, int out_dtype,
                              void* const* d_out_all, void* const* d_ws_all, size_t ws_bytes) {
  (void)comm; (void)in_dtypes; (void)out_dtype; (void)d_out_all;
  if (nranks != NR || (unsigned)rank >= (unsigned)NR || n_in != 4 || ws_bytes < WS_NEED ||
      in_sizes[0] != (int)SHARD || in_sizes[1] != (int)SHARD || out_size != NR * MS * NS) {
    fprintf(stderr, "kernel_launch: unexpected problem shape\n");
    return;
  }
  Params p;
  p.x = (const unsigned char*)d_in[0];
  p.w = (const unsigned char*)d_in[1];
  p.sx = (const float*)d_in[2];
  p.sw = (const float*)d_in[3];
  p.out = (float*)d_out;
  for (int i = 0; i < NR; ++i) p.ws[i] = (char*)d_ws_all[i];
  p.rank = rank;
  hipLaunchKernelGGL(ag_gemm_kernel, dim3(256), dim3(512), 0, stream, p);
}
```

```cpp
#include <hip/hip_runtime.h>
#include <rccl/rccl.h>
#include <cstdio>

typedef unsigned int u32;
typedef unsigned long long u64;
typedef u32 u32x4 __attribute__((ext_vector_type(4)));
typedef int i32x8 __attribute__((ext_vector_type(8)));
typedef float f32x4 __attribute__((ext_vector_type(4)));
typedef float f32x2 __attribute__((ext_vector_type(2)));
typedef u32 u32x2 __attribute__((ext_vector_type(2)));
typedef __attribute__((address_space(1))) u32 gu32;

namespace {

constexpr int NR = 8;
constexpr int KD = 4096;
constexpr int MS = 512;
constexpr int NS = 512;
constexpr u32 SHARD = (u32)MS * KD;
constexpr u32 TILE_B = 16 * KD;
constexpr int NT = MS / 16;
constexpr u32 BUF_B = (u32)NR * SHARD;
constexpr int NBUF = 3;
constexpr size_t FLAG_OFF = (size_t)NBUF * BUF_B;
constexpr size_t FLAG_B = (size_t)NR * 8 * 128;
constexpr size_t CNT_OFF = FLAG_OFF + FLAG_B;
constexpr size_t CNT_B = 256 * 8 * 4;
constexpr size_t TMO_OFF = CNT_OFF + CNT_B;
constexpr size_t DBG_OFF = TMO_OFF + 4096;
constexpr size_t DBG_B = 256 * 128 * 8;
constexpr size_t WS_NEED = DBG_OFF + DBG_B;
constexpr u64 DEADLINE = 100000000ull;
constexpr int NSLOT = 3;
#define PUSH_W 6
#define PUSH_H 2
#define PUSH_DT 114
#define PUSH_T0 30
#define PUSH_G 2
#define PUSH_STAG 0
#define W_DELAY 150
#define W_DT 100
#define W_SLOT 50
#define SELF_DT 60
#define SELF_T0 200
#define ROT 1
#define POLL_MODE 0
#define OUT_AUX 17
#define ZERO_AUX 17
constexpr int TR_ROW = 528;
constexpr int TR_B = 16 * TR_ROW;

struct Params {
  char* tgt[NR];
  const unsigned char* x;
  const unsigned char* w;
  const float* sx;
  const float* sw;
  float* out;
  int rank;
};

#define AUX_SYS 17

#define STAMP(i) do {} while (0)
#define STAMPL(i) do {} while (0)

__global__ void __launch_bounds__(512, 2) ag_gemm_kernel(Params p) {
  __shared__ __attribute__((aligned(16))) char smem[8 * TR_B];
  __shared__ u32 turn[2];
  const int lane = threadIdx.x & 63;
  const int wv = __builtin_amdgcn_readfirstlane(threadIdx.x >> 6);
  const int xg = blockIdx.x & 7;
  const int ci = blockIdx.x >> 3;
  int rank = p.rank;
  char* myws = p.tgt[NR - 1];
  char* pws = p.tgt[ci & 7];
  const unsigned char* xp = p.x;
  asm volatile("" : "+s"(rank), "+s"(myws), "+s"(pws), "+s"(xp));
  const int m = lane & 15, q = lane >> 4;

  gu32* cntp = (gu32*)(myws + CNT_OFF) + blockIdx.x * 8 + wv;

  if (__builtin_expect(ci < 8, 1)) {
    const u64 tstart = __builtin_amdgcn_s_memrealtime();
    if (wv == 0) __builtin_amdgcn_s_setprio(3);
    else __builtin_amdgcn_s_barrier();
    auto rx = __builtin_amdgcn_make_buffer_rsrc((void*)xp, 0, (int)SHARD, 0x00020000);
    const u32 voff = (u32)(lane >> 5) * KD + (lane & 31) * 16;
    u32x4 e[4][8];
#pragma unroll
    for (int j = 0; j < 8; ++j) e[0][j] = __builtin_amdgcn_raw_buffer_load_b128(rx, voff + j * 2 * KD, wv * 4 * TILE_B + ((xg + wv * 4 * ROT) & 7) * 512, 0);
    u32 call = __hip_atomic_load(cntp, __ATOMIC_RELAXED, __HIP_MEMORY_SCOPE_AGENT);
#pragma unroll
    for (int u = 1; u < 4; ++u) {
      __builtin_amdgcn_sched_barrier(0);
#pragma unroll
      for (int j = 0; j < 8; ++j) e[u][j] = __builtin_amdgcn_raw_buffer_load_b128(rx, voff + j * 2 * KD, (wv * 4 + u) * TILE_B + ((xg + (wv * 4 + u) * ROT) & 7) * 512, 0);
    }
    if (wv == 0) {
      if (lane < 2) __hip_atomic_store(&turn[lane], 0u, __ATOMIC_RELAXED, __HIP_MEMORY_SCOPE_WORKGROUP);
      asm volatile("s_waitcnt lgkmcnt(0)" ::: "memory");
      __builtin_amdgcn_s_barrier();
    }
    char* tw = smem + wv * TR_B + (lane >> 5) * TR_ROW + (lane & 31) * 16;
    const char* tr = smem + wv * TR_B + m * TR_ROW + q * 16;
    const u32 soff = lane * 16;
    call = __builtin_amdgcn_readfirstlane(call) + 1;
    char* dst = pws + (size_t)(call % NBUF) * BUF_B + (size_t)rank * SHARD + wv * 4 * TILE_B;
#pragma unroll
    for (int u = 0; u < 4; ++u) {
      auto rd = __builtin_amdgcn_make_buffer_rsrc((void*)(dst + u * TILE_B + ((xg + (wv * 4 + u) * ROT) & 7) * 8192), 0, 8192, 0x00020000);
#pragma unroll
      for (int j = 0; j < 8; ++j) *(u32x4*)(tw + j * 2 * TR_ROW) = e[u][j];
#pragma unroll
      for (int i = 0; i < 8; ++i) { e[u][i] = *(const u32x4*)(tr + i * 64); e[u][i][0] = ~e[u][i][0]; }
      if (u == 0 && ci < 7 && wv > 0) {
        while (__hip_atomic_load(&turn[0], __ATOMIC_RELAXED, __HIP_MEMORY_SCOPE_WORKGROUP) < (u32)wv) __builtin_amdgcn_s_sleep(1);
      }
      STAMPL(u);
#pragma unroll
      for (int i = 0; i < 8; ++i) {
        if ((SELF_DT || ci < 7) && i % PUSH_G == 0) {
          const int due = (ci < 7 ? PUSH_T0 : SELF_T0) + (((wv * 4 + u) * 8 + i) * 8 + (PUSH_STAG ? ((xg * PUSH_STAG) & 7) * PUSH_G : 0)) * (ci < 7 ? PUSH_DT : SELF_DT) / 64;
          while ((int)(__builtin_amdgcn_s_memrealtime() - tstart) < due) __builtin_amdgcn_s_sleep(1);
        }
        if (u * 8 + i >= PUSH_W) asm volatile("s_waitcnt vmcnt(%0)" ::"n"(PUSH_W - 1) : "memory");
        __builtin_amdgcn_raw_buffer_store_b128(e[u][i], rd, soff + i * 1024, 0, AUX_SYS);
        __builtin_amdgcn_sched_barrier(0);
        if (u * 8 + i == 31 - PUSH_H) __hip_atomic_store(&turn[0], (u32)wv + 1, __ATOMIC_RELAXED, __HIP_MEMORY_SCOPE_WORKGROUP);
      }
    }
    asm volatile("s_waitcnt vmcnt(0)" ::: "memory");
    STAMPL(4);
    if (lane == 0) {
      const u32 done = __hip_atomic_fetch_add(&turn[1], 1u, __ATOMIC_RELAXED, __HIP_MEMORY_SCOPE_WORKGROUP);
      if (done == 7) __hip_atomic_store((gu32*)(pws + FLAG_OFF) + (rank * 8 + xg) * 32, call, __ATOMIC_RELAXED, __HIP_MEMORY_SCOPE_SYSTEM);
      __hip_atomic_store(cntp, call, __ATOMIC_RELAXED, __HIP_MEMORY_SCOPE_AGENT);
    }
    return;
  }

  STAMP(0);
  const int g = (ci - 8) & 7;
  const int slot = (ci - 8) >> 3;
  const int src = (rank + xg) & 7;
  u32 call = __hip_atomic_load(cntp, __ATOMIC_RELAXED, __HIP_MEMORY_SCOPE_AGENT) + 1;

  const u64 tw0 = __builtin_amdgcn_s_memrealtime();
  i32x8 bf[4][4];
  {
    auto rw = __builtin_amdgcn_make_buffer_rsrc((void*)p.w, 0, (int)SHARD, 0x00020000);
    const u32 voff = (u32)(wv * 512 + q * 16) * NS + g * 64 + m * 4;
#pragma unroll
    for (int ks = 0; ks < 4; ++ks)
#pragma unroll
      for (int h = 0; h < 2; ++h) {
        if (W_DT || (ks == 0 && h == 0)) {
          const int due = W_DELAY + slot * W_SLOT + (ks * 2 + h) * W_DT;
          while ((int)(__builtin_amdgcn_s_memrealtime() - tw0) < due) __builtin_amdgcn_s_sleep(4);
        }
#pragma unroll
        for (int r = 0; r < 4; ++r) {
          u32 d0 = __builtin_amdgcn_raw_buffer_load_b32(rw, voff + (r * 4 + 0) * NS, (ks * 128 + h * 64) * NS, 0);
          u32 d1 = __builtin_amdgcn_raw_buffer_load_b32(rw, voff + (r * 4 + 1) * NS, (ks * 128 + h * 64) * NS, 0);
          u32 d2 = __builtin_amdgcn_raw_buffer_load_b32(rw, voff + (r * 4 + 2) * NS, (ks * 128 + h * 64) * NS, 0);
          u32 d3 = __builtin_amdgcn_raw_buffer_load_b32(rw, voff + (r * 4 + 3) * NS, (ks * 128 + h * 64) * NS, 0);
          u32 t0 = __builtin_amdgcn_perm(d1, d0, 0x05010400u), t1 = __builtin_amdgcn_perm(d1, d0, 0x07030602u);
          u32 t2 = __builtin_amdgcn_perm(d3, d2, 0x05010400u), t3 = __builtin_amdgcn_perm(d3, d2, 0x07030602u);
          bf[0][ks][h * 4 + r] = (int)__builtin_amdgcn_perm(t2, t0, 0x05040100u);
          bf[1][ks][h * 4 + r] = (int)__builtin_amdgcn_perm(t2, t0, 0x07060302u);
          bf[2][ks][h * 4 + r] = (int)__builtin_amdgcn_perm(t3, t1, 0x05040100u);
          bf[3][ks][h * 4 + r] = (int)__builtin_amdgcn_perm(t3, t1, 0x07060302u);
        }
      }
  }
  call = __builtin_amdgcn_readfirstlane(call);
  if (lane == 0) __hip_atomic_store(cntp, call, __ATOMIC_RELAXED, __HIP_MEMORY_SCOPE_AGENT);
  const float scale = p.sx[0] * p.sw[0];
  STAMP(1);

  auto rxg = __builtin_amdgcn_make_buffer_rsrc((void*)(myws + (size_t)(call % NBUF) * BUF_B), 0, (int)BUF_B, 0x00020000);
  auto rzr = __builtin_amdgcn_make_buffer_rsrc((void*)(myws + (size_t)((call + NBUF - 1) % NBUF) * BUF_B), 0, (int)BUF_B, 0x00020000);
  auto rout = __builtin_amdgcn_make_buffer_rsrc((void*)p.out, 0, NR * MS * NS * 4, 0x00020000);
  gu32* flags = (gu32*)(myws + FLAG_OFF);
  gu32* tmo = (gu32*)(myws + TMO_OFF);
  const u32 voff_g = (u32)wv * 8192 + lane * 16;
  const u32 voff_z = (u32)g * 8192 + threadIdx.x * 16;
  const u32 voff_o = ((wv * 2 + ((lane >> 4) & 1)) * NS + g * 64 + (lane & 15) * 4) * 4;

  int it = 0;
  for (int t = slot; t < NT; t += NSLOT, ++it) {
    u32x4 a[8];
    bool bad = false;
    const u32 toff = src * SHARD + t * TILE_B;
    __builtin_amdgcn_raw_buffer_store_b128(u32x4{0, 0, 0, 0}, rzr, voff_z + toff, 0, ZERO_AUX);
    {
      gu32* f = flags + (src * 8 + ((wv - t * ROT) & 7)) * 32;
      const u64 t0 = __builtin_amdgcn_s_memrealtime();
      u32 n = 0;
      bool sure = false;
      auto slow = [&]() {
        if ((++n & 31u) == 0) {
          if (__hip_atomic_load(f, __ATOMIC_RELAXED, __HIP_MEMORY_SCOPE_SYSTEM) >= call) { sure = true; return; }
          if (__hip_atomic_load(tmo, __ATOMIC_RELAXED, __HIP_MEMORY_SCOPE_AGENT) == call ||
              __builtin_amdgcn_s_memrealtime() - t0 > DEADLINE) {
            __hip_atomic_store(tmo, call, __ATOMIC_RELAXED, __HIP_MEMORY_SCOPE_AGENT);
            bad = true; sure = true;
            return;
          }
        }
        __builtin_amdgcn_s_sleep(1);
      };
      if (POLL_MODE != 1) {
        while (!sure) {
          const u32 v = __builtin_amdgcn_raw_buffer_load_b32(rxg, POLL_MODE == 3 ? wv * 8192 + 8176 : voff_g + (POLL_MODE == 0 ? 7 : 0) * 1024, toff, AUX_SYS);
          if (__builtin_amdgcn_ballot_w64(v == 0) == 0) break;
          slow();
        }
      }
      STAMP(2 + 3 * it);
      while (true) {
        const bool was_sure = sure;
#pragma unroll
        for (int i = 0; i < 8; ++i)
          a[i] = __builtin_amdgcn_raw_buffer_load_b128(rxg, voff_g + i * 1024, toff, AUX_SYS);
        __builtin_amdgcn_sched_barrier(0);
        u32 mn = a[0][0];
#pragma unroll
        for (int i = 1; i < 8; ++i) mn = mn < a[i][0] ? mn : a[i][0];
        if (was_sure || __builtin_amdgcn_ballot_w64(mn == 0) == 0) break;
        slow();
      }
    }
    f32x4 acc[4];
    const float init = bad ? __builtin_nanf("") : 0.f;
#pragma unroll
    for (int c = 0; c < 4; ++c) acc[c] = f32x4{init, init, init, init};
#pragma unroll
    for (int ks = 0; ks < 4; ++ks) {
      i32x8 af;
#pragma unroll
      for (int e = 0; e < 4; ++e) { af[e] = (int)a[2 * ks][e]; af[4 + e] = (int)a[2 * ks + 1][e]; }
      af[0] = ~af[0]; af[4] = ~af[4];
#pragma unroll
      for (int c = 0; c < 4; ++c)
        acc[c] = __builtin_amdgcn_mfma_scale_f32_16x16x128_f8f6f4(af, bf[c][ks], acc[c], 0, 1, 0, 0, 0, 0);
    }
    STAMP(3 + 3 * it);
    float* rb = (float*)smem + (it & 1) * 8192;
#pragma unroll
    for (int j = 0; j < 4; ++j)
      *(f32x4*)(rb + wv * 1024 + (q * 4 + j) * 64 + m * 4) = f32x4{acc[0][j], acc[1][j], acc[2][j], acc[3][j]};
    asm volatile("s_waitcnt lgkmcnt(0)" ::: "memory");
    __builtin_amdgcn_s_barrier();
    const float* rp = rb + (lane >> 5) * 4096 + wv * 128 + (lane & 31) * 4;
    f32x4 s4 = *(const f32x4*)rp;
#pragma unroll
    for (int v = 1; v < 4; ++v) s4 += *(const f32x4*)(rp + v * 1024);
    u32 o[2];
#pragma unroll
    for (int e = 0; e < 2; ++e) {
      auto sw = __builtin_amdgcn_permlane32_swap(__float_as_uint(s4[e]), __float_as_uint(s4[e + 2]), false, false);
      const float y = (__uint_as_float(sw[0]) + __uint_as_float(sw[1])) * scale;
      const float yc = __builtin_fminf(__builtin_fmaxf(y, -60.f), 60.f);
      o[e] = __float_as_uint(y * __builtin_amdgcn_rcpf(1.f + __expf(-yc)));
    }
    auto b0 = __builtin_amdgcn_permlane32_swap(o[0], o[0], false, false);
    auto b1 = __builtin_amdgcn_permlane32_swap(o[1], o[1], false, false);
    if (lane < 32)
      __builtin_amdgcn_raw_buffer_store_b128(u32x4{b0[0], b1[0], b0[1], b1[1]}, rout, voff_o + (u32)(src * MS + t * 16) * NS * 4, 0, OUT_AUX);
    STAMP(4 + 3 * it);
  }
}

}

extern "C" void kernel_launch(int rank, int nranks, ncclComm_t comm, hipStream_t stream,
                              void* const* d_in, const int* in_sizes, int n_in,
                              void* d_out, int out_size, const int* in_dtypes, int out_dtype,
                              void* const* d_out_all, void* const* d_ws_all, size_t ws_bytes) {
  (void)comm; (void)in_dtypes; (void)out_dtype; (void)d_out_all;
  if (nranks != NR || (unsigned)rank >= (unsigned)NR || n_in != 4 || ws_bytes < WS_NEED ||
      in_sizes[0] != (int)SHARD || in_sizes[1] != (int)SHARD || out_size != NR * MS * NS) {
    fprintf(stderr, "kernel_launch: unexpected problem shape\n");
    return;
  }
  Params p;
  p.x = (const unsigned char*)d_in[0];
  p.w = (const unsigned char*)d_in[1];
  p.sx = (const float*)d_in[2];
  p.sw = (const float*)d_in[3];
  p.out = (float*)d_out;
  for (int i = 0; i < NR; ++i) p.tgt[i] = (char*)d_ws_all[(rank + 1 + i) % NR];
  p.rank = rank;
  hipLaunchKernelGGL(ag_gemm_kernel, dim3(256), dim3(512), 0, stream, p);
}
```

```cpp
#include <hip/hip_runtime.h>
#include <rccl/rccl.h>
#include <cstdio>

typedef unsigned int u32;
typedef unsigned long long u64;
typedef u32 u32x4 __attribute__((ext_vector_type(4)));
typedef int i32x8 __attribute__((ext_vector_type(8)));
typedef float f32x4 __attribute__((ext_vector_type(4)));
typedef float f32x2 __attribute__((ext_vector_type(2)));
typedef u32 u32x2 __attribute__((ext_vector_type(2)));
typedef __attribute__((address_space(1))) u32 gu32;

namespace {

constexpr int NR = 8;
constexpr int KD = 4096;
constexpr int MS = 512;
constexpr int NS = 512;
constexpr u32 SHARD = (u32)MS * KD;
constexpr u32 TILE_B = 16 * KD;
constexpr int NT = MS / 16;
constexpr u32 BUF_B = (u32)NR * SHARD;
constexpr int NBUF = 3;
constexpr size_t FLAG_OFF = (size_t)NBUF * BUF_B;
constexpr size_t FLAG_B = (size_t)NR * 8 * 128;
constexpr size_t CNT_OFF = FLAG_OFF + FLAG_B;
constexpr size_t CNT_B = 256 * 8 * 4;
constexpr size_t TMO_OFF = CNT_OFF + CNT_B;
constexpr size_t DBG_OFF = TMO_OFF + 4096;
constexpr size_t DBG_B = 256 * 128 * 8;
constexpr size_t WS_NEED = DBG_OFF + DBG_B;
constexpr u64 DEADLINE = 100000000ull;
constexpr int NSLOT = 3;
#define PUSH_W 7
#define PUSH_H 2
#define PUSH_DT 110
#define PUSH_T0 30
#define PUSH_G 2
#define PUSH_STAG 0
#define W_DELAY 150
#define W_DT 100
#define W_SLOT 50
#define SELF_DT 60
#define SELF_T0 200
#define PUSH_PRE 300
#define PUSH_LEADT 3
#define PUSH_DT0 20
#define PUSH_AUX 17
#define PROBE 0
#define OUT_AUX 17
#define ZERO_AUX 17
constexpr int TR_ROW = 528;
constexpr int TR_B = 16 * TR_ROW;

struct Params {
  char* tgt[NR];
  const unsigned char* x;
  const unsigned char* w;
  const float* sx;
  const float* sw;
  float* out;
  int rank;
};

#define AUX_SYS 17

#define STAMP(i) do {} while (0)
#define STAMPL(i) do {} while (0)

__global__ void __launch_bounds__(512, 2) ag_gemm_kernel(Params p) {
  __shared__ __attribute__((aligned(16))) char smem[8 * TR_B];
  __shared__ u32 turn[2];
  const int lane = threadIdx.x & 63;
  const int wv = __builtin_amdgcn_readfirstlane(threadIdx.x >> 6);
  const int xg = blockIdx.x & 7;
  const int ci = blockIdx.x >> 3;
  int rank = p.rank;
  char* myws = p.tgt[NR - 1];
  char* pws = p.tgt[ci & 7];
  const unsigned char* xp = p.x;
  asm volatile("" : "+s"(rank), "+s"(myws), "+s"(pws), "+s"(xp));
  const int m = lane & 15, q = lane >> 4;

  gu32* cntp = (gu32*)(myws + CNT_OFF) + blockIdx.x * 8 + wv;

  if (__builtin_expect(ci < 8, 1)) {
    const u64 tstart = __builtin_amdgcn_s_memrealtime();
    if (wv > 0) __builtin_amdgcn_s_barrier();
    auto rx = __builtin_amdgcn_make_buffer_rsrc((void*)xp, 0, (int)SHARD, 0x00020000);
    const u32 voff = (u32)(lane >> 5) * KD + (lane & 31) * 16;
    const int kb = (xg + wv) & 7;
    const int dt = ci < 7 ? PUSH_DT : SELF_DT, t0 = ci < 7 ? PUSH_T0 : SELF_T0;
    char* tw = smem + wv * TR_B + (lane >> 5) * TR_ROW + (lane & 31) * 16;
    const char* tr = smem + wv * TR_B + m * TR_ROW + q * 16;
    const u32 soff = lane * 16;
    u32 call = 0;
    char* dst = nullptr;
#pragma unroll 1
    for (int u = 0; u < 4; ++u) {
      const int t = wv + 8 * u;
      if (t > 0) {
        const int due = t0 + t * dt - (t < PUSH_LEADT ? t * (dt - PUSH_DT0) + t0 : PUSH_PRE);
        while ((int)(__builtin_amdgcn_s_memrealtime() - tstart) < due) __builtin_amdgcn_s_sleep(2);
      }
      u32x4 e[8];
#pragma unroll
      for (int j = 0; j < 8; ++j) e[j] = __builtin_amdgcn_raw_buffer_load_b128(rx, voff + j * 2 * KD, t * TILE_B + kb * 512, 0);
      if (u == 0) {
        call = __hip_atomic_load(cntp, __ATOMIC_RELAXED, __HIP_MEMORY_SCOPE_AGENT);
        if (wv == 0) {
          if (lane < 2) __hip_atomic_store(&turn[lane], 0u, __ATOMIC_RELAXED, __HIP_MEMORY_SCOPE_WORKGROUP);
          asm volatile("s_waitcnt lgkmcnt(0)" ::: "memory");
          __builtin_amdgcn_s_barrier();
        }
      }
#pragma unroll
      for (int j = 0; j < 8; ++j) *(u32x4*)(tw + j * 2 * TR_ROW) = e[j];
#pragma unroll
      for (int i = 0; i < 8; ++i) { e[i] = *(const u32x4*)(tr + i * 64); e[i][0] = ~e[i][0]; }
      if (u == 0) {
        call = __builtin_amdgcn_readfirstlane(call) + 1;
        dst = pws + (size_t)(call % NBUF) * BUF_B + (size_t)rank * SHARD + kb * 8192;
      }
      auto rd = __builtin_amdgcn_make_buffer_rsrc((void*)(dst + t * TILE_B), 0, 8192, 0x00020000);
#pragma unroll
      for (int i = 0; i < 8; ++i) {
        if (i >= PUSH_W) asm volatile("s_waitcnt vmcnt(%0)" ::"n"(PUSH_W - 1) : "memory");
        else if (t > 0) {
          while ((int)__hip_atomic_load(&turn[0], __ATOMIC_RELAXED, __HIP_MEMORY_SCOPE_WORKGROUP) < t * 8 + i - (PUSH_W - 1)) __builtin_amdgcn_s_sleep(1);
        }
        if (i % PUSH_G == 0) {
          const int due = t0 + (t * 8 + i) * dt / 8;
          while ((int)(__builtin_amdgcn_s_memrealtime() - tstart) < due) __builtin_amdgcn_s_sleep(1);
        }
        if (i == 0) STAMPL(u);
        __builtin_amdgcn_raw_buffer_store_b128(e[i], rd, soff + i * 1024, 0, PUSH_AUX);
        __builtin_amdgcn_sched_barrier(0);
      }
#pragma unroll
      for (int k = PUSH_W - 1; k >= 0; --k) {
        asm volatile("s_waitcnt vmcnt(%0)" ::"n"(k) : "memory");
        __hip_atomic_fetch_max(&turn[0], (u32)(t * 8 + 8 - k), __ATOMIC_RELAXED, __HIP_MEMORY_SCOPE_WORKGROUP);
      }
    }
    STAMPL(4);
    if (lane == 0) {
      const u32 done = __hip_atomic_fetch_add(&turn[1], 1u, __ATOMIC_RELAXED, __HIP_MEMORY_SCOPE_WORKGROUP);
      if (done == 7) __hip_atomic_store((gu32*)(pws + FLAG_OFF) + (rank * 8 + xg) * 32, call, __ATOMIC_RELAXED, __HIP_MEMORY_SCOPE_SYSTEM);
      __hip_atomic_store(cntp, call, __ATOMIC_RELAXED, __HIP_MEMORY_SCOPE_AGENT);
    }
    return;
  }

  STAMP(0);
  const int g = (ci - 8) & 7;
  const int slot = (ci - 8) >> 3;
  const int src = (rank + xg) & 7;
  u32 call = __hip_atomic_load(cntp, __ATOMIC_RELAXED, __HIP_MEMORY_SCOPE_AGENT) + 1;

  const u64 tw0 = __builtin_amdgcn_s_memrealtime();
  i32x8 bf[4][4];
  {
    auto rw = __builtin_amdgcn_make_buffer_rsrc((void*)p.w, 0, (int)SHARD, 0x00020000);
    const u32 voff = (u32)(wv * 512 + q * 16) * NS + g * 64 + m * 4;
#pragma unroll
    for (int ks = 0; ks < 4; ++ks)
#pragma unroll
      for (int h = 0; h < 2; ++h) {
        if (W_DT || (ks == 0 && h == 0)) {
          const int due = W_DELAY + slot * W_SLOT + (ks * 2 + h) * W_DT;
          while ((int)(__builtin_amdgcn_s_memrealtime() - tw0) < due) __builtin_amdgcn_s_sleep(4);
        }
#pragma unroll
        for (int r = 0; r < 4; ++r) {
          u32 d0 = __builtin_amdgcn_raw_buffer_load_b32(rw, voff + (r * 4 + 0) * NS, (ks * 128 + h * 64) * NS, 0);
          u32 d1 = __builtin_amdgcn_raw_buffer_load_b32(rw, voff + (r * 4 + 1) * NS, (ks * 128 + h * 64) * NS, 0);
          u32 d2 = __builtin_amdgcn_raw_buffer_load_b32(rw, voff + (r * 4 + 2) * NS, (ks * 128 + h * 64) * NS, 0);
          u32 d3 = __builtin_amdgcn_raw_buffer_load_b32(rw, voff + (r * 4 + 3) * NS, (ks * 128 + h * 64) * NS, 0);
          u32 t0 = __builtin_amdgcn_perm(d1, d0, 0x05010400u), t1 = __builtin_amdgcn_perm(d1, d0, 0x07030602u);
          u32 t2 = __builtin_amdgcn_perm(d3, d2, 0x05010400u), t3 = __builtin_amdgcn_perm(d3, d2, 0x07030602u);
          bf[0][ks][h * 4 + r] = (int)__builtin_amdgcn_perm(t2, t0, 0x05040100u);
          bf[1][ks][h * 4 + r] = (int)__builtin_amdgcn_perm(t2, t0, 0x07060302u);
          bf[2][ks][h * 4 + r] = (int)__builtin_amdgcn_perm(t3, t1, 0x05040100u);
          bf[3][ks][h * 4 + r] = (int)__builtin_amdgcn_perm(t3, t1, 0x07060302u);
        }
      }
  }
  call = __builtin_amdgcn_readfirstlane(call);
  if (lane == 0) __hip_atomic_store(cntp, call, __ATOMIC_RELAXED, __HIP_MEMORY_SCOPE_AGENT);
  const float scale = p.sx[0] * p.sw[0];
  STAMP(1);

  auto rxg = __builtin_amdgcn_make_buffer_rsrc((void*)(myws + (size_t)(call % NBUF) * BUF_B), 0, (int)BUF_B, 0x00020000);
  auto rzr = __builtin_amdgcn_make_buffer_rsrc((void*)(myws + (size_t)((call + NBUF - 1) % NBUF) * BUF_B), 0, (int)BUF_B, 0x00020000);
  auto rout = __builtin_amdgcn_make_buffer_rsrc((void*)p.out, 0, NR * MS * NS * 4, 0x00020000);
  gu32* flags = (gu32*)(myws + FLAG_OFF);
  gu32* tmo = (gu32*)(myws + TMO_OFF);
  const u32 voff_g = (u32)wv * 8192 + lane * 16;
  const u32 voff_z = (u32)g * 8192 + threadIdx.x * 16;
  const u32 voff_o = ((wv * 2 + ((lane >> 4) & 1)) * NS + g * 64 + (lane & 15) * 4) * 4;

  u32x4 a[8];
#pragma unroll
  for (int i = 0; i < 8; ++i) a[i] = __builtin_amdgcn_raw_buffer_load_b128(rxg, voff_g + i * 1024, src * SHARD + slot * TILE_B, AUX_SYS);
  int it = 0;
  for (int t = slot; t < NT; t += NSLOT, ++it) {
    const u32 toff = src * SHARD + t * TILE_B;
    __builtin_amdgcn_raw_buffer_store_b128(u32x4{0, 0, 0, 0}, rzr, voff_z + toff, 0, ZERO_AUX);
    gu32* f = flags + (src * 8 + ((wv - t) & 7)) * 32;
    const u64 t0 = __builtin_amdgcn_s_memrealtime();
    u32 n = 0;
    bool sure = false;
    auto slow = [&]() {
      if ((++n & 31u) == 0) {
        if (__hip_atomic_load(f, __ATOMIC_RELAXED, __HIP_MEMORY_SCOPE_SYSTEM) >= call) { sure = true; return; }
        if (__hip_atomic_load(tmo, __ATOMIC_RELAXED, __HIP_MEMORY_SCOPE_AGENT) == call ||
            __builtin_amdgcn_s_memrealtime() - t0 > DEADLINE) {
          __hip_atomic_store(tmo, call, __ATOMIC_RELAXED, __HIP_MEMORY_SCOPE_AGENT);
          sure = true;
          return;
        }
      }
      __builtin_amdgcn_s_sleep(1);
    };
    if (__builtin_amdgcn_ballot_w64(a[PROBE][0] == 0) != 0) {
      while (!sure) {
        const u32 v = __builtin_amdgcn_raw_buffer_load_b32(rxg, wv * 8192 + PROBE * 1024 + 1008, toff, AUX_SYS);
        if (__builtin_amdgcn_ballot_w64(v == 0) == 0) break;
        slow();
      }
#pragma unroll
      for (int i = 0; i < 8; ++i) a[i] = __builtin_amdgcn_raw_buffer_load_b128(rxg, voff_g + i * 1024, toff, AUX_SYS);
    }
    STAMP(2 + 3 * it);
    f32x4 acc[4];
#pragma unroll
    for (int c = 0; c < 4; ++c) acc[c] = f32x4{0.f, 0.f, 0.f, 0.f};
#pragma unroll
    for (int ks = 0; ks < 4; ++ks) {
      while (!sure) {
        const u32 mn = a[2 * ks][0] < a[2 * ks + 1][0] ? a[2 * ks][0] : a[2 * ks + 1][0];
        if (__builtin_amdgcn_ballot_w64(mn == 0) == 0) break;
        slow();
#pragma unroll
        for (int i = 2 * ks; i < 8; ++i) a[i] = __builtin_amdgcn_raw_buffer_load_b128(rxg, voff_g + i * 1024, toff, AUX_SYS);
      }
      i32x8 af;
#pragma unroll
      for (int e = 0; e < 4; ++e) { af[e] = (int)a[2 * ks][e]; af[4 + e] = (int)a[2 * ks + 1][e]; }
      af[0] = ~af[0]; af[4] = ~af[4];
#pragma unroll
      for (int c = 0; c < 4; ++c)
        acc[c] = __builtin_amdgcn_mfma_scale_f32_16x16x128_f8f6f4(af, bf[c][ks], acc[c], 0, 1, 0, 0, 0, 0);
    }
    if (t + NSLOT < NT) {
#pragma unroll
      for (int i = 0; i < 8; ++i) a[i] = __builtin_amdgcn_raw_buffer_load_b128(rxg, voff_g + i * 1024, toff + NSLOT * TILE_B, AUX_SYS);
    }
    STAMP(3 + 3 * it);
    float* rb = (float*)smem + (it & 1) * 8192;
#pragma unroll
    for (int j = 0; j < 4; ++j)
      *(f32x4*)(rb + wv * 1024 + (q * 4 + j) * 64 + m * 4) = f32x4{acc[0][j], acc[1][j], acc[2][j], acc[3][j]};
    asm volatile("s_waitcnt lgkmcnt(0)" ::: "memory");
    __builtin_amdgcn_s_barrier();
    const float* rp = rb + (lane >> 5) * 4096 + wv * 128 + (lane & 31) * 4;
    f32x4 s4 = *(const f32x4*)rp;
#pragma unroll
    for (int v = 1; v < 4; ++v) s4 += *(const f32x4*)(rp + v * 1024);
    u32 o[2];
#pragma unroll
    for (int e = 0; e < 2; ++e) {
      auto sw = __builtin_amdgcn_permlane32_swap(__float_as_uint(s4[e]), __float_as_uint(s4[e + 2]), false, false);
      const float y = (__uint_as_float(sw[0]) + __uint_as_float(sw[1])) * scale;
      const float yc = __builtin_fminf(__builtin_fmaxf(y, -60.f), 60.f);
      o[e] = __float_as_uint(y * __builtin_amdgcn_rcpf(1.f + __expf(-yc)));
    }
    auto b0 = __builtin_amdgcn_permlane32_swap(o[0], o[0], false, false);
    auto b1 = __builtin_amdgcn_permlane32_swap(o[1], o[1], false, false);
    if (lane < 32)
      __builtin_amdgcn_raw_buffer_store_b128(u32x4{b0[0], b1[0], b0[1], b1[1]}, rout, voff_o + (u32)(src * MS + t * 16) * NS * 4, 0, OUT_AUX);
    STAMP(4 + 3 * it);
  }
}

}

extern "C" void kernel_launch(int rank, int nranks, ncclComm_t comm, hipStream_t stream,
                              void* const* d_in, const int* in_sizes, int n_in,
                              void* d_out, int out_size, const int* in_dtypes, int out_dtype,
                              void* const* d_out_all, void* const* d_ws_all, size_t ws_bytes) {
  (void)comm; (void)in_dtypes; (void)out_dtype; (void)d_out_all;
  if (nranks != NR || (unsigned)rank >= (unsigned)NR || n_in != 4 || ws_bytes < WS_NEED ||
      in_sizes[0] != (int)SHARD || in_sizes[1] != (int)SHARD || out_size != NR * MS * NS) {
    fprintf(stderr, "kernel_launch: unexpected problem shape\n");
    return;
  }
  Params p;
  p.x = (const unsigned char*)d_in[0];
  p.w = (const unsigned char*)d_in[1];
  p.sx = (const float*)d_in[2];
  p.sw = (const float*)d_in[3];
  p.out = (float*)d_out;
  for (int i = 0; i < NR; ++i) p.tgt[i] = (char*)d_ws_all[(rank + 1 + i) % NR];
  p.rank = rank;
  hipLaunchKernelGGL(ag_gemm_kernel, dim3(256), dim3(512), 0, stream, p);
}
```

```cpp
#define PRSTAT
#include <hip/hip_runtime.h>
#include <rccl/rccl.h>
#include <cstdio>

typedef unsigned int u32;
typedef unsigned long long u64;
typedef u32 u32x4 __attribute__((ext_vector_type(4)));
typedef int i32x8 __attribute__((ext_vector_type(8)));
typedef float f32x4 __attribute__((ext_vector_type(4)));
typedef float f32x2 __attribute__((ext_vector_type(2)));
typedef u32 u32x2 __attribute__((ext_vector_type(2)));
typedef __attribute__((address_space(1))) u32 gu32;

namespace {

constexpr int NR = 8;
constexpr int KD = 4096;
constexpr int MS = 512;
constexpr int NS = 512;
constexpr u32 SHARD = (u32)MS * KD;
constexpr u32 TILE_B = 16 * KD;
constexpr int NT = MS / 16;
constexpr u32 BUF_B = (u32)NR * SHARD;
constexpr int NBUF = 3;
constexpr size_t FLAG_OFF = (size_t)NBUF * BUF_B;
constexpr size_t FLAG_B = (size_t)NR * 8 * 128;
constexpr size_t CNT_OFF = FLAG_OFF + FLAG_B;
constexpr size_t CNT_B = 256 * 8 * 4;
constexpr size_t TMO_OFF = CNT_OFF + CNT_B;
constexpr size_t DBG_OFF = TMO_OFF + 4096;
constexpr size_t DBG_B = 256 * 128 * 8;
constexpr size_t WS_NEED = DBG_OFF + DBG_B;
constexpr u64 DEADLINE = 100000000ull;
constexpr int NSLOT = 3;
#define PUSH_W 7
#define PUSH_H 2
#define PUSH_DT 110
#define PUSH_T0 30
#define PUSH_G 2
#define PUSH_STAG 0
#define W_DELAY 150
#define W_DT 100
#define W_SLOT 50
#define SELF_DT 60
#define SELF_T0 200
#define PUSH_PRE 300
#define PUSH_LEADT 3
#define PUSH_DT0 20
#define PUSH_AUX 17
#define PROBE 0
#define OUT_AUX 17
#define ZERO_AUX 17
constexpr int TR_ROW = 528;
constexpr int TR_B = 16 * TR_ROW;

struct Params {
  char* tgt[NR];
  const unsigned char* x;
  const unsigned char* w;
  const float* sx;
  const float* sw;
  float* out;
  int rank;
};

#define AUX_SYS 17

#define STAMP(i) do {} while (0)
#define STAMPL(i) do {} while (0)

typedef __attribute__((address_space(1))) u64 gu64;
__device__ __forceinline__ void prstat(char* myws, int wv, u64 tstart, u64 tfs, u64 tend) {
  gu64* st = (gu64*)(myws + DBG_OFF) + blockIdx.x * 16;
  if (wv == 0) { __hip_atomic_store(st, tstart, __ATOMIC_RELAXED, __HIP_MEMORY_SCOPE_AGENT); __hip_atomic_store(st + 1, tfs, __ATOMIC_RELAXED, __HIP_MEMORY_SCOPE_AGENT); }
  __hip_atomic_store(st + 8 + wv, tend, __ATOMIC_RELAXED, __HIP_MEMORY_SCOPE_AGENT);
}
__device__ __noinline__ void prreport(char* myws, int rank, u32 call, u64 tstart) {
  gu64* st = (gu64*)(myws + DBG_OFF);
  gu64* pv = st + 256 * 16;
  const u64 prev = __hip_atomic_load(pv, __ATOMIC_RELAXED, __HIP_MEMORY_SCOPE_AGENT);
  if (tstart - prev > 20000 || (call % 200) == 0) {
    while (__builtin_amdgcn_s_memrealtime() - tstart < 9000) __builtin_amdgcn_s_sleep(8);
    int b = 1 << 30, ls = -(1 << 30), fs = ls, ack[8], sr[8];
    for (int i = 0; i < 8; ++i) { ack[i] = ls; sr[i] = ls; }
#pragma unroll 1
    for (int c = 0; c < 32; ++c)
#pragma unroll
      for (int x = 0; x < 8; ++x) {
        gu64* q = st + (c * 8 + x) * 16;
        const int s0 = (int)(__hip_atomic_load(q, __ATOMIC_RELAXED, __HIP_MEMORY_SCOPE_AGENT) - tstart);
        b = s0 < b ? s0 : b; ls = s0 > ls ? s0 : ls;
        if (c < 7) { const int f = (int)(__hip_atomic_load(q + 1, __ATOMIC_RELAXED, __HIP_MEMORY_SCOPE_AGENT) - tstart); fs = f > fs ? f : fs; }
        for (int w = 0; w < 8; ++w) {
          const int e = (int)(__hip_atomic_load(q + 8 + w, __ATOMIC_RELAXED, __HIP_MEMORY_SCOPE_AGENT) - tstart);
          if (c < 7) ack[x] = e > ack[x] ? e : ack[x];
          if (c >= 8) sr[x] = e > sr[x] ? e : sr[x];
        }
      }
    printf("PR r%d call %u base %llu gap %llu lstart %d fs %d ackx %d %d %d %d %d %d %d %d src %d %d %d %d %d %d %d %d\n", rank, call, (tstart + b) % 1000000000ull, tstart - prev, ls - b, fs - b,
           ack[0] - b, ack[1] - b, ack[2] - b, ack[3] - b, ack[4] - b, ack[5] - b, ack[6] - b, ack[7] - b,
           sr[0] - b, sr[1] - b, sr[2] - b, sr[3] - b, sr[4] - b, sr[5] - b, sr[6] - b, sr[7] - b);
  }
  __hip_atomic_store(pv, __builtin_amdgcn_s_memrealtime(), __ATOMIC_RELAXED, __HIP_MEMORY_SCOPE_AGENT);
}

__global__ void __launch_bounds__(512, 2) ag_gemm_kernel(Params p) {
  __shared__ __attribute__((aligned(16))) char smem[8 * TR_B];
  __shared__ u32 turn[2];
  const int lane = threadIdx.x & 63;
  const int wv = __builtin_amdgcn_readfirstlane(threadIdx.x >> 6);
  const int xg = blockIdx.x & 7;
  const int ci = blockIdx.x >> 3;
  int rank = p.rank;
  char* myws = p.tgt[NR - 1];
  char* pws = p.tgt[ci & 7];
  const unsigned char* xp = p.x;
  asm volatile("" : "+s"(rank), "+s"(myws), "+s"(pws), "+s"(xp));
  const int m = lane & 15, q = lane >> 4;

  gu32* cntp = (gu32*)(myws + CNT_OFF) + blockIdx.x * 8 + wv;

  if (__builtin_expect(ci < 8, 1)) {
    const u64 tstart = __builtin_amdgcn_s_memrealtime();
    if (wv > 0) __builtin_amdgcn_s_barrier();
    auto rx = __builtin_amdgcn_make_buffer_rsrc((void*)xp, 0, (int)SHARD, 0x00020000);
    const u32 voff = (u32)(lane >> 5) * KD + (lane & 31) * 16;
    const int kb = (xg + wv) & 7;
    const int dt = ci < 7 ? PUSH_DT : SELF_DT, t0 = ci < 7 ? PUSH_T0 : SELF_T0;
    char* tw = smem + wv * TR_B + (lane >> 5) * TR_ROW + (lane & 31) * 16;
    const char* tr = smem + wv * TR_B + m * TR_ROW + q * 16;
    const u32 soff = lane * 16;
    u32 call = 0;
    char* dst = nullptr;
    u64 tfs = 0;
#pragma unroll 1
    for (int u = 0; u < 4; ++u) {
      const int t = wv + 8 * u;
      if (t > 0) {
        const int due = t0 + t * dt - (t < PUSH_LEADT ? t * (dt - PUSH_DT0) + t0 : PUSH_PRE);
        while ((int)(__builtin_amdgcn_s_memrealtime() - tstart) < due) __builtin_amdgcn_s_sleep(2);
      }
      u32x4 e[8];
#pragma unroll
      for (int j = 0; j < 8; ++j) e[j] = __builtin_amdgcn_raw_buffer_load_b128(rx, voff + j * 2 * KD, t * TILE_B + kb * 512, 0);
      if (u == 0) {
        call = __hip_atomic_load(cntp, __ATOMIC_RELAXED, __HIP_MEMORY_SCOPE_AGENT);
        if (wv == 0) {
          if (lane < 2) __hip_atomic_store(&turn[lane], 0u, __ATOMIC_RELAXED, __HIP_MEMORY_SCOPE_WORKGROUP);
          asm volatile("s_waitcnt lgkmcnt(0)" ::: "memory");
          __builtin_amdgcn_s_barrier();
        }
      }
#pragma unroll
      for (int j = 0; j < 8; ++j) *(u32x4*)(tw + j * 2 * TR_ROW) = e[j];
#pragma unroll
      for (int i = 0; i < 8; ++i) { e[i] = *(const u32x4*)(tr + i * 64); e[i][0] = ~e[i][0]; }
      if (u == 0) {
        call = __builtin_amdgcn_readfirstlane(call) + 1;
        dst = pws + (size_t)(call % NBUF) * BUF_B + (size_t)rank * SHARD + kb * 8192;
      }
      auto rd = __builtin_amdgcn_make_buffer_rsrc((void*)(dst + t * TILE_B), 0, 8192, 0x00020000);
#pragma unroll
      for (int i = 0; i < 8; ++i) {
        if (i >= PUSH_W) asm volatile("s_waitcnt vmcnt(%0)" ::"n"(PUSH_W - 1) : "memory");
        else if (t > 0) {
          while ((int)__hip_atomic_load(&turn[0], __ATOMIC_RELAXED, __HIP_MEMORY_SCOPE_WORKGROUP) < t * 8 + i - (PUSH_W - 1)) __builtin_amdgcn_s_sleep(1);
        }
        if (i % PUSH_G == 0) {
          const int due = t0 + (t * 8 + i) * dt / 8;
          while ((int)(__builtin_amdgcn_s_memrealtime() - tstart) < due) __builtin_amdgcn_s_sleep(1);
        }
        if (i == 0) STAMPL(u);
        if (i == 0 && t == 0) tfs = __builtin_amdgcn_s_memrealtime();
        __builtin_amdgcn_raw_buffer_store_b128(e[i], rd, soff + i * 1024, 0, PUSH_AUX);
        __builtin_amdgcn_sched_barrier(0);
      }
#pragma unroll
      for (int k = PUSH_W - 1; k >= 0; --k) {
        asm volatile("s_waitcnt vmcnt(%0)" ::"n"(k) : "memory");
        __hip_atomic_fetch_max(&turn[0], (u32)(t * 8 + 8 - k), __ATOMIC_RELAXED, __HIP_MEMORY_SCOPE_WORKGROUP);
      }
    }
    STAMPL(4);
    const u64 tack = __builtin_amdgcn_s_memrealtime();
    if (lane == 0) {
      const u32 done = __hip_atomic_fetch_add(&turn[1], 1u, __ATOMIC_RELAXED, __HIP_MEMORY_SCOPE_WORKGROUP);
      if (done == 7) __hip_atomic_store((gu32*)(pws + FLAG_OFF) + (rank * 8 + xg) * 32, call, __ATOMIC_RELAXED, __HIP_MEMORY_SCOPE_SYSTEM);
      __hip_atomic_store(cntp, call, __ATOMIC_RELAXED, __HIP_MEMORY_SCOPE_AGENT);
      prstat(myws, wv, tstart, tfs, tack);
      if (blockIdx.x == 56 && wv == 0) prreport(myws, rank, call, tstart);
    }
    return;
  }

  STAMP(0);
  const int g = (ci - 8) & 7;
  const int slot = (ci - 8) >> 3;
  const int src = (rank + xg) & 7;
  u32 call = __hip_atomic_load(cntp, __ATOMIC_RELAXED, __HIP_MEMORY_SCOPE_AGENT) + 1;

  const u64 tw0 = __builtin_amdgcn_s_memrealtime();
  i32x8 bf[4][4];
  {
    auto rw = __builtin_amdgcn_make_buffer_rsrc((void*)p.w, 0, (int)SHARD, 0x00020000);
    const u32 voff = (u32)(wv * 512 + q * 16) * NS + g * 64 + m * 4;
#pragma unroll
    for (int ks = 0; ks < 4; ++ks)
#pragma unroll
      for (int h = 0; h < 2; ++h) {
        if (W_DT || (ks == 0 && h == 0)) {
          const int due = W_DELAY + slot * W_SLOT + (ks * 2 + h) * W_DT;
          while ((int)(__builtin_amdgcn_s_memrealtime() - tw0) < due) __builtin_amdgcn_s_sleep(4);
        }
#pragma unroll
        for (int r = 0; r < 4; ++r) {
          u32 d0 = __builtin_amdgcn_raw_buffer_load_b32(rw, voff + (r * 4 + 0) * NS, (ks * 128 + h * 64) * NS, 0);
          u32 d1 = __builtin_amdgcn_raw_buffer_load_b32(rw, voff + (r * 4 + 1) * NS, (ks * 128 + h * 64) * NS, 0);
          u32 d2 = __builtin_amdgcn_raw_buffer_load_b32(rw, voff + (r * 4 + 2) * NS, (ks * 128 + h * 64) * NS, 0);
          u32 d3 = __builtin_amdgcn_raw_buffer_load_b32(rw, voff + (r * 4 + 3) * NS, (ks * 128 + h * 64) * NS, 0);
          u32 t0 = __builtin_amdgcn_perm(d1, d0, 0x05010400u), t1 = __builtin_amdgcn_perm(d1, d0, 0x07030602u);
          u32 t2 = __builtin_amdgcn_perm(d3, d2, 0x05010400u), t3 = __builtin_amdgcn_perm(d3, d2, 0x07030602u);
          bf[0][ks][h * 4 + r] = (int)__builtin_amdgcn_perm(t2, t0, 0x05040100u);
          bf[1][ks][h * 4 + r] = (int)__builtin_amdgcn_perm(t2, t0, 0x07060302u);
          bf[2][ks][h * 4 + r] = (int)__builtin_amdgcn_perm(t3, t1, 0x05040100u);
          bf[3][ks][h * 4 + r] = (int)__builtin_amdgcn_perm(t3, t1, 0x07060302u);
        }
      }
  }
  call = __builtin_amdgcn_readfirstlane(call);
  if (lane == 0) __hip_atomic_store(cntp, call, __ATOMIC_RELAXED, __HIP_MEMORY_SCOPE_AGENT);
  const float scale = p.sx[0] * p.sw[0];
  STAMP(1);

  auto rxg = __builtin_amdgcn_make_buffer_rsrc((void*)(myws + (size_t)(call % NBUF) * BUF_B), 0, (int)BUF_B, 0x00020000);
  auto rzr = __builtin_amdgcn_make_buffer_rsrc((void*)(myws + (size_t)((call + NBUF - 1) % NBUF) * BUF_B), 0, (int)BUF_B, 0x00020000);
  auto rout = __builtin_amdgcn_make_buffer_rsrc((void*)p.out, 0, NR * MS * NS * 4, 0x00020000);
  gu32* flags = (gu32*)(myws + FLAG_OFF);
  gu32* tmo = (gu32*)(myws + TMO_OFF);
  const u32 voff_g = (u32)wv * 8192 + lane * 16;
  const u32 voff_z = (u32)g * 8192 + threadIdx.x * 16;
  const u32 voff_o = ((wv * 2 + ((lane >> 4) & 1)) * NS + g * 64 + (lane & 15) * 4) * 4;

  u32x4 a[8];
#pragma unroll
  for (int i = 0; i < 8; ++i) a[i] = __builtin_amdgcn_raw_buffer_load_b128(rxg, voff_g + i * 1024, src * SHARD + slot * TILE_B, AUX_SYS);
  int it = 0;
  for (int t = slot; t < NT; t += NSLOT, ++it) {
    const u32 toff = src * SHARD + t * TILE_B;
    __builtin_amdgcn_raw_buffer_store_b128(u32x4{0, 0, 0, 0}, rzr, voff_z + toff, 0, ZERO_AUX);
    gu32* f = flags + (src * 8 + ((wv - t) & 7)) * 32;
    const u64 t0 = __builtin_amdgcn_s_memrealtime();
    u32 n = 0;
    bool sure = false;
    auto slow = [&]() {
      if ((++n & 31u) == 0) {
        if (__hip_atomic_load(f, __ATOMIC_RELAXED, __HIP_MEMORY_SCOPE_SYSTEM) >= call) { sure = true; return; }
        if (__hip_atomic_load(tmo, __ATOMIC_RELAXED, __HIP_MEMORY_SCOPE_AGENT) == call ||
            __builtin_amdgcn_s_memrealtime() - t0 > DEADLINE) {
          __hip_atomic_store(tmo, call, __ATOMIC_RELAXED, __HIP_MEMORY_SCOPE_AGENT);
          sure = true;
          return;
        }
      }
      __builtin_amdgcn_s_sleep(1);
    };
    if (__builtin_amdgcn_ballot_w64(a[PROBE][0] == 0) != 0) {
      while (!sure) {
        const u32 v = __builtin_amdgcn_raw_buffer_load_b32(rxg, wv * 8192 + PROBE * 1024 + 1008, toff, AUX_SYS);
        if (__builtin_amdgcn_ballot_w64(v == 0) == 0) break;
        slow();
      }
#pragma unroll
      for (int i = 0; i < 8; ++i) a[i] = __builtin_amdgcn_raw_buffer_load_b128(rxg, voff_g + i * 1024, toff, AUX_SYS);
    }
    STAMP(2 + 3 * it);
    f32x4 acc[4];
#pragma unroll
    for (int c = 0; c < 4; ++c) acc[c] = f32x4{0.f, 0.f, 0.f, 0.f};
#pragma unroll
    for (int ks = 0; ks < 4; ++ks) {
      while (!sure) {
        const u32 mn = a[2 * ks][0] < a[2 * ks + 1][0] ? a[2 * ks][0] : a[2 * ks + 1][0];
        if (__builtin_amdgcn_ballot_w64(mn == 0) == 0) break;
        slow();
#pragma unroll
        for (int i = 2 * ks; i < 8; ++i) a[i] = __builtin_amdgcn_raw_buffer_load_b128(rxg, voff_g + i * 1024, toff, AUX_SYS);
      }
      i32x8 af;
#pragma unroll
      for (int e = 0; e < 4; ++e) { af[e] = (int)a[2 * ks][e]; af[4 + e] = (int)a[2 * ks + 1][e]; }
      af[0] = ~af[0]; af[4] = ~af[4];
#pragma unroll
      for (int c = 0; c < 4; ++c)
        acc[c] = __builtin_amdgcn_mfma_scale_f32_16x16x128_f8f6f4(af, bf[c][ks], acc[c], 0, 1, 0, 0, 0, 0);
    }
    if (t + NSLOT < NT) {
#pragma unroll
      for (int i = 0; i < 8; ++i) a[i] = __builtin_amdgcn_raw_buffer_load_b128(rxg, voff_g + i * 1024, toff + NSLOT * TILE_B, AUX_SYS);
    }
    STAMP(3 + 3 * it);
    float* rb = (float*)smem + (it & 1) * 8192;
#pragma unroll
    for (int j = 0; j < 4; ++j)
      *(f32x4*)(rb + wv * 1024 + (q * 4 + j) * 64 + m * 4) = f32x4{acc[0][j], acc[1][j], acc[2][j], acc[3][j]};
    asm volatile("s_waitcnt lgkmcnt(0)" ::: "memory");
    __builtin_amdgcn_s_barrier();
    const float* rp = rb + (lane >> 5) * 4096 + wv * 128 + (lane & 31) * 4;
    f32x4 s4 = *(const f32x4*)rp;
#pragma unroll
    for (int v = 1; v < 4; ++v) s4 += *(const f32x4*)(rp + v * 1024);
    u32 o[2];
#pragma unroll
    for (int e = 0; e < 2; ++e) {
      auto sw = __builtin_amdgcn_permlane32_swap(__float_as_uint(s4[e]), __float_as_uint(s4[e + 2]), false, false);
      const float y = (__uint_as_float(sw[0]) + __uint_as_float(sw[1])) * scale;
      const float yc = __builtin_fminf(__builtin_fmaxf(y, -60.f), 60.f);
      o[e] = __float_as_uint(y * __builtin_amdgcn_rcpf(1.f + __expf(-yc)));
    }
    auto b0 = __builtin_amdgcn_permlane32_swap(o[0], o[0], false, false);
    auto b1 = __builtin_amdgcn_permlane32_swap(o[1], o[1], false, false);
    if (lane < 32)
      __builtin_amdgcn_raw_buffer_store_b128(u32x4{b0[0], b1[0], b0[1], b1[1]}, rout, voff_o + (u32)(src * MS + t * 16) * NS * 4, 0, OUT_AUX);
    STAMP(4 + 3 * it);
  }
  if (lane == 0) prstat(myws, wv, tw0, 0, __builtin_amdgcn_s_memrealtime());
}

}

extern "C" void kernel_launch(int rank, int nranks, ncclComm_t comm, hipStream_t stream,
                              void* const* d_in, const int* in_sizes, int n_in,
                              void* d_out, int out_size, const int* in_dtypes, int out_dtype,
                              void* const* d_out_all, void* const* d_ws_all, size_t ws_bytes) {
  (void)comm; (void)in_dtypes; (void)out_dtype; (void)d_out_all;
  if (nranks != NR || (unsigned)rank >= (unsigned)NR || n_in != 4 || ws_bytes < WS_NEED ||
      in_sizes[0] != (int)SHARD || in_sizes[1] != (int)SHARD || out_size != NR * MS * NS) {
    fprintf(stderr, "kernel_launch: unexpected problem shape\n");
    return;
  }
  Params p;
  p.x = (const unsigned char*)d_in[0];
  p.w = (const unsigned char*)d_in[1];
  p.sx = (const float*)d_in[2];
  p.sw = (const float*)d_in[3];
  p.out = (float*)d_out;
  for (int i = 0; i < NR; ++i) p.tgt[i] = (char*)d_ws_all[(rank + 1 + i) % NR];
  p.rank = rank;
  hipLaunchKernelGGL(ag_gemm_kernel, dim3(256), dim3(512), 0, stream, p);
}
```

```cpp
#define PRSTAT 1
#include <hip/hip_runtime.h>
#include <rccl/rccl.h>
#include <cstdio>

typedef unsigned int u32;
typedef unsigned long long u64;
typedef u32 u32x4 __attribute__((ext_vector_type(4)));
typedef int i32x8 __attribute__((ext_vector_type(8)));
typedef float f32x4 __attribute__((ext_vector_type(4)));
typedef float f32x2 __attribute__((ext_vector_type(2)));
typedef u32 u32x2 __attribute__((ext_vector_type(2)));
typedef __attribute__((address_space(1))) u32 gu32;

namespace {

constexpr int NR = 8;
constexpr int KD = 4096;
constexpr int MS = 512;
constexpr int NS = 512;
constexpr u32 SHARD = (u32)MS * KD;
constexpr u32 TILE_B = 16 * KD;
constexpr int NT = MS / 16;
constexpr u32 BUF_B = (u32)NR * SHARD;
constexpr int NBUF = 3;
constexpr size_t FLAG_OFF = (size_t)NBUF * BUF_B;
constexpr size_t FLAG_B = (size_t)NR * 8 * 128;
constexpr size_t CNT_OFF = FLAG_OFF + FLAG_B;
constexpr size_t CNT_B = 256 * 8 * 4;
constexpr size_t TMO_OFF = CNT_OFF + CNT_B;
constexpr size_t DBG_OFF = TMO_OFF + 4096;
constexpr size_t DBG_B = 256 * 128 * 8;
constexpr size_t WS_NEED = DBG_OFF + DBG_B;
constexpr u64 DEADLINE = 100000000ull;
constexpr int NSLOT = 3;
#define PUSH_W 7
#define PUSH_H 2
#define PUSH_DT 110
#define PUSH_T0 30
#define PUSH_G 2
#define PUSH_STAG 0
#define W_DELAY 150
#define W_DT 100
#define W_SLOT 50
#define SELF_DT 60
#define SELF_T0 200
#define PUSH_PRE 300
#define PUSH_LEADT 3
#define PUSH_DT0 20
#define PUSH_AUX 17
#define PROBE 0
#define OUT_AUX 17
#define ZERO_AUX 17
constexpr int TR_ROW = 528;
constexpr int TR_B = 16 * TR_ROW;

struct Params {
  char* tgt[NR];
  const unsigned char* x;
  const unsigned char* w;
  const float* sx;
  const float* sw;
  float* out;
  int rank;
};

#define AUX_SYS 17

#define STAMP(i) do {} while (0)
#define STAMPL(i) do {} while (0)

typedef __attribute__((address_space(1))) u64 gu64;
__device__ __forceinline__ void prstat(char* myws, int wv, u64 tstart, u64 tfs, u64 tend) {
  gu64* st = (gu64*)(myws + DBG_OFF) + blockIdx.x * 16;
  if (wv == 0) { __hip_atomic_store(st, tstart, __ATOMIC_RELAXED, __HIP_MEMORY_SCOPE_AGENT); __hip_atomic_store(st + 1, tfs, __ATOMIC_RELAXED, __HIP_MEMORY_SCOPE_AGENT); }
  __hip_atomic_store(st + 8 + wv, tend, __ATOMIC_RELAXED, __HIP_MEMORY_SCOPE_AGENT);
}
__device__ __forceinline__ int wmax(int v) {
  for (int o = 32; o; o >>= 1) { const int w = __shfl_xor(v, o); v = w > v ? w : v; }
  return v;
}
__device__ __forceinline__ void prreport(char* myws, int lane, u64 tstart) {
  gu64* st = (gu64*)(myws + DBG_OFF);
  gu64* pv = st + 256 * 16;
  const u64 prev = __hip_atomic_load(pv, __ATOMIC_RELAXED, __HIP_MEMORY_SCOPE_AGENT);
  if (tstart - prev > 20000) {
    while (__builtin_amdgcn_s_memrealtime() - tstart < 8000) __builtin_amdgcn_s_sleep(8);
    int nb = -(1 << 30), ls = nb, fs = nb, ack = nb, sr = nb;
#pragma unroll 1
    for (int k = 0; k < 4; ++k) {
      const int bb = k * 64 + lane;
      gu64* q = st + bb * 16;
      const int s0 = (int)(__hip_atomic_load(q, __ATOMIC_RELAXED, __HIP_MEMORY_SCOPE_AGENT) - tstart);
      nb = -s0 > nb ? -s0 : nb; ls = s0 > ls ? s0 : ls;
      if (bb < 56) { const int f = (int)(__hip_atomic_load(q + 1, __ATOMIC_RELAXED, __HIP_MEMORY_SCOPE_AGENT) - tstart); fs = f > fs ? f : fs; }
      for (int w = 0; w < 8; ++w) {
        const int e = (int)(__hip_atomic_load(q + 8 + w, __ATOMIC_RELAXED, __HIP_MEMORY_SCOPE_AGENT) - tstart);
        if (bb < 56) ack = e > ack ? e : ack;
        if (bb >= 64) sr = e > sr ? e : sr;
      }
    }
    nb = wmax(nb); ls = wmax(ls) + nb; fs = wmax(fs) + nb; ack = wmax(ack) + nb; sr = wmax(sr) + nb;
    auto qz = [](int v, int lo) { int r = (v - lo) / 25; return r < 0 ? 0 : r > 63 ? 63 : r; };
    int target = 10000 - nb;
    if (PRSTAT == 2) target += 400 * qz(sr, 3400) + 25600 * qz(ack, 3400);
    if (PRSTAT == 3) target += 400 * qz(ack, 3400) + 25600 * qz(sr, 3400);
    if (PRSTAT == 4) target += 400 * qz(ls, 0) + 25600 * qz(fs, 0);
    if (PRSTAT == 5) target += 400 * qz(ack, 3400) + 25600 * (63 - qz(sr, 3400));
    while ((int)(__builtin_amdgcn_s_memrealtime() - tstart) < target) __builtin_amdgcn_s_sleep(1);
  }
  if (lane == 0) __hip_atomic_store(pv, __builtin_amdgcn_s_memrealtime(), __ATOMIC_RELAXED, __HIP_MEMORY_SCOPE_AGENT);
}

__global__ void __launch_bounds__(512, 2) ag_gemm_kernel(Params p) {
  __shared__ __attribute__((aligned(16))) char smem[8 * TR_B];
  __shared__ u32 turn[2];
  const int lane = threadIdx.x & 63;
  const int wv = __builtin_amdgcn_readfirstlane(threadIdx.x >> 6);
  const int xg = blockIdx.x & 7;
  const int ci = blockIdx.x >> 3;
  int rank = p.rank;
  char* myws = p.tgt[NR - 1];
  char* pws = p.tgt[ci & 7];
  const unsigned char* xp = p.x;
  asm volatile("" : "+s"(rank), "+s"(myws), "+s"(pws), "+s"(xp));
  const int m = lane & 15, q = lane >> 4;

  gu32* cntp = (gu32*)(myws + CNT_OFF) + blockIdx.x * 8 + wv;

  if (__builtin_expect(ci < 8, 1)) {
    const u64 tstart = __builtin_amdgcn_s_memrealtime();
    if (wv > 0) __builtin_amdgcn_s_barrier();
    auto rx = __builtin_amdgcn_make_buffer_rsrc((void*)xp, 0, (int)SHARD, 0x00020000);
    const u32 voff = (u32)(lane >> 5) * KD + (lane & 31) * 16;
    const int kb = (xg + wv) & 7;
    const int dt = ci < 7 ? PUSH_DT : SELF_DT, t0 = ci < 7 ? PUSH_T0 : SELF_T0;
    char* tw = smem + wv * TR_B + (lane >> 5) * TR_ROW + (lane & 31) * 16;
    const char* tr = smem + wv * TR_B + m * TR_ROW + q * 16;
    const u32 soff = lane * 16;
    u32 call = 0;
    char* dst = nullptr;
    u64 tfs = 0;
#pragma unroll 1
    for (int u = 0; u < 4; ++u) {
      const int t = wv + 8 * u;
      if (t > 0) {
        const int due = t0 + t * dt - (t < PUSH_LEADT ? t * (dt - PUSH_DT0) + t0 : PUSH_PRE);
        while ((int)(__builtin_amdgcn_s_memrealtime() - tstart) < due) __builtin_amdgcn_s_sleep(2);
      }
      u32x4 e[8];
#pragma unroll
      for (int j = 0; j < 8; ++j) e[j] = __builtin_amdgcn_raw_buffer_load_b128(rx, voff + j * 2 * KD, t * TILE_B + kb * 512, 0);
      if (u == 0) {
        call = __hip_atomic_load(cntp, __ATOMIC_RELAXED, __HIP_MEMORY_SCOPE_AGENT);
        if (wv == 0) {
          if (lane < 2) __hip_atomic_store(&turn[lane], 0u, __ATOMIC_RELAXED, __HIP_MEMORY_SCOPE_WORKGROUP);
          asm volatile("s_waitcnt lgkmcnt(0)" ::: "memory");
          __builtin_amdgcn_s_barrier();
        }
      }
#pragma unroll
      for (int j = 0; j < 8; ++j) *(u32x4*)(tw + j * 2 * TR_ROW) = e[j];
#pragma unroll
      for (int i = 0; i < 8; ++i) { e[i] = *(const u32x4*)(tr + i * 64); e[i][0] = ~e[i][0]; }
      if (u == 0) {
        call = __builtin_amdgcn_readfirstlane(call) + 1;
        dst = pws + (size_t)(call % NBUF) * BUF_B + (size_t)rank * SHARD + kb * 8192;
      }
      auto rd = __builtin_amdgcn_make_buffer_rsrc((void*)(dst + t * TILE_B), 0, 8192, 0x00020000);
#pragma unroll
      for (int i = 0; i < 8; ++i) {
        if (i >= PUSH_W) asm volatile("s_waitcnt vmcnt(%0)" ::"n"(PUSH_W - 1) : "memory");
        else if (t > 0) {
          while ((int)__hip_atomic_load(&turn[0], __ATOMIC_RELAXED, __HIP_MEMORY_SCOPE_WORKGROUP) < t * 8 + i - (PUSH_W - 1)) __builtin_amdgcn_s_sleep(1);
        }
        if (i % PUSH_G == 0) {
          const int due = t0 + (t * 8 + i) * dt / 8;
          while ((int)(__builtin_amdgcn_s_memrealtime() - tstart) < due) __builtin_amdgcn_s_sleep(1);
        }
        if (i == 0) STAMPL(u);
        if (i == 0 && t == 0) tfs = __builtin_amdgcn_s_memrealtime();
        __builtin_amdgcn_raw_buffer_store_b128(e[i], rd, soff + i * 1024, 0, PUSH_AUX);
        __builtin_amdgcn_sched_barrier(0);
      }
#pragma unroll
      for (int k = PUSH_W - 1; k >= 0; --k) {
        asm volatile("s_waitcnt vmcnt(%0)" ::"n"(k) : "memory");
        __hip_atomic_fetch_max(&turn[0], (u32)(t * 8 + 8 - k), __ATOMIC_RELAXED, __HIP_MEMORY_SCOPE_WORKGROUP);
      }
    }
    STAMPL(4);
    const u64 tack = __builtin_amdgcn_s_memrealtime();
    if (lane == 0) {
      const u32 done = __hip_atomic_fetch_add(&turn[1], 1u, __ATOMIC_RELAXED, __HIP_MEMORY_SCOPE_WORKGROUP);
      if (done == 7) __hip_atomic_store((gu32*)(pws + FLAG_OFF) + (rank * 8 + xg) * 32, call, __ATOMIC_RELAXED, __HIP_MEMORY_SCOPE_SYSTEM);
      __hip_atomic_store(cntp, call, __ATOMIC_RELAXED, __HIP_MEMORY_SCOPE_AGENT);
      prstat(myws, wv, tstart, tfs, tack);
    }
    if (blockIdx.x == 56 && wv == 0) prreport(myws, lane, tstart);
    return;
  }

  STAMP(0);
  const int g = (ci - 8) & 7;
  const int slot = (ci - 8) >> 3;
  const int src = (rank + xg) & 7;
  u32 call = __hip_atomic_load(cntp, __ATOMIC_RELAXED, __HIP_MEMORY_SCOPE_AGENT) + 1;

  const u64 tw0 = __builtin_amdgcn_s_memrealtime();
  i32x8 bf[4][4];
  {
    auto rw = __builtin_amdgcn_make_buffer_rsrc((void*)p.w, 0, (int)SHARD, 0x00020000);
    const u32 voff = (u32)(wv * 512 + q * 16) * NS + g * 64 + m * 4;
#pragma unroll
    for (int ks = 0; ks < 4; ++ks)
#pragma unroll
      for (int h = 0; h < 2; ++h) {
        if (W_DT || (ks == 0 && h == 0)) {
          const int due = W_DELAY + slot * W_SLOT + (ks * 2 + h) * W_DT;
          while ((int)(__builtin_amdgcn_s_memrealtime() - tw0) < due) __builtin_amdgcn_s_sleep(4);
        }
#pragma unroll
        for (int r = 0; r < 4; ++r) {
          u32 d0 = __builtin_amdgcn_raw_buffer_load_b32(rw, voff + (r * 4 + 0) * NS, (ks * 128 + h * 64) * NS, 0);
          u32 d1 = __builtin_amdgcn_raw_buffer_load_b32(rw, voff + (r * 4 + 1) * NS, (ks * 128 + h * 64) * NS, 0);
          u32 d2 = __builtin_amdgcn_raw_buffer_load_b32(rw, voff + (r * 4 + 2) * NS, (ks * 128 + h * 64) * NS, 0);
          u32 d3 = __builtin_amdgcn_raw_buffer_load_b32(rw, voff + (r * 4 + 3) * NS, (ks * 128 + h * 64) * NS, 0);
          u32 t0 = __builtin_amdgcn_perm(d1, d0, 0x05010400u), t1 = __builtin_amdgcn_perm(d1, d0, 0x07030602u);
          u32 t2 = __builtin_amdgcn_perm(d3, d2, 0x05010400u), t3 = __builtin_amdgcn_perm(d3, d2, 0x07030602u);
          bf[0][ks][h * 4 + r] = (int)__builtin_amdgcn_perm(t2, t0, 0x05040100u);
          bf[1][ks][h * 4 + r] = (int)__builtin_amdgcn_perm(t2, t0, 0x07060302u);
          bf[2][ks][h * 4 + r] = (int)__builtin_amdgcn_perm(t3, t1, 0x05040100u);
          bf[3][ks][h * 4 + r] = (int)__builtin_amdgcn_perm(t3, t1, 0x07060302u);
        }
      }
  }
  call = __builtin_amdgcn_readfirstlane(call);
  if (lane == 0) __hip_atomic_store(cntp, call, __ATOMIC_RELAXED, __HIP_MEMORY_SCOPE_AGENT);
  const float scale = p.sx[0] * p.sw[0];
  STAMP(1);

  auto rxg = __builtin_amdgcn_make_buffer_rsrc((void*)(myws + (size_t)(call % NBUF) * BUF_B), 0, (int)BUF_B, 0x00020000);
  auto rzr = __builtin_amdgcn_make_buffer_rsrc((void*)(myws + (size_t)((call + NBUF - 1) % NBUF) * BUF_B), 0, (int)BUF_B, 0x00020000);
  auto rout = __builtin_amdgcn_make_buffer_rsrc((void*)p.out, 0, NR * MS * NS * 4, 0x00020000);
  gu32* flags = (gu32*)(myws + FLAG_OFF);
  gu32* tmo = (gu32*)(myws + TMO_OFF);
  const u32 voff_g = (u32)wv * 8192 + lane * 16;
  const u32 voff_z = (u32)g * 8192 + threadIdx.x * 16;
  const u32 voff_o = ((wv * 2 + ((lane >> 4) & 1)) * NS + g * 64 + (lane & 15) * 4) * 4;

  u32x4 a[8];
#pragma unroll
  for (int i = 0; i < 8; ++i) a[i] = __builtin_amdgcn_raw_buffer_load_b128(rxg, voff_g + i * 1024, src * SHARD + slot * TILE_B, AUX_SYS);
  int it = 0;
  for (int t = slot; t < NT; t += NSLOT, ++it) {
    const u32 toff = src * SHARD + t * TILE_B;
    __builtin_amdgcn_raw_buffer_store_b128(u32x4{0, 0, 0, 0}, rzr, voff_z + toff, 0, ZERO_AUX);
    gu32* f = flags + (src * 8 + ((wv - t) & 7)) * 32;
    const u64 t0 = __builtin_amdgcn_s_memrealtime();
    u32 n = 0;
    bool sure = false;
    auto slow = [&]() {
      if ((++n & 31u) == 0) {
        if (__hip_atomic_load(f, __ATOMIC_RELAXED, __HIP_MEMORY_SCOPE_SYSTEM) >= call) { sure = true; return; }
        if (__hip_atomic_load(tmo, __ATOMIC_RELAXED, __HIP_MEMORY_SCOPE_AGENT) == call ||
            __builtin_amdgcn_s_memrealtime() - t0 > DEADLINE) {
          __hip_atomic_store(tmo, call, __ATOMIC_RELAXED, __HIP_MEMORY_SCOPE_AGENT);
          sure = true;
          return;
        }
      }
      __builtin_amdgcn_s_sleep(1);
    };
    if (__builtin_amdgcn_ballot_w64(a[PROBE][0] == 0) != 0) {
      while (!sure) {
        const u32 v = __builtin_amdgcn_raw_buffer_load_b32(rxg, wv * 8192 + PROBE * 1024 + 1008, toff, AUX_SYS);
        if (__builtin_amdgcn_ballot_w64(v == 0) == 0) break;
        slow();
      }
#pragma unroll
      for (int i = 0; i < 8; ++i) a[i] = __builtin_amdgcn_raw_buffer_load_b128(rxg, voff_g + i * 1024, toff, AUX_SYS);
    }
    STAMP(2 + 3 * it);
    f32x4 acc[4];
#pragma unroll
    for (int c = 0; c < 4; ++c) acc[c] = f32x4{0.f, 0.f, 0.f, 0.f};
#pragma unroll
    for (int ks = 0; ks < 4; ++ks) {
      while (!sure) {
        const u32 mn = a[2 * ks][0] < a[2 * ks + 1][0] ? a[2 * ks][0] : a[2 * ks + 1][0];
        if (__builtin_amdgcn_ballot_w64(mn == 0) == 0) break;
        slow();
#pragma unroll
        for (int i = 2 * ks; i < 8; ++i) a[i] = __builtin_amdgcn_raw_buffer_load_b128(rxg, voff_g + i * 1024, toff, AUX_SYS);
      }
      i32x8 af;
#pragma unroll
      for (int e = 0; e < 4; ++e) { af[e] = (int)a[2 * ks][e]; af[4 + e] = (int)a[2 * ks + 1][e]; }
      af[0] = ~af[0]; af[4] = ~af[4];
#pragma unroll
      for (int c = 0; c < 4; ++c)
        acc[c] = __builtin_amdgcn_mfma_scale_f32_16x16x128_f8f6f4(af, bf[c][ks], acc[c], 0, 1, 0, 0, 0, 0);
    }
    if (t + NSLOT < NT) {
#pragma unroll
      for (int i = 0; i < 8; ++i) a[i] = __builtin_amdgcn_raw_buffer_load_b128(rxg, voff_g + i * 1024, toff + NSLOT * TILE_B, AUX_SYS);
    }
    STAMP(3 + 3 * it);
    float* rb = (float*)smem + (it & 1) * 8192;
#pragma unroll
    for (int j = 0; j < 4; ++j)
      *(f32x4*)(rb + wv * 1024 + (q * 4 + j) * 64 + m * 4) = f32x4{acc[0][j], acc[1][j], acc[2][j], acc[3][j]};
    asm volatile("s_waitcnt lgkmcnt(0)" ::: "memory");
    __builtin_amdgcn_s_barrier();
    const float* rp = rb + (lane >> 5) * 4096 + wv * 128 + (lane & 31) * 4;
    f32x4 s4 = *(const f32x4*)rp;
#pragma unroll
    for (int v = 1; v < 4; ++v) s4 += *(const f32x4*)(rp + v * 1024);
    u32 o[2];
#pragma unroll
    for (int e = 0; e < 2; ++e) {
      auto sw = __builtin_amdgcn_permlane32_swap(__float_as_uint(s4[e]), __float_as_uint(s4[e + 2]), false, false);
      const float y = (__uint_as_float(sw[0]) + __uint_as_float(sw[1])) * scale;
      const float yc = __builtin_fminf(__builtin_fmaxf(y, -60.f), 60.f);
      o[e] = __float_as_uint(y * __builtin_amdgcn_rcpf(1.f + __expf(-yc)));
    }
    auto b0 = __builtin_amdgcn_permlane32_swap(o[0], o[0], false, false);
    auto b1 = __builtin_amdgcn_permlane32_swap(o[1], o[1], false, false);
    if (lane < 32)
      __builtin_amdgcn_raw_buffer_store_b128(u32x4{b0[0], b1[0], b0[1], b1[1]}, rout, voff_o + (u32)(src * MS + t * 16) * NS * 4, 0, OUT_AUX);
    STAMP(4 + 3 * it);
  }
  if (lane == 0) prstat(myws, wv, tw0, 0, __builtin_amdgcn_s_memrealtime());
}

}

extern "C" void kernel_launch(int rank, int nranks, ncclComm_t comm, hipStream_t stream,
                              void* const* d_in, const int* in_sizes, int n_in,
                              void* d_out, int out_size, const int* in_dtypes, int out_dtype,
                              void* const* d_out_all, void* const* d_ws_all, size_t ws_bytes) {
  (void)comm; (void)in_dtypes; (void)out_dtype; (void)d_out_all;
  if (nranks != NR || (unsigned)rank >= (unsigned)NR || n_in != 4 || ws_bytes < WS_NEED ||
      in_sizes[0] != (int)SHARD || in_sizes[1] != (int)SHARD || out_size != NR * MS * NS) {
    fprintf(stderr, "kernel_launch: unexpected problem shape\n");
    return;
  }
  Params p;
  p.x = (const unsigned char*)d_in[0];
  p.w = (const unsigned char*)d_in[1];
  p.sx = (const float*)d_in[2];
  p.sw = (const float*)d_in[3];
  p.out = (float*)d_out;
  for (int i = 0; i < NR; ++i) p.tgt[i] = (char*)d_ws_all[(rank + 1 + i) % NR];
  p.rank = rank;
  hipLaunchKernelGGL(ag_gemm_kernel, dim3(256), dim3(512), 0, stream, p);
}
```

```cpp
#define PRSTAT 6
#include <hip/hip_runtime.h>
#include <rccl/rccl.h>
#include <cstdio>

typedef unsigned int u32;
typedef unsigned long long u64;
typedef u32 u32x4 __attribute__((ext_vector_type(4)));
typedef int i32x8 __attribute__((ext_vector_type(8)));
typedef float f32x4 __attribute__((ext_vector_type(4)));
typedef float f32x2 __attribute__((ext_vector_type(2)));
typedef u32 u32x2 __attribute__((ext_vector_type(2)));
typedef __attribute__((address_space(1))) u32 gu32;

namespace {

constexpr int NR = 8;
constexpr int KD = 4096;
constexpr int MS = 512;
constexpr int NS = 512;
constexpr u32 SHARD = (u32)MS * KD;
constexpr u32 TILE_B = 16 * KD;
constexpr int NT = MS / 16;
constexpr u32 BUF_B = (u32)NR * SHARD;
constexpr int NBUF = 3;
constexpr size_t FLAG_OFF = (size_t)NBUF * BUF_B;
constexpr size_t FLAG_B = (size_t)NR * 8 * 128;
constexpr size_t CNT_OFF = FLAG_OFF + FLAG_B;
constexpr size_t CNT_B = 256 * 8 * 4;
constexpr size_t TMO_OFF = CNT_OFF + CNT_B;
constexpr size_t DBG_OFF = TMO_OFF + 4096;
constexpr size_t DBG_B = 256 * 128 * 8;
constexpr size_t WS_NEED = DBG_OFF + DBG_B;
constexpr u64 DEADLINE = 100000000ull;
constexpr int NSLOT = 3;
#define PUSH_W 7
#define PUSH_H 2
#define PUSH_DT 110
#define PUSH_T0 30
#define PUSH_G 2
#define PUSH_STAG 0
#define W_DELAY 150
#define W_DT 100
#define W_SLOT 50
#define SELF_DT 60
#define SELF_T0 200
#define PUSH_PRE 300
#define PUSH_LEADT 3
#define PUSH_DT0 20
#define PUSH_AUX 17
#define PROBE 0
#define OUT_AUX 17
#define ZERO_AUX 17
constexpr int TR_ROW = 528;
constexpr int TR_B = 16 * TR_ROW;

struct Params {
  char* tgt[NR];
  const unsigned char* x;
  const unsigned char* w;
  const float* sx;
  const float* sw;
  float* out;
  int rank;
};

#define AUX_SYS 17

#define STAMP(i) do {} while (0)
#define STAMPL(i) do {} while (0)

typedef __attribute__((address_space(1))) u64 gu64;
__device__ __forceinline__ void prstat(char* myws, int wv, u64 tstart, u64 tfs, u64 tend) {
  gu64* st = (gu64*)(myws + DBG_OFF) + blockIdx.x * 16;
  if (wv == 0) { __hip_atomic_store(st, tstart, __ATOMIC_RELAXED, __HIP_MEMORY_SCOPE_AGENT); __hip_atomic_store(st + 1, tfs, __ATOMIC_RELAXED, __HIP_MEMORY_SCOPE_AGENT); }
  __hip_atomic_store(st + 8 + wv, tend, __ATOMIC_RELAXED, __HIP_MEMORY_SCOPE_AGENT);
}
__device__ __forceinline__ int wmax(int v) {
  for (int o = 32; o; o >>= 1) { const int w = __shfl_xor(v, o); v = w > v ? w : v; }
  return v;
}
__device__ __forceinline__ void prreport(char* myws, int lane, u64 tstart) {
  gu64* st = (gu64*)(myws + DBG_OFF);
  gu64* pv = st + 256 * 16;
  const u64 prev = __hip_atomic_load(pv, __ATOMIC_RELAXED, __HIP_MEMORY_SCOPE_AGENT);
  if (tstart - prev > 20000) {
    while (__builtin_amdgcn_s_memrealtime() - tstart < 8000) __builtin_amdgcn_s_sleep(8);
    int nb = -(1 << 30), ls = nb, fs = nb, ack = nb, sr = nb;
#pragma unroll 1
    for (int k = 0; k < 4; ++k) {
      const int bb = k * 64 + lane;
      gu64* q = st + bb * 16;
      const int s0 = (int)(__hip_atomic_load(q, __ATOMIC_RELAXED, __HIP_MEMORY_SCOPE_AGENT) - tstart);
      nb = -s0 > nb ? -s0 : nb; ls = s0 > ls ? s0 : ls;
      if (bb < 56) { const int f = (int)(__hip_atomic_load(q + 1, __ATOMIC_RELAXED, __HIP_MEMORY_SCOPE_AGENT) - tstart); fs = f > fs ? f : fs; }
      for (int w = 0; w < 8; ++w) {
        const int e = (int)(__hip_atomic_load(q + 8 + w, __ATOMIC_RELAXED, __HIP_MEMORY_SCOPE_AGENT) - tstart);
        if (bb < 56) ack = e > ack ? e : ack;
        if (bb >= 64) sr = e > sr ? e : sr;
      }
    }
    nb = wmax(nb); ls = wmax(ls) + nb; fs = wmax(fs) + nb; ack = wmax(ack) + nb; sr = wmax(sr) + nb;
    auto qz = [](int v, int lo) { int r = (v - lo) / 25; return r < 0 ? 0 : r > 63 ? 63 : r; };
    int target = 10000 - nb;
    if (PRSTAT == 2) target += 400 * qz(sr, 3400) + 25600 * qz(ack, 3400);
    if (PRSTAT == 3) target += 400 * qz(ack, 3400) + 25600 * qz(sr, 3400);
    if (PRSTAT == 4) target += 400 * qz(ls, 0) + 25600 * qz(fs, 0);
    if (PRSTAT == 6) target += 400 * qz(fs, 0) + 25600 * qz(ack, 3400);
    if (PRSTAT == 7) target += 400 * qz(fs, 800) + 25600 * qz(ack, 3400);
    if (PRSTAT == 8) target += 400 * qz(fs, 1600) + 25600 * qz(ack, 3400);
    if (PRSTAT == 5) target += 400 * qz(ack, 3400) + 25600 * (63 - qz(sr, 3400));
    while ((int)(__builtin_amdgcn_s_memrealtime() - tstart) < target) __builtin_amdgcn_s_sleep(1);
  }
  if (lane == 0) __hip_atomic_store(pv, __builtin_amdgcn_s_memrealtime(), __ATOMIC_RELAXED, __HIP_MEMORY_SCOPE_AGENT);
}

__global__ void __launch_bounds__(512, 2) ag_gemm_kernel(Params p) {
  __shared__ __attribute__((aligned(16))) char smem[8 * TR_B];
  __shared__ u32 turn[2];
  const int lane = threadIdx.x & 63;
  const int wv = __builtin_amdgcn_readfirstlane(threadIdx.x >> 6);
  const int xg = blockIdx.x & 7;
  const int ci = blockIdx.x >> 3;
  int rank = p.rank;
  char* myws = p.tgt[NR - 1];
  char* pws = p.tgt[ci & 7];
  const unsigned char* xp = p.x;
  asm volatile("" : "+s"(rank), "+s"(myws), "+s"(pws), "+s"(xp));
  const int m = lane & 15, q = lane >> 4;

  gu32* cntp = (gu32*)(myws + CNT_OFF) + blockIdx.x * 8 + wv;

  if (__builtin_expect(ci < 8, 1)) {
    const u64 tstart = __builtin_amdgcn_s_memrealtime();
    if (wv > 0) __builtin_amdgcn_s_barrier();
    auto rx = __builtin_amdgcn_make_buffer_rsrc((void*)xp, 0, (int)SHARD, 0x00020000);
    const u32 voff = (u32)(lane >> 5) * KD + (lane & 31) * 16;
    const int kb = (xg + wv) & 7;
    const int dt = ci < 7 ? PUSH_DT : SELF_DT, t0 = ci < 7 ? PUSH_T0 : SELF_T0;
    char* tw = smem + wv * TR_B + (lane >> 5) * TR_ROW + (lane & 31) * 16;
    const char* tr = smem + wv * TR_B + m * TR_ROW + q * 16;
    const u32 soff = lane * 16;
    u32 call = 0;
    char* dst = nullptr;
    u64 tfs = 0;
#pragma unroll 1
    for (int u = 0; u < 4; ++u) {
      const int t = wv + 8 * u;
      if (t > 0) {
        const int due = t0 + t * dt - (t < PUSH_LEADT ? t * (dt - PUSH_DT0) + t0 : PUSH_PRE);
        while ((int)(__builtin_amdgcn_s_memrealtime() - tstart) < due) __builtin_amdgcn_s_sleep(2);
      }
      u32x4 e[8];
#pragma unroll
      for (int j = 0; j < 8; ++j) e[j] = __builtin_amdgcn_raw_buffer_load_b128(rx, voff + j * 2 * KD, t * TILE_B + kb * 512, 0);
      if (u == 0) {
        call = __hip_atomic_load(cntp, __ATOMIC_RELAXED, __HIP_MEMORY_SCOPE_AGENT);
        if (wv == 0) {
          if (lane < 2) __hip_atomic_store(&turn[lane], 0u, __ATOMIC_RELAXED, __HIP_MEMORY_SCOPE_WORKGROUP);
          asm volatile("s_waitcnt lgkmcnt(0)" ::: "memory");
          __builtin_amdgcn_s_barrier();
        }
      }
#pragma unroll
      for (int j = 0; j < 8; ++j) *(u32x4*)(tw + j * 2 * TR_ROW) = e[j];
#pragma unroll
      for (int i = 0; i < 8; ++i) { e[i] = *(const u32x4*)(tr + i * 64); e[i][0] = ~e[i][0]; }
      if (u == 0) {
        call = __builtin_amdgcn_readfirstlane(call) + 1;
        dst = pws + (size_t)(call % NBUF) * BUF_B + (size_t)rank * SHARD + kb * 8192;
      }
      auto rd = __builtin_amdgcn_make_buffer_rsrc((void*)(dst + t * TILE_B), 0, 8192, 0x00020000);
#pragma unroll
      for (int i = 0; i < 8; ++i) {
        if (i >= PUSH_W) asm volatile("s_waitcnt vmcnt(%0)" ::"n"(PUSH_W - 1) : "memory");
        else if (t > 0) {
          while ((int)__hip_atomic_load(&turn[0], __ATOMIC_RELAXED, __HIP_MEMORY_SCOPE_WORKGROUP) < t * 8 + i - (PUSH_W - 1)) __builtin_amdgcn_s_sleep(1);
        }
        if (i % PUSH_G == 0) {
          const int due = t0 + (t * 8 + i) * dt / 8;
          while ((int)(__builtin_amdgcn_s_memrealtime() - tstart) < due) __builtin_amdgcn_s_sleep(1);
        }
        if (i == 0) STAMPL(u);
        if (i == 0 && t == (PRSTAT == 7 ? 8 : PRSTAT == 8 ? 16 : 0)) tfs = __builtin_amdgcn_s_memrealtime();
        __builtin_amdgcn_raw_buffer_store_b128(e[i], rd, soff + i * 1024, 0, PUSH_AUX);
        __builtin_amdgcn_sched_barrier(0);
      }
#pragma unroll
      for (int k = PUSH_W - 1; k >= 0; --k) {
        asm volatile("s_waitcnt vmcnt(%0)" ::"n"(k) : "memory");
        __hip_atomic_fetch_max(&turn[0], (u32)(t * 8 + 8 - k), __ATOMIC_RELAXED, __HIP_MEMORY_SCOPE_WORKGROUP);
      }
      if (PRSTAT == 6 && t == 0) tfs = __builtin_amdgcn_s_memrealtime();
    }
    STAMPL(4);
    const u64 tack = __builtin_amdgcn_s_memrealtime();
    if (lane == 0) {
      const u32 done = __hip_atomic_fetch_add(&turn[1], 1u, __ATOMIC_RELAXED, __HIP_MEMORY_SCOPE_WORKGROUP);
      if (done == 7) __hip_atomic_store((gu32*)(pws + FLAG_OFF) + (rank * 8 + xg) * 32, call, __ATOMIC_RELAXED, __HIP_MEMORY_SCOPE_SYSTEM);
      __hip_atomic_store(cntp, call, __ATOMIC_RELAXED, __HIP_MEMORY_SCOPE_AGENT);
      prstat(myws, wv, tstart, tfs, tack);
    }
    if (blockIdx.x == 56 && wv == 0) prreport(myws, lane, tstart);
    return;
  }

  STAMP(0);
  const int g = (ci - 8) & 7;
  const int slot = (ci - 8) >> 3;
  const int src = (rank + xg) & 7;
  u32 call = __hip_atomic_load(cntp, __ATOMIC_RELAXED, __HIP_MEMORY_SCOPE_AGENT) + 1;

  const u64 tw0 = __builtin_amdgcn_s_memrealtime();
  i32x8 bf[4][4];
  {
    auto rw = __builtin_amdgcn_make_buffer_rsrc((void*)p.w, 0, (int)SHARD, 0x00020000);
    const u32 voff = (u32)(wv * 512 + q * 16) * NS + g * 64 + m * 4;
#pragma unroll
    for (int ks = 0; ks < 4; ++ks)
#pragma unroll
      for (int h = 0; h < 2; ++h) {
        if (W_DT || (ks == 0 && h == 0)) {
          const int due = W_DELAY + slot * W_SLOT + (ks * 2 + h) * W_DT;
          while ((int)(__builtin_amdgcn_s_memrealtime() - tw0) < due) __builtin_amdgcn_s_sleep(4);
        }
#pragma unroll
        for (int r = 0; r < 4; ++r) {
          u32 d0 = __builtin_amdgcn_raw_buffer_load_b32(rw, voff + (r * 4 + 0) * NS, (ks * 128 + h * 64) * NS, 0);
          u32 d1 = __builtin_amdgcn_raw_buffer_load_b32(rw, voff + (r * 4 + 1) * NS, (ks * 128 + h * 64) * NS, 0);
          u32 d2 = __builtin_amdgcn_raw_buffer_load_b32(rw, voff + (r * 4 + 2) * NS, (ks * 128 + h * 64) * NS, 0);
          u32 d3 = __builtin_amdgcn_raw_buffer_load_b32(rw, voff + (r * 4 + 3) * NS, (ks * 128 + h * 64) * NS, 0);
          u32 t0 = __builtin_amdgcn_perm(d1, d0, 0x05010400u), t1 = __builtin_amdgcn_perm(d1, d0, 0x07030602u);
          u32 t2 = __builtin_amdgcn_perm(d3, d2, 0x05010400u), t3 = __builtin_amdgcn_perm(d3, d2, 0x07030602u);
          bf[0][ks][h * 4 + r] = (int)__builtin_amdgcn_perm(t2, t0, 0x05040100u);
          bf[1][ks][h * 4 + r] = (int)__builtin_amdgcn_perm(t2, t0, 0x07060302u);
          bf[2][ks][h * 4 + r] = (int)__builtin_amdgcn_perm(t3, t1, 0x05040100u);
          bf[3][ks][h * 4 + r] = (int)__builtin_amdgcn_perm(t3, t1, 0x07060302u);
        }
      }
  }
  call = __builtin_amdgcn_readfirstlane(call);
  if (lane == 0) __hip_atomic_store(cntp, call, __ATOMIC_RELAXED, __HIP_MEMORY_SCOPE_AGENT);
  const float scale = p.sx[0] * p.sw[0];
  STAMP(1);

  auto rxg = __builtin_amdgcn_make_buffer_rsrc((void*)(myws + (size_t)(call % NBUF) * BUF_B), 0, (int)BUF_B, 0x00020000);
  auto rzr = __builtin_amdgcn_make_buffer_rsrc((void*)(myws + (size_t)((call + NBUF - 1) % NBUF) * BUF_B), 0, (int)BUF_B, 0x00020000);
  auto rout = __builtin_amdgcn_make_buffer_rsrc((void*)p.out, 0, NR * MS * NS * 4, 0x00020000);
  gu32* flags = (gu32*)(myws + FLAG_OFF);
  gu32* tmo = (gu32*)(myws + TMO_OFF);
  const u32 voff_g = (u32)wv * 8192 + lane * 16;
  const u32 voff_z = (u32)g * 8192 + threadIdx.x * 16;
  const u32 voff_o = ((wv * 2 + ((lane >> 4) & 1)) * NS + g * 64 + (lane & 15) * 4) * 4;

  u32x4 a[8];
#pragma unroll
  for (int i = 0; i < 8; ++i) a[i] = __builtin_amdgcn_raw_buffer_load_b128(rxg, voff_g + i * 1024, src * SHARD + slot * TILE_B, AUX_SYS);
  int it = 0;
  for (int t = slot; t < NT; t += NSLOT, ++it) {
    const u32 toff = src * SHARD + t * TILE_B;
    __builtin_amdgcn_raw_buffer_store_b128(u32x4{0, 0, 0, 0}, rzr, voff_z + toff, 0, ZERO_AUX);
    gu32* f = flags + (src * 8 + ((wv - t) & 7)) * 32;
    const u64 t0 = __builtin_amdgcn_s_memrealtime();
    u32 n = 0;
    bool sure = false;
    auto slow = [&]() {
      if ((++n & 31u) == 0) {
        if (__hip_atomic_load(f, __ATOMIC_RELAXED, __HIP_MEMORY_SCOPE_SYSTEM) >= call) { sure = true; return; }
        if (__hip_atomic_load(tmo, __ATOMIC_RELAXED, __HIP_MEMORY_SCOPE_AGENT) == call ||
            __builtin_amdgcn_s_memrealtime() - t0 > DEADLINE) {
          __hip_atomic_store(tmo, call, __ATOMIC_RELAXED, __HIP_MEMORY_SCOPE_AGENT);
          sure = true;
          return;
        }
      }
      __builtin_amdgcn_s_sleep(1);
    };
    if (__builtin_amdgcn_ballot_w64(a[PROBE][0] == 0) != 0) {
      while (!sure) {
        const u32 v = __builtin_amdgcn_raw_buffer_load_b32(rxg, wv * 8192 + PROBE * 1024 + 1008, toff, AUX_SYS);
        if (__builtin_amdgcn_ballot_w64(v == 0) == 0) break;
        slow();
      }
#pragma unroll
      for (int i = 0; i < 8; ++i) a[i] = __builtin_amdgcn_raw_buffer_load_b128(rxg, voff_g + i * 1024, toff, AUX_SYS);
    }
    STAMP(2 + 3 * it);
    f32x4 acc[4];
#pragma unroll
    for (int c = 0; c < 4; ++c) acc[c] = f32x4{0.f, 0.f, 0.f, 0.f};
#pragma unroll
    for (int ks = 0; ks < 4; ++ks) {
      while (!sure) {
        const u32 mn = a[2 * ks][0] < a[2 * ks + 1][0] ? a[2 * ks][0] : a[2 * ks + 1][0];
        if (__builtin_amdgcn_ballot_w64(mn == 0) == 0) break;
        slow();
#pragma unroll
        for (int i = 2 * ks; i < 8; ++i) a[i] = __builtin_amdgcn_raw_buffer_load_b128(rxg, voff_g + i * 1024, toff, AUX_SYS);
      }
      i32x8 af;
#pragma unroll
      for (int e = 0; e < 4; ++e) { af[e] = (int)a[2 * ks][e]; af[4 + e] = (int)a[2 * ks + 1][e]; }
      af[0] = ~af[0]; af[4] = ~af[4];
#pragma unroll
      for (int c = 0; c < 4; ++c)
        acc[c] = __builtin_amdgcn_mfma_scale_f32_16x16x128_f8f6f4(af, bf[c][ks], acc[c], 0, 1, 0, 0, 0, 0);
    }
    if (t + NSLOT < NT) {
#pragma unroll
      for (int i = 0; i < 8; ++i) a[i] = __builtin_amdgcn_raw_buffer_load_b128(rxg, voff_g + i * 1024, toff + NSLOT * TILE_B, AUX_SYS);
    }
    STAMP(3 + 3 * it);
    float* rb = (float*)smem + (it & 1) * 8192;
#pragma unroll
    for (int j = 0; j < 4; ++j)
      *(f32x4*)(rb + wv * 1024 + (q * 4 + j) * 64 + m * 4) = f32x4{acc[0][j], acc[1][j], acc[2][j], acc[3][j]};
    asm volatile("s_waitcnt lgkmcnt(0)" ::: "memory");
    __builtin_amdgcn_s_barrier();
    const float* rp = rb + (lane >> 5) * 4096 + wv * 128 + (lane & 31) * 4;
    f32x4 s4 = *(const f32x4*)rp;
#pragma unroll
    for (int v = 1; v < 4; ++v) s4 += *(const f32x4*)(rp + v * 1024);
    u32 o[2];
#pragma unroll
    for (int e = 0; e < 2; ++e) {
      auto sw = __builtin_amdgcn_permlane32_swap(__float_as_uint(s4[e]), __float_as_uint(s4[e + 2]), false, false);
      const float y = (__uint_as_float(sw[0]) + __uint_as_float(sw[1])) * scale;
      const float yc = __builtin_fminf(__builtin_fmaxf(y, -60.f), 60.f);
      o[e] = __float_as_uint(y * __builtin_amdgcn_rcpf(1.f + __expf(-yc)));
    }
    auto b0 = __builtin_amdgcn_permlane32_swap(o[0], o[0], false, false);
    auto b1 = __builtin_amdgcn_permlane32_swap(o[1], o[1], false, false);
    if (lane < 32)
      __builtin_amdgcn_raw_buffer_store_b128(u32x4{b0[0], b1[0], b0[1], b1[1]}, rout, voff_o + (u32)(src * MS + t * 16) * NS * 4, 0, OUT_AUX);
    STAMP(4 + 3 * it);
  }
  if (lane == 0) prstat(myws, wv, tw0, 0, __builtin_amdgcn_s_memrealtime());
}

}

extern "C" void kernel_launch(int rank, int nranks, ncclComm_t comm, hipStream_t stream,
                              void* const* d_in, const int* in_sizes, int n_in,
                              void* d_out, int out_size, const int* in_dtypes, int out_dtype,
                              void* const* d_out_all, void* const* d_ws_all, size_t ws_bytes) {
  (void)comm; (void)in_dtypes; (void)out_dtype; (void)d_out_all;
  if (nranks != NR || (unsigned)rank >= (unsigned)NR || n_in != 4 || ws_bytes < WS_NEED ||
      in_sizes[0] != (int)SHARD || in_sizes[1] != (int)SHARD || out_size != NR * MS * NS) {
    fprintf(stderr, "kernel_launch: unexpected problem shape\n");
    return;
  }
  Params p;
  p.x = (const unsigned char*)d_in[0];
  p.w = (const unsigned char*)d_in[1];
  p.sx = (const float*)d_in[2];
  p.sw = (const float*)d_in[3];
  p.out = (float*)d_out;
  for (int i = 0; i < NR; ++i) p.tgt[i] = (char*)d_ws_all[(rank + 1 + i) % NR];
  p.rank = rank;
  hipLaunchKernelGGL(ag_gemm_kernel, dim3(256), dim3(512), 0, stream, p);
}
```

```cpp
#define PRSTAT 6
#include <hip/hip_runtime.h>
#include <rccl/rccl.h>
#include <cstdio>

typedef unsigned int u32;
typedef unsigned long long u64;
typedef u32 u32x4 __attribute__((ext_vector_type(4)));
typedef int i32x8 __attribute__((ext_vector_type(8)));
typedef float f32x4 __attribute__((ext_vector_type(4)));
typedef float f32x2 __attribute__((ext_vector_type(2)));
typedef u32 u32x2 __attribute__((ext_vector_type(2)));
typedef __attribute__((address_space(1))) u32 gu32;

namespace {

constexpr int NR = 8;
constexpr int KD = 4096;
constexpr int MS = 512;
constexpr int NS = 512;
constexpr u32 SHARD = (u32)MS * KD;
constexpr u32 TILE_B = 16 * KD;
constexpr int NT = MS / 16;
constexpr u32 BUF_B = (u32)NR * SHARD;
constexpr int NBUF = 3;
constexpr size_t FLAG_OFF = (size_t)NBUF * BUF_B;
constexpr size_t FLAG_B = (size_t)NR * 8 * 128;
constexpr size_t CNT_OFF = FLAG_OFF + FLAG_B;
constexpr size_t CNT_B = 256 * 8 * 4;
constexpr size_t TMO_OFF = CNT_OFF + CNT_B;
constexpr size_t DBG_OFF = TMO_OFF + 4096;
constexpr size_t DBG_B = 256 * 128 * 8;
constexpr size_t WS_NEED = DBG_OFF + DBG_B;
constexpr u64 DEADLINE = 100000000ull;
constexpr int NSLOT = 3;
#define PUSH_W 7
#define PUSH_H 2
#define PUSH_DT 110
#define PUSH_T0 30
#define PUSH_G 2
#define PUSH_STAG 0
#define W_DELAY 150
#define W_DT 100
#define W_SLOT 50
#define SELF_DT 60
#define SELF_T0 200
#define PUSH_PRE 300
#define PUSH_LEADT 3
#define PUSH_DT0 20
#define TOUCH 2
#define TOUCH_N 32
#define PUSH_X 0
#define PUSH_CATCH 1
#define PUSH_AUX 17
#define PROBE 0
#define OUT_AUX 17
#define ZERO_AUX 17
constexpr int TR_ROW = 528;
constexpr int TR_B = 16 * TR_ROW;

struct Params {
  char* tgt[NR];
  const unsigned char* x;
  const unsigned char* w;
  const float* sx;
  const float* sw;
  float* out;
  int rank;
};

#define AUX_SYS 17

#define STAMP(i) do {} while (0)
#define STAMPL(i) do {} while (0)

typedef __attribute__((address_space(1))) u64 gu64;
__device__ __forceinline__ void prstat(char* myws, int wv, u64 tstart, u64 tfs, u64 tend) {
  gu64* st = (gu64*)(myws + DBG_OFF) + blockIdx.x * 16;
  if (wv == 0) { __hip_atomic_store(st, tstart, __ATOMIC_RELAXED, __HIP_MEMORY_SCOPE_AGENT); __hip_atomic_store(st + 1, tfs, __ATOMIC_RELAXED, __HIP_MEMORY_SCOPE_AGENT); }
  __hip_atomic_store(st + 8 + wv, tend, __ATOMIC_RELAXED, __HIP_MEMORY_SCOPE_AGENT);
}
__device__ __forceinline__ int wmax(int v) {
  for (int o = 32; o; o >>= 1) { const int w = __shfl_xor(v, o); v = w > v ? w : v; }
  return v;
}
__device__ __forceinline__ void prreport(char* myws, int lane, u64 tstart) {
  gu64* st = (gu64*)(myws + DBG_OFF);
  gu64* pv = st + 256 * 16;
  const u64 prev = __hip_atomic_load(pv, __ATOMIC_RELAXED, __HIP_MEMORY_SCOPE_AGENT);
  if (tstart - prev > 20000) {
    while (__builtin_amdgcn_s_memrealtime() - tstart < 8000) __builtin_amdgcn_s_sleep(8);
    int nb = -(1 << 30), ls = nb, fs = nb, ack = nb, sr = nb;
#pragma unroll 1
    for (int k = 0; k < 4; ++k) {
      const int bb = k * 64 + lane;
      gu64* q = st + bb * 16;
      const int s0 = (int)(__hip_atomic_load(q, __ATOMIC_RELAXED, __HIP_MEMORY_SCOPE_AGENT) - tstart);
      nb = -s0 > nb ? -s0 : nb; ls = s0 > ls ? s0 : ls;
      if (bb < 56) { const int f = (int)(__hip_atomic_load(q + 1, __ATOMIC_RELAXED, __HIP_MEMORY_SCOPE_AGENT) - tstart); fs = f > fs ? f : fs; }
      for (int w = 0; w < 8; ++w) {
        const int e = (int)(__hip_atomic_load(q + 8 + w, __ATOMIC_RELAXED, __HIP_MEMORY_SCOPE_AGENT) - tstart);
        if (bb < 56) ack = e > ack ? e : ack;
        if (bb >= 64) sr = e > sr ? e : sr;
      }
    }
    nb = wmax(nb); ls = wmax(ls) + nb; fs = wmax(fs) + nb; ack = wmax(ack) + nb; sr = wmax(sr) + nb;
    auto qz = [](int v, int lo) { int r = (v - lo) / 25; return r < 0 ? 0 : r > 63 ? 63 : r; };
    int target = 10000 - nb;
    if (PRSTAT == 2) target += 400 * qz(sr, 3400) + 25600 * qz(ack, 3400);
    if (PRSTAT == 3) target += 400 * qz(ack, 3400) + 25600 * qz(sr, 3400);
    if (PRSTAT == 4) target += 400 * qz(ls, 0) + 25600 * qz(fs, 0);
    if (PRSTAT == 6) target += 400 * qz(fs, 0) + 25600 * qz(ack, 3400);
    if (PRSTAT == 7) target += 400 * qz(fs, 800) + 25600 * qz(ack, 3400);
    if (PRSTAT == 8) target += 400 * qz(fs, 1600) + 25600 * qz(ack, 3400);
    if (PRSTAT == 5) target += 400 * qz(ack, 3400) + 25600 * (63 - qz(sr, 3400));
    while ((int)(__builtin_amdgcn_s_memrealtime() - tstart) < target) __builtin_amdgcn_s_sleep(1);
  }
  if (lane == 0) __hip_atomic_store(pv, __builtin_amdgcn_s_memrealtime(), __ATOMIC_RELAXED, __HIP_MEMORY_SCOPE_AGENT);
}

__global__ void __launch_bounds__(512, 2) ag_gemm_kernel(Params p) {
  __shared__ __attribute__((aligned(16))) char smem[8 * TR_B];
  __shared__ u32 turn[4];
  const int lane = threadIdx.x & 63;
  const int wv = __builtin_amdgcn_readfirstlane(threadIdx.x >> 6);
  const int xg = blockIdx.x & 7;
  const int ci = blockIdx.x >> 3;
  int rank = p.rank;
  char* myws = p.tgt[NR - 1];
  char* pws = p.tgt[ci & 7];
  const unsigned char* xp = p.x;
  asm volatile("" : "+s"(rank), "+s"(myws), "+s"(pws), "+s"(xp));
  const int m = lane & 15, q = lane >> 4;

  gu32* cntp = (gu32*)(myws + CNT_OFF) + blockIdx.x * 8 + wv;

  if (__builtin_expect(ci < 8, 1)) {
    const u64 tstart = __builtin_amdgcn_s_memrealtime();
    if (wv > 0) __builtin_amdgcn_s_barrier();
    auto rx = __builtin_amdgcn_make_buffer_rsrc((void*)xp, 0, (int)SHARD, 0x00020000);
    const u32 voff = (u32)(lane >> 5) * KD + (lane & 31) * 16;
    const int kb = (xg + wv) & 7;
    const int dt = ci < 7 ? PUSH_DT : SELF_DT, t0 = ci < 7 ? PUSH_T0 : SELF_T0;
    char* tw = smem + wv * TR_B + (lane >> 5) * TR_ROW + (lane & 31) * 16;
    const char* tr = smem + wv * TR_B + m * TR_ROW + q * 16;
    const u32 soff = lane * 16;
    u32 call = 0;
    int shift = 0;
    char* dst = nullptr;
    u64 tfs = 0;
#pragma unroll 1
    for (int u = 0; u < 4; ++u) {
      const int t = wv + 8 * u;
      if (t > 0) {
        const int due = t0 + t * dt - (t < PUSH_LEADT ? t * (dt - PUSH_DT0) + t0 : PUSH_PRE);
        while ((int)(__builtin_amdgcn_s_memrealtime() - tstart) < due) __builtin_amdgcn_s_sleep(2);
      }
      u32x4 e[8];
#pragma unroll
      for (int j = 0; j < 8; ++j) e[j] = __builtin_amdgcn_raw_buffer_load_b128(rx, voff + j * 2 * KD, t * TILE_B + kb * 512, 0);
      if (u == 0) {
        call = __hip_atomic_load(cntp, __ATOMIC_RELAXED, __HIP_MEMORY_SCOPE_AGENT);
        if (wv == 0) {
          if (lane < 4) __hip_atomic_store(&turn[lane], 0u, __ATOMIC_RELAXED, __HIP_MEMORY_SCOPE_WORKGROUP);
          asm volatile("s_waitcnt lgkmcnt(0)" ::: "memory");
          __builtin_amdgcn_s_barrier();
        }
      }
#pragma unroll
      for (int j = 0; j < 8; ++j) *(u32x4*)(tw + j * 2 * TR_ROW) = e[j];
#pragma unroll
      for (int i = 0; i < 8; ++i) { e[i] = *(const u32x4*)(tr + i * 64); e[i][0] = ~e[i][0]; }
      if (u == 0) {
        call = __builtin_amdgcn_readfirstlane(call) + 1;
        dst = pws + (size_t)(call % NBUF) * BUF_B + (size_t)rank * SHARD + kb * 8192;
      }
      auto rd = __builtin_amdgcn_make_buffer_rsrc((void*)(dst + t * TILE_B), 0, 8192, 0x00020000);
#pragma unroll
      for (int i = 0; i < 8; ++i) {
        if (i >= PUSH_W) asm volatile("s_waitcnt vmcnt(%0)" ::"n"(PUSH_W - 1) : "memory");
        else if (t > 0) {
          while ((int)__hip_atomic_load(&turn[0], __ATOMIC_RELAXED, __HIP_MEMORY_SCOPE_WORKGROUP) < t * 8 + i - (PUSH_W - 1)) __builtin_amdgcn_s_sleep(1);
        }
        if (i % PUSH_G == 0) {
          const int due = t0 + (t * 8 + i) * dt / 8;
          while ((int)(__builtin_amdgcn_s_memrealtime() - tstart) < due) __builtin_amdgcn_s_sleep(1);
        }
        if (i == 0) STAMPL(u);
        if (i == 0 && t == (PRSTAT == 7 ? 8 : PRSTAT == 8 ? 16 : 0)) tfs = __builtin_amdgcn_s_memrealtime();
        __builtin_amdgcn_raw_buffer_store_b128(e[i], rd, soff + i * 1024, 0, PUSH_AUX);
        __builtin_amdgcn_sched_barrier(0);
      }
#pragma unroll
      for (int k = PUSH_W - 1; k >= 0; --k) {
        asm volatile("s_waitcnt vmcnt(%0)" ::"n"(k) : "memory");
        __hip_atomic_fetch_max(&turn[0], (u32)(t * 8 + 8 - k), __ATOMIC_RELAXED, __HIP_MEMORY_SCOPE_WORKGROUP);
      }
      if (PRSTAT == 6 && t == 0) tfs = __builtin_amdgcn_s_memrealtime();
    }
    STAMPL(4);
    const u64 tack = __builtin_amdgcn_s_memrealtime();
    if (lane == 0) {
      const u32 done = __hip_atomic_fetch_add(&turn[1], 1u, __ATOMIC_RELAXED, __HIP_MEMORY_SCOPE_WORKGROUP);
      if (done == 7) __hip_atomic_store((gu32*)(pws + FLAG_OFF) + (rank * 8 + xg) * 32, call, __ATOMIC_RELAXED, __HIP_MEMORY_SCOPE_SYSTEM);
      __hip_atomic_store(cntp, call, __ATOMIC_RELAXED, __HIP_MEMORY_SCOPE_AGENT);
      prstat(myws, wv, tstart, tfs, tack);
    }
    if (blockIdx.x == 56 && wv == 0) prreport(myws, lane, tstart);
    return;
  }

  STAMP(0);
  const int g = (ci - 8) & 7;
  const int slot = (ci - 8) >> 3;
  const int src = (rank + xg) & 7;
  if (ci < 15 && wv < NBUF) {
    auto rt = __builtin_amdgcn_make_buffer_rsrc((void*)(pws + (size_t)wv * BUF_B + (size_t)rank * SHARD), 0, (int)SHARD, 0x00020000);
    const u32 v = __builtin_amdgcn_raw_buffer_load_b32(rt, lane * (SHARD / TOUCH_N), 0, AUX_SYS);
    asm volatile("" ::"v"(v));
  }
  u32 call = __hip_atomic_load(cntp, __ATOMIC_RELAXED, __HIP_MEMORY_SCOPE_AGENT) + 1;

  const u64 tw0 = __builtin_amdgcn_s_memrealtime();
  i32x8 bf[4][4];
  {
    auto rw = __builtin_amdgcn_make_buffer_rsrc((void*)p.w, 0, (int)SHARD, 0x00020000);
    const u32 voff = (u32)(wv * 512 + q * 16) * NS + g * 64 + m * 4;
#pragma unroll
    for (int ks = 0; ks < 4; ++ks)
#pragma unroll
      for (int h = 0; h < 2; ++h) {
        if (W_DT || (ks == 0 && h == 0)) {
          const int due = W_DELAY + slot * W_SLOT + (ks * 2 + h) * W_DT;
          while ((int)(__builtin_amdgcn_s_memrealtime() - tw0) < due) __builtin_amdgcn_s_sleep(4);
        }
#pragma unroll
        for (int r = 0; r < 4; ++r) {
          u32 d0 = __builtin_amdgcn_raw_buffer_load_b32(rw, voff + (r * 4 + 0) * NS, (ks * 128 + h * 64) * NS, 0);
          u32 d1 = __builtin_amdgcn_raw_buffer_load_b32(rw, voff + (r * 4 + 1) * NS, (ks * 128 + h * 64) * NS, 0);
          u32 d2 = __builtin_amdgcn_raw_buffer_load_b32(rw, voff + (r * 4 + 2) * NS, (ks * 128 + h * 64) * NS, 0);
          u32 d3 = __builtin_amdgcn_raw_buffer_load_b32(rw, voff + (r * 4 + 3) * NS, (ks * 128 + h * 64) * NS, 0);
          u32 t0 = __builtin_amdgcn_perm(d1, d0, 0x05010400u), t1 = __builtin_amdgcn_perm(d1, d0, 0x07030602u);
          u32 t2 = __builtin_amdgcn_perm(d3, d2, 0x05010400u), t3 = __builtin_amdgcn_perm(d3, d2, 0x07030602u);
          bf[0][ks][h * 4 + r] = (int)__builtin_amdgcn_perm(t2, t0, 0x05040100u);
          bf[1][ks][h * 4 + r] = (int)__builtin_amdgcn_perm(t2, t0, 0x07060302u);
          bf[2][ks][h * 4 + r] = (int)__builtin_amdgcn_perm(t3, t1, 0x05040100u);
          bf[3][ks][h * 4 + r] = (int)__builtin_amdgcn_perm(t3, t1, 0x07060302u);
        }
      }
  }
  call = __builtin_amdgcn_readfirstlane(call);
  if (lane == 0) __hip_atomic_store(cntp, call, __ATOMIC_RELAXED, __HIP_MEMORY_SCOPE_AGENT);
  const float scale = p.sx[0] * p.sw[0];
  STAMP(1);

  auto rxg = __builtin_amdgcn_make_buffer_rsrc((void*)(myws + (size_t)(call % NBUF) * BUF_B), 0, (int)BUF_B, 0x00020000);
  auto rzr = __builtin_amdgcn_make_buffer_rsrc((void*)(myws + (size_t)((call + NBUF - 1) % NBUF) * BUF_B), 0, (int)BUF_B, 0x00020000);
  auto rout = __builtin_amdgcn_make_buffer_rsrc((void*)p.out, 0, NR * MS * NS * 4, 0x00020000);
  gu32* flags = (gu32*)(myws + FLAG_OFF);
  gu32* tmo = (gu32*)(myws + TMO_OFF);
  const u32 voff_g = (u32)wv * 8192 + lane * 16;
  const u32 voff_z = (u32)g * 8192 + threadIdx.x * 16;
  const u32 voff_o = ((wv * 2 + ((lane >> 4) & 1)) * NS + g * 64 + (lane & 15) * 4) * 4;

  u32x4 a[8];
#pragma unroll
  for (int i = 0; i < 8; ++i) a[i] = __builtin_amdgcn_raw_buffer_load_b128(rxg, voff_g + i * 1024, src * SHARD + slot * TILE_B, AUX_SYS);
  int it = 0;
  for (int t = slot; t < NT; t += NSLOT, ++it) {
    const u32 toff = src * SHARD + t * TILE_B;
    __builtin_amdgcn_raw_buffer_store_b128(u32x4{0, 0, 0, 0}, rzr, voff_z + toff, 0, ZERO_AUX);
    gu32* f = flags + (src * 8 + ((wv - t) & 7)) * 32;
    const u64 t0 = __builtin_amdgcn_s_memrealtime();
    u32 n = 0;
    bool sure = false;
    auto slow = [&]() {
      if ((++n & 31u) == 0) {
        if (__hip_atomic_load(f, __ATOMIC_RELAXED, __HIP_MEMORY_SCOPE_SYSTEM) >= call) { sure = true; return; }
        if (__hip_atomic_load(tmo, __ATOMIC_RELAXED, __HIP_MEMORY_SCOPE_AGENT) == call ||
            __builtin_amdgcn_s_memrealtime() - t0 > DEADLINE) {
          __hip_atomic_store(tmo, call, __ATOMIC_RELAXED, __HIP_MEMORY_SCOPE_AGENT);
          sure = true;
          return;
        }
      }
      __builtin_amdgcn_s_sleep(1);
    };
    if (__builtin_amdgcn_ballot_w64(a[PROBE][0] == 0) != 0) {
      while (!sure) {
        const u32 v = __builtin_amdgcn_raw_buffer_load_b32(rxg, wv * 8192 + PROBE * 1024 + 1008, toff, AUX_SYS);
        if (__builtin_amdgcn_ballot_w64(v == 0) == 0) break;
        slow();
      }
#pragma unroll
      for (int i = 0; i < 8; ++i) a[i] = __builtin_amdgcn_raw_buffer_load_b128(rxg, voff_g + i * 1024, toff, AUX_SYS);
    }
    STAMP(2 + 3 * it);
    f32x4 acc[4];
#pragma unroll
    for (int c = 0; c < 4; ++c) acc[c] = f32x4{0.f, 0.f, 0.f, 0.f};
#pragma unroll
    for (int ks = 0; ks < 4; ++ks) {
      while (!sure) {
        const u32 mn = a[2 * ks][0] < a[2 * ks + 1][0] ? a[2 * ks][0] : a[2 * ks + 1][0];
        if (__builtin_amdgcn_ballot_w64(mn == 0) == 0) break;
        slow();
#pragma unroll
        for (int i = 2 * ks; i < 8; ++i) a[i] = __builtin_amdgcn_raw_buffer_load_b128(rxg, voff_g + i * 1024, toff, AUX_SYS);
      }
      i32x8 af;
#pragma unroll
      for (int e = 0; e < 4; ++e) { af[e] = (int)a[2 * ks][e]; af[4 + e] = (int)a[2 * ks + 1][e]; }
      af[0] = ~af[0]; af[4] = ~af[4];
#pragma unroll
      for (int c = 0; c < 4; ++c)
        acc[c] = __builtin_amdgcn_mfma_scale_f32_16x16x128_f8f6f4(af, bf[c][ks], acc[c], 0, 1, 0, 0, 0, 0);
    }
    if (t + NSLOT < NT) {
#pragma unroll
      for (int i = 0; i < 8; ++i) a[i] = __builtin_amdgcn_raw_buffer_load_b128(rxg, voff_g + i * 1024, toff + NSLOT * TILE_B, AUX_SYS);
    }
    STAMP(3 + 3 * it);
    float* rb = (float*)smem + (it & 1) * 8192;
#pragma unroll
    for (int j = 0; j < 4; ++j)
      *(f32x4*)(rb + wv * 1024 + (q * 4 + j) * 64 + m * 4) = f32x4{acc[0][j], acc[1][j], acc[2][j], acc[3][j]};
    asm volatile("s_waitcnt lgkmcnt(0)" ::: "memory");
    __builtin_amdgcn_s_barrier();
    const float* rp = rb + (lane >> 5) * 4096 + wv * 128 + (lane & 31) * 4;
    f32x4 s4 = *(const f32x4*)rp;
#pragma unroll
    for (int v = 1; v < 4; ++v) s4 += *(const f32x4*)(rp + v * 1024);
    u32 o[2];
#pragma unroll
    for (int e = 0; e < 2; ++e) {
      auto sw = __builtin_amdgcn_permlane32_swap(__float_as_uint(s4[e]), __float_as_uint(s4[e + 2]), false, false);
      const float y = (__uint_as_float(sw[0]) + __uint_as_float(sw[1])) * scale;
      const float yc = __builtin_fminf(__builtin_fmaxf(y, -60.f), 60.f);
      o[e] = __float_as_uint(y * __builtin_amdgcn_rcpf(1.f + __expf(-yc)));
    }
    auto b0 = __builtin_amdgcn_permlane32_swap(o[0], o[0], false, false);
    auto b1 = __builtin_amdgcn_permlane32_swap(o[1], o[1], false, false);
    if (lane < 32)
      __builtin_amdgcn_raw_buffer_store_b128(u32x4{b0[0], b1[0], b0[1], b1[1]}, rout, voff_o + (u32)(src * MS + t * 16) * NS * 4, 0, OUT_AUX);
    STAMP(4 + 3 * it);
  }
  if (lane == 0) prstat(myws, wv, tw0, 0, __builtin_amdgcn_s_memrealtime());
}

}

extern "C" void kernel_launch(int rank, int nranks, ncclComm_t comm, hipStream_t stream,
                              void* const* d_in, const int* in_sizes, int n_in,
                              void* d_out, int out_size, const int* in_dtypes, int out_dtype,
                              void* const* d_out_all, void* const* d_ws_all, size_t ws_bytes) {
  (void)comm; (void)in_dtypes; (void)out_dtype; (void)d_out_all;
  if (nranks != NR || (unsigned)rank >= (unsigned)NR || n_in != 4 || ws_bytes < WS_NEED ||
      in_sizes[0] != (int)SHARD || in_sizes[1] != (int)SHARD || out_size != NR * MS * NS) {
    fprintf(stderr, "kernel_launch: unexpected problem shape\n");
    return;
  }
  Params p;
  p.x = (const unsigned char*)d_in[0];
  p.w = (const unsigned char*)d_in[1];
  p.sx = (const float*)d_in[2];
  p.sw = (const float*)d_in[3];
  p.out = (float*)d_out;
  for (int i = 0; i < NR; ++i) p.tgt[i] = (char*)d_ws_all[(rank + 1 + i) % NR];
  p.rank = rank;
  hipLaunchKernelGGL(ag_gemm_kernel, dim3(256), dim3(512), 0, stream, p);
}
```

```cpp
#define DIAG
#define DIAG_EXIT
#include <hip/hip_runtime.h>
#include <rccl/rccl.h>
#include <cstdio>

typedef unsigned int u32;
typedef unsigned long long u64;
typedef u32 u32x4 __attribute__((ext_vector_type(4)));
typedef int i32x8 __attribute__((ext_vector_type(8)));
typedef float f32x4 __attribute__((ext_vector_type(4)));
typedef float f32x2 __attribute__((ext_vector_type(2)));
typedef u32 u32x2 __attribute__((ext_vector_type(2)));
typedef __attribute__((address_space(1))) u32 gu32;

namespace {

constexpr int NR = 8;
constexpr int KD = 4096;
constexpr int MS = 512;
constexpr int NS = 512;
constexpr u32 SHARD = (u32)MS * KD;
constexpr u32 TILE_B = 16 * KD;
constexpr int NT = MS / 16;
constexpr u32 BUF_B = (u32)NR * SHARD;
constexpr int NBUF = 3;
constexpr size_t FLAG_OFF = (size_t)NBUF * BUF_B;
constexpr size_t FLAG_B = (size_t)NR * 8 * 128;
constexpr size_t CNT_OFF = FLAG_OFF + FLAG_B;
constexpr size_t CNT_B = 256 * 8 * 4;
constexpr size_t TMO_OFF = CNT_OFF + CNT_B;
constexpr size_t DBG_OFF = TMO_OFF + 4096;
constexpr size_t DBG_B = 256 * 128 * 8;
constexpr size_t WS_NEED = DBG_OFF + DBG_B;
constexpr u64 DEADLINE = 100000000ull;
constexpr int NSLOT = 3;
#define PUSH_W 7
#define PUSH_H 2
#define PUSH_DT 110
#define PUSH_T0 30
#define PUSH_G 2
#define PUSH_STAG 0
#define W_DELAY 150
#define W_DT 100
#define W_SLOT 50
#define SELF_DT 60
#define SELF_T0 200
#define PUSH_PRE 300
#define PUSH_LEADT 3
#define PUSH_DT0 20
#define TOUCH 0
#define TOUCH_N 32
#define PUSH_X 0
#define PUSH_CATCH 1
#define PUSH_AUX 17
#define PROBE 0
#define OUT_AUX 17
#define ZERO_AUX 17
constexpr int TR_ROW = 528;
constexpr int TR_B = 16 * TR_ROW;

struct Params {
  char* tgt[NR];
  const unsigned char* x;
  const unsigned char* w;
  const float* sx;
  const float* sw;
  float* out;
  int rank;
};

#define AUX_SYS 17

#define STAMP(i) do { if (lane == 0 && wv == 0) dbg[i] = __builtin_amdgcn_s_memrealtime(); } while (0)
#define STAMPL(i) do { tl[i] = __builtin_amdgcn_s_memrealtime(); } while (0)

__global__ void __launch_bounds__(512, 2) ag_gemm_kernel(Params p) {
  __shared__ __attribute__((aligned(16))) char smem[8 * TR_B];
  __shared__ u32 turn[4];
  const int lane = threadIdx.x & 63;
  const int wv = __builtin_amdgcn_readfirstlane(threadIdx.x >> 6);
  const int xg = blockIdx.x & 7;
  const int ci = blockIdx.x >> 3;
  int rank = p.rank;
  char* myws = p.tgt[NR - 1];
  char* pws = p.tgt[ci & 7];
  const unsigned char* xp = p.x;
  asm volatile("" : "+s"(rank), "+s"(myws), "+s"(pws), "+s"(xp));
  auto* dbg = (__attribute__((address_space(1))) u64*)(myws + DBG_OFF) + blockIdx.x * 128;
  const int m = lane & 15, q = lane >> 4;

  gu32* cntp = (gu32*)(myws + CNT_OFF) + blockIdx.x * 8 + wv;

  if (__builtin_expect(ci < 8, 1)) {
    u64 tl[8];
    const u64 tstart = __builtin_amdgcn_s_memrealtime();
    if (wv > 0) __builtin_amdgcn_s_barrier();
    auto rx = __builtin_amdgcn_make_buffer_rsrc((void*)xp, 0, (int)SHARD, 0x00020000);
    const u32 voff = (u32)(lane >> 5) * KD + (lane & 31) * 16;
    const int kb = (xg + wv) & 7;
    const int dt = ci < 7 ? PUSH_DT : SELF_DT, t0 = ci < 7 ? PUSH_T0 : SELF_T0;
    char* tw = smem + wv * TR_B + (lane >> 5) * TR_ROW + (lane & 31) * 16;
    const char* tr = smem + wv * TR_B + m * TR_ROW + q * 16;
    const u32 soff = lane * 16;
    u32 call = 0;
    int shift = 0;
    char* dst = nullptr;
#pragma unroll 1
    for (int u = 0; u < 4; ++u) {
      const int t = wv + 8 * u;
      if (t > 0) {
        const int due = t0 + t * dt - (t < PUSH_LEADT ? t * (dt - PUSH_DT0) + t0 : PUSH_PRE);
        while ((int)(__builtin_amdgcn_s_memrealtime() - tstart) < due) __builtin_amdgcn_s_sleep(2);
      }
      u32x4 e[8];
#pragma unroll
      for (int j = 0; j < 8; ++j) e[j] = __builtin_amdgcn_raw_buffer_load_b128(rx, voff + j * 2 * KD, t * TILE_B + kb * 512, 0);
      if (u == 0) {
        call = __hip_atomic_load(cntp, __ATOMIC_RELAXED, __HIP_MEMORY_SCOPE_AGENT);
        if (wv == 0) {
          if (lane < 4) __hip_atomic_store(&turn[lane], 0u, __ATOMIC_RELAXED, __HIP_MEMORY_SCOPE_WORKGROUP);
          asm volatile("s_waitcnt lgkmcnt(0)" ::: "memory");
          __builtin_amdgcn_s_barrier();
        }
      }
#pragma unroll
      for (int j = 0; j < 8; ++j) *(u32x4*)(tw + j * 2 * TR_ROW) = e[j];
#pragma unroll
      for (int i = 0; i < 8; ++i) { e[i] = *(const u32x4*)(tr + i * 64); e[i][0] = ~e[i][0]; }
      if (u == 0) {
        call = __builtin_amdgcn_readfirstlane(call) + 1;
        dst = pws + (size_t)(call % NBUF) * BUF_B + (size_t)rank * SHARD + kb * 8192;
      }
      auto rd = __builtin_amdgcn_make_buffer_rsrc((void*)(dst + t * TILE_B), 0, 8192, 0x00020000);
#pragma unroll
      for (int i = 0; i < 8; ++i) {
        if (i >= PUSH_W) asm volatile("s_waitcnt vmcnt(%0)" ::"n"(PUSH_W - 1) : "memory");
        else if (t > 0) {
          while ((int)__hip_atomic_load(&turn[0], __ATOMIC_RELAXED, __HIP_MEMORY_SCOPE_WORKGROUP) < t * 8 + i - (PUSH_W - 1)) __builtin_amdgcn_s_sleep(1);
        }
        if (i % PUSH_G == 0) {
          const int due = t0 + (t * 8 + i) * dt / 8;
          while ((int)(__builtin_amdgcn_s_memrealtime() - tstart) < due) __builtin_amdgcn_s_sleep(1);
        }
        if (i == 0) STAMPL(u);
        __builtin_amdgcn_raw_buffer_store_b128(e[i], rd, soff + i * 1024, 0, PUSH_AUX);
        __builtin_amdgcn_sched_barrier(0);
      }
#pragma unroll
      for (int k = PUSH_W - 1; k >= 0; --k) {
        asm volatile("s_waitcnt vmcnt(%0)" ::"n"(k) : "memory");
        __hip_atomic_fetch_max(&turn[0], (u32)(t * 8 + 8 - k), __ATOMIC_RELAXED, __HIP_MEMORY_SCOPE_WORKGROUP);
      }
    }
    STAMPL(4);
    if (lane == 0) {
      const u32 done = __hip_atomic_fetch_add(&turn[1], 1u, __ATOMIC_RELAXED, __HIP_MEMORY_SCOPE_WORKGROUP);
      if (done == 7) __hip_atomic_store((gu32*)(pws + FLAG_OFF) + (rank * 8 + xg) * 32, call, __ATOMIC_RELAXED, __HIP_MEMORY_SCOPE_SYSTEM);
      __hip_atomic_store(cntp, call, __ATOMIC_RELAXED, __HIP_MEMORY_SCOPE_AGENT);
      if (wv == 0) { dbg[2] = dbg[0]; dbg[3] = call; dbg[0] = tstart; dbg[1] = __builtin_amdgcn_s_getreg((31 << 11) | 20); }
      for (int i = 0; i < 5; ++i) dbg[8 + wv * 8 + i] = tl[i];
    }
    return;
  }

  STAMP(0);
  const int g = (ci - 8) & 7;
  const int slot = (ci - 8) >> 3;
  const int src = (rank + xg) & 7;
  u32 call = __hip_atomic_load(cntp, __ATOMIC_RELAXED, __HIP_MEMORY_SCOPE_AGENT) + 1;

  const u64 tw0 = __builtin_amdgcn_s_memrealtime();
  i32x8 bf[4][4];
  {
    auto rw = __builtin_amdgcn_make_buffer_rsrc((void*)p.w, 0, (int)SHARD, 0x00020000);
    const u32 voff = (u32)(wv * 512 + q * 16) * NS + g * 64 + m * 4;
#pragma unroll
    for (int ks = 0; ks < 4; ++ks)
#pragma unroll
      for (int h = 0; h < 2; ++h) {
        if (W_DT || (ks == 0 && h == 0)) {
          const int due = W_DELAY + slot * W_SLOT + (ks * 2 + h) * W_DT;
          while ((int)(__builtin_amdgcn_s_memrealtime() - tw0) < due) __builtin_amdgcn_s_sleep(4);
        }
#pragma unroll
        for (int r = 0; r < 4; ++r) {
          u32 d0 = __builtin_amdgcn_raw_buffer_load_b32(rw, voff + (r * 4 + 0) * NS, (ks * 128 + h * 64) * NS, 0);
          u32 d1 = __builtin_amdgcn_raw_buffer_load_b32(rw, voff + (r * 4 + 1) * NS, (ks * 128 + h * 64) * NS, 0);
          u32 d2 = __builtin_amdgcn_raw_buffer_load_b32(rw, voff + (r * 4 + 2) * NS, (ks * 128 + h * 64) * NS, 0);
          u32 d3 = __builtin_amdgcn_raw_buffer_load_b32(rw, voff + (r * 4 + 3) * NS, (ks * 128 + h * 64) * NS, 0);
          u32 t0 = __builtin_amdgcn_perm(d1, d0, 0x05010400u), t1 = __builtin_amdgcn_perm(d1, d0, 0x07030602u);
          u32 t2 = __builtin_amdgcn_perm(d3, d2, 0x05010400u), t3 = __builtin_amdgcn_perm(d3, d2, 0x07030602u);
          bf[0][ks][h * 4 + r] = (int)__builtin_amdgcn_perm(t2, t0, 0x05040100u);
          bf[1][ks][h * 4 + r] = (int)__builtin_amdgcn_perm(t2, t0, 0x07060302u);
          bf[2][ks][h * 4 + r] = (int)__builtin_amdgcn_perm(t3, t1, 0x05040100u);
          bf[3][ks][h * 4 + r] = (int)__builtin_amdgcn_perm(t3, t1, 0x07060302u);
        }
      }
  }
  call = __builtin_amdgcn_readfirstlane(call);
  if (lane == 0) __hip_atomic_store(cntp, call, __ATOMIC_RELAXED, __HIP_MEMORY_SCOPE_AGENT);
  const float scale = p.sx[0] * p.sw[0];
  STAMP(1);

  auto rxg = __builtin_amdgcn_make_buffer_rsrc((void*)(myws + (size_t)(call % NBUF) * BUF_B), 0, (int)BUF_B, 0x00020000);
  auto rzr = __builtin_amdgcn_make_buffer_rsrc((void*)(myws + (size_t)((call + NBUF - 1) % NBUF) * BUF_B), 0, (int)BUF_B, 0x00020000);
  auto rout = __builtin_amdgcn_make_buffer_rsrc((void*)p.out, 0, NR * MS * NS * 4, 0x00020000);
  gu32* flags = (gu32*)(myws + FLAG_OFF);
  gu32* tmo = (gu32*)(myws + TMO_OFF);
  const u32 voff_g = (u32)wv * 8192 + lane * 16;
  const u32 voff_z = (u32)g * 8192 + threadIdx.x * 16;
  const u32 voff_o = ((wv * 2 + ((lane >> 4) & 1)) * NS + g * 64 + (lane & 15) * 4) * 4;

  u32x4 a[8];
#pragma unroll
  for (int i = 0; i < 8; ++i) a[i] = __builtin_amdgcn_raw_buffer_load_b128(rxg, voff_g + i * 1024, src * SHARD + slot * TILE_B, AUX_SYS);
  int it = 0;
  for (int t = slot; t < NT; t += NSLOT, ++it) {
    const u32 toff = src * SHARD + t * TILE_B;
    __builtin_amdgcn_raw_buffer_store_b128(u32x4{0, 0, 0, 0}, rzr, voff_z + toff, 0, ZERO_AUX);
    gu32* f = flags + (src * 8 + ((wv - t) & 7)) * 32;
    const u64 t0 = __builtin_amdgcn_s_memrealtime();
    u32 n = 0;
    bool sure = false;
    auto slow = [&]() {
      if ((++n & 31u) == 0) {
        if (__hip_atomic_load(f, __ATOMIC_RELAXED, __HIP_MEMORY_SCOPE_SYSTEM) >= call) { sure = true; return; }
        if (__hip_atomic_load(tmo, __ATOMIC_RELAXED, __HIP_MEMORY_SCOPE_AGENT) == call ||
            __builtin_amdgcn_s_memrealtime() - t0 > DEADLINE) {
          __hip_atomic_store(tmo, call, __ATOMIC_RELAXED, __HIP_MEMORY_SCOPE_AGENT);
          sure = true;
          return;
        }
      }
      __builtin_amdgcn_s_sleep(1);
    };
    if (__builtin_amdgcn_ballot_w64(a[PROBE][0] == 0) != 0) {
      while (!sure) {
        const u32 v = __builtin_amdgcn_raw_buffer_load_b32(rxg, wv * 8192 + PROBE * 1024 + 1008, toff, AUX_SYS);
        if (__builtin_amdgcn_ballot_w64(v == 0) == 0) break;
        slow();
      }
#pragma unroll
      for (int i = 0; i < 8; ++i) a[i] = __builtin_amdgcn_raw_buffer_load_b128(rxg, voff_g + i * 1024, toff, AUX_SYS);
    }
    STAMP(2 + 3 * it);
    f32x4 acc[4];
#pragma unroll
    for (int c = 0; c < 4; ++c) acc[c] = f32x4{0.f, 0.f, 0.f, 0.f};
#pragma unroll
    for (int ks = 0; ks < 4; ++ks) {
      while (!sure) {
        const u32 mn = a[2 * ks][0] < a[2 * ks + 1][0] ? a[2 * ks][0] : a[2 * ks + 1][0];
        if (__builtin_amdgcn_ballot_w64(mn == 0) == 0) break;
        slow();
#pragma unroll
        for (int i = 2 * ks; i < 8; ++i) a[i] = __builtin_amdgcn_raw_buffer_load_b128(rxg, voff_g + i * 1024, toff, AUX_SYS);
      }
      i32x8 af;
#pragma unroll
      for (int e = 0; e < 4; ++e) { af[e] = (int)a[2 * ks][e]; af[4 + e] = (int)a[2 * ks + 1][e]; }
      af[0] = ~af[0]; af[4] = ~af[4];
#pragma unroll
      for (int c = 0; c < 4; ++c)
        acc[c] = __builtin_amdgcn_mfma_scale_f32_16x16x128_f8f6f4(af, bf[c][ks], acc[c], 0, 1, 0, 0, 0, 0);
    }
    if (t + NSLOT < NT) {
#pragma unroll
      for (int i = 0; i < 8; ++i) a[i] = __builtin_amdgcn_raw_buffer_load_b128(rxg, voff_g + i * 1024, toff + NSLOT * TILE_B, AUX_SYS);
    }
    STAMP(3 + 3 * it);
    float* rb = (float*)smem + (it & 1) * 8192;
#pragma unroll
    for (int j = 0; j < 4; ++j)
      *(f32x4*)(rb + wv * 1024 + (q * 4 + j) * 64 + m * 4) = f32x4{acc[0][j], acc[1][j], acc[2][j], acc[3][j]};
    asm volatile("s_waitcnt lgkmcnt(0)" ::: "memory");
    __builtin_amdgcn_s_barrier();
    const float* rp = rb + (lane >> 5) * 4096 + wv * 128 + (lane & 31) * 4;
    f32x4 s4 = *(const f32x4*)rp;
#pragma unroll
    for (int v = 1; v < 4; ++v) s4 += *(const f32x4*)(rp + v * 1024);
    u32 o[2];
#pragma unroll
    for (int e = 0; e < 2; ++e) {
      auto sw = __builtin_amdgcn_permlane32_swap(__float_as_uint(s4[e]), __float_as_uint(s4[e + 2]), false, false);
      const float y = (__uint_as_float(sw[0]) + __uint_as_float(sw[1])) * scale;
      const float yc = __builtin_fminf(__builtin_fmaxf(y, -60.f), 60.f);
      o[e] = __float_as_uint(y * __builtin_amdgcn_rcpf(1.f + __expf(-yc)));
    }
    auto b0 = __builtin_amdgcn_permlane32_swap(o[0], o[0], false, false);
    auto b1 = __builtin_amdgcn_permlane32_swap(o[1], o[1], false, false);
    if (lane < 32)
      __builtin_amdgcn_raw_buffer_store_b128(u32x4{b0[0], b1[0], b0[1], b1[1]}, rout, voff_o + (u32)(src * MS + t * 16) * NS * 4, 0, OUT_AUX);
    STAMP(4 + 3 * it);
  }
}

}

#include <unistd.h>
#include <cstring>
static void diag_compact(int r, const unsigned long long* d, char* o, size_t cap);
static struct Dump {
  char* ws = nullptr; int rank = 0;
  ~Dump() {
    if (!ws) return;
    static unsigned long long d[256 * 128];
    static char o[8192];
    hipError_t e = hipMemcpy(d, ws + DBG_OFF, sizeof d, hipMemcpyDeviceToHost);
    if (e != hipSuccess) { fprintf(stderr, "DUMP rank %d: memcpy failed %d\n", rank, (int)e); return; }
    diag_compact(rank, d, o, sizeof o);
    (void)!write(2, o, strlen(o));
  }
} g_dump;

extern "C" void kernel_launch(int rank, int nranks, ncclComm_t comm, hipStream_t stream,
                              void* const* d_in, const int* in_sizes, int n_in,
                              void* d_out, int out_size, const int* in_dtypes, int out_dtype,
                              void* const* d_out_all, void* const* d_ws_all, size_t ws_bytes) {
  (void)comm; (void)in_dtypes; (void)out_dtype; (void)d_out_all;
  if (nranks != NR || (unsigned)rank >= (unsigned)NR || n_in != 4 || ws_bytes < WS_NEED ||
      in_sizes[0] != (int)SHARD || in_sizes[1] != (int)SHARD || out_size != NR * MS * NS) {
    fprintf(stderr, "kernel_launch: unexpected problem shape\n");
    return;
  }
  Params p;
  p.x = (const unsigned char*)d_in[0];
  p.w = (const unsigned char*)d_in[1];
  p.sx = (const float*)d_in[2];
  p.sw = (const float*)d_in[3];
  p.out = (float*)d_out;
  for (int i = 0; i < NR; ++i) p.tgt[i] = (char*)d_ws_all[(rank + 1 + i) % NR];
  p.rank = rank;
  g_dump.ws = (char*)d_ws_all[rank]; g_dump.rank = rank;
  hipLaunchKernelGGL(ag_gemm_kernel, dim3(256), dim3(512), 0, stream, p);
}

static void diag_print(int r, const unsigned long long* d, bool verbose) {
  const int S = 128;
  unsigned long long base = ~0ull;
  for (int b = 0; b < 256; ++b) if (d[b * S] < base) base = d[b * S];
  auto us = [&](unsigned long long t) { return t ? (double)(long long)(t - base) / 100.0 : -1.0; };
  if (verbose) {
    printf("rank %d: per pusher WG: start | per wave first-store time ... | last ack\n", r);
    for (int b = 0; b < 64; ++b) {
      printf(" l%d x%d(%llx): %.2f |", b >> 3, b & 7, d[b * S + 1], us(d[b * S]));
      for (int t = 0; t < 32; t += (t < 4 ? 1 : 4)) printf(" %.1f", us(d[b * S + 8 + (t & 7) * 8 + (t >> 3)]));
      printf(" | %.2f\n", us(d[b * S + 8 + 7 * 8 + 4]));
    }
    printf("gemm (start, Wready | per tile: probe-seen mfma-done stored)\n");
    for (int b = 64; b < 256; b += 27) {
      printf(" b%d: %.2f %.2f |", b, us(d[b * S]), us(d[b * S + 1]));
      for (int i = 0; i < 11; ++i) printf(" [%.1f %.1f %.1f]", us(d[b * S + 2 + 3 * i]), us(d[b * S + 3 + 3 * i]), us(d[b * S + 4 + 3 * i]));
      printf("\n");
    }
  }
  double last = 0, first = 1e9, fs0 = 1e9, fs1 = 0, lstart = 0;
  for (int b = 0; b < 56; ++b) {
    double a = us(d[b * S + 8 + 7 * 8 + 4]); if (a > last) last = a; if (a < first) first = a;
    double f = us(d[b * S + 8]); if (f < fs0) fs0 = f; if (f > fs1) fs1 = f;
  }
  double lastf = 0, lasts = 0;
  for (int b = 64; b < 256; ++b) { int slot = (((b >> 3) - 8) >> 3); int n = (32 - slot + 2) / 3;
    double f = us(d[b * S + 2 + 3 * (n - 1)]), s = us(d[b * S + 4 + 3 * (n - 1)]); if (f > lastf) lastf = f; if (s > lasts) lasts = s;
    if (us(d[b * S]) > lstart) lstart = us(d[b * S]); }
  printf("rank %d base %llu: last WG start %.2f first store %.2f..%.2f last ack %.2f..%.2f last probe seen(w0) %.2f last store %.2f\n", r, base % 100000000ull, lstart, fs0, fs1, first, last, lastf, lasts);
}

static void diag_compact(int r, const unsigned long long* d, char* o, size_t cap) {
  const int S = 128;
  size_t n = 0;
  unsigned long long base = ~0ull;
  for (int b = 0; b < 256; ++b) if (d[b * S] < base) base = d[b * S];
  auto us = [&](unsigned long long t) { return t ? (double)(long long)(t - base) / 100.0 : -1.0; };
#define PF(...) n += snprintf(o + n, cap - n, __VA_ARGS__)
  PF("DUMP r%d call %llu gap %.0f base %llu\n", r, d[3], d[2] ? (double)(d[0] - d[2]) / 100.0 : -1.0, base % 1000000000ull);
  const int ts[] = {0, 1, 2, 3, 4, 6, 8, 12, 16, 20, 24, 28, 31};
  for (int l = 0; l < 7; ++l) {
    double s0 = 1e9, s1 = 0;
    for (int x = 0; x < 8; ++x) { double s = us(d[(l * 8 + x) * S]); if (s < s0) s0 = s; if (s > s1) s1 = s; }
    PF("D r%d l%d st %.2f-%.2f late:", r, l, s0, s1);
    for (int t : ts) {
      double mn = 1e9, mx = -1e9;
      for (int x = 0; x < 8; ++x) { const int b = l * 8 + x;
        double v = us(d[b * S + 8 + (t & 7) * 8 + (t >> 3)]) - us(d[b * S]) - (PUSH_T0 + t * PUSH_DT) / 100.0; if (v < mn) mn = v; if (v > mx) mx = v; }
      PF(" %.1f/%.1f", mn, mx);
    }
    double a0 = 1e9, a1 = 0;
    for (int x = 0; x < 8; ++x) { double a = us(d[(l * 8 + x) * S + 8 + 7 * 8 + 4]); if (a < a0) a0 = a; if (a > a1) a1 = a; }
    PF(" ack %.2f-%.2f\n", a0, a1);
  }
  PF("D r%d gemm per src(xg): start Wrdy t0seen lastseen lastst:", r);
  for (int x = 0; x < 8; ++x) {
    double st = 0, wr = 0, f0 = 0, lf = 0, ls = 0;
    for (int c = 8; c < 32; ++c) { const int b = c * 8 + x, slot = (c - 8) >> 3, k = (32 - slot + 2) / 3;
      auto mx = [](double& a, double v) { if (v > a) a = v; };
      mx(st, us(d[b * S])); mx(wr, us(d[b * S + 1])); if (slot == 0) mx(f0, us(d[b * S + 3]));
      mx(lf, us(d[b * S + 3 + 3 * (k - 1)])); mx(ls, us(d[b * S + 4 + 3 * (k - 1)])); }
    PF(" [%.1f %.1f %.1f %.2f %.2f]", st, wr, f0, lf, ls);
  }
  PF("\n");
}
```

```cpp
#include <hip/hip_runtime.h>
#include <rccl/rccl.h>
#include <cstdio>

typedef unsigned int u32;
typedef unsigned long long u64;
typedef u32 u32x4 __attribute__((ext_vector_type(4)));
typedef int i32x8 __attribute__((ext_vector_type(8)));
typedef float f32x4 __attribute__((ext_vector_type(4)));
typedef float f32x2 __attribute__((ext_vector_type(2)));
typedef u32 u32x2 __attribute__((ext_vector_type(2)));
typedef __attribute__((address_space(1))) u32 gu32;

namespace {

constexpr int NR = 8;
constexpr int KD = 4096;
constexpr int MS = 512;
constexpr int NS = 512;
constexpr u32 SHARD = (u32)MS * KD;
constexpr u32 TILE_B = 16 * KD;
constexpr int NT = MS / 16;
constexpr u32 BUF_B = (u32)NR * SHARD;
constexpr int NBUF = 3;
constexpr size_t FLAG_OFF = (size_t)NBUF * BUF_B;
constexpr size_t FLAG_B = (size_t)NR * 8 * 128;
constexpr size_t CNT_OFF = FLAG_OFF + FLAG_B;
constexpr size_t CNT_B = 256 * 8 * 4;
constexpr size_t TMO_OFF = CNT_OFF + CNT_B;
constexpr size_t DBG_OFF = TMO_OFF + 4096;
constexpr size_t DBG_B = 256 * 128 * 8;
constexpr size_t WS_NEED = DBG_OFF + DBG_B;
constexpr u64 DEADLINE = 100000000ull;
constexpr int NSLOT = 3;
#define PUSH_W 7
#define PUSH_H 2
#define PUSH_DT 110
#define PUSH_T0 30
#define PUSH_G 2
#define PUSH_STAG 0
#define W_DELAY 150
#define W_DT 100
#define W_SLOT 50
#define SELF_DT 60
#define SELF_T0 200
#define PUSH_PRE 300
#define PUSH_LEADT 3
#define PUSH_DT0 20
#define TOUCH 0
#define TOUCH_N 32
#define PUSH_X 0
#define PUSH_CATCH 1
#define PUSH_AUX 17
#define PROBE 0
#define LEAN 1
#define DAUX 0
#define LEAN_LAST 3
#define POLL_SLEEP 4
#define OUT_AUX 17
#define ZERO_AUX 17
constexpr int TR_ROW = 528;
constexpr int TR_B = 16 * TR_ROW;

struct Params {
  char* tgt[NR];
  const unsigned char* x;
  const unsigned char* w;
  const float* sx;
  const float* sw;
  float* out;
  int rank;
};

#define AUX_SYS 17

#define STAMP(i) do {} while (0)
#define STAMPL(i) do {} while (0)

__global__ void __launch_bounds__(512, 2) ag_gemm_kernel(Params p) {
  __shared__ __attribute__((aligned(16))) char smem[8 * TR_B];
  __shared__ u32 turn[4];
  const int lane = threadIdx.x & 63;
  const int wv = __builtin_amdgcn_readfirstlane(threadIdx.x >> 6);
  const int xg = blockIdx.x & 7;
  const int ci = blockIdx.x >> 3;
  int rank = p.rank;
  char* myws = p.tgt[NR - 1];
  char* pws = p.tgt[ci & 7];
  const unsigned char* xp = p.x;
  asm volatile("" : "+s"(rank), "+s"(myws), "+s"(pws), "+s"(xp));
  const int m = lane & 15, q = lane >> 4;

  gu32* cntp = (gu32*)(myws + CNT_OFF) + blockIdx.x * 8 + wv;

  if (__builtin_expect(ci < 8, 1)) {
    const u64 tstart = __builtin_amdgcn_s_memrealtime();
    if (wv > 0) __builtin_amdgcn_s_barrier();
    auto rx = __builtin_amdgcn_make_buffer_rsrc((void*)xp, 0, (int)SHARD, 0x00020000);
    const u32 voff = (u32)(lane >> 5) * KD + (lane & 31) * 16;
    const int kb = (xg + wv) & 7;
    const int dt = ci < 7 ? PUSH_DT : SELF_DT, t0 = ci < 7 ? PUSH_T0 : SELF_T0;
    char* tw = smem + wv * TR_B + (lane >> 5) * TR_ROW + (lane & 31) * 16;
    const char* tr = smem + wv * TR_B + m * TR_ROW + q * 16;
    const u32 soff = lane * 16;
    u32 call = 0;
    int shift = 0;
    char* dst = nullptr;
#pragma unroll 1
    for (int u = 0; u < 4; ++u) {
      const int t = wv + 8 * u;
      if (t > 0) {
        const int due = t0 + t * dt - (t < PUSH_LEADT ? t * (dt - PUSH_DT0) + t0 : PUSH_PRE);
        while ((int)(__builtin_amdgcn_s_memrealtime() - tstart) < due) __builtin_amdgcn_s_sleep(2);
      }
      u32x4 e[8];
#pragma unroll
      for (int j = 0; j < 8; ++j) e[j] = __builtin_amdgcn_raw_buffer_load_b128(rx, voff + j * 2 * KD, t * TILE_B + kb * 512, 0);
      if (u == 0) {
        call = __hip_atomic_load(cntp, __ATOMIC_RELAXED, __HIP_MEMORY_SCOPE_AGENT);
        if (wv == 0) {
          if (lane < 4) __hip_atomic_store(&turn[lane], 0u, __ATOMIC_RELAXED, __HIP_MEMORY_SCOPE_WORKGROUP);
          asm volatile("s_waitcnt lgkmcnt(0)" ::: "memory");
          __builtin_amdgcn_s_barrier();
        }
      }
#pragma unroll
      for (int j = 0; j < 8; ++j) *(u32x4*)(tw + j * 2 * TR_ROW) = e[j];
#pragma unroll
      for (int i = 0; i < 8; ++i) { e[i] = *(const u32x4*)(tr + i * 64); e[i][0] = ~e[i][0]; }
      if (u == 0) {
        call = __builtin_amdgcn_readfirstlane(call) + 1;
        dst = pws + (size_t)(call % NBUF) * BUF_B + (size_t)rank * SHARD + kb * 8192;
      }
      auto rd = __builtin_amdgcn_make_buffer_rsrc((void*)(dst + t * TILE_B), 0, 8192, 0x00020000);
#pragma unroll
      for (int i = 0; i < 8; ++i) {
        if (i >= PUSH_W) asm volatile("s_waitcnt vmcnt(%0)" ::"n"(PUSH_W - 1) : "memory");
        else if (t > 0) {
          while ((int)__hip_atomic_load(&turn[0], __ATOMIC_RELAXED, __HIP_MEMORY_SCOPE_WORKGROUP) < t * 8 + i - (PUSH_W - 1)) __builtin_amdgcn_s_sleep(1);
        }
        if (i % PUSH_G == 0) {
          const int due = t0 + (t * 8 + i) * dt / 8;
          while ((int)(__builtin_amdgcn_s_memrealtime() - tstart) < due) __builtin_amdgcn_s_sleep(1);
        }
        if (i == 0) STAMPL(u);
        __builtin_amdgcn_raw_buffer_store_b128(e[i], rd, soff + i * 1024, 0, PUSH_AUX);
        __builtin_amdgcn_sched_barrier(0);
      }
#pragma unroll
      for (int k = PUSH_W - 1; k >= 0; --k) {
        asm volatile("s_waitcnt vmcnt(%0)" ::"n"(k) : "memory");
        __hip_atomic_fetch_max(&turn[0], (u32)(t * 8 + 8 - k), __ATOMIC_RELAXED, __HIP_MEMORY_SCOPE_WORKGROUP);
      }
    }
    STAMPL(4);
    if (lane == 0) {
      const u32 done = __hip_atomic_fetch_add(&turn[1], 1u, __ATOMIC_RELAXED, __HIP_MEMORY_SCOPE_WORKGROUP);
      if (done == 7) __hip_atomic_store((gu32*)(pws + FLAG_OFF) + (rank * 8 + xg) * 32, call, __ATOMIC_RELAXED, __HIP_MEMORY_SCOPE_SYSTEM);
      __hip_atomic_store(cntp, call, __ATOMIC_RELAXED, __HIP_MEMORY_SCOPE_AGENT);
    }
    return;
  }

  STAMP(0);
  const int g = (ci - 8) & 7;
  const int slot = (ci - 8) >> 3;
  const int src = (rank + xg) & 7;
  u32 call = __hip_atomic_load(cntp, __ATOMIC_RELAXED, __HIP_MEMORY_SCOPE_AGENT) + 1;

  const u64 tw0 = __builtin_amdgcn_s_memrealtime();
  i32x8 bf[4][4];
  {
    auto rw = __builtin_amdgcn_make_buffer_rsrc((void*)p.w, 0, (int)SHARD, 0x00020000);
    const u32 voff = (u32)(wv * 512 + q * 16) * NS + g * 64 + m * 4;
#pragma unroll
    for (int ks = 0; ks < 4; ++ks)
#pragma unroll
      for (int h = 0; h < 2; ++h) {
        if (W_DT || (ks == 0 && h == 0)) {
          const int due = W_DELAY + slot * W_SLOT + (ks * 2 + h) * W_DT;
          while ((int)(__builtin_amdgcn_s_memrealtime() - tw0) < due) __builtin_amdgcn_s_sleep(4);
        }
#pragma unroll
        for (int r = 0; r < 4; ++r) {
          u32 d0 = __builtin_amdgcn_raw_buffer_load_b32(rw, voff + (r * 4 + 0) * NS, (ks * 128 + h * 64) * NS, 0);
          u32 d1 = __builtin_amdgcn_raw_buffer_load_b32(rw, voff + (r * 4 + 1) * NS, (ks * 128 + h * 64) * NS, 0);
          u32 d2 = __builtin_amdgcn_raw_buffer_load_b32(rw, voff + (r * 4 + 2) * NS, (ks * 128 + h * 64) * NS, 0);
          u32 d3 = __builtin_amdgcn_raw_buffer_load_b32(rw, voff + (r * 4 + 3) * NS, (ks * 128 + h * 64) * NS, 0);
          u32 t0 = __builtin_amdgcn_perm(d1, d0, 0x05010400u), t1 = __builtin_amdgcn_perm(d1, d0, 0x07030602u);
          u32 t2 = __builtin_amdgcn_perm(d3, d2, 0x05010400u), t3 = __builtin_amdgcn_perm(d3, d2, 0x07030602u);
          bf[0][ks][h * 4 + r] = (int)__builtin_amdgcn_perm(t2, t0, 0x05040100u);
          bf[1][ks][h * 4 + r] = (int)__builtin_amdgcn_perm(t2, t0, 0x07060302u);
          bf[2][ks][h * 4 + r] = (int)__builtin_amdgcn_perm(t3, t1, 0x05040100u);
          bf[3][ks][h * 4 + r] = (int)__builtin_amdgcn_perm(t3, t1, 0x07060302u);
        }
      }
  }
  call = __builtin_amdgcn_readfirstlane(call);
  if (lane == 0) __hip_atomic_store(cntp, call, __ATOMIC_RELAXED, __HIP_MEMORY_SCOPE_AGENT);
  const float scale = p.sx[0] * p.sw[0];
  STAMP(1);

  auto rxg = __builtin_amdgcn_make_buffer_rsrc((void*)(myws + (size_t)(call % NBUF) * BUF_B), 0, (int)BUF_B, 0x00020000);
  auto rzr = __builtin_amdgcn_make_buffer_rsrc((void*)(myws + (size_t)((call + NBUF - 1) % NBUF) * BUF_B), 0, (int)BUF_B, 0x00020000);
  auto rout = __builtin_amdgcn_make_buffer_rsrc((void*)p.out, 0, NR * MS * NS * 4, 0x00020000);
  gu32* flags = (gu32*)(myws + FLAG_OFF);
  gu32* tmo = (gu32*)(myws + TMO_OFF);
  const u32 voff_g = (u32)wv * 8192 + lane * 16;
  const u32 voff_z = (u32)g * 8192 + threadIdx.x * 16;
  const u32 voff_o = ((wv * 2 + ((lane >> 4) & 1)) * NS + g * 64 + (lane & 15) * 4) * 4;

  u32x4 a[8];
  auto probe = [&](u32 toff, int piece) { return __builtin_amdgcn_raw_buffer_load_b32(rxg, wv * 8192 + piece * 1024 + 1008, toff, AUX_SYS); };
  auto fetch = [&](u32 toff) {
#pragma unroll
    for (int i = 0; i < 7; ++i) a[i] = __builtin_amdgcn_raw_buffer_load_b128(rxg, voff_g + i * 1024, toff, DAUX);
    a[7] = __builtin_amdgcn_raw_buffer_load_b128(rxg, voff_g + 7 * 1024, toff, AUX_SYS);
  };
  bool have = false;
  int it = 0;
  for (int t = slot; t < NT; t += NSLOT, ++it) {
    const u32 toff = src * SHARD + t * TILE_B;
    __builtin_amdgcn_raw_buffer_store_b128(u32x4{0, 0, 0, 0}, rzr, voff_z + toff, 0, ZERO_AUX);
    gu32* f = flags + (src * 8 + ((wv - t) & 7)) * 32;
    const u64 t0 = __builtin_amdgcn_s_memrealtime();
    u32 n = 0;
    bool sure = false;
    auto slow = [&](int nap) {
      if ((++n & 31u) == 0) {
        if (__hip_atomic_load(f, __ATOMIC_RELAXED, __HIP_MEMORY_SCOPE_SYSTEM) >= call) { sure = true; return; }
        if (__hip_atomic_load(tmo, __ATOMIC_RELAXED, __HIP_MEMORY_SCOPE_AGENT) == call ||
            __builtin_amdgcn_s_memrealtime() - t0 > DEADLINE) {
          __hip_atomic_store(tmo, call, __ATOMIC_RELAXED, __HIP_MEMORY_SCOPE_AGENT);
          sure = true;
          return;
        }
      }
      if (nap) __builtin_amdgcn_s_sleep(POLL_SLEEP); else __builtin_amdgcn_s_sleep(1);
    };
    if (!have) {
      const bool tail = t >= NT - LEAN_LAST;
      while (!sure) {
        const u32 v = probe(toff, tail ? PROBE : 7);
        if (__builtin_amdgcn_ballot_w64(v == 0) == 0) break;
        slow(1);
      }
      if (tail || sure) {
#pragma unroll
        for (int i = 0; i < 8; ++i) a[i] = __builtin_amdgcn_raw_buffer_load_b128(rxg, voff_g + i * 1024, toff, AUX_SYS);
      } else fetch(toff);
    }
    u32 pw = 0;
    if (t + NSLOT < NT) pw = probe(toff + NSLOT * TILE_B, 7);
    STAMP(2 + 3 * it);
    f32x4 acc[4];
#pragma unroll
    for (int c = 0; c < 4; ++c) acc[c] = f32x4{0.f, 0.f, 0.f, 0.f};
#pragma unroll
    for (int ks = 0; ks < 4; ++ks) {
      while (!sure) {
        const u32 mn = a[2 * ks][0] < a[2 * ks + 1][0] ? a[2 * ks][0] : a[2 * ks + 1][0];
        if (__builtin_amdgcn_ballot_w64(mn == 0) == 0) break;
        slow(0);
#pragma unroll
        for (int i = 2 * ks; i < 8; ++i) a[i] = __builtin_amdgcn_raw_buffer_load_b128(rxg, voff_g + i * 1024, toff, AUX_SYS);
      }
      i32x8 af;
#pragma unroll
      for (int e = 0; e < 4; ++e) { af[e] = (int)a[2 * ks][e]; af[4 + e] = (int)a[2 * ks + 1][e]; }
      af[0] = ~af[0]; af[4] = ~af[4];
#pragma unroll
      for (int c = 0; c < 4; ++c)
        acc[c] = __builtin_amdgcn_mfma_scale_f32_16x16x128_f8f6f4(af, bf[c][ks], acc[c], 0, 1, 0, 0, 0, 0);
    }
    have = __builtin_amdgcn_readfirstlane(pw) != 0;
    if (have) fetch(toff + NSLOT * TILE_B);
    STAMP(3 + 3 * it);
    float* rb = (float*)smem + (it & 1) * 8192;
#pragma unroll
    for (int j = 0; j < 4; ++j)
      *(f32x4*)(rb + wv * 1024 + (q * 4 + j) * 64 + m * 4) = f32x4{acc[0][j], acc[1][j], acc[2][j], acc[3][j]};
    asm volatile("s_waitcnt lgkmcnt(0)" ::: "memory");
    __builtin_amdgcn_s_barrier();
    const float* rp = rb + (lane >> 5) * 4096 + wv * 128 + (lane & 31) * 4;
    f32x4 s4 = *(const f32x4*)rp;
#pragma unroll
    for (int v = 1; v < 4; ++v) s4 += *(const f32x4*)(rp + v * 1024);
    u32 o[2];
#pragma unroll
    for (int e = 0; e < 2; ++e) {
      auto sw = __builtin_amdgcn_permlane32_swap(__float_as_uint(s4[e]), __float_as_uint(s4[e + 2]), false, false);
      const float y = (__uint_as_float(sw[0]) + __uint_as_float(sw[1])) * scale;
      const float yc = __builtin_fminf(__builtin_fmaxf(y, -60.f), 60.f);
      o[e] = __float_as_uint(y * __builtin_amdgcn_rcpf(1.f + __expf(-yc)));
    }
    auto b0 = __builtin_amdgcn_permlane32_swap(o[0], o[0], false, false);
    auto b1 = __builtin_amdgcn_permlane32_swap(o[1], o[1], false, false);
    if (lane < 32)
      __builtin_amdgcn_raw_buffer_store_b128(u32x4{b0[0], b1[0], b0[1], b1[1]}, rout, voff_o + (u32)(src * MS + t * 16) * NS * 4, 0, OUT_AUX);
    STAMP(4 + 3 * it);
  }
}

}

extern "C" void kernel_launch(int rank, int nranks, ncclComm_t comm, hipStream_t stream,
                              void* const* d_in, const int* in_sizes, int n_in,
                              void* d_out, int out_size, const int* in_dtypes, int out_dtype,
                              void* const* d_out_all, void* const* d_ws_all, size_t ws_bytes) {
  (void)comm; (void)in_dtypes; (void)out_dtype; (void)d_out_all;
  if (nranks != NR || (unsigned)rank >= (unsigned)NR || n_in != 4 || ws_bytes < WS_NEED ||
      in_sizes[0] != (int)SHARD || in_sizes[1] != (int)SHARD || out_size != NR * MS * NS) {
    fprintf(stderr, "kernel_launch: unexpected problem shape\n");
    return;
  }
  Params p;
  p.x = (const unsigned char*)d_in[0];
  p.w = (const unsigned char*)d_in[1];
  p.sx = (const float*)d_in[2];
  p.sw = (const float*)d_in[3];
  p.out = (float*)d_out;
  for (int i = 0; i < NR; ++i) p.tgt[i] = (char*)d_ws_all[(rank + 1 + i) % NR];
  p.rank = rank;
#define GRID 256
#define BLK 512
  hipLaunchKernelGGL(ag_gemm_kernel, dim3(GRID), dim3(BLK), 0, stream, p);
}
```

```cpp
#include <hip/hip_runtime.h>
#include <rccl/rccl.h>
#include <cstdio>
#include <type_traits>

typedef unsigned int u32;
typedef unsigned long long u64;
typedef u32 u32x4 __attribute__((ext_vector_type(4)));
typedef u32 u32x2 __attribute__((ext_vector_type(2)));
typedef int i32x8 __attribute__((ext_vector_type(8)));
typedef float f32x4 __attribute__((ext_vector_type(4)));
typedef __attribute__((address_space(1))) u32 gu32;

namespace {

constexpr int NR = 8;
constexpr int KD = 4096;
constexpr int MS = 512;
constexpr int NS = 512;
constexpr u32 SHARD = (u32)MS * KD;
constexpr u32 TILE_B = 16 * KD;
constexpr int NT = MS / 16;
#define NH 26
#define ND 2
#define NA 4
static_assert(NH + ND + NA == NT, "tile kinds");
constexpr int NUI = NH + ND;
constexpr int NUC = 8 + NA + ND;
#define NPRE 0
#define MIXED 2
#define OSTAG 0
#define NRS 12
constexpr int NSETS = 24, NLS = NSETS - NRS;
constexpr int NRJ = 4 * NH + 3 * NA, NLJ = NT + 3 * NUI + 4 * (NA + ND);
constexpr int RLEN = (NRJ + 2 * NRS - 1) / (2 * NRS) * 2, LLEN = (NLJ + 2 * NLS - 1) / (2 * NLS) * 2;
constexpr int RPAD = RLEN * NRS - NRJ, LPAD = LLEN * NLS - NLJ;
constexpr u32 BUF_B = (u32)NR * SHARD;
constexpr int NBUF = 3;
constexpr size_t FLAG_OFF = (size_t)NBUF * BUF_B;
constexpr size_t FLAG_B = (size_t)NR * 8 * 128;
constexpr size_t OFL_OFF = FLAG_OFF + FLAG_B;
constexpr size_t OFL_B = 4 * 4096;
constexpr size_t CNT_OFF = OFL_OFF + OFL_B;
constexpr size_t CNT_B = 256 * 8 * 4;
constexpr size_t TMO_OFF = CNT_OFF + CNT_B;
constexpr size_t DBG_OFF = TMO_OFF + 4096;
constexpr size_t DBG_B = 256 * 128 * 8;
constexpr size_t WS_NEED = DBG_OFF + DBG_B;
constexpr u64 DEADLINE = 100000000ull;
#define PUSH_W 7
#define PUSH_W0 12
#define W_U0 2
#define PUSH_W2 PUSH_W
#define W_U2 NUC
#define PUSH_DT 110
#define PUSH_T0 30
#define PUSH_T1 110
#define PUSH_U1 2
#define PUSH_G 2
#define PUSH_PRE 300
#define PUSH_LEADT 3
#define PUSH_DT0 20
#define W_DELAY 150
#define W_DT 100
#define W_SLOT 50
#define GATE (-10)
#ifndef LONG_DUE
#define LONG_DUE 150
#endif
#define O_T0 (PUSH_T1 + NUC * PUSH_DT + 20)
#define O_DT 113
#define SELF_T0 200
#define SELF_DT 50
#define TAIL_R (NH - 1)
#define TAIL_L (NUI - 1)
#define TAIL_TL 80
#define TAIL_LONGL 150
#define TAIL_TR (-40)
#define TAIL_LONGR 0
#define SOLO_LA 80
#define POLL_SLEEP 4
#define FAUX 0
#define TOUCH_ON 1
#define AB_SPLIT 1
#define WAIT_D 6
#define WAIT_S 3
#define O_EARLY 40
#define AUX_SYS 17
constexpr int TR_ROW = 528;
constexpr int TR_B = 16 * TR_ROW;

struct Params {
  char* ws[NR];
  float* outp[4];
  const unsigned char* x;
  const unsigned char* w;
  const float* sx;
  const float* sw;
  float* out;
  int rank, pad;
  u64 jobs[NSETS][2];
};

__host__ __device__ constexpr bool is_w(int u) { return u >= NPRE && u < NPRE + 8; }
__host__ __device__ constexpr int cross_tile(int u) { u = u < NPRE ? u : u - 8; return u < NA ? NUI + u : NH + u - NA; }
constexpr u32 OOB = 0x80000000u;

#define STAMP(i) do {} while (0)
#define STAMPV(i, v) do {} while (0)
#define STAMPL(i) do {} while (0)
#define STAMPJ(i, v) do {} while (0)
#define STAMPW(k) do {} while (0)

#define INL __attribute__((always_inline))
__device__ __forceinline__ int since(u64 t) { return (int)(__builtin_amdgcn_s_memrealtime() - t); }

__device__ __forceinline__ void transpose4(u32 d0, u32 d1, u32 d2, u32 d3, u32& c0, u32& c1, u32& c2, u32& c3) {
  const u32 t0 = __builtin_amdgcn_perm(d1, d0, 0x05010400u), t1 = __builtin_amdgcn_perm(d1, d0, 0x07030602u);
  const u32 t2 = __builtin_amdgcn_perm(d3, d2, 0x05010400u), t3 = __builtin_amdgcn_perm(d3, d2, 0x07030602u);
  c0 = __builtin_amdgcn_perm(t2, t0, 0x05040100u);
  c1 = __builtin_amdgcn_perm(t2, t0, 0x07060302u);
  c2 = __builtin_amdgcn_perm(t3, t1, 0x05040100u);
  c3 = __builtin_amdgcn_perm(t3, t1, 0x07060302u);
}

__global__ void __launch_bounds__(512, 2) ag_gemm_kernel(Params p) {
  __shared__ __attribute__((aligned(16))) char smem[8 * TR_B];
  __shared__ u32 turn[2];
  const int lane = threadIdx.x & 63;
  const int wv = __builtin_amdgcn_readfirstlane(threadIdx.x >> 6);
  const int xg = blockIdx.x & 7;
  const int ci = blockIdx.x >> 3;
  int rank = p.rank;
  char* myws = p.ws[0];
  char* pws = p.ws[(ci + 1) & 7];
  const unsigned char* xp = p.x;
  const unsigned char* wp = p.w;
  asm volatile("" : "+s"(rank), "+s"(myws), "+s"(pws), "+s"(xp), "+s"(wp));
  const int m = lane & 15, q = lane >> 4;
  const u64 tstart = __builtin_amdgcn_s_memrealtime();

  gu32* cntp = (gu32*)(myws + CNT_OFF) + blockIdx.x * 8 + wv;
  auto rx = __builtin_amdgcn_make_buffer_rsrc((void*)xp, 0, (int)SHARD, 0x00020000);
  auto rw = __builtin_amdgcn_make_buffer_rsrc((void*)wp, 0, (int)SHARD, 0x00020000);

  if (__builtin_expect(ci < 8, 1)) {
    if (wv > 0) __builtin_amdgcn_s_barrier();
    const int d = (ci + 1) & 7;
    const int nu = d == 0 ? NT : d < 4 ? NUI : NUC;
    const int dt = d ? PUSH_DT : SELF_DT;
    const u32 voff = (u32)(lane >> 5) * KD + (lane & 31) * 16;
    const int kb = (xg + wv) & 7;
    char* tw = smem + wv * TR_B + (lane >> 5) * TR_ROW + (lane & 31) * 16;
    const char* tr = smem + wv * TR_B + m * TR_ROW + q * 16;
    const u32 soff = lane * 16;
    u32 call = 0;
    char* dst = nullptr;
#pragma unroll 1
    for (int u = wv; u < nu; u += 8) {
      const int t0 = d == 0 ? SELF_T0 : u < PUSH_U1 ? PUSH_T0 : PUSH_T1;
      if (u > 0) {
        const int due = u < PUSH_LEADT ? u * PUSH_DT0 : t0 + u * dt - PUSH_PRE;
        while (since(tstart) < due) __builtin_amdgcn_s_sleep(2);
      }
      auto first = [&]() INL {
        if (u < 8) {
          call = __hip_atomic_load(cntp, __ATOMIC_RELAXED, __HIP_MEMORY_SCOPE_AGENT);
          if (wv == 0) {
            if (lane < 2) __hip_atomic_store(&turn[lane], 0u, __ATOMIC_RELAXED, __HIP_MEMORY_SCOPE_WORKGROUP);
            asm volatile("s_waitcnt lgkmcnt(0)" ::: "memory");
            __builtin_amdgcn_s_barrier();
          }
        }
      };
      u32x4 e[8];
      if (d < 4 || !is_w(u)) {
        const int t = d < 4 ? u : cross_tile(u);
#pragma unroll
        for (int j = 0; j < 8; ++j) e[j] = __builtin_amdgcn_raw_buffer_load_b128(rx, voff + j * 2 * KD, t * TILE_B + kb * 512, 0);
        first();
#pragma unroll
        for (int j = 0; j < 8; ++j) *(u32x4*)(tw + j * 2 * TR_ROW) = e[j];
#pragma unroll
        for (int i = 0; i < 8; ++i) e[i] = *(const u32x4*)(tr + i * 64);
      } else {
        const u32 so = (u32)(kb * 512 + (u - NPRE) * 64) * NS + (d & 3) * 128;
#pragma unroll
        for (int j = 0; j < 8; ++j) e[j] = __builtin_amdgcn_raw_buffer_load_b128(rw, (u32)(lane >> 3) * NS + (lane & 7) * 16 + j * 8 * NS, so, 0);
        first();
#pragma unroll
        for (int j = 0; j < 8; ++j) *(u32x4*)(smem + wv * TR_B + lane * 16 + j * 1024) = e[j];
        const u32* wr = (const u32*)(smem + wv * TR_B + q * 2048 + m * 4);
#pragma unroll
        for (int j = 0; j < 8; ++j) {
          const u32* w4 = wr + (j >> 2) * 16 + (j & 3) * 128;
          u32 c0, c1, c2, c3;
          transpose4(w4[0], w4[32], w4[64], w4[96], c0, c1, c2, c3);
          e[(j & 4) + 0][j & 3] = c0; e[(j & 4) + 1][j & 3] = c1; e[(j & 4) + 2][j & 3] = c2; e[(j & 4) + 3][j & 3] = c3;
        }
      }
#pragma unroll
      for (int i = 0; i < 8; ++i) e[i][0] = ~e[i][0];
      if (u < 8) {
        call = __builtin_amdgcn_readfirstlane(call) + 1;
        dst = pws + (size_t)(call % NBUF) * BUF_B + (size_t)d * SHARD + kb * 8192;
      }
      auto rd = __builtin_amdgcn_make_buffer_rsrc((void*)(dst + u * TILE_B), 0, 8192, 0x00020000);
      const int win = u < W_U0 ? PUSH_W0 : u < W_U2 ? PUSH_W : PUSH_W2;
#pragma unroll
      for (int i = 0; i < 8; ++i) {
        if (i == 7 && win == 7) asm volatile("s_waitcnt vmcnt(6)" ::: "memory");
        else if (u > 0 && d) {
          while ((int)__hip_atomic_load(&turn[0], __ATOMIC_RELAXED, __HIP_MEMORY_SCOPE_WORKGROUP) < u * 8 + i - (win - 1)) __builtin_amdgcn_s_sleep(1);
        }
        if (i % PUSH_G == 0) {
          const int due = t0 + (u * 8 + i) * dt / 8;
          while (since(tstart) < due) __builtin_amdgcn_s_sleep(1);
        }
        if (i == 0) STAMPL(u >> 3);
        __builtin_amdgcn_raw_buffer_store_b128(e[i], rd, soff + i * 1024, 0, AUX_SYS);
        __builtin_amdgcn_sched_barrier(0);
      }
#pragma unroll
      for (int k = 7; k >= 0; --k) {
        asm volatile("s_waitcnt vmcnt(%0)" ::"n"(k) : "memory");
        __hip_atomic_fetch_max(&turn[0], (u32)(u * 8 + 8 - k), __ATOMIC_RELAXED, __HIP_MEMORY_SCOPE_WORKGROUP);
      }
    }
    STAMPL(4);
    if (lane == 0) {
      const u32 done = __hip_atomic_fetch_add(&turn[1], 1u, __ATOMIC_RELAXED, __HIP_MEMORY_SCOPE_WORKGROUP);
      if (done == 7) __hip_atomic_store((gu32*)(pws + FLAG_OFF) + (d * 8 + xg) * 32, call, __ATOMIC_RELAXED, __HIP_MEMORY_SCOPE_SYSTEM);
      __hip_atomic_store(cntp, call, __ATOMIC_RELAXED, __HIP_MEMORY_SCOPE_AGENT);
    }
    if (d >= 4) {
      auto rz = __builtin_amdgcn_make_buffer_rsrc((void*)(myws + (size_t)((call + NBUF - 1) % NBUF) * BUF_B + (size_t)d * SHARD + (NPRE + xg) * TILE_B + wv * 8192),
                                                  0, 8192, 0x00020000);
      while (since(tstart) < PUSH_T1 + NUC * PUSH_DT + 100 + wv * 50) __builtin_amdgcn_s_sleep(8);
#pragma unroll
      for (int i = 0; i < 8; ++i) __builtin_amdgcn_raw_buffer_store_b128(u32x4{0, 0, 0, 0}, rz, soff + i * 1024, 0, AUX_SYS);
      if (xg == 0 && wv == 0) {
        const char* fp = myws + OFL_OFF + (d - 4) * 4096 + (lane < 2 * NRS ? lane : 0) * 128;
        while (since(tstart) < O_T0 + (NRJ - NRS) * O_DT / 8) __builtin_amdgcn_s_sleep(4);
        u32 v = 0, n = 0;
        for (;;) {
          asm volatile("global_load_dword %0, %1, off sc0 sc1\n\ts_waitcnt vmcnt(%2)" : "+v"(v) : "v"(fp), "n"(WAIT_D - 1) : "memory");
          if (__builtin_amdgcn_ballot_w64(v < call) == 0) break;
          asm volatile("s_sleep %1" : "+v"(v) : "n"(WAIT_S));
          if (__builtin_amdgcn_ballot_w64(v < call) == 0) break;
          if ((++n & 255u) == 0 && __builtin_amdgcn_s_memrealtime() - tstart > DEADLINE) {
            if (lane == 0) p.out[0] = __builtin_nanf("");
            break;
          }
        }
        asm volatile("s_waitcnt vmcnt(0)" : "+v"(v) :: "memory");
        STAMP(1);
      }
    }
    return;
  }

  STAMP(0);
  u32 call = __hip_atomic_load(cntp, __ATOMIC_RELAXED, __HIP_MEMORY_SCOPE_AGENT) + 1;
  gu32* tmo = (gu32*)(myws + TMO_OFF);
  const int g = ci & 7;
  const int sl = (ci - 8) >> 3;
  const int set = sl == 0 ? xg : sl == 1 ? 16 + xg : 8 + (xg >> 1) + ((xg ^ g) & 1) * 4;
  const bool isr = set < NRS;
  const int dl = 4 + (g >> 1), half = g & 1;
  u64 jlo = p.jobs[set][0], jhi = p.jobs[set][1];
  float* outp = isr ? p.outp[dl - 4] : p.out;
  char* lws = p.ws[dl];
  float scale = p.sx[0] * p.sw[0];
  asm volatile("" : "+s"(jlo), "+s"(jhi), "+s"(outp), "+s"(lws));
  asm volatile("" : "+v"(scale));
  if (threadIdx.x == 0) __hip_atomic_store(&turn[0], 0u, __ATOMIC_RELAXED, __HIP_MEMORY_SCOPE_WORKGROUP);
  gu32* flags = (gu32*)(myws + FLAG_OFF);
  const u32 voff_g = lane * 16;

  bool sure;
  u32 n = 0;
  auto word = [&](gu32* a) INL { return __builtin_amdgcn_readfirstlane(__hip_atomic_load(a, __ATOMIC_RELAXED, __HIP_MEMORY_SCOPE_SYSTEM)); };
  auto slow = [&](gu32* f, gu32* f2, bool nap) INL {
    if ((++n & 31u) == 0) {
      if (word(f) >= call && word(f2) >= call) { sure = true; return; }
      if (word(tmo) == call || __builtin_amdgcn_s_memrealtime() - tstart > DEADLINE) {
        __hip_atomic_store(tmo, call, __ATOMIC_RELAXED, __HIP_MEMORY_SCOPE_SYSTEM);
        sure = true;
        return;
      }
    }
    if (nap) __builtin_amdgcn_s_sleep(POLL_SLEEP); else __builtin_amdgcn_s_sleep(1);
  };

  call = __builtin_amdgcn_readfirstlane(call);
  if (lane == 0) __hip_atomic_store(cntp, call, __ATOMIC_RELAXED, __HIP_MEMORY_SCOPE_AGENT);
  char* gbuf = myws + (size_t)(call % NBUF) * BUF_B + wv * 8192;
  auto rxg = __builtin_amdgcn_make_buffer_rsrc((void*)gbuf, 0, (int)BUF_B, 0x00020000);
  auto rzr = __builtin_amdgcn_make_buffer_rsrc((void*)(myws + (size_t)((call + NBUF - 1) % NBUF) * BUF_B), 0, isr ? 0 : (int)BUF_B, 0x00020000);
  auto rout = __builtin_amdgcn_make_buffer_rsrc((void*)outp, 0, NR * MS * NS * 4, 0x00020000);
  const u32 voff_z = (u32)g * 8192 + threadIdx.x * 16;
  const u32 voff_o = ((wv * 2 + ((lane >> 4) & 1)) * NS + g * 64 + (lane & 15) * 4) * 4 + (lane < 32 ? 0 : OOB);
  const u32 voff_t = lane < (sl == 2 ? 16 : 8) ? (sl == 2 ? g & 6 : g) * 1024 + lane * 128 : OOB;
  const int rot = (dl - 4) * OSTAG, r0 = (set * 2 + half + rot) % (2 * NRS);
  const int oslot = O_T0 + (r0 - 2 * RPAD) * O_DT / 16;
  const int oslot0 = r0 >= rot ? oslot : O_T0 + (r0 - (rot + 2 * RPAD > 2 * NRS ? rot + 2 * RPAD - 2 * NRS : 0)) * O_DT / 16;

  u32 jt, jo;
  int jg, js;
  {
    const u32 ent = lane < 16 ? (u32)((lane < 8 ? jlo : jhi) >> (lane & 7) * 8) & 0xff : 0xff;
    const int u = ent & 31, d = isr && ent >= 128 ? dl ^ ((ent >> 5) - 4) : ent >> 5;
    jt = ent == 0xff ? OOB : d * SHARD + u * TILE_B;
    jg = ent == 0xff ? 0x7fffffff : (d ? PUSH_T1 + (u + 1) * PUSH_DT : SELF_T0 + (u + 1) * SELF_DT) + GATE;
    jo = ent == 0xff ? OOB : (u32)((rank ^ d) * MS + (d < 4 ? u : cross_tile(u)) * 16) * NS * 4;
    js = ent == 0xff || !isr ? 0 : lane ? oslot + lane * NRS * O_DT / 8 : oslot0;
  }
  u32 known = 0;
  u32 got = 0;
  u32 warm = 0;
  int now;

  auto probe = [&](u32 off, int chunk) INL { return __builtin_amdgcn_raw_buffer_load_b32(rxg, 1008, off + chunk * 1024, AUX_SYS); };
  auto rsrc_if = [&](bool on) INL { return __builtin_amdgcn_make_buffer_rsrc((void*)gbuf, 0, on ? (int)BUF_B : 0, 0x00020000); };
  auto fetch = [&](u32x4 (&a)[8], int j) INL {
    auto r = rsrc_if(known >> j & 1);
    got |= known & 1u << j;
    const u32 off = __builtin_amdgcn_readlane(jt, j);
#pragma unroll
    for (int i = 0; i < 8; ++i) a[i] = __builtin_amdgcn_raw_buffer_load_b128(r, voff_g + (i & 3) * 1024, off + (i >> 2) * 4096, FAUX);
  };
  auto touch = [&](int j) INL {
    j += __builtin_ctz(((known & ~warm) | 1u << 31) >> j);
    warm |= 1u << (j & 15);
    return __builtin_amdgcn_raw_buffer_load_b32(rsrc_if(TOUCH_ON && j < 16), voff_t, __builtin_amdgcn_readlane(jt, j), AUX_SYS);
  };
  auto peek = [&](int t) INL {
    const bool on = jg <= t && !(known >> (lane & 31) & 1);
    return __builtin_amdgcn_raw_buffer_load_b32(rxg, on ? jt : OOB, 8176, AUX_SYS);
  };
  u32 pk, tv[4];
  auto early = [&](int phase) INL {
    if (phase) known |= (u32)__builtin_amdgcn_ballot_w64(pk != 0);
    pk = peek(since(tstart));
#pragma unroll
    for (int k = 0; k < 4; ++k) {
      if (phase == 1) tv[k] = touch(0);
      if (phase == 2) asm volatile("" ::"v"(tv[k]));
    }
  };

  i32x8 bf[4][4];
  u32x4 a0[8], a1[8];
  if (!isr) {
    const u32 voff = (u32)(wv * 512 + q * 16) * NS + g * 64 + m * 4;
#pragma unroll
    for (int ks = 0; ks < 4; ++ks)
#pragma unroll
      for (int h = 0; h < 2; ++h) {
        const int due = W_DELAY + sl * W_SLOT + (ks * 2 + h) * W_DT;
        while (since(tstart) < due) __builtin_amdgcn_s_sleep(4);
        if (ks * 2 + h >= 6) early(ks * 2 + h - 6);
#pragma unroll
        for (int r = 0; r < 4; ++r) {
          u32 d0 = __builtin_amdgcn_raw_buffer_load_b32(rw, voff + (r * 4 + 0) * NS, (ks * 128 + h * 64) * NS, 0);
          u32 d1 = __builtin_amdgcn_raw_buffer_load_b32(rw, voff + (r * 4 + 1) * NS, (ks * 128 + h * 64) * NS, 0);
          u32 d2 = __builtin_amdgcn_raw_buffer_load_b32(rw, voff + (r * 4 + 2) * NS, (ks * 128 + h * 64) * NS, 0);
          u32 d3 = __builtin_amdgcn_raw_buffer_load_b32(rw, voff + (r * 4 + 3) * NS, (ks * 128 + h * 64) * NS, 0);
          u32 c0, c1, c2, c3;
          transpose4(d0, d1, d2, d3, c0, c1, c2, c3);
          bf[0][ks][h * 4 + r] = (int)c0; bf[1][ks][h * 4 + r] = (int)c1; bf[2][ks][h * 4 + r] = (int)c2; bf[3][ks][h * 4 + r] = (int)c3;
        }
      }
  } else {
#pragma unroll
    for (int ks = 0; ks < 4; ++ks) {
      const int u = NPRE + ks * 2;
      const u32 off = dl * SHARD + u * TILE_B + half * 4096;
      gu32* f = flags + (dl * 8 + ((wv - u) & 7)) * 32;
      gu32* f2 = flags + (dl * 8 + ((wv - u - 1) & 7)) * 32;
      if (ks >= 2) early(ks - 2);
      while (since(tstart) < PUSH_T1 + (u + 2) * PUSH_DT - (1 - half) * PUSH_DT / 2 + GATE) __builtin_amdgcn_s_sleep(4);
      sure = false;
      while (!sure) {
        if (__builtin_amdgcn_ballot_w64(probe(off + TILE_B, 3) == 0) == 0) break;
        slow(f, f2, true);
      }
      for (;;) {
#pragma unroll
        for (int i = 0; i < 8; ++i) a0[i] = __builtin_amdgcn_raw_buffer_load_b128(rxg, voff_g + (i & 3) * 1024, off + (i >> 2) * TILE_B, AUX_SYS);
        if (sure) break;
        u32 mn = a0[0][0];
#pragma unroll
        for (int i = 1; i < 8; ++i) mn = a0[i][0] < mn ? a0[i][0] : mn;
        if (__builtin_amdgcn_ballot_w64(mn == 0) == 0) break;
        slow(f, f2, true);
      }
#pragma unroll
      for (int i = 0; i < 8; ++i)
#pragma unroll
        for (int e = 0; e < 4; ++e) bf[i & 3][ks][(i >> 2) * 4 + e] = (int)(e ? a0[i][e] : ~a0[i][e]);
    }
  }
  early(2);

  known |= (u32)__builtin_amdgcn_ballot_w64(pk != 0);
  int it = 0;
  u32 o[2] = {0, 0};
  auto put = [&](int j) INL {
    auto b0 = __builtin_amdgcn_permlane32_swap(o[0], o[0], false, false);
    auto b1 = __builtin_amdgcn_permlane32_swap(o[1], o[1], false, false);
    __builtin_amdgcn_raw_buffer_store_b128(u32x4{b0[0], b1[0], b0[1], b1[1]}, rout, voff_o | __builtin_amdgcn_readlane(jo, j), 0, AUX_SYS);
    __builtin_amdgcn_sched_barrier(0);
    STAMPJ(3 + 2 * j, since(tstart));
  };
  auto slot = [&](int j) INL {
    const int due = __builtin_amdgcn_readlane(js, j);
    if (now < due) do { __builtin_amdgcn_s_sleep(1); now = since(tstart); } while (now < due);
  };
  auto hole = [&](u32x4 (&a)[8], int i0, int i1) INL {
    u32 mn = a[i0][0];
#pragma unroll
    for (int i = i0 + 1; i < i1; ++i) mn = a[i][0] < mn ? a[i][0] : mn;
    return __builtin_amdgcn_ballot_w64(mn == 0) != 0;
  };
  auto refill = [&](u32x4 (&a)[8], int i0, u32 off) INL {
#pragma unroll
    for (int i = i0; i < 8; ++i)
      if (hole(a, i, i + 1)) a[i] = __builtin_amdgcn_raw_buffer_load_b128(rxg, voff_g + (i & 3) * 1024, off + (i >> 2) * 4096, AUX_SYS);
  };
  f32x4 acc[4];
  auto mul = [&](u32x4 (&a)[8], int ks) INL {
    i32x8 af;
#pragma unroll
    for (int e = 0; e < 4; ++e) { af[e] = (int)a[2 * ks][e]; af[4 + e] = (int)a[2 * ks + 1][e]; }
    af[0] = ~af[0]; af[4] = ~af[4];
#pragma unroll
    for (int c = 0; c < 4; ++c)
      acc[c] = __builtin_amdgcn_mfma_scale_f32_16x16x128_f8f6f4(af, bf[c][ks], ks ? acc[c] : f32x4{0.f, 0.f, 0.f, 0.f}, 0, 1, 0, 0, 0, 0);
  };
  auto share = [&]() INL {
    float* rb = (float*)smem + (it & 1) * 8192;
#pragma unroll
    for (int j = 0; j < 4; ++j)
      *(f32x4*)(rb + wv * 1024 + (q * 4 + j) * 64 + m * 4) = f32x4{acc[0][j], acc[1][j], acc[2][j], acc[3][j]};
    __builtin_amdgcn_sched_barrier(0);
  };
  auto gather = [&]() INL {
    asm volatile("s_waitcnt lgkmcnt(0)" ::: "memory");
    __builtin_amdgcn_s_barrier();
    const float* rp = (float*)smem + (it & 1) * 8192 + (lane >> 5) * 4096 + wv * 128 + (lane & 31) * 4;
    f32x4 s4 = *(const f32x4*)rp;
#pragma unroll
    for (int v = 1; v < 4; ++v) s4 += *(const f32x4*)(rp + v * 1024);
    return s4;
  };
  auto finish = [&](f32x4 s4) INL {
#pragma unroll
    for (int e = 0; e < 2; ++e) {
      auto sw = __builtin_amdgcn_permlane32_swap(__float_as_uint(s4[e]), __float_as_uint(s4[e + 2]), false, false);
      const float y = (__uint_as_float(sw[0]) + __uint_as_float(sw[1])) * scale;
      const float yc = __builtin_fminf(__builtin_fmaxf(y, -60.f), 60.f);
      o[e] = __float_as_uint(y * __builtin_amdgcn_rcpf(1.f + __expf(-yc)));
    }
  };
  auto wipe = [&]() INL { __builtin_amdgcn_raw_buffer_store_b128(u32x4{0, 0, 0, 0}, rzr, voff_z + __builtin_amdgcn_readlane(jt, it), 0, AUX_SYS); };
  auto step = [&](auto first_wave, u32x4 (&a)[8], u32x4 (&an)[8], u32& tv) INL {
    wipe();
    const int gate = __builtin_amdgcn_readlane(jg, it);
    sure = gate == 0x7fffffff;
    for (bool first = true; !sure; first = false) {
      if (__builtin_expect(!hole(a, 0, 8), 1)) break;
      const u32 off = __builtin_amdgcn_readlane(jt, it);
      gu32* f = flags + (off / SHARD * 8 + ((wv - off / TILE_B) & 7)) * 32;
      if (!first) slow(f, f, false);
      else if (__builtin_amdgcn_readlane(pk, it) == 0) {
        while (since(tstart) < gate) __builtin_amdgcn_s_sleep(2);
        if (since(tstart) < gate + LONG_DUE)
          while (!sure) {
            if (__builtin_amdgcn_ballot_w64(probe(off, 7) == 0) == 0) break;
            slow(f, f, true);
          }
      }
      refill(a, 0, off);
      __builtin_amdgcn_s_waitcnt(0x0f70);
    }
    now = since(tstart);
    STAMPJ(2 + 2 * it, now);
#pragma unroll
    for (int ks = 0; ks < 4; ++ks) mul(a, ks);
    share();
    known |= (u32)__builtin_amdgcn_ballot_w64(pk != 0) | 1u << it;
    if (__builtin_expect((known & ~got) >> (it + 1) & 1, 0)) fetch(an, it + 1);
    __builtin_amdgcn_sched_barrier(0);
    auto ahead = [&]() INL {
      fetch(a, it + 2);
      __builtin_amdgcn_sched_barrier(0);
      asm volatile("" ::"v"(tv));
      tv = touch(it + 3);
      __builtin_amdgcn_sched_barrier(0);
    };
    if (first_wave()) ahead();
    const f32x4 s4 = gather();
    now += O_EARLY;
    slot((it - 1) & 63);
    pk = peek(now);
    __builtin_amdgcn_sched_barrier(0);
    put((it - 1) & 63);
    if (!first_wave()) ahead();
    finish(s4);
    ++it;
  };
  auto solo = [&](u32x4 (&a)[8]) INL {
    const int jp = (it - 1) & 63;
    int due = __builtin_amdgcn_readlane(js, jp);
    auto flush = [&]() INL {
      now = since(tstart);
      if (now >= due) { put(jp); due = 0x7fffffff; }
    };
    const u32 off = __builtin_amdgcn_readlane(jt, it);
    gu32* f = flags + (off / SHARD * 8 + ((wv - off / TILE_B) & 7)) * 32;
    sure = off == OOB;
    if (!sure && hole(a, 0, 8)) {
      if (__builtin_amdgcn_readlane(pk, it) == 0) {
        const bool tail = (off / TILE_B & 31) >= (isr ? TAIL_R : TAIL_L);
        const int gate = __builtin_amdgcn_readlane(jg, it) - (!tail ? 0 : isr ? TAIL_TR : TAIL_TL);
        for (flush(); now < gate; flush()) __builtin_amdgcn_s_sleep(1);
        if (now < gate + (!tail ? LONG_DUE : isr ? TAIL_LONGR : TAIL_LONGL))
          while (!sure) {
            if (__builtin_amdgcn_ballot_w64(probe(off, tail ? 0 : 7) == 0) == 0) break;
            slow(f, f, true);
            flush();
          }
      }
#pragma unroll
      for (int i = 0; i < 8; ++i) a[i] = __builtin_amdgcn_raw_buffer_load_b128(rxg, voff_g + (i & 3) * 1024, off + (i >> 2) * 4096, AUX_SYS);
      __builtin_amdgcn_sched_barrier(0);
      now = since(tstart);
      const int j = now + SOLO_LA >= due ? jp : 63;
      slot(j);
      put(j);
      if (j == jp) due = 0x7fffffff;
    }
#pragma unroll
    for (int ks = 0; ks < 4; ++ks) {
      while (!sure && hole(a, 2 * ks, 2 * ks + 2)) {
        slow(f, f, false);
        refill(a, 2 * ks, off);
        flush();
        __builtin_amdgcn_s_waitcnt(0x0f70);
      }
      if (ks == 3) STAMPJ(2 + 2 * it, since(tstart));
      mul(a, ks);
    }
    share();
    wipe();
    const f32x4 s4 = gather();
    now = 0;
    if (due != 0x7fffffff) { slot(jp); put(jp); }
    finish(s4);
    slot(it ? it : 63);
    put(it ? it : 63);
    ++it;
  };
  const int len = isr ? RLEN : LLEN;
  auto run = [&](auto first_wave) INL {
    fetch(a0, 0);
    pk = peek(since(tstart));
    __builtin_amdgcn_s_waitcnt(0x0f70);
    for (;;) {
      solo(a0);
      if (it == len) break;
      known |= (u32)__builtin_amdgcn_ballot_w64(pk != 0) | 1u;
      fetch(a0, 1);
      fetch(a1, 2);
      pk = 0;
      __builtin_amdgcn_s_waitcnt(0x0f70);
      do { step(first_wave, a0, a1, tv[0]); step(first_wave, a1, a0, tv[1]); } while (it < len - 1);
    }
  };
  if (AB_SPLIT && wv >= 4) run(std::false_type{}); else run(std::true_type{});
  if (isr) {
    asm volatile("s_waitcnt vmcnt(0)" ::: "memory");
    if (lane == 0 && __hip_atomic_fetch_add(&turn[0], 1u, __ATOMIC_RELAXED, __HIP_MEMORY_SCOPE_WORKGROUP) == 7)
      __hip_atomic_store((gu32*)(lws + OFL_OFF) + ((dl - 4) * 32 + set * 2 + half) * 32, call, __ATOMIC_RELAXED, __HIP_MEMORY_SCOPE_SYSTEM);
    STAMPJ(60, since(tstart));
  }
}

void build_jobs(u64 jobs[NSETS][2]) {
  unsigned char lst[2][256];
  int n[2] = {RPAD, LPAD};
  for (int i = 0; i < 256; ++i) lst[0][i] = lst[1][i] = 0xff;
  auto add = [&](int k, int d, int u) { lst[k][n[k]++] = (unsigned char)(d << 5 | u); };
  for (int u = 0, v = 0; u < NH; ++u) {
    for (; v < NH && SELF_T0 + (v + 1) * SELF_DT <= PUSH_T1 + (u + 1) * PUSH_DT; ++v) add(0, 0, v);
    for (int d = 1; d < 4; ++d) add(0, d, u);
    if (u < NUC && !is_w(u) && cross_tile(u) >= NUI) for (int d = 5; d < 8; ++d) add(0, d, u);
  }
  unsigned char ex[64];
  int ne = 0, ie = 0;
  for (int u = 0; u < 2 * NUC; ++u) ex[ne++] = (unsigned char)u;
  for (int u = 0; u < NUC; ++u) if (!is_w(u)) for (int d = 4; d < 8; ++d) ex[ne++] = (unsigned char)(d << 5 | u);
  for (int u = 2 * NUC; u < NT; ++u) ex[ne++] = (unsigned char)u;
  for (int u = 0; u < NUI; ++u) {
    for (int d = 1; d < 4; ++d) add(1, d, u);
    for (int k = 0; k < (u < NUC ? 2 : 4) && ie < ne; ++k) lst[1][n[1]++] = ex[ie++];
  }
  for (int s = 0; s < NSETS; ++s) {
    const int k = s >= NRS, ns = k ? NLS : NRS, s0 = k ? s - NRS : s;
    jobs[s][0] = jobs[s][1] = ~0ull;
    for (int j = 0; j < 16 && s0 + j * ns < n[k]; ++j) ((unsigned char*)jobs[s])[j] = lst[k][s0 + j * ns];
  }
}
static_assert(RLEN <= 16 && LLEN <= 16 && NRS <= 16 && 2 * NUC <= NT, "job lists");

}

extern "C" void kernel_launch(int rank, int nranks, ncclComm_t comm, hipStream_t stream,
                              void* const* d_in, const int* in_sizes, int n_in,
                              void* d_out, int out_size, const int* in_dtypes, int out_dtype,
                              void* const* d_out_all, void* const* d_ws_all, size_t ws_bytes) {
  (void)comm; (void)in_dtypes; (void)out_dtype;
  if (nranks != NR || (unsigned)rank >= (unsigned)NR || n_in != 4 || ws_bytes < WS_NEED ||
      in_sizes[0] != (int)SHARD || in_sizes[1] != (int)SHARD || out_size != NR * MS * NS) {
    fprintf(stderr, "kernel_launch: unexpected problem shape\n");
    return;
  }
  Params p;
  p.x = (const unsigned char*)d_in[0];
  p.w = (const unsigned char*)d_in[1];
  p.sx = (const float*)d_in[2];
  p.sw = (const float*)d_in[3];
  p.out = (float*)d_out;
  for (int d = 0; d < NR; ++d) p.ws[d] = (char*)d_ws_all[rank ^ d];
  for (int i = 0; i < 4; ++i) p.outp[i] = (float*)d_out_all[rank ^ (4 + i)];
  p.rank = rank;
  p.pad = 0;
  build_jobs(p.jobs);
  hipLaunchKernelGGL(ag_gemm_kernel, dim3(256), dim3(512), 0, stream, p);
}
```

```cpp
#include <hip/hip_runtime.h>
#include <rccl/rccl.h>
#include <cstdio>
#include <type_traits>

typedef unsigned int u32;
typedef unsigned long long u64;
typedef u32 u32x4 __attribute__((ext_vector_type(4)));
typedef u32 u32x2 __attribute__((ext_vector_type(2)));
typedef int i32x8 __attribute__((ext_vector_type(8)));
typedef float f32x4 __attribute__((ext_vector_type(4)));
typedef __attribute__((address_space(1))) u32 gu32;

namespace {

constexpr int NR = 8;
constexpr int KD = 4096;
constexpr int MS = 512;
constexpr int NS = 512;
constexpr u32 SHARD = (u32)MS * KD;
constexpr u32 TILE_B = 16 * KD;
constexpr int NT = MS / 16;
#define NH 26
#define ND 2
#define NA 4
static_assert(NH + ND + NA == NT, "tile kinds");
constexpr int NUI = NH + ND;
constexpr int NUC = 8 + NA + ND;
#define NPRE 0
#define MIXED 2
#define OSTAG 0
#define NRS 12
constexpr int NSETS = 24, NLS = NSETS - NRS;
constexpr int NRJ = 4 * NH + 3 * NA, NLJ = NT + 3 * NUI + 4 * (NA + ND);
constexpr int RLEN = (NRJ + 2 * NRS - 1) / (2 * NRS) * 2, LLEN = (NLJ + 2 * NLS - 1) / (2 * NLS) * 2;
constexpr int RPAD = RLEN * NRS - NRJ, LPAD = LLEN * NLS - NLJ;
constexpr u32 BUF_B = (u32)NR * SHARD;
constexpr int NBUF = 3;
constexpr size_t FLAG_OFF = (size_t)NBUF * BUF_B;
constexpr size_t FLAG_B = (size_t)NR * 8 * 128;
constexpr size_t OFL_OFF = FLAG_OFF + FLAG_B;
constexpr size_t OFL_B = 4 * 4096;
constexpr size_t CNT_OFF = OFL_OFF + OFL_B;
constexpr size_t CNT_B = 256 * 8 * 4;
constexpr size_t TMO_OFF = CNT_OFF + CNT_B;
constexpr size_t PACE_OFF = TMO_OFF + 4096;
constexpr size_t DBG_OFF = PACE_OFF + 4096;
constexpr size_t DBG_B = 256 * 128 * 8;
constexpr size_t WS_NEED = DBG_OFF + DBG_B;
constexpr u64 DEADLINE = 100000000ull;
#define PUSH_W 7
#define PUSH_W0 12
#define W_U0 2
#define PUSH_W2 PUSH_W
#define W_U2 NUC
#define PUSH_DT 110
#define PUSH_T0 30
#define PUSH_T1 110
#define PUSH_U1 2
#define CONS 2
#define PUSH_T1A 90
#define CONS_UP 2
#define CONS_UR 5
#define CONS_M 5
#define CONS_UP2 11
#define O_M 5
#define PUSH_G 2
#define PUSH_PRE 300
#define PUSH_LEADT 3
#define PUSH_DT0 20
#define W_DELAY 150
#define W_DT 100
#define W_SLOT 50
#define GATE (-10)
#ifndef LONG_DUE
#define LONG_DUE 150
#endif
#define O_T0 (PUSH_T1 + NUC * PUSH_DT + O_M)
#define O_DT 113
#define SELF_T0 200
#define SELF_DT 50
#define TAIL_R (NH - 1)
#define TAIL_L (NUI - 1)
#define TAIL_TL 80
#define TAIL_LONGL 150
#define TAIL_TR (-40)
#define TAIL_LONGR 0
#define SOLO_LA 80
#define POLL_SLEEP 4
#define FAUX 0
#define TOUCH_ON 1
#define AB_SPLIT 1
#define WAIT_D 6
#define WAIT_S 3
#define OWARM 0
#define OWARM_AUX 17
#define PUT_LATE 0
#define O_EARLY 25
#define AUX_SYS 17
constexpr int TR_ROW = 528;
constexpr int TR_B = 16 * TR_ROW;

struct Params {
  char* ws[NR];
  float* outp[4];
  const unsigned char* x;
  const unsigned char* w;
  const float* sx;
  const float* sw;
  float* out;
  int rank, pad;
  u64 jobs[NSETS][2];
};

__host__ __device__ constexpr bool is_w(int u) { return u >= NPRE && u < NPRE + 8; }
__host__ __device__ constexpr int cross_tile(int u) { u = u < NPRE ? u : u - 8; return u < NA ? NUI + u : NH + u - NA; }
constexpr u32 OOB = 0x80000000u;

#define STAMP(i) do {} while (0)
#define STAMPV(i, v) do {} while (0)
#define STAMPL(i) do {} while (0)
#define STAMPJ(i, v) do {} while (0)
#define STAMPW(k) do {} while (0)

#define INL __attribute__((always_inline))
__device__ __forceinline__ int since(u64 t) { return (int)(__builtin_amdgcn_s_memrealtime() - t); }

__device__ __forceinline__ int origin(u32 v, u64 tstart, int dflt) {
  int sum = 0;
  bool ok = true;
#pragma unroll
  for (int k = 0; k < 8; ++k) {
    const int r = (int)(__builtin_amdgcn_readlane(v, k) - (u32)tstart);
    ok = ok && r > -100 && r < 1000;
    sum += r;
  }
  return ok ? sum >> 3 : dflt;
}

__device__ __forceinline__ void transpose4(u32 d0, u32 d1, u32 d2, u32 d3, u32& c0, u32& c1, u32& c2, u32& c3) {
  const u32 t0 = __builtin_amdgcn_perm(d1, d0, 0x05010400u), t1 = __builtin_amdgcn_perm(d1, d0, 0x07030602u);
  const u32 t2 = __builtin_amdgcn_perm(d3, d2, 0x05010400u), t3 = __builtin_amdgcn_perm(d3, d2, 0x07030602u);
  c0 = __builtin_amdgcn_perm(t2, t0, 0x05040100u);
  c1 = __builtin_amdgcn_perm(t2, t0, 0x07060302u);
  c2 = __builtin_amdgcn_perm(t3, t1, 0x05040100u);
  c3 = __builtin_amdgcn_perm(t3, t1, 0x07060302u);
}

__global__ void __launch_bounds__(512, 2) ag_gemm_kernel(Params p) {
  __shared__ __attribute__((aligned(16))) char smem[8 * TR_B];
  __shared__ u32 turn[2];
  const int lane = threadIdx.x & 63;
  const int wv = __builtin_amdgcn_readfirstlane(threadIdx.x >> 6);
  const int xg = blockIdx.x & 7;
  const int ci = blockIdx.x >> 3;
  int rank = p.rank;
  char* myws = p.ws[0];
  char* pws = p.ws[(ci + 1) & 7];
  const unsigned char* xp = p.x;
  const unsigned char* wp = p.w;
  asm volatile("" : "+s"(rank), "+s"(myws), "+s"(pws), "+s"(xp), "+s"(wp));
  const int m = lane & 15, q = lane >> 4;
  const u64 tstart = __builtin_amdgcn_s_memrealtime();

  gu32* cntp = (gu32*)(myws + CNT_OFF) + blockIdx.x * 8 + wv;
  auto rx = __builtin_amdgcn_make_buffer_rsrc((void*)xp, 0, (int)SHARD, 0x00020000);
  auto rw = __builtin_amdgcn_make_buffer_rsrc((void*)wp, 0, (int)SHARD, 0x00020000);

  if (__builtin_expect(ci < 8, 1)) {
    if (wv > 0) __builtin_amdgcn_s_barrier();
    const int d = (ci + 1) & 7;
    const int nu = d == 0 ? NT : d < 4 ? NUI : NUC;
    const int dt = d ? PUSH_DT : SELF_DT;
    const u32 voff = (u32)(lane >> 5) * KD + (lane & 31) * 16;
    const int kb = (xg + wv) & 7;
    char* tw = smem + wv * TR_B + (lane >> 5) * TR_ROW + (lane & 31) * 16;
    const char* tr = smem + wv * TR_B + m * TR_ROW + q * 16;
    const u32 soff = lane * 16;
    u32 call = 0;
    char* dst = nullptr;
    gu32* pace = (gu32*)(myws + PACE_OFF) + d * 32;
    int t1 = -1;
#pragma unroll 1
    for (int u = wv; u < nu; u += 8) {
      int t0 = d == 0 ? SELF_T0 : u < PUSH_U1 ? PUSH_T0 : (CONS & 1) ? PUSH_T1A : PUSH_T1;
      if (u > 0) {
        const int due = u < PUSH_LEADT ? u * PUSH_DT0 : t0 + u * dt - PUSH_PRE;
        while (since(tstart) < due) __builtin_amdgcn_s_sleep(2);
      }
      auto first = [&]() INL {
        if (u < 8) {
          call = __hip_atomic_load(cntp, __ATOMIC_RELAXED, __HIP_MEMORY_SCOPE_AGENT);
          if (wv == 0) {
            if (lane < 2) __hip_atomic_store(&turn[lane], 0u, __ATOMIC_RELAXED, __HIP_MEMORY_SCOPE_WORKGROUP);
            asm volatile("s_waitcnt lgkmcnt(0)" ::: "memory");
            __builtin_amdgcn_s_barrier();
          }
        }
      };
      u32x4 e[8];
      if (d < 4 || !is_w(u)) {
        const int t = d < 4 ? u : cross_tile(u);
#pragma unroll
        for (int j = 0; j < 8; ++j) e[j] = __builtin_amdgcn_raw_buffer_load_b128(rx, voff + j * 2 * KD, t * TILE_B + kb * 512, 0);
        first();
#pragma unroll
        for (int j = 0; j < 8; ++j) *(u32x4*)(tw + j * 2 * TR_ROW) = e[j];
#pragma unroll
        for (int i = 0; i < 8; ++i) e[i] = *(const u32x4*)(tr + i * 64);
      } else {
        const u32 so = (u32)(kb * 512 + (u - NPRE) * 64) * NS + (d & 3) * 128;
#pragma unroll
        for (int j = 0; j < 8; ++j) e[j] = __builtin_amdgcn_raw_buffer_load_b128(rw, (u32)(lane >> 3) * NS + (lane & 7) * 16 + j * 8 * NS, so, 0);
        first();
#pragma unroll
        for (int j = 0; j < 8; ++j) *(u32x4*)(smem + wv * TR_B + lane * 16 + j * 1024) = e[j];
        const u32* wr = (const u32*)(smem + wv * TR_B + q * 2048 + m * 4);
#pragma unroll
        for (int j = 0; j < 8; ++j) {
          const u32* w4 = wr + (j >> 2) * 16 + (j & 3) * 128;
          u32 c0, c1, c2, c3;
          transpose4(w4[0], w4[32], w4[64], w4[96], c0, c1, c2, c3);
          e[(j & 4) + 0][j & 3] = c0; e[(j & 4) + 1][j & 3] = c1; e[(j & 4) + 2][j & 3] = c2; e[(j & 4) + 3][j & 3] = c3;
        }
      }
#pragma unroll
      for (int i = 0; i < 8; ++i) e[i][0] = ~e[i][0];
      if (u < 8) {
        call = __builtin_amdgcn_readfirstlane(call) + 1;
        dst = pws + (size_t)(call % NBUF) * BUF_B + (size_t)d * SHARD + kb * 8192;
      }
      auto rd = __builtin_amdgcn_make_buffer_rsrc((void*)(dst + u * TILE_B), 0, 8192, 0x00020000);
      if ((CONS & 1) && d && u >= CONS_UR) {
        if (t1 < 0) {
          while (since(tstart) < t0 + u * dt - 90) __builtin_amdgcn_s_sleep(2);
          t1 = origin(__hip_atomic_load(pace + (lane & 7), __ATOMIC_RELAXED, __HIP_MEMORY_SCOPE_AGENT), tstart, PUSH_T1 - CONS_M) + CONS_M;
        }
        t0 = t1;
      }
      const int win = u < W_U0 ? PUSH_W0 : u < W_U2 ? PUSH_W : PUSH_W2;
#pragma unroll
      for (int i = 0; i < 8; ++i) {
        if (i == 7 && win == 7) asm volatile("s_waitcnt vmcnt(6)" ::: "memory");
        else if (u > 0 && d) {
          while ((int)__hip_atomic_load(&turn[0], __ATOMIC_RELAXED, __HIP_MEMORY_SCOPE_WORKGROUP) < u * 8 + i - (win - 1)) __builtin_amdgcn_s_sleep(1);
        }
        if (i % PUSH_G == 0) {
          const int due = t0 + (u * 8 + i) * dt / 8;
          while (since(tstart) < due) __builtin_amdgcn_s_sleep(1);
        }
        if (i == 0 && CONS && (u == CONS_UP || u == CONS_UP2) && lane == 0)
          __hip_atomic_store(pace + (u == CONS_UP ? 0 : 256) + xg, (u32)__builtin_amdgcn_s_memrealtime() - u * dt, __ATOMIC_RELAXED, __HIP_MEMORY_SCOPE_AGENT);
        if (i == 0) STAMPL(u >> 3);
        __builtin_amdgcn_raw_buffer_store_b128(e[i], rd, soff + i * 1024, 0, AUX_SYS);
        __builtin_amdgcn_sched_barrier(0);
      }
#pragma unroll
      for (int k = 7; k >= 0; --k) {
        asm volatile("s_waitcnt vmcnt(%0)" ::"n"(k) : "memory");
        __hip_atomic_fetch_max(&turn[0], (u32)(u * 8 + 8 - k), __ATOMIC_RELAXED, __HIP_MEMORY_SCOPE_WORKGROUP);
      }
    }
    STAMPL(4);
    if (lane == 0) {
      const u32 done = __hip_atomic_fetch_add(&turn[1], 1u, __ATOMIC_RELAXED, __HIP_MEMORY_SCOPE_WORKGROUP);
      if (done == 7) __hip_atomic_store((gu32*)(pws + FLAG_OFF) + (d * 8 + xg) * 32, call, __ATOMIC_RELAXED, __HIP_MEMORY_SCOPE_SYSTEM);
      __hip_atomic_store(cntp, call, __ATOMIC_RELAXED, __HIP_MEMORY_SCOPE_AGENT);
    }
    if (d >= 4) {
      auto rz = __builtin_amdgcn_make_buffer_rsrc((void*)(myws + (size_t)((call + NBUF - 1) % NBUF) * BUF_B + (size_t)d * SHARD + (NPRE + xg) * TILE_B + wv * 8192),
                                                  0, 8192, 0x00020000);
      while (since(tstart) < PUSH_T1 + NUC * PUSH_DT + 100 + wv * 50) __builtin_amdgcn_s_sleep(8);
#pragma unroll
      for (int i = 0; i < 8; ++i) __builtin_amdgcn_raw_buffer_store_b128(u32x4{0, 0, 0, 0}, rz, soff + i * 1024, 0, AUX_SYS);
      if (xg == 0 && wv == 0) {
        const char* fp = myws + OFL_OFF + (d - 4) * 4096 + (lane < 2 * NRS ? lane : 0) * 128;
        while (since(tstart) < O_T0 + (NRJ - NRS) * O_DT / 8) __builtin_amdgcn_s_sleep(4);
        u32 v = 0, n = 0;
        for (;;) {
          asm volatile("global_load_dword %0, %1, off sc0 sc1\n\ts_waitcnt vmcnt(%2)" : "+v"(v) : "v"(fp), "n"(WAIT_D - 1) : "memory");
          if (__builtin_amdgcn_ballot_w64(v < call) == 0) break;
          asm volatile("s_sleep %1" : "+v"(v) : "n"(WAIT_S));
          if (__builtin_amdgcn_ballot_w64(v < call) == 0) break;
          if ((++n & 255u) == 0 && __builtin_amdgcn_s_memrealtime() - tstart > DEADLINE) {
            if (lane == 0) p.out[0] = __builtin_nanf("");
            break;
          }
        }
        asm volatile("s_waitcnt vmcnt(0)" : "+v"(v) :: "memory");
        STAMP(1);
      }
    }
    return;
  }

  STAMP(0);
  u32 call = __hip_atomic_load(cntp, __ATOMIC_RELAXED, __HIP_MEMORY_SCOPE_AGENT) + 1;
  gu32* tmo = (gu32*)(myws + TMO_OFF);
  const int g = ci & 7;
  const int sl = (ci - 8) >> 3;
  const int set = sl == 0 ? xg : sl == 1 ? 16 + xg : 8 + (xg >> 1) + ((xg ^ g) & 1) * 4;
  const bool isr = set < NRS;
  const int dl = 4 + (g >> 1), half = g & 1;
  u64 jlo = p.jobs[set][0], jhi = p.jobs[set][1];
  float* outp = isr ? p.outp[dl - 4] : p.out;
  char* lws = p.ws[dl];
  float scale = p.sx[0] * p.sw[0];
  asm volatile("" : "+s"(jlo), "+s"(jhi), "+s"(outp), "+s"(lws));
  asm volatile("" : "+v"(scale));
  if (threadIdx.x == 0) __hip_atomic_store(&turn[0], 0u, __ATOMIC_RELAXED, __HIP_MEMORY_SCOPE_WORKGROUP);
  gu32* flags = (gu32*)(myws + FLAG_OFF);
  const u32 voff_g = lane * 16;

  bool sure;
  u32 n = 0;
  auto word = [&](gu32* a) INL { return __builtin_amdgcn_readfirstlane(__hip_atomic_load(a, __ATOMIC_RELAXED, __HIP_MEMORY_SCOPE_SYSTEM)); };
  auto slow = [&](gu32* f, gu32* f2, bool nap) INL {
    if ((++n & 31u) == 0) {
      if (word(f) >= call && word(f2) >= call) { sure = true; return; }
      if (word(tmo) == call || __builtin_amdgcn_s_memrealtime() - tstart > DEADLINE) {
        __hip_atomic_store(tmo, call, __ATOMIC_RELAXED, __HIP_MEMORY_SCOPE_SYSTEM);
        sure = true;
        return;
      }
    }
    if (nap) __builtin_amdgcn_s_sleep(POLL_SLEEP); else __builtin_amdgcn_s_sleep(1);
  };

  call = __builtin_amdgcn_readfirstlane(call);
  if (lane == 0) __hip_atomic_store(cntp, call, __ATOMIC_RELAXED, __HIP_MEMORY_SCOPE_AGENT);
  char* gbuf = myws + (size_t)(call % NBUF) * BUF_B + wv * 8192;
  auto rxg = __builtin_amdgcn_make_buffer_rsrc((void*)gbuf, 0, (int)BUF_B, 0x00020000);
  auto rzr = __builtin_amdgcn_make_buffer_rsrc((void*)(myws + (size_t)((call + NBUF - 1) % NBUF) * BUF_B), 0, isr ? 0 : (int)BUF_B, 0x00020000);
  auto rout = __builtin_amdgcn_make_buffer_rsrc((void*)outp, 0, NR * MS * NS * 4, 0x00020000);
  const u32 voff_z = (u32)g * 8192 + threadIdx.x * 16;
  const u32 voff_o = ((wv * 2 + ((lane >> 4) & 1)) * NS + g * 64 + (lane & 15) * 4) * 4 + (lane < 32 ? 0 : OOB);
  const u32 voff_t = lane < (sl == 2 ? 16 : 8) ? (sl == 2 ? g & 6 : g) * 1024 + lane * 128 : OOB;
  const int rot = (dl - 4) * OSTAG, r0 = (set * 2 + half + rot) % (2 * NRS);
  const int oslot = O_T0 + (r0 - 2 * RPAD) * O_DT / 16;
  const int oslot0 = r0 >= rot ? oslot : O_T0 + (r0 - (rot + 2 * RPAD > 2 * NRS ? rot + 2 * RPAD - 2 * NRS : 0)) * O_DT / 16;

  u32 jt, jo;
  int jg, js;
  {
    const u32 ent = lane < 16 ? (u32)((lane < 8 ? jlo : jhi) >> (lane & 7) * 8) & 0xff : 0xff;
    const int u = ent & 31, d = isr && ent >= 128 ? dl ^ ((ent >> 5) - 4) : ent >> 5;
    jt = ent == 0xff ? OOB : d * SHARD + u * TILE_B;
    jg = ent == 0xff ? 0x7fffffff : (d ? PUSH_T1 + (u + 1) * PUSH_DT : SELF_T0 + (u + 1) * SELF_DT) + GATE;
    jo = ent == 0xff ? OOB : (u32)((rank ^ d) * MS + (d < 4 ? u : cross_tile(u)) * 16) * NS * 4;
    js = ent == 0xff || !isr ? 0 : lane ? oslot + lane * NRS * O_DT / 8 : oslot0;
  }
  u32 known = 0;
  u32 got = 0;
  u32 warm = 0;
  int now;

  auto probe = [&](u32 off, int chunk) INL { return __builtin_amdgcn_raw_buffer_load_b32(rxg, 1008, off + chunk * 1024, AUX_SYS); };
  auto rsrc_if = [&](bool on) INL { return __builtin_amdgcn_make_buffer_rsrc((void*)gbuf, 0, on ? (int)BUF_B : 0, 0x00020000); };
  auto fetch = [&](u32x4 (&a)[8], int j) INL {
    auto r = rsrc_if(known >> j & 1);
    got |= known & 1u << j;
    const u32 off = __builtin_amdgcn_readlane(jt, j);
#pragma unroll
    for (int i = 0; i < 8; ++i) a[i] = __builtin_amdgcn_raw_buffer_load_b128(r, voff_g + (i & 3) * 1024, off + (i >> 2) * 4096, FAUX);
  };
  auto touch = [&](int j) INL {
    j += __builtin_ctz(((known & ~warm) | 1u << 31) >> j);
    warm |= 1u << (j & 15);
    return __builtin_amdgcn_raw_buffer_load_b32(rsrc_if(TOUCH_ON && j < 16), voff_t, __builtin_amdgcn_readlane(jt, j), AUX_SYS);
  };
  auto peek = [&](int t) INL {
    const bool on = jg <= t && !(known >> (lane & 31) & 1);
    return __builtin_amdgcn_raw_buffer_load_b32(rxg, on ? jt : OOB, 8176, AUX_SYS);
  };
  u32 pk, tv[4];
  auto early = [&](int phase) INL {
    if (phase) known |= (u32)__builtin_amdgcn_ballot_w64(pk != 0);
    pk = peek(since(tstart));
#pragma unroll
    for (int k = 0; k < 4; ++k) {
      if (phase == 1) tv[k] = touch(0);
      if (phase == 2) asm volatile("" ::"v"(tv[k]));
    }
  };

  i32x8 bf[4][4];
  u32x4 a0[8], a1[8];
  if (!isr) {
    const u32 voff = (u32)(wv * 512 + q * 16) * NS + g * 64 + m * 4;
#pragma unroll
    for (int ks = 0; ks < 4; ++ks)
#pragma unroll
      for (int h = 0; h < 2; ++h) {
        const int due = W_DELAY + sl * W_SLOT + (ks * 2 + h) * W_DT;
        while (since(tstart) < due) __builtin_amdgcn_s_sleep(4);
        if (ks * 2 + h >= 6) early(ks * 2 + h - 6);
#pragma unroll
        for (int r = 0; r < 4; ++r) {
          u32 d0 = __builtin_amdgcn_raw_buffer_load_b32(rw, voff + (r * 4 + 0) * NS, (ks * 128 + h * 64) * NS, 0);
          u32 d1 = __builtin_amdgcn_raw_buffer_load_b32(rw, voff + (r * 4 + 1) * NS, (ks * 128 + h * 64) * NS, 0);
          u32 d2 = __builtin_amdgcn_raw_buffer_load_b32(rw, voff + (r * 4 + 2) * NS, (ks * 128 + h * 64) * NS, 0);
          u32 d3 = __builtin_amdgcn_raw_buffer_load_b32(rw, voff + (r * 4 + 3) * NS, (ks * 128 + h * 64) * NS, 0);
          u32 c0, c1, c2, c3;
          transpose4(d0, d1, d2, d3, c0, c1, c2, c3);
          bf[0][ks][h * 4 + r] = (int)c0; bf[1][ks][h * 4 + r] = (int)c1; bf[2][ks][h * 4 + r] = (int)c2; bf[3][ks][h * 4 + r] = (int)c3;
        }
      }
  } else {
#pragma unroll
    for (int ks = 0; ks < 4; ++ks) {
      const int u = NPRE + ks * 2;
      const u32 off = dl * SHARD + u * TILE_B + half * 4096;
      gu32* f = flags + (dl * 8 + ((wv - u) & 7)) * 32;
      gu32* f2 = flags + (dl * 8 + ((wv - u - 1) & 7)) * 32;
      if (ks >= 2) early(ks - 2);
      while (since(tstart) < PUSH_T1 + (u + 2) * PUSH_DT - (1 - half) * PUSH_DT / 2 + GATE) __builtin_amdgcn_s_sleep(4);
      sure = false;
      while (!sure) {
        if (__builtin_amdgcn_ballot_w64(probe(off + TILE_B, 3) == 0) == 0) break;
        slow(f, f2, true);
      }
      for (;;) {
#pragma unroll
        for (int i = 0; i < 8; ++i) a0[i] = __builtin_amdgcn_raw_buffer_load_b128(rxg, voff_g + (i & 3) * 1024, off + (i >> 2) * TILE_B, AUX_SYS);
        if (sure) break;
        u32 mn = a0[0][0];
#pragma unroll
        for (int i = 1; i < 8; ++i) mn = a0[i][0] < mn ? a0[i][0] : mn;
        if (__builtin_amdgcn_ballot_w64(mn == 0) == 0) break;
        slow(f, f2, true);
      }
#pragma unroll
      for (int i = 0; i < 8; ++i)
#pragma unroll
        for (int e = 0; e < 4; ++e) bf[i & 3][ks][(i >> 2) * 4 + e] = (int)(e ? a0[i][e] : ~a0[i][e]);
    }
  }
  early(2);

  known |= (u32)__builtin_amdgcn_ballot_w64(pk != 0);
  int it = 0;
  u32 o[2] = {0, 0};
  auto put = [&](int j) INL {
    auto b0 = __builtin_amdgcn_permlane32_swap(o[0], o[0], false, false);
    auto b1 = __builtin_amdgcn_permlane32_swap(o[1], o[1], false, false);
    __builtin_amdgcn_raw_buffer_store_b128(u32x4{b0[0], b1[0], b0[1], b1[1]}, rout, voff_o | __builtin_amdgcn_readlane(jo, j), 0, AUX_SYS);
    __builtin_amdgcn_sched_barrier(0);
    STAMPJ(3 + 2 * j, since(tstart));
  };
  auto slot = [&](int j) INL {
    const int due = __builtin_amdgcn_readlane(js, j);
    if (now < due) do { __builtin_amdgcn_s_sleep(1); now = since(tstart); } while (now < due);
  };
  auto hole = [&](u32x4 (&a)[8], int i0, int i1) INL {
    u32 mn = a[i0][0];
#pragma unroll
    for (int i = i0 + 1; i < i1; ++i) mn = a[i][0] < mn ? a[i][0] : mn;
    return __builtin_amdgcn_ballot_w64(mn == 0) != 0;
  };
  auto refill = [&](u32x4 (&a)[8], int i0, u32 off) INL {
#pragma unroll
    for (int i = i0; i < 8; ++i)
      if (hole(a, i, i + 1)) a[i] = __builtin_amdgcn_raw_buffer_load_b128(rxg, voff_g + (i & 3) * 1024, off + (i >> 2) * 4096, AUX_SYS);
  };
  f32x4 acc[4];
  auto mul = [&](u32x4 (&a)[8], int ks) INL {
    i32x8 af;
#pragma unroll
    for (int e = 0; e < 4; ++e) { af[e] = (int)a[2 * ks][e]; af[4 + e] = (int)a[2 * ks + 1][e]; }
    af[0] = ~af[0]; af[4] = ~af[4];
#pragma unroll
    for (int c = 0; c < 4; ++c)
      acc[c] = __builtin_amdgcn_mfma_scale_f32_16x16x128_f8f6f4(af, bf[c][ks], ks ? acc[c] : f32x4{0.f, 0.f, 0.f, 0.f}, 0, 1, 0, 0, 0, 0);
  };
  auto share = [&]() INL {
    float* rb = (float*)smem + (it & 1) * 8192;
#pragma unroll
    for (int j = 0; j < 4; ++j)
      *(f32x4*)(rb + wv * 1024 + (q * 4 + j) * 64 + m * 4) = f32x4{acc[0][j], acc[1][j], acc[2][j], acc[3][j]};
    __builtin_amdgcn_sched_barrier(0);
  };
  auto gather = [&]() INL {
    asm volatile("s_waitcnt lgkmcnt(0)" ::: "memory");
    __builtin_amdgcn_s_barrier();
    const float* rp = (float*)smem + (it & 1) * 8192 + (lane >> 5) * 4096 + wv * 128 + (lane & 31) * 4;
    f32x4 s4 = *(const f32x4*)rp;
#pragma unroll
    for (int v = 1; v < 4; ++v) s4 += *(const f32x4*)(rp + v * 1024);
    return s4;
  };
  auto finish = [&](f32x4 s4) INL {
#pragma unroll
    for (int e = 0; e < 2; ++e) {
      auto sw = __builtin_amdgcn_permlane32_swap(__float_as_uint(s4[e]), __float_as_uint(s4[e + 2]), false, false);
      const float y = (__uint_as_float(sw[0]) + __uint_as_float(sw[1])) * scale;
      const float yc = __builtin_fminf(__builtin_fmaxf(y, -60.f), 60.f);
      o[e] = __float_as_uint(y * __builtin_amdgcn_rcpf(1.f + __expf(-yc)));
    }
  };
  auto wipe = [&]() INL { __builtin_amdgcn_raw_buffer_store_b128(u32x4{0, 0, 0, 0}, rzr, voff_z + __builtin_amdgcn_readlane(jt, it), 0, AUX_SYS); };
  auto step = [&](auto first_wave, u32x4 (&a)[8], u32x4 (&an)[8], u32& tv) INL {
    wipe();
    const int gate = __builtin_amdgcn_readlane(jg, it);
    sure = gate == 0x7fffffff;
    for (bool first = true; !sure; first = false) {
      if (__builtin_expect(!hole(a, 0, 8), 1)) break;
      const u32 off = __builtin_amdgcn_readlane(jt, it);
      gu32* f = flags + (off / SHARD * 8 + ((wv - off / TILE_B) & 7)) * 32;
      if (!first) slow(f, f, false);
      else if (__builtin_amdgcn_readlane(pk, it) == 0) {
        while (since(tstart) < gate) __builtin_amdgcn_s_sleep(2);
        if (since(tstart) < gate + LONG_DUE)
          while (!sure) {
            if (__builtin_amdgcn_ballot_w64(probe(off, 7) == 0) == 0) break;
            slow(f, f, true);
          }
      }
      refill(a, 0, off);
      __builtin_amdgcn_s_waitcnt(0x0f70);
    }
    now = since(tstart);
    STAMPJ(2 + 2 * it, now);
#pragma unroll
    for (int ks = 0; ks < 4; ++ks) mul(a, ks);
    share();
    known |= (u32)__builtin_amdgcn_ballot_w64(pk != 0) | 1u << it;
    if (__builtin_expect((known & ~got) >> (it + 1) & 1, 0)) fetch(an, it + 1);
    __builtin_amdgcn_sched_barrier(0);
    auto ahead = [&]() INL {
      fetch(a, it + 2);
      __builtin_amdgcn_sched_barrier(0);
      asm volatile("" ::"v"(tv));
      tv = touch(it + 3);
      __builtin_amdgcn_sched_barrier(0);
    };
    if (first_wave()) ahead();
    now += O_EARLY;
    slot(it - 1);
    pk = peek(now);
    __builtin_amdgcn_sched_barrier(0);
    put(it - 1);
    const f32x4 s4 = gather();
    if (!first_wave()) ahead();
    finish(s4);
    ++it;
  };
  auto solo = [&](u32x4 (&a)[8]) INL {
    const int jp = (it - 1) & 63;
    int due = __builtin_amdgcn_readlane(js, jp);
    auto flush = [&]() INL {
      now = since(tstart);
      if (now >= due) { put(jp); due = 0x7fffffff; }
    };
    const u32 off = __builtin_amdgcn_readlane(jt, it);
    gu32* f = flags + (off / SHARD * 8 + ((wv - off / TILE_B) & 7)) * 32;
    sure = off == OOB;
    if (!sure && hole(a, 0, 8)) {
      if (__builtin_amdgcn_readlane(pk, it) == 0) {
        const bool tail = (off / TILE_B & 31) >= (isr ? TAIL_R : TAIL_L);
        const int gate = __builtin_amdgcn_readlane(jg, it) - (!tail ? 0 : isr ? TAIL_TR : TAIL_TL);
        for (flush(); now < gate; flush()) __builtin_amdgcn_s_sleep(1);
        if (now < gate + (!tail ? LONG_DUE : isr ? TAIL_LONGR : TAIL_LONGL))
          while (!sure) {
            if (__builtin_amdgcn_ballot_w64(probe(off, tail ? 0 : 7) == 0) == 0) break;
            slow(f, f, true);
            flush();
          }
      }
#pragma unroll
      for (int i = 0; i < 8; ++i) a[i] = __builtin_amdgcn_raw_buffer_load_b128(rxg, voff_g + (i & 3) * 1024, off + (i >> 2) * 4096, AUX_SYS);
      __builtin_amdgcn_sched_barrier(0);
      now = since(tstart);
      const int j = now + SOLO_LA >= due ? jp : 63;
      slot(j);
      put(j);
      if (j == jp) due = 0x7fffffff;
    }
#pragma unroll
    for (int ks = 0; ks < 4; ++ks) {
      while (!sure && hole(a, 2 * ks, 2 * ks + 2)) {
        slow(f, f, false);
        refill(a, 2 * ks, off);
        flush();
        __builtin_amdgcn_s_waitcnt(0x0f70);
      }
      if (ks == 3) STAMPJ(2 + 2 * it, since(tstart));
      mul(a, ks);
    }
    share();
    wipe();
    now = 0;
    if (due != 0x7fffffff) { slot(jp); put(jp); }
    finish(gather());
    slot(it ? it : 63);
    put(it ? it : 63);
    ++it;
  };
  const int len = isr ? RLEN : LLEN;
  auto run = [&](auto first_wave) INL {
    fetch(a0, 0);
    pk = peek(since(tstart));
    __builtin_amdgcn_s_waitcnt(0x0f70);
    for (;;) {
      solo(a0);
      if (it == len) break;
      known |= (u32)__builtin_amdgcn_ballot_w64(pk != 0) | 1u;
      fetch(a0, 1);
      fetch(a1, 2);
      pk = 0;
      if ((CONS & 2) && isr) {
        while (since(tstart) < PUSH_T1 + CONS_UP2 * PUSH_DT + 150) __builtin_amdgcn_s_sleep(2);
        const int sh = origin(__hip_atomic_load((gu32*)(myws + PACE_OFF) + 256 + dl * 32 + (lane & 7), __ATOMIC_RELAXED, __HIP_MEMORY_SCOPE_AGENT), tstart, PUSH_T1 + 50) - PUSH_T1;
        js = js ? js + sh : 0;
      }
      __builtin_amdgcn_s_waitcnt(0x0f70);
      do { step(first_wave, a0, a1, tv[0]); step(first_wave, a1, a0, tv[1]); } while (it < len - 1);
    }
  };
  if (AB_SPLIT && wv >= 4) run(std::false_type{}); else run(std::true_type{});
  if (isr) {
    asm volatile("s_waitcnt vmcnt(0)" ::: "memory");
    if (lane == 0 && __hip_atomic_fetch_add(&turn[0], 1u, __ATOMIC_RELAXED, __HIP_MEMORY_SCOPE_WORKGROUP) == 7)
      __hip_atomic_store((gu32*)(lws + OFL_OFF) + ((dl - 4) * 32 + set * 2 + half) * 32, call, __ATOMIC_RELAXED, __HIP_MEMORY_SCOPE_SYSTEM);
    STAMPJ(60, since(tstart));
  }
}

void build_jobs(u64 jobs[NSETS][2]) {
  unsigned char lst[2][256];
  int n[2] = {RPAD, LPAD};
  for (int i = 0; i < 256; ++i) lst[0][i] = lst[1][i] = 0xff;
  auto add = [&](int k, int d, int u) { lst[k][n[k]++] = (unsigned char)(d << 5 | u); };
  for (int u = 0, v = 0; u < NH; ++u) {
    for (; v < NH && SELF_T0 + (v + 1) * SELF_DT <= PUSH_T1 + (u + 1) * PUSH_DT; ++v) add(0, 0, v);
    for (int d = 1; d < 4; ++d) add(0, d, u);
    if (u < NUC && !is_w(u) && cross_tile(u) >= NUI) for (int d = 5; d < 8; ++d) add(0, d, u);
  }
  unsigned char ex[64];
  int ne = 0, ie = 0;
  for (int u = 0; u < 2 * NUC; ++u) ex[ne++] = (unsigned char)u;
  for (int u = 0; u < NUC; ++u) if (!is_w(u)) for (int d = 4; d < 8; ++d) ex[ne++] = (unsigned char)(d << 5 | u);
  for (int u = 2 * NUC; u < NT; ++u) ex[ne++] = (unsigned char)u;
  for (int u = 0; u < NUI; ++u) {
    for (int d = 1; d < 4; ++d) add(1, d, u);
    for (int k = 0; k < (u < NUC ? 2 : 4) && ie < ne; ++k) lst[1][n[1]++] = ex[ie++];
  }
  for (int s = 0; s < NSETS; ++s) {
    const int k = s >= NRS, ns = k ? NLS : NRS, s0 = k ? s - NRS : s;
    jobs[s][0] = jobs[s][1] = ~0ull;
    for (int j = 0; j < 16 && s0 + j * ns < n[k]; ++j) ((unsigned char*)jobs[s])[j] = lst[k][s0 + j * ns];
  }
}
static_assert(RLEN <= 16 && LLEN <= 16 && NRS <= 16 && 2 * NUC <= NT, "job lists");

}

extern "C" void kernel_launch(int rank, int nranks, ncclComm_t comm, hipStream_t stream,
                              void* const* d_in, const int* in_sizes, int n_in,
                              void* d_out, int out_size, const int* in_dtypes, int out_dtype,
                              void* const* d_out_all, void* const* d_ws_all, size_t ws_bytes) {
  (void)comm; (void)in_dtypes; (void)out_dtype;
  if (nranks != NR || (unsigned)rank >= (unsigned)NR || n_in != 4 || ws_bytes < WS_NEED ||
      in_sizes[0] != (int)SHARD || in_sizes[1] != (int)SHARD || out_size != NR * MS * NS) {
    fprintf(stderr, "kernel_launch: unexpected problem shape\n");
    return;
  }
  Params p;
  p.x = (const unsigned char*)d_in[0];
  p.w = (const unsigned char*)d_in[1];
  p.sx = (const float*)d_in[2];
  p.sw = (const float*)d_in[3];
  p.out = (float*)d_out;
  for (int d = 0; d < NR; ++d) p.ws[d] = (char*)d_ws_all[rank ^ d];
  for (int i = 0; i < 4; ++i) p.outp[i] = (float*)d_out_all[rank ^ (4 + i)];
  p.rank = rank;
  p.pad = 0;
  build_jobs(p.jobs);
  hipLaunchKernelGGL(ag_gemm_kernel, dim3(256), dim3(512), 0, stream, p);
}
```

```cpp
#include <hip/hip_runtime.h>
#include <rccl/rccl.h>
#include <cstdio>
#include <type_traits>

typedef unsigned int u32;
typedef unsigned long long u64;
typedef u32 u32x4 __attribute__((ext_vector_type(4)));
typedef u32 u32x2 __attribute__((ext_vector_type(2)));
typedef int i32x8 __attribute__((ext_vector_type(8)));
typedef float f32x4 __attribute__((ext_vector_type(4)));
typedef __attribute__((address_space(1))) u32 gu32;

namespace {

constexpr int NR = 8;
constexpr int KD = 4096;
constexpr int MS = 512;
constexpr int NS = 512;
constexpr u32 SHARD = (u32)MS * KD;
constexpr u32 TILE_B = 16 * KD;
constexpr int NT = MS / 16;
#define NH 26
#define ND 2
#define NA 4
static_assert(NH + ND + NA == NT, "tile kinds");
constexpr int NUI = NH + ND;
constexpr int NUC = 8 + NA + ND;
#define NPRE 1
#define MIXED 2
#define OSTAG 0
#define NRS 12
constexpr int NSETS = 24, NLS = NSETS - NRS;
constexpr int NRJ = 4 * NH + 3 * NA, NLJ = NT + 3 * NUI + 4 * (NA + ND);
constexpr int RLEN = (NRJ + 2 * NRS - 1) / (2 * NRS) * 2, LLEN = (NLJ + 2 * NLS - 1) / (2 * NLS) * 2;
constexpr int RPAD = RLEN * NRS - NRJ, LPAD = LLEN * NLS - NLJ;
constexpr u32 BUF_B = (u32)NR * SHARD;
constexpr int NBUF = 3;
constexpr size_t FLAG_OFF = (size_t)NBUF * BUF_B;
constexpr size_t FLAG_B = (size_t)NR * 8 * 128;
constexpr size_t OFL_OFF = FLAG_OFF + FLAG_B;
constexpr size_t OFL_B = 4 * 4096;
constexpr size_t CNT_OFF = OFL_OFF + OFL_B;
constexpr size_t CNT_B = 256 * 8 * 4;
constexpr size_t TMO_OFF = CNT_OFF + CNT_B;
constexpr size_t PACE_OFF = TMO_OFF + 4096;
constexpr size_t DBG_OFF = PACE_OFF + 4096;
constexpr size_t DBG_B = 256 * 128 * 8;
constexpr size_t WS_NEED = DBG_OFF + DBG_B;
constexpr u64 DEADLINE = 100000000ull;
#define PUSH_W 7
#define PUSH_W0 7
#define W_U0 2
#define PUSH_W2 PUSH_W
#define W_U2 NUC
#define PUSH_DT 110
#define PUSH_T0 60
#define PUSH_T1 60
#define PUSH_U1 2
#define CONS 2
#define PUSH_T1A 90
#define CONS_UP 2
#define CONS_UR 5
#define CONS_M 5
#define CONS_UP2 11
#define O_M 5
#define PUSH_G 2
#define PUSH_PRE 300
#define PUSH_LEADT 3
#define NLEAD 2
#define PUSH_DT0 20
#define W_DELAY 150
#define W_DT 100
#define W_SLOT 50
#define GATE (-10)
#ifndef LONG_DUE
#define LONG_DUE 150
#endif
#define O_T0 (PUSH_T1 + NUC * PUSH_DT + O_M)
#define O_DT 113
#define SELF_T0 200
#define SELF_DT 50
#define TAIL_R (NH - 1)
#define TAIL_L (NUI - 1)
#define TAIL_TL 80
#define TAIL_LONGL 150
#define TAIL_TR (-40)
#define TAIL_LONGR 0
#define SOLO_LA 80
#define POLL_SLEEP 4
#define FAUX 0
#define TOUCH_ON 1
#define AB_SPLIT 1
#define WAIT_D 6
#define WAIT_S 3
#define CPF 0
#define PWARM 0
#define OWARM 0
#define OWARM_AUX 17
#define PUT_LATE 0
#define O_EARLY 25
#define AUX_SYS 17
#define AUX_AGENT 16
constexpr int TR_ROW = 528;
constexpr int TR_B = 16 * TR_ROW;

struct Params {
  char* ws[NR];
  float* outp[4];
  const unsigned char* x;
  const unsigned char* w;
  const float* sx;
  const float* sw;
  float* out;
  int rank, pad;
  u64 jobs[NSETS][2];
};

__host__ __device__ constexpr bool is_w(int u) { return u >= NPRE && u < NPRE + 8; }
__host__ __device__ constexpr int cross_tile(int u) { u = u < NPRE ? u : u - 8; return u < NA ? NUI + u : NH + u - NA; }
constexpr u32 OOB = 0x80000000u;

#define STAMP(i) do {} while (0)
#define STAMPV(i, v) do {} while (0)
#define STAMPL(i) do {} while (0)
#define STAMPJ(i, v) do {} while (0)
#define STAMPW(k) do {} while (0)

#define INL __attribute__((always_inline))
__device__ __forceinline__ int since(u64 t) { return (int)(__builtin_amdgcn_s_memrealtime() - t); }

__device__ __forceinline__ int origin(u32 v, u64 tstart, int dflt) {
  int sum = 0;
  bool ok = true;
#pragma unroll
  for (int k = 0; k < 8; ++k) {
    const int r = (int)(__builtin_amdgcn_readlane(v, k) - (u32)tstart);
    ok = ok && r > -100 && r < 1000;
    sum += r;
  }
  return ok ? sum >> 3 : dflt;
}

__device__ __forceinline__ void transpose4(u32 d0, u32 d1, u32 d2, u32 d3, u32& c0, u32& c1, u32& c2, u32& c3) {
  const u32 t0 = __builtin_amdgcn_perm(d1, d0, 0x05010400u), t1 = __builtin_amdgcn_perm(d1, d0, 0x07030602u);
  const u32 t2 = __builtin_amdgcn_perm(d3, d2, 0x05010400u), t3 = __builtin_amdgcn_perm(d3, d2, 0x07030602u);
  c0 = __builtin_amdgcn_perm(t2, t0, 0x05040100u);
  c1 = __builtin_amdgcn_perm(t2, t0, 0x07060302u);
  c2 = __builtin_amdgcn_perm(t3, t1, 0x05040100u);
  c3 = __builtin_amdgcn_perm(t3, t1, 0x07060302u);
}

__global__ void __launch_bounds__(512, 2) ag_gemm_kernel(Params p) {
  __shared__ __attribute__((aligned(16))) char smem[8 * TR_B];
  __shared__ u32 turn[2];
  const int lane = threadIdx.x & 63;
  const int wv = __builtin_amdgcn_readfirstlane(threadIdx.x >> 6);
  const int xg = blockIdx.x & 7;
  const int ci = blockIdx.x >> 3;
  int rank = p.rank;
  char* myws = p.ws[0];
  char* pws = p.ws[(ci + 1) & 7];
  const unsigned char* xp = p.x;
  const unsigned char* wp = p.w;
  const int d = (ci + 1) & 7;
  const int kb = (xg + wv) & 7;
  u32 so, sst, vo;
  auto source = [&](int u) INL {
    const bool w = d >= 4 && is_w(u);
    u32 sw_ = (u32)(kb * 512 + (u - NPRE) * 64) * NS + (d & 3) * 128, sx_ = (d < 4 ? u : cross_tile(u)) * TILE_B + kb * 512;
    asm("" : "+s"(sw_), "+s"(sx_));
    so = w ? sw_ : sx_;
    sst = w ? 8 * NS : 2 * KD;
    const int sh = w ? 3 : 5;
    vo = (u32)(lane >> sh) * (sst >> (6 - sh)) + (lane & ((1 << sh) - 1)) * 16;
  };
  __builtin_assume(wv < 8);
  source(wv);
  asm volatile("" : "+s"(so), "+s"(sst));
  asm volatile("" : "+v"(vo));
  asm volatile("" : "+s"(rank), "+s"(myws), "+s"(pws), "+s"(xp), "+s"(wp));
  const int m = lane & 15, q = lane >> 4;
  const u64 tstart = __builtin_amdgcn_s_memrealtime();

  gu32* cntp = (gu32*)(myws + CNT_OFF) + blockIdx.x * 8 + wv;
  auto rw = __builtin_amdgcn_make_buffer_rsrc((void*)wp, 0, (int)SHARD, 0x00020000);

  if (__builtin_expect(ci < 8, 1)) {
    const int dt = d ? PUSH_DT : SELF_DT;
    auto start = [&](int u) INL { return d == 0 ? SELF_T0 : u < PUSH_U1 ? PUSH_T0 : (CONS & 1) ? PUSH_T1A : PUSH_T1; };
    auto ready = [&](int u) INL {
      const int due = u < PUSH_LEADT ? u * PUSH_DT0 : start(u) + u * dt - PUSH_PRE;
      while (since(tstart) < due) __builtin_amdgcn_s_sleep(2);
    };
    u32x4 e[8];
    auto load = [&](int u) INL {
      auto r = __builtin_amdgcn_make_buffer_rsrc((void*)(sst == 8 * NS ? wp : xp), 0, (int)SHARD, 0x00020000);
#pragma unroll
      for (int j = 0; j < 8; ++j) e[j] = __builtin_amdgcn_raw_buffer_load_b128(r, vo, so + j * sst, 0);
    };
    if (__builtin_expect(wv >= NLEAD, 0)) { __builtin_amdgcn_s_barrier(); ready(wv); }
    load(wv);
    u32 cv = __builtin_amdgcn_raw_buffer_load_b32(__builtin_amdgcn_make_buffer_rsrc((void*)cntp, 0, 4, 0x00020000), 0, 0, AUX_AGENT);
    __builtin_amdgcn_sched_barrier(0);
    if (wv < NLEAD) {
      if (lane < 2) __hip_atomic_store(&turn[lane], 0u, __ATOMIC_RELAXED, __HIP_MEMORY_SCOPE_WORKGROUP);
      asm volatile("s_waitcnt lgkmcnt(0)" ::: "memory");
      __builtin_amdgcn_s_barrier();
    }
    const int nu = d == 0 ? NT : d < 4 ? NUI : NUC;
    char* tw = smem + wv * TR_B + (lane >> 5) * TR_ROW + (lane & 31) * 16;
    const char* tr = smem + wv * TR_B + m * TR_ROW + q * 16;
    const u32 soff = lane * 16;
    u32 call = 0;
    char* dst = nullptr;
    gu32* pace = (gu32*)(myws + PACE_OFF) + d * 32;
    int t1 = -1;
#pragma unroll 1
    for (int u = wv;;) {
      int t0 = start(u);
      if (sst != 8 * NS) {
#pragma unroll
        for (int j = 0; j < 8; ++j) *(u32x4*)(tw + j * 2 * TR_ROW) = e[j];
#pragma unroll
        for (int i = 0; i < 8; ++i) e[i] = *(const u32x4*)(tr + i * 64);
      } else {
#pragma unroll
        for (int j = 0; j < 8; ++j) *(u32x4*)(smem + wv * TR_B + lane * 16 + j * 1024) = e[j];
        const u32* wr = (const u32*)(smem + wv * TR_B + q * 2048 + m * 4);
#pragma unroll
        for (int j = 0; j < 8; ++j) {
          const u32* w4 = wr + (j >> 2) * 16 + (j & 3) * 128;
          u32 c0, c1, c2, c3;
          transpose4(w4[0], w4[32], w4[64], w4[96], c0, c1, c2, c3);
          e[(j & 4) + 0][j & 3] = c0; e[(j & 4) + 1][j & 3] = c1; e[(j & 4) + 2][j & 3] = c2; e[(j & 4) + 3][j & 3] = c3;
        }
      }
#pragma unroll
      for (int i = 0; i < 8; ++i) e[i][0] = ~e[i][0];
      if (u < 8) {
        asm volatile("" : "+v"(cv));
        call = __builtin_amdgcn_readfirstlane(cv) + 1;
        dst = pws + (size_t)(call % NBUF) * BUF_B + (size_t)d * SHARD + kb * 8192;
      }
      auto rd = __builtin_amdgcn_make_buffer_rsrc((void*)(dst + u * TILE_B), 0, 8192, 0x00020000);
      if ((CONS & 1) && d && u >= CONS_UR) {
        if (t1 < 0) {
          while (since(tstart) < t0 + u * dt - 90) __builtin_amdgcn_s_sleep(2);
          t1 = origin(__hip_atomic_load(pace + (lane & 7), __ATOMIC_RELAXED, __HIP_MEMORY_SCOPE_AGENT), tstart, PUSH_T1 - CONS_M) + CONS_M;
        }
        t0 = t1;
      }
      const int win = u < W_U0 ? PUSH_W0 : u < W_U2 ? PUSH_W : PUSH_W2;
#pragma unroll
      for (int i = 0; i < 8; ++i) {
        if (i == 7 && win == 7) asm volatile("s_waitcnt vmcnt(6)" ::: "memory");
        else if (u > 0 && d) {
          while ((int)__hip_atomic_load(&turn[0], __ATOMIC_RELAXED, __HIP_MEMORY_SCOPE_WORKGROUP) < u * 8 + i - (win - 1)) __builtin_amdgcn_s_sleep(1);
        }
        if (i % PUSH_G == 0) {
          const int due = t0 + (u * 8 + i) * dt / 8;
          while (since(tstart) < due) __builtin_amdgcn_s_sleep(1);
        }
        if (i == 0 && CONS && (u == CONS_UP || u == CONS_UP2) && lane == 0)
          __hip_atomic_store(pace + (u == CONS_UP ? 0 : 256) + xg, (u32)__builtin_amdgcn_s_memrealtime() - u * dt, __ATOMIC_RELAXED, __HIP_MEMORY_SCOPE_AGENT);
        if (i == 0) STAMPL(u >> 3);
        __builtin_amdgcn_raw_buffer_store_b128(e[i], rd, soff + i * 1024, 0, AUX_SYS);
        __builtin_amdgcn_sched_barrier(0);
      }
#pragma unroll
      for (int k = 7; k >= 0; --k) {
        asm volatile("s_waitcnt vmcnt(%0)" ::"n"(k) : "memory");
        __hip_atomic_fetch_max(&turn[0], (u32)(u * 8 + 8 - k), __ATOMIC_RELAXED, __HIP_MEMORY_SCOPE_WORKGROUP);
      }
      u += 8;
      if (u >= nu) break;
      source(u);
      ready(u);
      load(u);
    }
    STAMPL(4);
    if (lane == 0) {
      const u32 done = __hip_atomic_fetch_add(&turn[1], 1u, __ATOMIC_RELAXED, __HIP_MEMORY_SCOPE_WORKGROUP);
      if (done == 7) __hip_atomic_store((gu32*)(pws + FLAG_OFF) + (d * 8 + xg) * 32, call, __ATOMIC_RELAXED, __HIP_MEMORY_SCOPE_SYSTEM);
      __hip_atomic_store(cntp, call, __ATOMIC_RELAXED, __HIP_MEMORY_SCOPE_AGENT);
    }
    if (d >= 4) {
      auto rz = __builtin_amdgcn_make_buffer_rsrc((void*)(myws + (size_t)((call + NBUF - 1) % NBUF) * BUF_B + (size_t)d * SHARD + (NPRE + xg) * TILE_B + wv * 8192),
                                                  0, 8192, 0x00020000);
      while (since(tstart) < PUSH_T1 + NUC * PUSH_DT + 100 + wv * 50) __builtin_amdgcn_s_sleep(8);
#pragma unroll
      for (int i = 0; i < 8; ++i) __builtin_amdgcn_raw_buffer_store_b128(u32x4{0, 0, 0, 0}, rz, soff + i * 1024, 0, AUX_SYS);
      if (xg == 0 && wv == 0) {
        const char* fp = myws + OFL_OFF + (d - 4) * 4096 + (lane < 2 * NRS ? lane : 0) * 128;
        while (since(tstart) < O_T0 + (NRJ - NRS) * O_DT / 8) __builtin_amdgcn_s_sleep(4);
        u32 v = 0, n = 0;
        for (;;) {
          asm volatile("global_load_dword %0, %1, off sc0 sc1\n\ts_waitcnt vmcnt(%2)" : "+v"(v) : "v"(fp), "n"(WAIT_D - 1) : "memory");
          if (__builtin_amdgcn_ballot_w64(v < call) == 0) break;
          asm volatile("s_sleep %1" : "+v"(v) : "n"(WAIT_S));
          if (__builtin_amdgcn_ballot_w64(v < call) == 0) break;
          if ((++n & 255u) == 0 && __builtin_amdgcn_s_memrealtime() - tstart > DEADLINE) {
            if (lane == 0) p.out[0] = __builtin_nanf("");
            break;
          }
        }
        asm volatile("s_waitcnt vmcnt(0)" : "+v"(v) :: "memory");
        STAMP(1);
      }
    }
    return;
  }

  STAMP(0);
  u32 call = __hip_atomic_load(cntp, __ATOMIC_RELAXED, __HIP_MEMORY_SCOPE_AGENT) + 1;
  gu32* tmo = (gu32*)(myws + TMO_OFF);
  const int g = ci & 7;
  const int sl = (ci - 8) >> 3;
  const int set = sl == 0 ? xg : sl == 1 ? 16 + xg : 8 + (xg >> 1) + ((xg ^ g) & 1) * 4;
  const bool isr = set < NRS;
  const int dl = 4 + (g >> 1), half = g & 1;
  u64 jlo = p.jobs[set][0], jhi = p.jobs[set][1];
  float* outp = isr ? p.outp[dl - 4] : p.out;
  char* lws = p.ws[dl];
  float scale = p.sx[0] * p.sw[0];
  asm volatile("" : "+s"(jlo), "+s"(jhi), "+s"(outp), "+s"(lws));
  asm volatile("" : "+v"(scale));
  if (threadIdx.x == 0) __hip_atomic_store(&turn[0], 0u, __ATOMIC_RELAXED, __HIP_MEMORY_SCOPE_WORKGROUP);
  gu32* flags = (gu32*)(myws + FLAG_OFF);
  const u32 voff_g = lane * 16;

  bool sure;
  u32 n = 0;
  auto word = [&](gu32* a) INL { return __builtin_amdgcn_readfirstlane(__hip_atomic_load(a, __ATOMIC_RELAXED, __HIP_MEMORY_SCOPE_SYSTEM)); };
  auto slow = [&](gu32* f, gu32* f2, bool nap) INL {
    if ((++n & 31u) == 0) {
      if (word(f) >= call && word(f2) >= call) { sure = true; return; }
      if (word(tmo) == call || __builtin_amdgcn_s_memrealtime() - tstart > DEADLINE) {
        __hip_atomic_store(tmo, call, __ATOMIC_RELAXED, __HIP_MEMORY_SCOPE_SYSTEM);
        sure = true;
        return;
      }
    }
    if (nap) __builtin_amdgcn_s_sleep(POLL_SLEEP); else __builtin_amdgcn_s_sleep(1);
  };

  call = __builtin_amdgcn_readfirstlane(call);
  if (lane == 0) __hip_atomic_store(cntp, call, __ATOMIC_RELAXED, __HIP_MEMORY_SCOPE_AGENT);
  char* gbuf = myws + (size_t)(call % NBUF) * BUF_B + wv * 8192;
  auto rxg = __builtin_amdgcn_make_buffer_rsrc((void*)gbuf, 0, (int)BUF_B, 0x00020000);
  auto rzr = __builtin_amdgcn_make_buffer_rsrc((void*)(myws + (size_t)((call + NBUF - 1) % NBUF) * BUF_B), 0, isr ? 0 : (int)BUF_B, 0x00020000);
  auto rout = __builtin_amdgcn_make_buffer_rsrc((void*)outp, 0, NR * MS * NS * 4, 0x00020000);
  const u32 voff_z = (u32)g * 8192 + threadIdx.x * 16;
  const u32 voff_o = ((wv * 2 + ((lane >> 4) & 1)) * NS + g * 64 + (lane & 15) * 4) * 4 + (lane < 32 ? 0 : OOB);
  const u32 voff_t = lane < (sl == 2 ? 16 : 8) ? (sl == 2 ? g & 6 : g) * 1024 + lane * 128 : OOB;
  const int rot = (dl - 4) * OSTAG, r0 = (set * 2 + half + rot) % (2 * NRS);
  const int oslot = O_T0 + (r0 - 2 * RPAD) * O_DT / 16;
  const int oslot0 = r0 >= rot ? oslot : O_T0 + (r0 - (rot + 2 * RPAD > 2 * NRS ? rot + 2 * RPAD - 2 * NRS : 0)) * O_DT / 16;

  u32 jt, jo;
  int jg, js;
  {
    const u32 ent = lane < 16 ? (u32)((lane < 8 ? jlo : jhi) >> (lane & 7) * 8) & 0xff : 0xff;
    const int u = ent & 31, d = isr && ent >= 128 ? dl ^ ((ent >> 5) - 4) : ent >> 5;
    jt = ent == 0xff ? OOB : d * SHARD + u * TILE_B;
    jg = ent == 0xff ? 0x7fffffff : (d ? PUSH_T1 + (u + 1) * PUSH_DT : SELF_T0 + (u + 1) * SELF_DT) + GATE;
    jo = ent == 0xff ? OOB : (u32)((rank ^ d) * MS + (d < 4 ? u : cross_tile(u)) * 16) * NS * 4;
    js = ent == 0xff || !isr ? 0 : lane ? oslot + lane * NRS * O_DT / 8 : oslot0;
  }
  u32 known = 0;
  u32 got = 0;
  u32 warm = 0;
  int now;

  auto probe = [&](u32 off, int chunk) INL { return __builtin_amdgcn_raw_buffer_load_b32(rxg, 1008, off + chunk * 1024, AUX_SYS); };
  auto rsrc_if = [&](bool on) INL { return __builtin_amdgcn_make_buffer_rsrc((void*)gbuf, 0, on ? (int)BUF_B : 0, 0x00020000); };
  auto fetch = [&](u32x4 (&a)[8], int j) INL {
    auto r = rsrc_if(known >> j & 1);
    got |= known & 1u << j;
    const u32 off = __builtin_amdgcn_readlane(jt, j);
#pragma unroll
    for (int i = 0; i < 8; ++i) a[i] = __builtin_amdgcn_raw_buffer_load_b128(r, voff_g + (i & 3) * 1024, off + (i >> 2) * 4096, FAUX);
  };
  auto touch = [&](int j) INL {
    j += __builtin_ctz(((known & ~warm) | 1u << 31) >> j);
    warm |= 1u << (j & 15);
    return __builtin_amdgcn_raw_buffer_load_b32(rsrc_if(TOUCH_ON && j < 16), voff_t, __builtin_amdgcn_readlane(jt, j), AUX_SYS);
  };
  auto peek = [&](int t) INL {
    const bool on = jg <= t && !(known >> (lane & 31) & 1);
    return __builtin_amdgcn_raw_buffer_load_b32(rxg, on ? jt : OOB, 8176, AUX_SYS);
  };
  u32 pk, tv[4];
  auto early = [&](int phase) INL {
    if (phase) known |= (u32)__builtin_amdgcn_ballot_w64(pk != 0);
    pk = peek(since(tstart));
#pragma unroll
    for (int k = 0; k < 4; ++k) {
      if (phase == 1) tv[k] = touch(0);
      if (phase == 2) asm volatile("" ::"v"(tv[k]));
    }
  };

  i32x8 bf[4][4];
  u32x4 a0[8], a1[8];
  if (!isr) {
    const u32 voff = (u32)(wv * 512 + q * 16) * NS + g * 64 + m * 4;
#pragma unroll
    for (int ks = 0; ks < 4; ++ks)
#pragma unroll
      for (int h = 0; h < 2; ++h) {
        const int due = W_DELAY + sl * W_SLOT + (ks * 2 + h) * W_DT;
        while (since(tstart) < due) __builtin_amdgcn_s_sleep(4);
        if (ks * 2 + h >= 6) early(ks * 2 + h - 6);
#pragma unroll
        for (int r = 0; r < 4; ++r) {
          u32 d0 = __builtin_amdgcn_raw_buffer_load_b32(rw, voff + (r * 4 + 0) * NS, (ks * 128 + h * 64) * NS, 0);
          u32 d1 = __builtin_amdgcn_raw_buffer_load_b32(rw, voff + (r * 4 + 1) * NS, (ks * 128 + h * 64) * NS, 0);
          u32 d2 = __builtin_amdgcn_raw_buffer_load_b32(rw, voff + (r * 4 + 2) * NS, (ks * 128 + h * 64) * NS, 0);
          u32 d3 = __builtin_amdgcn_raw_buffer_load_b32(rw, voff + (r * 4 + 3) * NS, (ks * 128 + h * 64) * NS, 0);
          u32 c0, c1, c2, c3;
          transpose4(d0, d1, d2, d3, c0, c1, c2, c3);
          bf[0][ks][h * 4 + r] = (int)c0; bf[1][ks][h * 4 + r] = (int)c1; bf[2][ks][h * 4 + r] = (int)c2; bf[3][ks][h * 4 + r] = (int)c3;
        }
      }
  } else {
#pragma unroll
    for (int ks = 0; ks < 4; ++ks) {
      const int u = NPRE + ks * 2;
      const u32 off = dl * SHARD + u * TILE_B + half * 4096;
      gu32* f = flags + (dl * 8 + ((wv - u) & 7)) * 32;
      gu32* f2 = flags + (dl * 8 + ((wv - u - 1) & 7)) * 32;
      if (ks >= 2) early(ks - 2);
      while (since(tstart) < PUSH_T1 + (u + 2) * PUSH_DT - (1 - half) * PUSH_DT / 2 + GATE) __builtin_amdgcn_s_sleep(4);
      sure = false;
      while (!sure) {
        if (__builtin_amdgcn_ballot_w64(probe(off + TILE_B, 3) == 0) == 0) break;
        slow(f, f2, true);
      }
      for (;;) {
#pragma unroll
        for (int i = 0; i < 8; ++i) a0[i] = __builtin_amdgcn_raw_buffer_load_b128(rxg, voff_g + (i & 3) * 1024, off + (i >> 2) * TILE_B, AUX_SYS);
        if (sure) break;
        u32 mn = a0[0][0];
#pragma unroll
        for (int i = 1; i < 8; ++i) mn = a0[i][0] < mn ? a0[i][0] : mn;
        if (__builtin_amdgcn_ballot_w64(mn == 0) == 0) break;
        slow(f, f2, true);
      }
#pragma unroll
      for (int i = 0; i < 8; ++i)
#pragma unroll
        for (int e = 0; e < 4; ++e) bf[i & 3][ks][(i >> 2) * 4 + e] = (int)(e ? a0[i][e] : ~a0[i][e]);
    }
  }
  early(2);

  known |= (u32)__builtin_amdgcn_ballot_w64(pk != 0);
  int it = 0;
  u32 o[2] = {0, 0};
  auto put = [&](int j) INL {
    auto b0 = __builtin_amdgcn_permlane32_swap(o[0], o[0], false, false);
    auto b1 = __builtin_amdgcn_permlane32_swap(o[1], o[1], false, false);
    __builtin_amdgcn_raw_buffer_store_b128(u32x4{b0[0], b1[0], b0[1], b1[1]}, rout, voff_o | __builtin_amdgcn_readlane(jo, j), 0, AUX_SYS);
    __builtin_amdgcn_sched_barrier(0);
    STAMPJ(3 + 2 * j, since(tstart));
  };
  auto slot = [&](int j) INL {
    const int due = __builtin_amdgcn_readlane(js, j);
    if (now < due) do { __builtin_amdgcn_s_sleep(1); now = since(tstart); } while (now < due);
  };
  auto hole = [&](u32x4 (&a)[8], int i0, int i1) INL {
    u32 mn = a[i0][0];
#pragma unroll
    for (int i = i0 + 1; i < i1; ++i) mn = a[i][0] < mn ? a[i][0] : mn;
    return __builtin_amdgcn_ballot_w64(mn == 0) != 0;
  };
  auto refill = [&](u32x4 (&a)[8], int i0, u32 off) INL {
#pragma unroll
    for (int i = i0; i < 8; ++i)
      if (hole(a, i, i + 1)) a[i] = __builtin_amdgcn_raw_buffer_load_b128(rxg, voff_g + (i & 3) * 1024, off + (i >> 2) * 4096, AUX_SYS);
  };
  f32x4 acc[4];
  auto mul = [&](u32x4 (&a)[8], int ks) INL {
    i32x8 af;
#pragma unroll
    for (int e = 0; e < 4; ++e) { af[e] = (int)a[2 * ks][e]; af[4 + e] = (int)a[2 * ks + 1][e]; }
    af[0] = ~af[0]; af[4] = ~af[4];
#pragma unroll
    for (int c = 0; c < 4; ++c)
      acc[c] = __builtin_amdgcn_mfma_scale_f32_16x16x128_f8f6f4(af, bf[c][ks], ks ? acc[c] : f32x4{0.f, 0.f, 0.f, 0.f}, 0, 1, 0, 0, 0, 0);
  };
  auto share = [&]() INL {
    float* rb = (float*)smem + (it & 1) * 8192;
#pragma unroll
    for (int j = 0; j < 4; ++j)
      *(f32x4*)(rb + wv * 1024 + (q * 4 + j) * 64 + m * 4) = f32x4{acc[0][j], acc[1][j], acc[2][j], acc[3][j]};
    __builtin_amdgcn_sched_barrier(0);
  };
  auto gather = [&]() INL {
    asm volatile("s_waitcnt lgkmcnt(0)" ::: "memory");
    __builtin_amdgcn_s_barrier();
    const float* rp = (float*)smem + (it & 1) * 8192 + (lane >> 5) * 4096 + wv * 128 + (lane & 31) * 4;
    f32x4 s4 = *(const f32x4*)rp;
#pragma unroll
    for (int v = 1; v < 4; ++v) s4 += *(const f32x4*)(rp + v * 1024);
    return s4;
  };
  auto finish = [&](f32x4 s4) INL {
#pragma unroll
    for (int e = 0; e < 2; ++e) {
      auto sw = __builtin_amdgcn_permlane32_swap(__float_as_uint(s4[e]), __float_as_uint(s4[e + 2]), false, false);
      const float y = (__uint_as_float(sw[0]) + __uint_as_float(sw[1])) * scale;
      const float yc = __builtin_fminf(__builtin_fmaxf(y, -60.f), 60.f);
      o[e] = __float_as_uint(y * __builtin_amdgcn_rcpf(1.f + __expf(-yc)));
    }
  };
  auto wipe = [&]() INL { __builtin_amdgcn_raw_buffer_store_b128(u32x4{0, 0, 0, 0}, rzr, voff_z + __builtin_amdgcn_readlane(jt, it), 0, AUX_SYS); };
  auto step = [&](auto first_wave, u32x4 (&a)[8], u32x4 (&an)[8], u32& tv) INL {
    wipe();
    const int gate = __builtin_amdgcn_readlane(jg, it);
    sure = gate == 0x7fffffff;
    for (bool first = true; !sure; first = false) {
      if (__builtin_expect(!hole(a, 0, 8), 1)) break;
      const u32 off = __builtin_amdgcn_readlane(jt, it);
      gu32* f = flags + (off / SHARD * 8 + ((wv - off / TILE_B) & 7)) * 32;
      if (!first) slow(f, f, false);
      else if (__builtin_amdgcn_readlane(pk, it) == 0) {
        while (since(tstart) < gate) __builtin_amdgcn_s_sleep(2);
        if (since(tstart) < gate + LONG_DUE)
          while (!sure) {
            if (__builtin_amdgcn_ballot_w64(probe(off, 7) == 0) == 0) break;
            slow(f, f, true);
          }
      }
      refill(a, 0, off);
      __builtin_amdgcn_s_waitcnt(0x0f70);
    }
    now = since(tstart);
    STAMPJ(2 + 2 * it, now);
#pragma unroll
    for (int ks = 0; ks < 4; ++ks) mul(a, ks);
    share();
    known |= (u32)__builtin_amdgcn_ballot_w64(pk != 0) | 1u << it;
    if (__builtin_expect((known & ~got) >> (it + 1) & 1, 0)) fetch(an, it + 1);
    __builtin_amdgcn_sched_barrier(0);
    auto ahead = [&]() INL {
      fetch(a, it + 2);
      __builtin_amdgcn_sched_barrier(0);
      asm volatile("" ::"v"(tv));
      tv = touch(it + 3);
      __builtin_amdgcn_sched_barrier(0);
    };
    if (first_wave()) ahead();
    now += O_EARLY;
    slot(it - 1);
    pk = peek(now);
    __builtin_amdgcn_sched_barrier(0);
    put(it - 1);
    const f32x4 s4 = gather();
    if (!first_wave()) ahead();
    finish(s4);
    ++it;
  };
  auto solo = [&](u32x4 (&a)[8]) INL {
    const int jp = (it - 1) & 63;
    int due = __builtin_amdgcn_readlane(js, jp);
    auto flush = [&]() INL {
      now = since(tstart);
      if (now >= due) { put(jp); due = 0x7fffffff; }
    };
    const u32 off = __builtin_amdgcn_readlane(jt, it);
    gu32* f = flags + (off / SHARD * 8 + ((wv - off / TILE_B) & 7)) * 32;
    sure = off == OOB;
    if (!sure && hole(a, 0, 8)) {
      if (__builtin_amdgcn_readlane(pk, it) == 0) {
        const bool tail = (off / TILE_B & 31) >= (isr ? TAIL_R : TAIL_L);
        const int gate = __builtin_amdgcn_readlane(jg, it) - (!tail ? 0 : isr ? TAIL_TR : TAIL_TL);
        for (flush(); now < gate; flush()) __builtin_amdgcn_s_sleep(1);
        if (now < gate + (!tail ? LONG_DUE : isr ? TAIL_LONGR : TAIL_LONGL))
          while (!sure) {
            if (__builtin_amdgcn_ballot_w64(probe(off, tail ? 0 : 7) == 0) == 0) break;
            slow(f, f, true);
            flush();
          }
      }
#pragma unroll
      for (int i = 0; i < 8; ++i) a[i] = __builtin_amdgcn_raw_buffer_load_b128(rxg, voff_g + (i & 3) * 1024, off + (i >> 2) * 4096, AUX_SYS);
      __builtin_amdgcn_sched_barrier(0);
      now = since(tstart);
      const int j = now + SOLO_LA >= due ? jp : 63;
      slot(j);
      put(j);
      if (j == jp) due = 0x7fffffff;
    }
#pragma unroll
    for (int ks = 0; ks < 4; ++ks) {
      while (!sure && hole(a, 2 * ks, 2 * ks + 2)) {
        slow(f, f, false);
        refill(a, 2 * ks, off);
        flush();
        __builtin_amdgcn_s_waitcnt(0x0f70);
      }
      if (ks == 3) STAMPJ(2 + 2 * it, since(tstart));
      mul(a, ks);
    }
    share();
    wipe();
    now = 0;
    if (due != 0x7fffffff) { slot(jp); put(jp); }
    finish(gather());
    slot(it ? it : 63);
    put(it ? it : 63);
    ++it;
  };
  const int len = isr ? RLEN : LLEN;
  auto run = [&](auto first_wave) INL {
    fetch(a0, 0);
    pk = peek(since(tstart));
    __builtin_amdgcn_s_waitcnt(0x0f70);
    for (;;) {
      solo(a0);
      if (it == len) break;
      known |= (u32)__builtin_amdgcn_ballot_w64(pk != 0) | 1u;
      fetch(a0, 1);
      fetch(a1, 2);
      pk = 0;
      if ((CONS & 2) && isr) {
        while (since(tstart) < PUSH_T1 + CONS_UP2 * PUSH_DT + 150) __builtin_amdgcn_s_sleep(2);
        const int sh = origin(__hip_atomic_load((gu32*)(myws + PACE_OFF) + 256 + dl * 32 + (lane & 7), __ATOMIC_RELAXED, __HIP_MEMORY_SCOPE_AGENT), tstart, PUSH_T1 + 50) - PUSH_T1;
        js = js ? js + sh : 0;
      }
      __builtin_amdgcn_s_waitcnt(0x0f70);
      do { step(first_wave, a0, a1, tv[0]); step(first_wave, a1, a0, tv[1]); } while (it < len - 1);
    }
  };
  if (AB_SPLIT && wv >= 4) run(std::false_type{}); else run(std::true_type{});
  if (isr) {
    asm volatile("s_waitcnt vmcnt(0)" ::: "memory");
    if (lane == 0 && __hip_atomic_fetch_add(&turn[0], 1u, __ATOMIC_RELAXED, __HIP_MEMORY_SCOPE_WORKGROUP) == 7)
      __hip_atomic_store((gu32*)(lws + OFL_OFF) + ((dl - 4) * 32 + set * 2 + half) * 32, call, __ATOMIC_RELAXED, __HIP_MEMORY_SCOPE_SYSTEM);
    STAMPJ(60, since(tstart));
  }
}

void build_jobs(u64 jobs[NSETS][2]) {
  unsigned char lst[2][256];
  int n[2] = {RPAD, LPAD};
  for (int i = 0; i < 256; ++i) lst[0][i] = lst[1][i] = 0xff;
  auto add = [&](int k, int d, int u) { lst[k][n[k]++] = (unsigned char)(d << 5 | u); };
  for (int u = 0, v = 0; u < NH; ++u) {
    for (; v < NH && SELF_T0 + (v + 1) * SELF_DT <= PUSH_T1 + (u + 1) * PUSH_DT; ++v) add(0, 0, v);
    for (int d = 1; d < 4; ++d) add(0, d, u);
    if (u < NUC && !is_w(u) && cross_tile(u) >= NUI) for (int d = 5; d < 8; ++d) add(0, d, u);
  }
  unsigned char ex[64];
  int ne = 0, ie = 0;
  for (int u = 0; u < 2 * NUC; ++u) ex[ne++] = (unsigned char)u;
  for (int u = 0; u < NUC; ++u) if (!is_w(u)) for (int d = 4; d < 8; ++d) ex[ne++] = (unsigned char)(d << 5 | u);
  for (int u = 2 * NUC; u < NT; ++u) ex[ne++] = (unsigned char)u;
  for (int u = 0; u < NUI; ++u) {
    for (int d = 1; d < 4; ++d) add(1, d, u);
    for (int k = 0; k < (u < NUC ? 2 : 4) && ie < ne; ++k) lst[1][n[1]++] = ex[ie++];
  }
  for (int s = 0; s < NSETS; ++s) {
    const int k = s >= NRS, ns = k ? NLS : NRS, s0 = k ? s - NRS : s;
    jobs[s][0] = jobs[s][1] = ~0ull;
    for (int j = 0; j < 16 && s0 + j * ns < n[k]; ++j) ((unsigned char*)jobs[s])[j] = lst[k][s0 + j * ns];
  }
}
static_assert(RLEN <= 16 && LLEN <= 16 && NRS <= 16 && 2 * NUC <= NT, "job lists");

}

extern "C" void kernel_launch(int rank, int nranks, ncclComm_t comm, hipStream_t stream,
                              void* const* d_in, const int* in_sizes, int n_in,
                              void* d_out, int out_size, const int* in_dtypes, int out_dtype,
                              void* const* d_out_all, void* const* d_ws_all, size_t ws_bytes) {
  (void)comm; (void)in_dtypes; (void)out_dtype;
  if (nranks != NR || (unsigned)rank >= (unsigned)NR || n_in != 4 || ws_bytes < WS_NEED ||
      in_sizes[0] != (int)SHARD || in_sizes[1] != (int)SHARD || out_size != NR * MS * NS) {
    fprintf(stderr, "kernel_launch: unexpected problem shape\n");
    return;
  }
  Params p;
  p.x = (const unsigned char*)d_in[0];
  p.w = (const unsigned char*)d_in[1];
  p.sx = (const float*)d_in[2];
  p.sw = (const float*)d_in[3];
  p.out = (float*)d_out;
  for (int d = 0; d < NR; ++d) p.ws[d] = (char*)d_ws_all[rank ^ d];
  for (int i = 0; i < 4; ++i) p.outp[i] = (float*)d_out_all[rank ^ (4 + i)];
  p.rank = rank;
  p.pad = 0;
  build_jobs(p.jobs);
  hipLaunchKernelGGL(ag_gemm_kernel, dim3(256), dim3(512), 0, stream, p);
}
```

```cpp
#include <hip/hip_runtime.h>
#include <rccl/rccl.h>
#include <cstdio>
#include <type_traits>

typedef unsigned int u32;
typedef unsigned long long u64;
typedef u32 u32x4 __attribute__((ext_vector_type(4)));
typedef u32 u32x2 __attribute__((ext_vector_type(2)));
typedef int i32x8 __attribute__((ext_vector_type(8)));
typedef float f32x4 __attribute__((ext_vector_type(4)));
typedef __attribute__((address_space(1))) u32 gu32;

namespace {

constexpr int NR = 8;
constexpr int KD = 4096;
constexpr int MS = 512;
constexpr int NS = 512;
constexpr u32 SHARD = (u32)MS * KD;
constexpr u32 TILE_B = 16 * KD;
constexpr int NT = MS / 16;
#define NH 26
#define ND 2
#define NA 4
static_assert(NH + ND + NA == NT, "tile kinds");
constexpr int NUI = NH + ND;
constexpr int NUC = 8 + NA + ND;
#define NPRE 1
#define NRS 12
constexpr int NSETS = 24, NLS = NSETS - NRS;
constexpr int NRJ = 4 * NH + 3 * NA, NLJ = NT + 3 * NUI + 4 * (NA + ND);
constexpr int RLEN = (NRJ + 2 * NRS - 1) / (2 * NRS) * 2, LLEN = (NLJ + 2 * NLS - 1) / (2 * NLS) * 2;
constexpr int RPAD = RLEN * NRS - NRJ, LPAD = LLEN * NLS - NLJ;
constexpr int NST = NRJ - NRS;
static_assert(NUC * 8 + NST <= NT * 8, "staging");
constexpr u32 BUF_B = (u32)NR * SHARD;
constexpr int NBUF = 3;
constexpr size_t FLAG_OFF = (size_t)NBUF * BUF_B;
constexpr size_t FLAG_B = (size_t)NR * 8 * 128;
constexpr size_t OFL_OFF = FLAG_OFF + FLAG_B;
constexpr size_t OFL_B = 4 * 4096;
constexpr size_t CNT_OFF = OFL_OFF + OFL_B;
constexpr size_t CNT_B = 256 * 8 * 4;
constexpr size_t TMO_OFF = CNT_OFF + CNT_B;
constexpr size_t PACE_OFF = TMO_OFF + 4096;
constexpr size_t DBG_OFF = PACE_OFF + 4096;
constexpr size_t DBG_B = 256 * 128 * 8;
constexpr size_t WS_NEED = DBG_OFF + DBG_B;
constexpr u64 DEADLINE = 100000000ull;
#define PUSH_DT 110
#define PUSH_T1 60
#define ST_X 0
#define BONUS_U 2
#define BONUS_SH 5
#define BONUS_MAX 2
#define PACE_U 20
#define O_M 5
#define PUSH_G 2
#define PUSH_PRE 300
#define ST_PRE 100
#define ST_PROBE 0
#define ST_SLEEP 4
#define PUSH_LEADT 3
#define NLEAD 2
#define PUSH_DT0 20
#define W_DELAY 150
#define W_DT 100
#define W_LAG 0
#define W_SLOT 50
#define GATE (-10)
#define R_GATE 80
#ifndef LONG_DUE
#define LONG_DUE 150
#endif
#define O_T0 (PUSH_T1 + NUC * PUSH_DT + NST * (PUSH_DT + ST_X) / 8 + O_M)
#define O_DT 113
#define SELF_T0 200
#define SELF_DT 50
#define TAIL_R (NH - 1)
#define TAIL_L (NUI - 1)
#define TAIL_T 50
#define PROBE_C 3
#define PROBE_D 3
#define PEEK_LA 25
#define POLL_SLEEP 4
#define FAUX 0
#define WAIT_D 6
#define WAIT_S 3
#define SENT 1
#define SN_G 60
#define SN_L 130
#define OAUX 17
#define WIPE0 1
#define SN_D 3
#define SN_S 4
#define SN_W 0
#define EARLY0 1
#define PUT1 1
#define PEEK0 1
#define AUX_SYS 17
#define AUX_AGENT 16
constexpr int TR_ROW = 528;
constexpr int TR_B = 16 * TR_ROW;

struct Params {
  char* ws[NR];
  float* outp[4];
  const unsigned char* x;
  const unsigned char* w;
  const float* sx;
  const float* sw;
  float* out;
  int rank, pad;
  u64 jobs[NSETS][2];
};

__host__ __device__ constexpr bool is_w(int u) { return u >= NPRE && u < NPRE + 8; }
__host__ __device__ constexpr int cross_tile(int u) { u = u < NPRE ? u : u - 8; return u < NA ? NUI + u : NH + u - NA; }
__device__ __forceinline__ void job_of(u32 ent, bool isr, int dl, int& d, int& u) {
  u = ent & 31;
  d = isr && ent >= 128 ? dl ^ ((ent >> 5) - 4) : ent >> 5;
}
__device__ __forceinline__ u32 row_of(int rank, int d, int u) { return (u32)((rank ^ d) * MS + (d < 4 ? u : cross_tile(u)) * 16); }
constexpr u32 OOB = 0x80000000u;

#define STAMP(i) do {} while (0)
#define STAMPV(i, v) do {} while (0)
#define STAMPL(i) do {} while (0)
#define STAMPJ(i, v) do {} while (0)

#define INL __attribute__((always_inline))
__device__ __forceinline__ int since(u64 t) { return (int)(__builtin_amdgcn_s_memrealtime() - t); }

__device__ __forceinline__ int origin(u32 v, u64 tstart, int dflt) {
  int sum = 0;
  bool ok = true;
#pragma unroll
  for (int k = 0; k < 8; ++k) {
    const int r = (int)(__builtin_amdgcn_readlane(v, k) - (u32)tstart);
    ok = ok && r > -100 && r < 1000;
    sum += r;
  }
  return ok ? sum >> 3 : dflt;
}

__device__ __forceinline__ void transpose4(u32 d0, u32 d1, u32 d2, u32 d3, u32& c0, u32& c1, u32& c2, u32& c3) {
  const u32 t0 = __builtin_amdgcn_perm(d1, d0, 0x05010400u), t1 = __builtin_amdgcn_perm(d1, d0, 0x07030602u);
  const u32 t2 = __builtin_amdgcn_perm(d3, d2, 0x05010400u), t3 = __builtin_amdgcn_perm(d3, d2, 0x07030602u);
  c0 = __builtin_amdgcn_perm(t2, t0, 0x05040100u);
  c1 = __builtin_amdgcn_perm(t2, t0, 0x07060302u);
  c2 = __builtin_amdgcn_perm(t3, t1, 0x05040100u);
  c3 = __builtin_amdgcn_perm(t3, t1, 0x07060302u);
}

__global__ void __launch_bounds__(512, 2) ag_gemm_kernel(Params p) {
  __shared__ __attribute__((aligned(16))) char smem[8 * TR_B];
  __shared__ u32 turn[3];
  const int lane = threadIdx.x & 63;
  const int wv = __builtin_amdgcn_readfirstlane(threadIdx.x >> 6);
  const int xg = blockIdx.x & 7;
  const int ci = blockIdx.x >> 3;
  int rank = p.rank;
  char* myws = p.ws[0];
  char* pws = p.ws[(ci + 1) & 7];
  const unsigned char* xp = p.x;
  const unsigned char* wp = p.w;
  const int d = (ci + 1) & 7;
  const int kb = (xg + wv) & 7;
  u32 so, sst, vo;
  auto source = [&](int u) INL {
    const bool w = d >= 4 && is_w(u);
    u32 sw_ = (u32)(kb * 512 + (u - NPRE) * 64) * NS + (d & 3) * 128, sx_ = (d < 4 ? u : cross_tile(u)) * TILE_B + kb * 512;
    asm("" : "+s"(sw_), "+s"(sx_));
    so = w ? sw_ : sx_;
    sst = w ? 8 * NS : 2 * KD;
    const int sh = w ? 3 : 5;
    vo = (u32)(lane >> sh) * (sst >> (6 - sh)) + (lane & ((1 << sh) - 1)) * 16;
  };
  __builtin_assume(wv < 8);
  source(wv);
  asm volatile("" : "+s"(so), "+s"(sst));
  asm volatile("" : "+v"(vo));
  asm volatile("" : "+s"(rank), "+s"(myws), "+s"(pws), "+s"(xp), "+s"(wp));
  const int m = lane & 15, q = lane >> 4;
  const u64 tstart = __builtin_amdgcn_s_memrealtime();

  gu32* cntp = (gu32*)(myws + CNT_OFF) + blockIdx.x * 8 + wv;
  auto rw = __builtin_amdgcn_make_buffer_rsrc((void*)wp, 0, (int)SHARD, 0x00020000);

  if (__builtin_expect(ci < 8, 1)) {
    auto due_of = [&](int u8) INL {
      return d == 0 ? SELF_T0 + u8 * SELF_DT / 8 : PUSH_T1 + (u8 * PUSH_DT + (ST_X && d >= 4 && u8 > NUC * 8 ? (u8 - NUC * 8) * ST_X : 0)) / 8;
    };
    auto until = [&](int t) INL { while (since(tstart) < t) __builtin_amdgcn_s_sleep(2); };
    u32x4 e[8];
    auto load = [&]() INL {
      auto r = __builtin_amdgcn_make_buffer_rsrc((void*)(sst == 8 * NS ? wp : xp), 0, (int)SHARD, 0x00020000);
#pragma unroll
      for (int j = 0; j < 8; ++j) e[j] = __builtin_amdgcn_raw_buffer_load_b128(r, vo, so + j * sst, 0);
    };
    auto rout = [&]() INL { return __builtin_amdgcn_make_buffer_rsrc((void*)p.out, 0, NR * MS * NS * 4, 0x00020000); };
    auto direct = [&](int s) INL {
      int jd, ju;
      job_of(((const u32*)p.jobs[s])[(RLEN - 1) >> 2] >> ((RLEN - 1) & 3) * 8 & 0xff, true, d, jd, ju);
      return (row_of(rank ^ d, jd, ju) + xg * 2) * (NS * 4) + (d & 3) * 512;
    };
    const u32 ovo = (lane >> 5) * (NS * 4) + (lane & 31) * 16;
    if (__builtin_expect(wv >= NLEAD, 0)) {
      __builtin_amdgcn_s_barrier();
      static_assert(is_w(7) && NLEAD <= 7, "the marks are in place before the peer has all of w");
      if (wv == 7 && d >= 4)
#pragma unroll 1
        for (int s = 0; s < NRS; ++s) __builtin_amdgcn_raw_buffer_store_b128(u32x4{~0u, ~0u, ~0u, ~0u}, rout(), ovo + direct(s), 0, AUX_SYS);
      until(wv < PUSH_LEADT ? wv * PUSH_DT0 : due_of(wv * 8) - PUSH_PRE);
    }
    load();
    u32 cv = __builtin_amdgcn_raw_buffer_load_b32(__builtin_amdgcn_make_buffer_rsrc((void*)cntp, 0, 4, 0x00020000), 0, 0, AUX_AGENT);
    __builtin_amdgcn_sched_barrier(0);
    if (wv < NLEAD) {
      if (lane < 3) __hip_atomic_store(&turn[lane], 0u, __ATOMIC_RELAXED, __HIP_MEMORY_SCOPE_WORKGROUP);
      asm volatile("s_waitcnt lgkmcnt(0)" ::: "memory");
      __builtin_amdgcn_s_barrier();
    }
    const int nx = d == 0 ? NT : d < 4 ? NUI : NUC;
    const int nu = d < 4 ? nx : NUC + (NST + 7 - kb) / 8;
    char* tw = smem + wv * TR_B + (lane >> 5) * TR_ROW + (lane & 31) * 16;
    const char* tr = smem + wv * TR_B + m * TR_ROW + q * 16;
    const u32 soff = lane * 16;
    u32 call = 0;
    u32 boff = 0;
    char* to = nullptr;
    u32 dvo = soff, dst = 1024;
#pragma unroll 1
    for (int u = wv;;) {
      if (sst == 2 * KD) {
#pragma unroll
        for (int j = 0; j < 8; ++j) *(u32x4*)(tw + j * 2 * TR_ROW) = e[j];
#pragma unroll
        for (int i = 0; i < 8; ++i) e[i] = *(const u32x4*)(tr + i * 64);
      } else if (sst == 8 * NS) {
#pragma unroll
        for (int j = 0; j < 8; ++j) *(u32x4*)(smem + wv * TR_B + lane * 16 + j * 1024) = e[j];
        const u32* wr = (const u32*)(smem + wv * TR_B + q * 2048 + m * 4);
#pragma unroll
        for (int j = 0; j < 8; ++j) {
          const u32* w4 = wr + (j >> 2) * 16 + (j & 3) * 128;
          u32 c0, c1, c2, c3;
          transpose4(w4[0], w4[32], w4[64], w4[96], c0, c1, c2, c3);
          e[(j & 4) + 0][j & 3] = c0; e[(j & 4) + 1][j & 3] = c1; e[(j & 4) + 2][j & 3] = c2; e[(j & 4) + 3][j & 3] = c3;
        }
      }
#pragma unroll
      for (int i = 0; i < 8; ++i) e[i][0] = ~e[i][0];
      if (u < 8) {
        asm volatile("" : "+v"(cv));
        call = __builtin_amdgcn_readfirstlane(cv) + 1;
        boff = call % NBUF * BUF_B + d * SHARD + kb * 8192;
      }
      if (u < nx) to = pws + (boff + u * TILE_B);
      auto rd = __builtin_amdgcn_make_buffer_rsrc((void*)to, 0, u < nx ? 8192 : 16 * NS * 4, 0x00020000);
      int bonus = 0;
#pragma unroll
      for (int i = 0; i < 8; ++i) {
        if (i == 7) { if (!bonus) asm volatile("s_waitcnt vmcnt(6)" ::: "memory"); }
        else if (u > 0 && d) {
          while ((int)__hip_atomic_load(&turn[0], __ATOMIC_RELAXED, __HIP_MEMORY_SCOPE_WORKGROUP) < u * 8 + i - 6) __builtin_amdgcn_s_sleep(1);
        }
        if (i % PUSH_G == 0) {
          const int due = due_of(u * 8 + i);
          int t;
          while ((t = since(tstart)) < due) __builtin_amdgcn_s_sleep(1);
          if (i == 6 && u >= BONUS_U) { bonus = (t - due) >> BONUS_SH; bonus = bonus > BONUS_MAX ? BONUS_MAX : bonus; }
          if (i == 0 && u == PACE_U)
            __builtin_amdgcn_raw_buffer_store_b32((u32)tstart + (u32)(t - due), __builtin_amdgcn_make_buffer_rsrc((void*)(myws + PACE_OFF + d * 128 + xg * 4), 0, 4, 0x00020000), soff, 0, AUX_AGENT);
        }
        if (i == 0) STAMPL(u >> 3);
        __builtin_amdgcn_raw_buffer_store_b128(e[i], rd, dvo + i * dst, 0, AUX_SYS);
        __builtin_amdgcn_sched_barrier(0);
      }
#pragma unroll
      for (int k = 7; k >= 0; --k) {
        asm volatile("s_waitcnt vmcnt(%0)" ::"n"(k) : "memory");
        __hip_atomic_fetch_max(&turn[0], (u32)(u * 8 + 8 - k + bonus), __ATOMIC_RELAXED, __HIP_MEMORY_SCOPE_WORKGROUP);
      }
      if (u < nx && u + 8 >= nx) {
        u32 done = 0;
        if (lane == 0) done = __hip_atomic_fetch_add(&turn[1], 1u, __ATOMIC_RELAXED, __HIP_MEMORY_SCOPE_WORKGROUP);
        if (__builtin_amdgcn_readfirstlane(done) == 7)
          __builtin_amdgcn_raw_buffer_store_b32(call, __builtin_amdgcn_make_buffer_rsrc((void*)(pws + FLAG_OFF + (d * 8 + xg) * 128), 0, 4, 0x00020000), soff, 0, AUX_SYS);
      }
      u += 8;
      if (u >= nu) break;
      if (u < nx) {
        source(u);
        until(due_of(u * 8) - PUSH_PRE);
        load();
      } else {
        sst = 0;
        dvo = ovo;
        dst = 2 * NS * 4;
        const int n = (u - NUC) * 8 + kb, j = n / NRS;
        int jd, ju;
        job_of(((const u32*)p.jobs[n % NRS])[j >> 2] >> (j & 3) * 8 & 0xff, true, d, jd, ju);
        to = (char*)p.outp[d - 4] + (row_of(rank, jd, ju) * (NS * 4) + (d & 3) * 512);
        auto rs = __builtin_amdgcn_make_buffer_rsrc((void*)(myws + (boff + u * TILE_B)), 0, 8192, 0x00020000);
        until(due_of(u * 8) - ST_PRE);
        for (;;) {
#pragma unroll
          for (int j = 0; j < 8; ++j) e[j] = __builtin_amdgcn_raw_buffer_load_b128(rs, soff + j * 1024, 0, AUX_SYS);
          u32 mn = e[0][0];
#pragma unroll
          for (int j = 1; j < 8; ++j) mn = e[j][0] < mn ? e[j][0] : mn;
          if (__builtin_amdgcn_ballot_w64(mn == 0) == 0 || __builtin_amdgcn_s_memrealtime() - tstart > DEADLINE) break;
          __builtin_amdgcn_s_sleep(ST_SLEEP);
        }
      }
    }
    STAMPL(4);
    if (lane == 0) {
      if (d >= 4 && __hip_atomic_fetch_add(&turn[2], 1u, __ATOMIC_RELAXED, __HIP_MEMORY_SCOPE_WORKGROUP) == 7)
        __hip_atomic_store((gu32*)(pws + OFL_OFF) + ((d - 4) * 32 + 2 * NRS + xg) * 32, call, __ATOMIC_RELAXED, __HIP_MEMORY_SCOPE_SYSTEM);
      __hip_atomic_store(cntp, call, __ATOMIC_RELAXED, __HIP_MEMORY_SCOPE_AGENT);
    }
    if (d >= 4) {
      const bool two = wv + NRS - 8 >= 8;
      auto ro = rout();
      const u32 oa = direct(wv), ob = direct(two ? wv + NRS - 8 : wv);
      until(O_T0 + (two ? SN_G : SN_L) + wv * O_DT / 8);
      u32 v = two ? ~0u : 0u, n = 0;
      auto expired = [&]() INL { return (++n & 63u) == 0 && __builtin_amdgcn_s_memrealtime() - tstart > DEADLINE; };
      while (__builtin_amdgcn_readfirstlane(v) == ~0u && !expired()) {
        asm volatile("buffer_load_dword %0, off, %1, %2 offset:256 sc0 sc1\n\ts_waitcnt vmcnt(%3)" : "+v"(v) : "s"(ro), "s"(oa), "n"(SN_D - 1) : "memory");
        if (__builtin_amdgcn_readfirstlane(v) != ~0u) break;
        asm volatile("s_sleep %1" : "+v"(v) : "n"(SN_S));
      }
      if (SN_W) __builtin_amdgcn_s_sleep(SN_W);
      for (;;) {
        const u32 b = __builtin_amdgcn_raw_buffer_load_b32(ro, ovo, ob, AUX_SYS), a = __builtin_amdgcn_raw_buffer_load_b32(ro, ovo, oa, AUX_SYS);
        if (__builtin_amdgcn_ballot_w64(a == ~0u || b == ~0u) == 0 || expired()) break;
      }
      asm volatile("" :: "v"(v));
    }
    {
      if (d >= 4 && xg == 0 && wv == 0) {
        const char* fp = myws + OFL_OFF + (d - 4) * 4096 + (SENT ? 2 * NRS + (lane & 7) : lane < 2 * NRS + 8 ? lane : 0) * 128;
        while (since(tstart) < O_T0) __builtin_amdgcn_s_sleep(4);
        u32 v = 0, n = 0;
        for (;;) {
          asm volatile("global_load_dword %0, %1, off sc0 sc1\n\ts_waitcnt vmcnt(%2)" : "+v"(v) : "v"(fp), "n"(WAIT_D - 1) : "memory");
          if (__builtin_amdgcn_ballot_w64(v < call) == 0) break;
          asm volatile("s_sleep %1" : "+v"(v) : "n"(WAIT_S));
          if (__builtin_amdgcn_ballot_w64(v < call) == 0) break;
          if ((++n & 255u) == 0 && __builtin_amdgcn_s_memrealtime() - tstart > DEADLINE) {
            if (lane == 0) p.out[0] = __builtin_nanf("");
            break;
          }
        }
        asm volatile("s_waitcnt vmcnt(0)" : "+v"(v) :: "memory");
        STAMP(1);
      }
    }
  }
  int role = ci;
  asm volatile("" : "+s"(role));
  if (role < 8) return;

  STAMP(0);
  u32 call = __hip_atomic_load(cntp, __ATOMIC_RELAXED, __HIP_MEMORY_SCOPE_AGENT) + 1;
  gu32* tmo = (gu32*)(myws + TMO_OFF);
  const int g = ci & 7;
  const int sl = (ci - 8) >> 3;
  const int set = sl == 0 ? xg : sl == 1 ? 16 + xg : 8 + (xg >> 1) + ((xg ^ g) & 1) * 4;
  const bool isr = set < NRS;
  const int dl = 4 + (g >> 1), half = g & 1;
  const int len = isr ? RLEN : LLEN;
  u64 jlo = p.jobs[set][0], jhi = p.jobs[set][1];
  float* outp = isr ? p.outp[dl - 4] : p.out;
  char* lws = p.ws[dl];
  float scale = p.sx[0] * p.sw[0];
  asm volatile("" : "+s"(jlo), "+s"(jhi), "+s"(outp), "+s"(lws));
  asm volatile("" : "+v"(scale));
  if (threadIdx.x == 0) __hip_atomic_store(&turn[0], 0u, __ATOMIC_RELAXED, __HIP_MEMORY_SCOPE_WORKGROUP);
  gu32* flags = (gu32*)(myws + FLAG_OFF);
  const u32 voff_g = lane * 16;

  bool sure;
  u32 n = 0;
  auto word = [&](gu32* a) INL { return __builtin_amdgcn_readfirstlane(__hip_atomic_load(a, __ATOMIC_RELAXED, __HIP_MEMORY_SCOPE_SYSTEM)); };
  auto slow = [&](gu32* f, gu32* f2, bool nap) INL {
    if ((++n & 31u) == 0) {
      if (word(f) >= call && word(f2) >= call) { sure = true; return; }
      if (word(tmo) == call || __builtin_amdgcn_s_memrealtime() - tstart > DEADLINE) {
        __hip_atomic_store(tmo, call, __ATOMIC_RELAXED, __HIP_MEMORY_SCOPE_SYSTEM);
        sure = true;
        return;
      }
    }
    if (nap) __builtin_amdgcn_s_sleep(POLL_SLEEP); else __builtin_amdgcn_s_sleep(1);
  };

  call = __builtin_amdgcn_readfirstlane(call);
  if (lane == 0) __hip_atomic_store(cntp, call, __ATOMIC_RELAXED, __HIP_MEMORY_SCOPE_AGENT);
  char* gbuf = myws + (size_t)(call % NBUF) * BUF_B + wv * 8192;
  auto rxg = __builtin_amdgcn_make_buffer_rsrc((void*)gbuf, 0, (int)BUF_B, 0x00020000);
  auto rzr = __builtin_amdgcn_make_buffer_rsrc((void*)(myws + (size_t)((call + NBUF - 1) % NBUF) * BUF_B), 0, (int)BUF_B, 0x00020000);
  char* obase = isr ? myws + (size_t)(call % NBUF) * BUF_B + (size_t)dl * SHARD + NUC * TILE_B : (char*)outp;
  u32 omark = isr ? ~0u : 0u;
  auto voff_out = [&](u32 pitch, u32 col) INL { return (wv * 2 + ((voff_g >> 8) & 1)) * pitch + col + (voff_g & 255) + (voff_g < 512 ? 0 : OOB); };
  u32 voff_o = isr ? voff_out(512, half * 256) : voff_out(NS * 4, g * 256);
  const u32 voff_z = (isr ? 0u : (u32)g * 8192) + threadIdx.x * 16;
  const u32 voff_t = lane < (sl == 2 ? 16 : 8) ? (sl == 2 ? g & 6 : g) * 1024 + lane * 128 : OOB;

  u32 jt, jo;
  int jg;
  {
    const u32 ent = lane < 16 ? (u32)((lane < 8 ? jlo : jhi) >> (lane & 7) * 8) & 0xff : 0xff;
    int d, u;
    job_of(ent, isr, dl, d, u);
    jt = ent == 0xff ? OOB : d * SHARD + u * TILE_B;
    jg = ent == 0xff ? 0x7fffffff : (d ? PUSH_T1 + (u + 1) * PUSH_DT : SELF_T0 + (u + 1) * SELF_DT) + (isr && u < TAIL_R ? R_GATE : GATE);
    jo = ent == 0xff ? OOB : isr && lane < RLEN - 1 ? (u32)(lane * NRS + set) * 8192 : row_of(rank, d, u) * (NS * 4);
  }
  u32 known = 0;
  u32 got = 0;
  u32 warm = 0;
  int now;

  auto probe = [&](u32 off, int chunk) INL { return __builtin_amdgcn_raw_buffer_load_b32(rxg, 1008, off + chunk * 1024, AUX_SYS); };
  auto rsrc_if = [&](bool on) INL { return __builtin_amdgcn_make_buffer_rsrc((void*)gbuf, 0, on ? (int)BUF_B : 0, 0x00020000); };
  auto fetch = [&](u32x4 (&a)[8], int j) INL {
    auto r = rsrc_if(known >> j & 1);
    got |= known & 1u << j;
    const u32 off = __builtin_amdgcn_readlane(jt, j);
#pragma unroll
    for (int i = 0; i < 8; ++i) a[i] = __builtin_amdgcn_raw_buffer_load_b128(r, voff_g + (i & 3) * 1024, off + (i >> 2) * 4096, FAUX);
  };
  auto touch = [&](int j) INL {
    j += __builtin_ctz(((known & ~warm) | 1u << 31) >> j);
    warm |= 1u << (j & 15);
    return __builtin_amdgcn_raw_buffer_load_b32(rsrc_if(j < 16), voff_t, __builtin_amdgcn_readlane(jt, j), AUX_SYS);
  };
  auto peek = [&](int t) INL {
    const bool on = jg <= t && !(known >> (lane & 31) & 1);
    return __builtin_amdgcn_raw_buffer_load_b32(rxg, on ? jt : OOB, 8176, AUX_SYS);
  };
  u32 pk, tv[4];
  u32x4 a0[8], a1[8];
  auto park = [&](int i) INL { return (u32x4*)(smem + (i >> 2) * 32768 + wv * 4096 + (i & 3) * 1024 + voff_g); };
  auto early = [&](int phase) INL {
    if (phase) known |= (u32)__builtin_amdgcn_ballot_w64(pk != 0);
    if (phase < 2) pk = peek(since(tstart));
    if (phase == 1) {
      fetch(a1, isr ? 0 : 31);
      warm = isr;
    }
#pragma unroll
    for (int k = 0; k < 4; ++k) {
      if (phase == 1) tv[k] = touch(0);
      if (phase == 2) asm volatile("" ::"v"(tv[k]));
    }
    if (phase == 1)
#pragma unroll
      for (int i = 0; i < 8; ++i) *park(i) = a1[i];
  };

  i32x8 bf[4][4];
  if (!isr) {
    const u32 voff = (u32)(wv * 512 + q * 16) * NS + g * 64 + m * 4;
#pragma unroll
    for (int ks = 0; ks < 4; ++ks)
#pragma unroll
      for (int h = 0; h < 2; ++h) {
        const int due = W_DELAY + sl * W_SLOT + (ks * 2 + h) * W_DT;
        while (since(tstart) < due) __builtin_amdgcn_s_sleep(4);
        if (ks * 2 + h >= 6) early(ks * 2 + h - 6);
#pragma unroll
        for (int r = 0; r < 4; ++r) {
          u32 d0 = __builtin_amdgcn_raw_buffer_load_b32(rw, voff + (r * 4 + 0) * NS, (ks * 128 + h * 64) * NS, 0);
          u32 d1 = __builtin_amdgcn_raw_buffer_load_b32(rw, voff + (r * 4 + 1) * NS, (ks * 128 + h * 64) * NS, 0);
          u32 d2 = __builtin_amdgcn_raw_buffer_load_b32(rw, voff + (r * 4 + 2) * NS, (ks * 128 + h * 64) * NS, 0);
          u32 d3 = __builtin_amdgcn_raw_buffer_load_b32(rw, voff + (r * 4 + 3) * NS, (ks * 128 + h * 64) * NS, 0);
          u32 c0, c1, c2, c3;
          transpose4(d0, d1, d2, d3, c0, c1, c2, c3);
          bf[0][ks][h * 4 + r] = (int)c0; bf[1][ks][h * 4 + r] = (int)c1; bf[2][ks][h * 4 + r] = (int)c2; bf[3][ks][h * 4 + r] = (int)c3;
        }
      }
  } else {
#pragma unroll
    for (int ks = 0; ks < 4; ++ks) {
      const int u = NPRE + ks * 2;
      const u32 off = dl * SHARD + u * TILE_B + half * 4096;
      gu32* f = flags + (dl * 8 + ((wv - u) & 7)) * 32;
      gu32* f2 = flags + (dl * 8 + ((wv - u - 1) & 7)) * 32;
      if (ks >= 2) early(ks - 2);
      while (since(tstart) < PUSH_T1 + (u + 2) * PUSH_DT - (1 - half) * PUSH_DT / 2 + (ks == 3 && W_LAG ? W_LAG : GATE)) __builtin_amdgcn_s_sleep(4);
      sure = false;
      while (!sure && !(ks == 3 && W_LAG)) {
        if (__builtin_amdgcn_ballot_w64(probe(off + TILE_B, 3) == 0) == 0) break;
        slow(f, f2, true);
      }
      for (;;) {
#pragma unroll
        for (int i = 0; i < 8; ++i) a1[i] = __builtin_amdgcn_raw_buffer_load_b128(rxg, voff_g + (i & 3) * 1024, off + (i >> 2) * TILE_B, AUX_SYS);
        if (sure) break;
        u32 mn = a1[0][0];
#pragma unroll
        for (int i = 1; i < 8; ++i) mn = a1[i][0] < mn ? a1[i][0] : mn;
        if (__builtin_amdgcn_ballot_w64(mn == 0) == 0) break;
        slow(f, f2, true);
      }
#pragma unroll
      for (int i = 0; i < 8; ++i)
#pragma unroll
        for (int e = 0; e < 4; ++e) bf[i & 3][ks][(i >> 2) * 4 + e] = (int)(e ? a1[i][e] : ~a1[i][e]);
    }
  }
  early(2);

  int it = 0;
  auto hole = [&](u32x4 (&a)[8], int i0, int i1) INL {
    u32 mn = a[i0][0];
#pragma unroll
    for (int i = i0 + 1; i < i1; ++i) mn = a[i][0] < mn ? a[i][0] : mn;
    return __builtin_amdgcn_ballot_w64(mn == 0) != 0;
  };
  auto refill = [&](u32x4 (&a)[8], int i0, u32 off) INL {
#pragma unroll
    for (int i = i0; i < 8; ++i)
      if (hole(a, i, i + 1)) a[i] = __builtin_amdgcn_raw_buffer_load_b128(rxg, voff_g + (i & 3) * 1024, off + (i >> 2) * 4096, AUX_SYS);
  };
  auto drop = [&](u32x4 (&a)[8]) INL {
#pragma unroll
    for (int i = 0; i < 8; ++i) asm volatile("" : "=v"(a[i]));
  };
  f32x4 acc[4];
  auto mul = [&](u32x4 (&a)[8], int ks) INL {
    i32x8 af;
#pragma unroll
    for (int e = 0; e < 4; ++e) { af[e] = (int)a[2 * ks][e]; af[4 + e] = (int)a[2 * ks + 1][e]; }
    af[0] = ~af[0]; af[4] = ~af[4];
#pragma unroll
    for (int c = 0; c < 4; ++c)
      acc[c] = __builtin_amdgcn_mfma_scale_f32_16x16x128_f8f6f4(af, bf[c][ks], ks ? acc[c] : f32x4{0.f, 0.f, 0.f, 0.f}, 0, 1, 0, 0, 0, 0);
  };
  auto share = [&]() INL {
    float* rb = (float*)smem + (it & 1) * 8192;
#pragma unroll
    for (int j = 0; j < 4; ++j)
      *(f32x4*)(rb + wv * 1024 + (q * 4 + j) * 64 + m * 4) = f32x4{acc[0][j], acc[1][j], acc[2][j], acc[3][j]};
    __builtin_amdgcn_sched_barrier(0);
  };
  auto gather = [&]() INL {
    asm volatile("s_waitcnt lgkmcnt(0)" ::: "memory");
    __builtin_amdgcn_s_barrier();
    const float* rp = (float*)smem + (it & 1) * 8192 + (lane >> 5) * 4096 + wv * 128 + (lane & 31) * 4;
    f32x4 s4 = *(const f32x4*)rp;
#pragma unroll
    for (int v = 1; v < 4; ++v) s4 += *(const f32x4*)(rp + v * 1024);
    return s4;
  };
  u32 o[2];
  auto finish = [&](f32x4 s4) INL {
#pragma unroll
    for (int e = 0; e < 2; ++e) {
      auto sw = __builtin_amdgcn_permlane32_swap(__float_as_uint(s4[e]), __float_as_uint(s4[e + 2]), false, false);
      const float y = (__uint_as_float(sw[0]) + __uint_as_float(sw[1])) * scale;
      const float yc = __builtin_fminf(__builtin_fmaxf(y, -60.f), 60.f);
      o[e] = __float_as_uint(y * __builtin_amdgcn_rcpf(1.f + __expf(-yc)));
    }
  };
  auto put = [&](bool last = false) INL {
    auto b0 = __builtin_amdgcn_permlane32_swap(o[0], o[0], false, false);
    auto b1 = __builtin_amdgcn_permlane32_swap(o[1], o[1], false, false);
    const u32 c0 = (b0[0] < 0xfffffffeu ? b0[0] : 0xfffffffeu) ^ omark;
    auto r = __builtin_amdgcn_make_buffer_rsrc((void*)obase, 0, NR * MS * NS * 4, 0x00020000);
    if (OAUX != AUX_SYS && last && !isr) __builtin_amdgcn_raw_buffer_store_b128(u32x4{c0, b1[0], b0[1], b1[1]}, r, voff_o | __builtin_amdgcn_readlane(jo, it), 0, OAUX);
    else __builtin_amdgcn_raw_buffer_store_b128(u32x4{c0, b1[0], b0[1], b1[1]}, r, voff_o | __builtin_amdgcn_readlane(jo, it), 0, AUX_SYS);
    __builtin_amdgcn_sched_barrier(0);
    STAMPJ(3 + 2 * it, since(tstart));
  };
  auto wipe = [&]() INL {
    const u32 k = it * 2 * NRS + set * 2 + half;
    const u32 z = !isr ? __builtin_amdgcn_readlane(jt, it) : k >= 64 + NST ? OOB : dl * SHARD + (k < 64 ? NPRE * TILE_B : (NUC - 8) * TILE_B) + k * 8192;
    __builtin_amdgcn_raw_buffer_store_b128(u32x4{0, 0, 0, 0}, rzr, voff_z + z, 0, AUX_SYS);
  };
  auto step = [&](auto first_wave, u32x4 (&a)[8], u32x4 (&an)[8], u32& tv) INL {
    wipe();
    const int gate = __builtin_amdgcn_readlane(jg, it);
    sure = gate == 0x7fffffff;
    for (bool first = true; !sure; first = false) {
      if (__builtin_expect(!hole(a, 0, 8), 1)) break;
      const u32 off = __builtin_amdgcn_readlane(jt, it);
      gu32* f = flags + (off / SHARD * 8 + ((wv - off / TILE_B) & 7)) * 32;
      if (!first) slow(f, f, false);
      else if (__builtin_amdgcn_readlane(pk, it) == 0) {
        while (since(tstart) < gate) __builtin_amdgcn_s_sleep(2);
        if (since(tstart) < gate + LONG_DUE)
          while (!sure) {
            if (__builtin_amdgcn_ballot_w64(probe(off, 7) == 0) == 0) break;
            slow(f, f, true);
          }
      }
      refill(a, 0, off);
      __builtin_amdgcn_s_waitcnt(0x0f70);
    }
    now = since(tstart);
    STAMPJ(2 + 2 * it, now);
#pragma unroll
    for (int ks = 0; ks < 4; ++ks) mul(a, ks);
    share();
    known |= (u32)__builtin_amdgcn_ballot_w64(pk != 0) | 1u << it;
    if (__builtin_expect((known & ~got) >> (it + 1) & 1, 0)) fetch(an, it + 1);
    __builtin_amdgcn_sched_barrier(0);
    auto ahead = [&]() INL {
      fetch(a, it + 2);
      __builtin_amdgcn_sched_barrier(0);
      asm volatile("" ::"v"(tv));
      tv = touch(it + 3);
      __builtin_amdgcn_sched_barrier(0);
    };
    if (first_wave()) ahead();
    pk = peek(now + PEEK_LA);
    __builtin_amdgcn_sched_barrier(0);
    const f32x4 s4 = gather();
    if (!first_wave()) ahead();
    finish(s4);
    put();
    ++it;
  };
  auto solo = [&](u32x4 (&a)[8], u32x4 (&b)[8]) INL {
    const u32 off = __builtin_amdgcn_readlane(jt, it);
    gu32* f = flags + (off / SHARD * 8 + ((wv - off / TILE_B) & 7)) * 32;
    sure = off == OOB;
    known |= (u32)__builtin_amdgcn_ballot_w64(pk != 0);
    if (WIPE0) wipe();
    const u32 pace = __builtin_amdgcn_raw_buffer_load_b32(__builtin_amdgcn_make_buffer_rsrc((void*)(myws + PACE_OFF + dl * 128), 0, it ? 32 : 0, 0x00020000), voff_g >> 2, 0, AUX_AGENT);
    pk = peek(it ? -1 : since(tstart));
    u32 v = 0;
    if (!sure && hole(a, 0, 8)) {
      if (!(known >> it & 1)) {
        __builtin_amdgcn_s_waitcnt(0x0f70);
        const bool tail = (off / TILE_B & 31) >= (isr ? TAIL_R : TAIL_L);
        const int gate = __builtin_amdgcn_readlane(jg, it) - (tail ? TAIL_T : 0);
        while ((now = since(tstart)) < gate) __builtin_amdgcn_s_sleep(1);
        if (now < gate + LONG_DUE) {
          const u32 po = off + (tail ? PROBE_C * 1024 : 7168);
          while (!sure) {
            asm volatile("buffer_load_dword %0, off, %1, %2 offset:1008 sc0 sc1\n\ts_waitcnt vmcnt(%3)" : "+v"(v) : "s"(rxg), "s"(po), "n"(PROBE_D - 1) : "memory");
            if (!tail) asm volatile("s_waitcnt vmcnt(0)" : "+v"(v));
            if (__builtin_amdgcn_readfirstlane(v)) break;
            slow(f, f, true);
          }
        }
      }
#pragma unroll
      for (int i = 0; i < 8; ++i) {
        a[i] = __builtin_amdgcn_raw_buffer_load_b128(rxg, voff_g + (i & 3) * 1024, off + (i >> 2) * 4096, AUX_SYS);
        __builtin_amdgcn_sched_barrier(0);
      }
    }
#pragma unroll
    for (int ks = 0; ks < 4; ++ks) {
      while (!sure && hole(a, 2 * ks, 2 * ks + 2)) {
        slow(f, f, false);
        refill(a, 2 * ks, off);
        __builtin_amdgcn_s_waitcnt(0x0f70);
      }
      if (ks == 3) STAMPJ(2 + 2 * it, since(tstart));
      mul(a, ks);
      if (ks == 0) asm volatile("" ::"v"(v));
    }
    share();
    known |= 1u;
    drop(a);
    drop(b);
    int due = 0;
    if (it == 0) {
      fetch(a, 1);
      fetch(b, 2);
    } else if (isr) {
      due = O_T0 + (set * 2 + half) * O_DT / 16 + origin(pace, tstart, 50);
      obase = (char*)outp;
      omark = 0;
      voff_o = voff_out(NS * 4, g * 256);
    }
    finish(gather());
    if (due) while (since(tstart) < due) __builtin_amdgcn_s_sleep(1);
    put(true);
    if (!WIPE0) wipe();
    __builtin_amdgcn_sched_barrier(0);
    __builtin_amdgcn_s_waitcnt(0x0f72 - WIPE0);
    ++it;
  };
  auto run = [&](auto first_wave) INL {
#pragma unroll
    for (int i = 0; i < 8; ++i) a0[i] = *park(i);
    pk = 0;
    __builtin_amdgcn_s_waitcnt(0x0f70);
    for (;;) {
      solo(a0, a1);
      if (it == len) break;
      do { step(first_wave, a0, a1, tv[0]); step(first_wave, a1, a0, tv[1]); } while (it < len - 1);
    }
  };
  int second = wv >> 2;
  if (second) run(std::false_type{});
  asm volatile("" : "+s"(second));
  if (!second) run(std::true_type{});
  if (!SENT && isr) {
    asm volatile("s_waitcnt vmcnt(0)" ::: "memory");
    if (lane == 0 && __hip_atomic_fetch_add(&turn[0], 1u, __ATOMIC_RELAXED, __HIP_MEMORY_SCOPE_WORKGROUP) == 7)
      __hip_atomic_store((gu32*)(lws + OFL_OFF) + ((dl - 4) * 32 + set * 2 + half) * 32, call, __ATOMIC_RELAXED, __HIP_MEMORY_SCOPE_SYSTEM);
    STAMPJ(60, since(tstart));
  }
}

void build_jobs(u64 jobs[NSETS][2]) {
  unsigned char lst[2][256];
  int n[2] = {0, LPAD};
  for (int i = 0; i < 256; ++i) lst[0][i] = lst[1][i] = 0xff;
  auto add = [&](int k, int d, int u) { lst[k][n[k]++] = (unsigned char)(d << 5 | u); if (k == 0 && n[0] == NST) n[0] += RPAD; };
  for (int u = 0, v = 0; u < NH; ++u) {
    for (; v < NH && SELF_T0 + (v + 1) * SELF_DT <= PUSH_T1 + (u + 1) * PUSH_DT; ++v) add(0, 0, v);
    for (int d = 1; d < 4; ++d) add(0, d, u);
    if (u < NUC && !is_w(u) && cross_tile(u) >= NUI) for (int d = 5; d < 8; ++d) add(0, d, u);
  }
  unsigned char ex[64];
  int ne = 0, ie = 0;
  for (int u = 0; u < 2 * NUC; ++u) ex[ne++] = (unsigned char)u;
  for (int u = 0; u < NUC; ++u) if (!is_w(u)) for (int d = 4; d < 8; ++d) ex[ne++] = (unsigned char)(d << 5 | u);
  for (int u = 2 * NUC; u < NT; ++u) ex[ne++] = (unsigned char)u;
  for (int u = 0; u < NUI; ++u) {
    for (int d = 1; d < 4; ++d) add(1, d, u);
    for (int k = 0; k < (u < NUC ? 2 : 4) && ie < ne; ++k) lst[1][n[1]++] = ex[ie++];
  }
  for (int s = 0; s < NSETS; ++s) {
    const int k = s >= NRS, ns = k ? NLS : NRS, s0 = k ? s - NRS : s;
    jobs[s][0] = jobs[s][1] = ~0ull;
    for (int j = 0; j < 16 && s0 + j * ns < n[k]; ++j) ((unsigned char*)jobs[s])[j] = lst[k][s0 + j * ns];
  }
}
static_assert(RLEN <= 16 && LLEN <= 16 && NRS <= 16 && 2 * NUC <= NT && 64 + NST <= 2 * NRS * RLEN, "job lists");

}

extern "C" void kernel_launch(int rank, int nranks, ncclComm_t comm, hipStream_t stream,
                              void* const* d_in, const int* in_sizes, int n_in,
                              void* d_out, int out_size, const int* in_dtypes, int out_dtype,
                              void* const* d_out_all, void* const* d_ws_all, size_t ws_bytes) {
  (void)comm; (void)in_dtypes; (void)out_dtype;
  if (nranks != NR || (unsigned)rank >= (unsigned)NR || n_in != 4 || ws_bytes < WS_NEED ||
      in_sizes[0] != (int)SHARD || in_sizes[1] != (int)SHARD || out_size != NR * MS * NS) {
    fprintf(stderr, "kernel_launch: unexpected problem shape\n");
    return;
  }
  Params p;
  p.x = (const unsigned char*)d_in[0];
  p.w = (const unsigned char*)d_in[1];
  p.sx = (const float*)d_in[2];
  p.sw = (const float*)d_in[3];
  p.out = (float*)d_out;
  for (int d = 0; d < NR; ++d) p.ws[d] = (char*)d_ws_all[rank ^ d];
  for (int i = 0; i < 4; ++i) p.outp[i] = (float*)d_out_all[rank ^ (4 + i)];
  p.rank = rank;
  p.pad = 0;
  build_jobs(p.jobs);
  hipLaunchKernelGGL(ag_gemm_kernel, dim3(256), dim3(512), 0, stream, p);
}
```

```cpp
#include <hip/hip_runtime.h>
#include <rccl/rccl.h>
#include <cstdio>
#include <type_traits>

typedef unsigned int u32;
typedef unsigned long long u64;
typedef u32 u32x4 __attribute__((ext_vector_type(4)));
typedef u32 u32x2 __attribute__((ext_vector_type(2)));
typedef int i32x8 __attribute__((ext_vector_type(8)));
typedef float f32x4 __attribute__((ext_vector_type(4)));
typedef __attribute__((address_space(1))) u32 gu32;

namespace {

constexpr int NR = 8;
constexpr int KD = 4096;
constexpr int MS = 512;
constexpr int NS = 512;
constexpr u32 SHARD = (u32)MS * KD;
constexpr u32 TILE_B = 16 * KD;
constexpr int NT = MS / 16;
#define NH 26
#define ND 2
#define NA 4
static_assert(NH + ND + NA == NT, "tile kinds");
constexpr int NUI = NH + ND;
constexpr int NUC = 8 + NA + ND;
#define NPRE 1
#define NRS 12
constexpr int NSETS = 24, NLS = NSETS - NRS;
constexpr int NRJ = 4 * NH + 3 * NA, NLJ = NT + 3 * NUI + 4 * (NA + ND);
constexpr int RLEN = (NRJ + 2 * NRS - 1) / (2 * NRS) * 2, LLEN = (NLJ + 2 * NLS - 1) / (2 * NLS) * 2;
constexpr int RPAD = RLEN * NRS - NRJ, LPAD = LLEN * NLS - NLJ;
constexpr int NST = NRJ - NRS;
static_assert(NUC * 8 + NST <= NT * 8, "staging");
constexpr u32 BUF_B = (u32)NR * SHARD;
constexpr int NBUF = 3;
constexpr size_t FLAG_OFF = (size_t)NBUF * BUF_B;
constexpr size_t FLAG_B = (size_t)NR * 8 * 128;
constexpr size_t OFL_OFF = FLAG_OFF + FLAG_B;
constexpr size_t OFL_B = 4 * 4096;
constexpr size_t CNT_OFF = OFL_OFF + OFL_B;
constexpr size_t CNT_B = 256 * 8 * 4;
constexpr size_t TMO_OFF = CNT_OFF + CNT_B;
constexpr size_t PACE_OFF = TMO_OFF + 4096;
constexpr size_t DBG_OFF = PACE_OFF + 4096;
constexpr size_t DBG_B = 256 * 128 * 8;
constexpr size_t WS_NEED = DBG_OFF + DBG_B;
constexpr u64 DEADLINE = 100000000ull;
#define PUSH_DT 110
#define PUSH_T1 45
#define I_DT PUSH_DT
#define I_T1 PUSH_T1
#define BONUS_U 2
#define BONUS_SH 5
#define BONUS_MAX 2
#define PACE_U 20
#define O_M 5
#define PUSH_G 2
#define PUSH_PRE 500
#define ST_PRE 100
#define ST_PROBE 0
#define ST_SLEEP 4
#define PUSH_LEADT 8
#define NLEAD 2
#define PUSH_DT0 55
#define W_DELAY 150
#define W_DT 100
#define W_LAG 0
#define W_SLOT 50
#define GATE (-10)
#define R_GATE 80
#ifndef LONG_DUE
#define LONG_DUE 150
#endif
#define O_T0 (PUSH_T1 + NUC * PUSH_DT + NST * PUSH_DT / 8 + O_M)
#define O_DT 113
#define SELF_T0 200
#define SELF_DT 50
#define TAIL_R (NH - 1)
#define TAIL_L (NUI - 1)
#define TAIL_T 50
#define PROBE_C 2
#define PROBE_D 3
#define PEEK_LA 25
#define POLL_SLEEP 4
#define FAUX 0
#define WAIT_D 6
#define WAIT_S 3
#define SENT 1
#define SN_G 60
#define SN_L 130
#define OAUX 17
#define WIPE0 1
#define FAST0 1
#define W1_SLEEP 0
#define BONUS0 0
#define WSPLIT 1
#define FOL_4 0
#define FOL_6 4
#define SN_D 3
#define SN_S 4
#define SN_W 0
#define PUT1 1
#define PEEK0 1
#define AUX_SYS 17
#define AUX_AGENT 16
constexpr int TR_ROW = 528;
constexpr int TR_B = 16 * TR_ROW;

struct Params {
  char* ws[NR];
  float* outp[4];
  const unsigned char* x;
  const unsigned char* w;
  const float* sx;
  const float* sw;
  float* out;
  int rank, pad;
  u64 jobs[NSETS][2];
};

__host__ __device__ constexpr bool is_w(int u) { return u >= NPRE && u < NPRE + 8; }
__host__ __device__ constexpr int cross_tile(int u) { u = u < NPRE ? u : u - 8; return u < NA ? NUI + u : NH + u - NA; }
__device__ __forceinline__ void job_of(u32 ent, bool isr, int dl, int& d, int& u) {
  u = ent & 31;
  d = isr && ent >= 128 ? dl ^ ((ent >> 5) - 4) : ent >> 5;
}
__device__ __forceinline__ u32 row_of(int rank, int d, int u) { return (u32)((rank ^ d) * MS + (d < 4 ? u : cross_tile(u)) * 16); }
constexpr u32 OOB = 0x80000000u;

#define STAMP(i) do {} while (0)
#define STAMPV(i, v) do {} while (0)
#define STAMPL(i) do {} while (0)
#define STAMPJ(i, v) do {} while (0)

#define INL __attribute__((always_inline))
__device__ __forceinline__ int since(u64 t) { return (int)(__builtin_amdgcn_s_memrealtime() - t); }

__device__ __forceinline__ int origin(u32 v, u64 tstart, int dflt) {
  int sum = 0;
  bool ok = true;
#pragma unroll
  for (int k = 0; k < 8; ++k) {
    const int r = (int)(__builtin_amdgcn_readlane(v, k) - (u32)tstart);
    ok = ok && r > -100 && r < 1000;
    sum += r;
  }
  return ok ? sum >> 3 : dflt;
}

__device__ __forceinline__ void transpose4(u32 d0, u32 d1, u32 d2, u32 d3, u32& c0, u32& c1, u32& c2, u32& c3) {
  const u32 t0 = __builtin_amdgcn_perm(d1, d0, 0x05010400u), t1 = __builtin_amdgcn_perm(d1, d0, 0x07030602u);
  const u32 t2 = __builtin_amdgcn_perm(d3, d2, 0x05010400u), t3 = __builtin_amdgcn_perm(d3, d2, 0x07030602u);
  c0 = __builtin_amdgcn_perm(t2, t0, 0x05040100u);
  c1 = __builtin_amdgcn_perm(t2, t0, 0x07060302u);
  c2 = __builtin_amdgcn_perm(t3, t1, 0x05040100u);
  c3 = __builtin_amdgcn_perm(t3, t1, 0x07060302u);
}

__global__ void __launch_bounds__(512, 2) ag_gemm_kernel(Params p) {
  __shared__ __attribute__((aligned(16))) char smem[2 * TILE_B];
  __shared__ u32 turn[4];
  const int lane = threadIdx.x & 63;
  const int wv = __builtin_amdgcn_readfirstlane(threadIdx.x >> 6);
  const int xg = blockIdx.x & 7;
  const int ci = blockIdx.x >> 3;
  int rank = p.rank;
  char* myws = p.ws[0];
  char* pws = p.ws[(ci + 1) & 7];
  const unsigned char* xp = p.x;
  const unsigned char* wp = p.w;
  const int d = (ci + 1) & 7;
  const int kb = (xg + wv) & 7;
  u32 so, sst, vo;
  auto source = [&](int u) INL {
    const bool w = d >= 4 && is_w(u);
    u32 sw_ = (u32)(kb * 512 + (u - NPRE) * 64) * NS + (d & 3) * 128, sx_ = (d < 4 ? u : cross_tile(u)) * TILE_B + kb * 512;
    asm("" : "+s"(sw_), "+s"(sx_));
    so = w ? sw_ : sx_;
    sst = w ? 8 * NS : 2 * KD;
    const int sh = w ? 3 : 5;
    vo = (u32)(lane >> sh) * (sst >> (6 - sh)) + (lane & ((1 << sh) - 1)) * 16;
  };
  __builtin_assume(wv < 8);
  source(wv);
  static_assert(NPRE > 0, "the first unit is a tile of x");
  sst = wv || !d ? sst : 64;
  vo = wv || !d ? vo : (lane & 15) * KD + (lane >> 4) * 16;
  asm volatile("" : "+s"(so), "+s"(sst));
  asm volatile("" : "+v"(vo));
  asm volatile("" : "+s"(rank), "+s"(myws), "+s"(pws), "+s"(xp), "+s"(wp));
  const int m = lane & 15, q = lane >> 4;
  const u64 tstart = __builtin_amdgcn_s_memrealtime();

  gu32* cntp = (gu32*)(myws + CNT_OFF) + blockIdx.x * 8 + wv;
  auto rw = __builtin_amdgcn_make_buffer_rsrc((void*)wp, 0, (int)SHARD, 0x00020000);

  if (__builtin_expect(ci < 8, 1)) {
    auto due_of = [&](int u8) INL {
      return d == 0 ? SELF_T0 + u8 * SELF_DT / 8 : d < 4 ? I_T1 + u8 * I_DT / 8 : PUSH_T1 + u8 * PUSH_DT / 8;
    };
    auto until = [&](int t) INL { while (since(tstart) < t) __builtin_amdgcn_s_sleep(2); };
    u32x4 e[8];
    auto load = [&]() INL {
      auto r = __builtin_amdgcn_make_buffer_rsrc((void*)(sst == 8 * NS ? wp : xp), 0, (int)SHARD, 0x00020000);
#pragma unroll
      for (int j = 0; j < 8; ++j) e[j] = __builtin_amdgcn_raw_buffer_load_b128(r, vo, so + j * sst, 0);
    };
    auto rout = [&]() INL { return __builtin_amdgcn_make_buffer_rsrc((void*)p.out, 0, NR * MS * NS * 4, 0x00020000); };
    auto direct = [&](int s) INL {
      int jd, ju;
      job_of(((const u32*)p.jobs[s])[(RLEN - 1) >> 2] >> ((RLEN - 1) & 3) * 8 & 0xff, true, d, jd, ju);
      return (row_of(rank ^ d, jd, ju) + xg * 2) * (NS * 4) + (d & 3) * 512;
    };
    const u32 ovo = (lane >> 5) * (NS * 4) + (lane & 31) * 16;
    if (__builtin_expect(wv >= NLEAD, 0)) {
      __hip_atomic_store(&turn[lane & 3], 0u, __ATOMIC_RELAXED, __HIP_MEMORY_SCOPE_WORKGROUP);
      asm volatile("s_waitcnt lgkmcnt(0)" ::: "memory");
      __builtin_amdgcn_s_barrier();
      static_assert(is_w(7) && NLEAD <= 7, "the marks are in place before the peer has all of w");
      if (wv == 7 && d >= 4)
#pragma unroll 1
        for (int s = 0; s < NRS; ++s) __builtin_amdgcn_raw_buffer_store_b128(u32x4{~0u, ~0u, ~0u, ~0u}, rout(), ovo + direct(s), 0, AUX_SYS);
      until(wv < PUSH_LEADT ? wv * PUSH_DT0 : due_of(wv * 8) - PUSH_PRE);
    }
    u32 cs;
    asm volatile("s_load_dword %0, %1, 0x0" : "=s"(cs) : "s"(cntp) : "memory");
    if (W1_SLEEP && wv == 1) __builtin_amdgcn_s_sleep(W1_SLEEP);
    load();
    __builtin_amdgcn_sched_barrier(0);
    if (wv < NLEAD) __builtin_amdgcn_s_barrier();
    const int nx = d == 0 ? NT : d < 4 ? NUI : NUC;
    const int nu = d < 4 ? nx : NUC + (NST + 7 - kb) / 8;
    char* tw = smem + wv * TR_B + (lane >> 5) * TR_ROW + (lane & 31) * 16;
    const char* tr = smem + wv * TR_B + m * TR_ROW + q * 16;
    const u32 soff = lane * 16;
    int u = wv;
    auto acks = [&](int bonus) INL {
#pragma unroll
      for (int k = 7; k >= 0; --k) {
        asm volatile("s_waitcnt vmcnt(%0)" ::"n"(k) : "memory");
        __hip_atomic_fetch_max(&turn[0], (u32)(u * 8 + 8 - k + bonus), __ATOMIC_RELAXED, __HIP_MEMORY_SCOPE_WORKGROUP);
      }
    };
    auto next = [&]() INL {
      source(u);
      until(due_of(u * 8) - PUSH_PRE);
      load();
    };
    asm volatile("s_waitcnt lgkmcnt(0)" : "+s"(cs) :: "memory");
    const u32 call = cs + 1;
    const u32 boff = call % NBUF * BUF_B + d * SHARD + kb * 8192;
    if (__builtin_expect(sst == 64, 1)) {
      auto rd = __builtin_amdgcn_make_buffer_rsrc((void*)(pws + boff), 0, 8192, 0x00020000);
#pragma unroll
      for (int i = 0; i < 8; ++i) {
        e[i][0] = ~e[i][0];
        __builtin_amdgcn_raw_buffer_store_b128(e[i], rd, soff + i * 1024, 0, AUX_SYS);
        __builtin_amdgcn_sched_barrier(0);
        if (i == 0) STAMPL(0);
        if (i == 7) STAMPL(5);
        __builtin_amdgcn_sched_barrier(0);
      }
      acks(BONUS0);
      u = 8;
      next();
    }
    char* to = nullptr;
    u32 dvo = soff, dst = 1024;
#pragma unroll 1
    for (;;) {
      int bonus = 0;
      u32 kind = __builtin_amdgcn_readfirstlane(sst);
      asm volatile("" : "+s"(kind));
      if (kind == 2 * KD) {
#pragma unroll
        for (int j = 0; j < 8; ++j) *(u32x4*)(tw + j * 2 * TR_ROW) = e[j];
#pragma unroll
        for (int i = 0; i < 8; ++i) e[i] = *(const u32x4*)(tr + i * 64);
      }
      asm volatile("" : "+s"(kind));
      if (kind == 8 * NS) {
#pragma unroll
        for (int j = 0; j < 8; ++j) *(u32x4*)(smem + wv * TR_B + lane * 16 + j * 1024) = e[j];
        const u32* wr = (const u32*)(smem + wv * TR_B + q * 2048 + m * 4);
#pragma unroll
        for (int j = 0; j < 8; ++j) {
          const u32* w4 = wr + (j >> 2) * 16 + (j & 3) * 128;
          u32 c0, c1, c2, c3;
          transpose4(w4[0], w4[32], w4[64], w4[96], c0, c1, c2, c3);
          e[(j & 4) + 0][j & 3] = c0; e[(j & 4) + 1][j & 3] = c1; e[(j & 4) + 2][j & 3] = c2; e[(j & 4) + 3][j & 3] = c3;
        }
      }
#pragma unroll
      for (int i = 0; i < 8; ++i) e[i][0] = ~e[i][0];
      if (u < nx) to = pws + (boff + u * TILE_B);
      auto rd = __builtin_amdgcn_make_buffer_rsrc((void*)to, 0, u < nx ? 8192 : 16 * NS * 4, 0x00020000);
#pragma unroll
      for (int i = 0; i < 8; ++i) {
        if (i == 7) { if (!bonus) asm volatile("s_waitcnt vmcnt(6)" ::: "memory"); }
        else if (u > 0 && d) {
          while ((int)__hip_atomic_load(&turn[0], __ATOMIC_RELAXED, __HIP_MEMORY_SCOPE_WORKGROUP) < u * 8 + i - 6) __builtin_amdgcn_s_sleep(1);
        }
        if (i % PUSH_G == 0) {
          const int due = due_of(u * 8 + i);
          int t;
          while ((t = since(tstart)) < due) __builtin_amdgcn_s_sleep(1);
          if (i == 6 && u >= BONUS_U) { bonus = (t - due) >> BONUS_SH; bonus = bonus > BONUS_MAX ? BONUS_MAX : bonus; }
          if (i == 0 && u == PACE_U)
            __builtin_amdgcn_raw_buffer_store_b32((u32)tstart + (u32)(t - due), __builtin_amdgcn_make_buffer_rsrc((void*)(myws + PACE_OFF + d * 128 + xg * 4), 0, 4, 0x00020000), soff, 0, AUX_AGENT);
        }
        if (i == 0) STAMPL(u >> 3);
        __builtin_amdgcn_raw_buffer_store_b128(e[i], rd, dvo + i * dst, 0, AUX_SYS);
        __builtin_amdgcn_sched_barrier(0);
      }
      acks(bonus);
      if (u < nx && u + 8 >= nx) {
        u32 done = 0;
        if (lane == 0) done = __hip_atomic_fetch_add(&turn[1], 1u, __ATOMIC_RELAXED, __HIP_MEMORY_SCOPE_WORKGROUP);
        if (__builtin_amdgcn_readfirstlane(done) == 7)
          __builtin_amdgcn_raw_buffer_store_b32(call, __builtin_amdgcn_make_buffer_rsrc((void*)(pws + FLAG_OFF + (d * 8 + xg) * 128), 0, 4, 0x00020000), soff, 0, AUX_SYS);
      }
      u += 8;
      if (u >= nu) break;
      if (u < nx) {
        next();
      } else {
        sst = 0;
        dvo = ovo;
        dst = 2 * NS * 4;
        const int n = (u - NUC) * 8 + kb, j = n / NRS;
        int jd, ju;
        job_of(((const u32*)p.jobs[n % NRS])[j >> 2] >> (j & 3) * 8 & 0xff, true, d, jd, ju);
        to = (char*)p.outp[d - 4] + (row_of(rank, jd, ju) * (NS * 4) + (d & 3) * 512);
        auto rs = __builtin_amdgcn_make_buffer_rsrc((void*)(myws + (boff + u * TILE_B)), 0, 8192, 0x00020000);
        until(due_of(u * 8) - ST_PRE);
        for (;;) {
#pragma unroll
          for (int j = 0; j < 8; ++j) e[j] = __builtin_amdgcn_raw_buffer_load_b128(rs, soff + j * 1024, 0, AUX_SYS);
          u32 mn = e[0][0];
#pragma unroll
          for (int j = 1; j < 8; ++j) mn = e[j][0] < mn ? e[j][0] : mn;
          if (__builtin_amdgcn_ballot_w64(mn == 0) == 0 || __builtin_amdgcn_s_memrealtime() - tstart > DEADLINE) break;
          __builtin_amdgcn_s_sleep(ST_SLEEP);
        }
      }
    }
    STAMPL(4);
    if (lane == 0) {
      if (d >= 4 && __hip_atomic_fetch_add(&turn[2], 1u, __ATOMIC_RELAXED, __HIP_MEMORY_SCOPE_WORKGROUP) == 7)
        __hip_atomic_store((gu32*)(pws + OFL_OFF) + ((d - 4) * 32 + 2 * NRS + xg) * 32, call, __ATOMIC_RELAXED, __HIP_MEMORY_SCOPE_SYSTEM);
      __hip_atomic_store(cntp, call, __ATOMIC_RELAXED, __HIP_MEMORY_SCOPE_AGENT);
    }
    if (d >= 4) {
      const bool two = wv + NRS - 8 >= 8;
      auto ro = rout();
      const u32 oa = direct(wv), ob = direct(two ? wv + NRS - 8 : wv);
      until(O_T0 + (two ? SN_G : SN_L) + wv * O_DT / 8);
      u32 v = two ? ~0u : 0u, n = 0;
      auto expired = [&]() INL { return (++n & 63u) == 0 && __builtin_amdgcn_s_memrealtime() - tstart > DEADLINE; };
      while (__builtin_amdgcn_readfirstlane(v) == ~0u && !expired()) {
        asm volatile("buffer_load_dword %0, off, %1, %2 offset:256 sc0 sc1\n\ts_waitcnt vmcnt(%3)" : "+v"(v) : "s"(ro), "s"(oa), "n"(SN_D - 1) : "memory");
        if (__builtin_amdgcn_readfirstlane(v) != ~0u) break;
        asm volatile("s_sleep %1" : "+v"(v) : "n"(SN_S));
      }
      if (SN_W) __builtin_amdgcn_s_sleep(SN_W);
      for (;;) {
        const u32 b = __builtin_amdgcn_raw_buffer_load_b32(ro, ovo, ob, AUX_SYS), a = __builtin_amdgcn_raw_buffer_load_b32(ro, ovo, oa, AUX_SYS);
        if (__builtin_amdgcn_ballot_w64(a == ~0u || b == ~0u) == 0 || expired()) break;
      }
      asm volatile("" :: "v"(v));
    }
    {
      if (d >= 4 && xg == 0 && wv == 0) {
        const char* fp = myws + OFL_OFF + (d - 4) * 4096 + (SENT ? 2 * NRS + (lane & 7) : lane < 2 * NRS + 8 ? lane : 0) * 128;
        while (since(tstart) < O_T0) __builtin_amdgcn_s_sleep(4);
        u32 v = 0, n = 0;
        for (;;) {
          asm volatile("global_load_dword %0, %1, off sc0 sc1\n\ts_waitcnt vmcnt(%2)" : "+v"(v) : "v"(fp), "n"(WAIT_D - 1) : "memory");
          if (__builtin_amdgcn_ballot_w64(v < call) == 0) break;
          asm volatile("s_sleep %1" : "+v"(v) : "n"(WAIT_S));
          if (__builtin_amdgcn_ballot_w64(v < call) == 0) break;
          if ((++n & 255u) == 0 && __builtin_amdgcn_s_memrealtime() - tstart > DEADLINE) {
            if (lane == 0) p.out[0] = __builtin_nanf("");
            break;
          }
        }
        asm volatile("s_waitcnt vmcnt(0)" : "+v"(v) :: "memory");
        STAMP(1);
      }
    }
  }
  int role = ci;
  asm volatile("" : "+s"(role));
  if (role < 8) return;

  STAMP(0);
  u32 call = __hip_atomic_load(cntp, __ATOMIC_RELAXED, __HIP_MEMORY_SCOPE_AGENT) + 1;
  gu32* tmo = (gu32*)(myws + TMO_OFF);
  const int g = ci & 7;
  const int sl = (ci - 8) >> 3;
  const int set = sl == 0 ? xg : sl == 1 ? 16 + xg : 8 + (xg >> 1) + ((xg ^ g) & 1) * 4;
  const bool isr = set < NRS;
  const int dl = 4 + (g >> 1), half = g & 1;
  const int len = isr ? RLEN : LLEN;
  u64 jlo = p.jobs[set][0], jhi = p.jobs[set][1];
  float* outp = isr ? p.outp[dl - 4] : p.out;
  char* lws = p.ws[dl];
  float scale = p.sx[0] * p.sw[0];
  asm volatile("" : "+s"(jlo), "+s"(jhi), "+s"(outp), "+s"(lws));
  asm volatile("" : "+v"(scale));
  if (!FAST0 && threadIdx.x == 0) __hip_atomic_store(&turn[0], 0u, __ATOMIC_RELAXED, __HIP_MEMORY_SCOPE_WORKGROUP);
  gu32* flags = (gu32*)(myws + FLAG_OFF);
  const u32 voff_g = lane * 16;

  bool sure;
  u32 n = 0;
  auto word = [&](gu32* a) INL { return __builtin_amdgcn_readfirstlane(__hip_atomic_load(a, __ATOMIC_RELAXED, __HIP_MEMORY_SCOPE_SYSTEM)); };
  auto slow = [&](gu32* f, gu32* f2, bool nap) INL {
    if ((++n & 31u) == 0) {
      if (word(f) >= call && word(f2) >= call) { sure = true; return; }
      if (word(tmo) == call || __builtin_amdgcn_s_memrealtime() - tstart > DEADLINE) {
        __hip_atomic_store(tmo, call, __ATOMIC_RELAXED, __HIP_MEMORY_SCOPE_SYSTEM);
        sure = true;
        return;
      }
    }
    if (nap) __builtin_amdgcn_s_sleep(POLL_SLEEP); else __builtin_amdgcn_s_sleep(1);
  };

  call = __builtin_amdgcn_readfirstlane(call);
  if (lane == 0) __hip_atomic_store(cntp, call, __ATOMIC_RELAXED, __HIP_MEMORY_SCOPE_AGENT);
  char* gbuf = myws + (size_t)(call % NBUF) * BUF_B + wv * 8192;
  auto rxg = __builtin_amdgcn_make_buffer_rsrc((void*)gbuf, 0, (int)BUF_B, 0x00020000);
  auto rzr = __builtin_amdgcn_make_buffer_rsrc((void*)(myws + (size_t)((call + NBUF - 1) % NBUF) * BUF_B), 0, (int)BUF_B, 0x00020000);
  char* obase = isr ? myws + (size_t)(call % NBUF) * BUF_B + (size_t)dl * SHARD + NUC * TILE_B : (char*)outp;
  u32 omark = isr ? ~0u : 0u;
  auto voff_out = [&](u32 pitch, u32 col) INL { return (wv * 2 + ((voff_g >> 8) & 1)) * pitch + col + (voff_g & 255) + (voff_g < 512 ? 0 : OOB); };
  u32 voff_o = isr ? voff_out(512, half * 256) : voff_out(NS * 4, g * 256);
  const u32 voff_z = (isr ? 0u : (u32)g * 8192) + threadIdx.x * 16;
  const u32 voff_t = lane < (sl == 2 ? 16 : 8) ? (sl == 2 ? g & 6 : g) * 1024 + lane * 128 : OOB;

  u32 jt, jo;
  int jg;
  {
    const u32 ent = lane < 16 ? (u32)((lane < 8 ? jlo : jhi) >> (lane & 7) * 8) & 0xff : 0xff;
    int d, u;
    job_of(ent, isr, dl, d, u);
    jt = ent == 0xff ? OOB : d * SHARD + u * TILE_B;
    jg = ent == 0xff ? 0x7fffffff : (d >= 4 ? PUSH_T1 + (u + 1) * PUSH_DT : d ? I_T1 + (u + 1) * I_DT : SELF_T0 + (u + 1) * SELF_DT) + (isr && u < TAIL_R ? R_GATE : GATE);
    jo = ent == 0xff ? OOB : isr && lane < RLEN - 1 ? (u32)(lane * NRS + set) * 8192 : row_of(rank, d, u) * (NS * 4);
  }
  u32 known = 0;
  u32 got = 0;
  u32 warm = 2;
  int now;

  auto probe = [&](u32 off, int chunk) INL { return __builtin_amdgcn_raw_buffer_load_b32(rxg, 1008, off + chunk * 1024, AUX_SYS); };
  auto rsrc_if = [&](bool on) INL { return __builtin_amdgcn_make_buffer_rsrc((void*)gbuf, 0, on ? (int)BUF_B : 0, 0x00020000); };
  auto fetch = [&](u32x4 (&a)[8], int j) INL {
    auto r = rsrc_if(known >> j & 1);
    got |= known & 1u << j;
    const u32 off = __builtin_amdgcn_readlane(jt, j);
#pragma unroll
    for (int i = 0; i < 8; ++i) a[i] = __builtin_amdgcn_raw_buffer_load_b128(r, voff_g + (i & 3) * 1024, off + (i >> 2) * 4096, FAUX);
  };
  auto touch = [&](int j) INL {
    j += __builtin_ctz(((known & ~warm) | 1u << 31) >> j);
    warm |= 1u << (j & 15);
    return __builtin_amdgcn_raw_buffer_load_b32(rsrc_if(j < 16), voff_t, __builtin_amdgcn_readlane(jt, j), AUX_SYS);
  };
  auto peek = [&](int t) INL {
    const bool on = jg <= t && !(known >> (lane & 31) & 1);
    return __builtin_amdgcn_raw_buffer_load_b32(rxg, on ? jt : OOB, 8176, AUX_SYS);
  };
  u32 pk, tv[4];
  u32x4 a0[8], a1[8];
  auto park = [&](int i) INL { return (u32x4*)(smem + (i >> 2) * 32768 + wv * 4096 + (i & 3) * 1024 + voff_g); };
  auto early = [&](int phase) INL {
    if (phase) known |= (u32)__builtin_amdgcn_ballot_w64(pk != 0);
    if (phase < 3) pk = peek(since(tstart));
    if (phase == 1 || phase == 2) fetch(a1, phase == 2 ? 1 : isr ? 0 : 31);
#pragma unroll
    for (int k = 0; k < 4; ++k) {
      if (phase == 2) tv[k] = touch(isr ? 2 : 0);
      if (phase == 3) asm volatile("" ::"v"(tv[k]));
    }
    if (phase == 1 || phase == 2)
#pragma unroll
      for (int i = 0; i < 8; ++i) *park(phase * 8 - 8 + i) = a1[i];
  };

  i32x8 bf[4][4];
  if (!isr) {
    const u32 voff = (u32)(wv * 512 + q * 16) * NS + g * 64 + m * 4;
#pragma unroll
    for (int ks = 0; ks < 4; ++ks)
#pragma unroll
      for (int h = 0; h < 2; ++h) {
        const int due = W_DELAY + sl * W_SLOT + (ks * 2 + h) * W_DT;
        while (since(tstart) < due) __builtin_amdgcn_s_sleep(4);
        if (ks * 2 + h >= 5) early(ks * 2 + h - 5);
#pragma unroll
        for (int r = 0; r < 4; ++r) {
          u32 d0 = __builtin_amdgcn_raw_buffer_load_b32(rw, voff + (r * 4 + 0) * NS, (ks * 128 + h * 64) * NS, 0);
          u32 d1 = __builtin_amdgcn_raw_buffer_load_b32(rw, voff + (r * 4 + 1) * NS, (ks * 128 + h * 64) * NS, 0);
          u32 d2 = __builtin_amdgcn_raw_buffer_load_b32(rw, voff + (r * 4 + 2) * NS, (ks * 128 + h * 64) * NS, 0);
          u32 d3 = __builtin_amdgcn_raw_buffer_load_b32(rw, voff + (r * 4 + 3) * NS, (ks * 128 + h * 64) * NS, 0);
          u32 c0, c1, c2, c3;
          transpose4(d0, d1, d2, d3, c0, c1, c2, c3);
          bf[0][ks][h * 4 + r] = (int)c0; bf[1][ks][h * 4 + r] = (int)c1; bf[2][ks][h * 4 + r] = (int)c2; bf[3][ks][h * 4 + r] = (int)c3;
        }
      }
  } else {
#pragma unroll
    for (int ks = 0; ks < 4; ++ks) {
      const int u = NPRE + ks * 2;
      const u32 off = dl * SHARD + u * TILE_B + half * 4096;
      gu32* f = flags + (dl * 8 + ((wv - u) & 7)) * 32;
      gu32* f2 = flags + (dl * 8 + ((wv - u - 1) & 7)) * 32;
      if (ks >= 1) early(ks - 1);
      while (since(tstart) < PUSH_T1 + (u + 2) * PUSH_DT - (1 - half) * PUSH_DT / 2 + (ks == 3 && W_LAG ? W_LAG : GATE)) __builtin_amdgcn_s_sleep(4);
      sure = false;
#pragma unroll
      for (int i = 0; i < 4; ++i) a1[i] = __builtin_amdgcn_raw_buffer_load_b128(rxg, voff_g + i * 1024, off, AUX_SYS);
      while (!sure && !(ks == 3 && W_LAG)) {
        if (__builtin_amdgcn_ballot_w64(probe(off + TILE_B, 3) == 0) == 0) break;
        slow(f, f2, true);
      }
      for (bool all = !WSPLIT || sure;; all = true) {
#pragma unroll
        for (int i = 0; i < 8; ++i)
          if (i >= 4 || all) a1[i] = __builtin_amdgcn_raw_buffer_load_b128(rxg, voff_g + (i & 3) * 1024, off + (i >> 2) * TILE_B, AUX_SYS);
        if (sure) break;
        u32 mn = a1[0][0];
#pragma unroll
        for (int i = 1; i < 8; ++i) mn = a1[i][0] < mn ? a1[i][0] : mn;
        if (__builtin_amdgcn_ballot_w64(mn == 0) == 0) break;
        slow(f, f2, true);
      }
#pragma unroll
      for (int i = 0; i < 8; ++i)
#pragma unroll
        for (int e = 0; e < 4; ++e) bf[i & 3][ks][(i >> 2) * 4 + e] = (int)(e ? a1[i][e] : ~a1[i][e]);
    }
  }
  early(3);

  int it = 0;
  auto hole = [&](u32x4 (&a)[8], int i0, int i1) INL {
    u32 mn = a[i0][0];
#pragma unroll
    for (int i = i0 + 1; i < i1; ++i) mn = a[i][0] < mn ? a[i][0] : mn;
    return __builtin_amdgcn_ballot_w64(mn == 0) != 0;
  };
  auto refill = [&](u32x4 (&a)[8], int i0, u32 off) INL {
#pragma unroll
    for (int i = i0; i < 8; ++i)
      if (hole(a, i, i + 1)) a[i] = __builtin_amdgcn_raw_buffer_load_b128(rxg, voff_g + (i & 3) * 1024, off + (i >> 2) * 4096, AUX_SYS);
  };
  auto drop = [&](u32x4 (&a)[8]) INL {
#pragma unroll
    for (int i = 0; i < 8; ++i) asm volatile("" : "=v"(a[i]));
  };
  f32x4 acc[4];
  auto mul = [&](u32x4 (&a)[8], int ks) INL {
    i32x8 af;
#pragma unroll
    for (int e = 0; e < 4; ++e) { af[e] = (int)a[2 * ks][e]; af[4 + e] = (int)a[2 * ks + 1][e]; }
    af[0] = ~af[0]; af[4] = ~af[4];
#pragma unroll
    for (int c = 0; c < 4; ++c)
      acc[c] = __builtin_amdgcn_mfma_scale_f32_16x16x128_f8f6f4(af, bf[c][ks], ks ? acc[c] : f32x4{0.f, 0.f, 0.f, 0.f}, 0, 1, 0, 0, 0, 0);
  };
  auto share = [&]() INL {
    float* rb = (float*)smem + (it & 1) * 8192;
#pragma unroll
    for (int j = 0; j < 4; ++j)
      *(f32x4*)(rb + wv * 1024 + (q * 4 + j) * 64 + m * 4) = f32x4{acc[0][j], acc[1][j], acc[2][j], acc[3][j]};
    __builtin_amdgcn_sched_barrier(0);
  };
  auto gather = [&]() INL {
    asm volatile("s_waitcnt lgkmcnt(0)" ::: "memory");
    __builtin_amdgcn_s_barrier();
    const float* rp = (float*)smem + (it & 1) * 8192 + (lane >> 5) * 4096 + wv * 128 + (lane & 31) * 4;
    f32x4 s4 = *(const f32x4*)rp;
#pragma unroll
    for (int v = 1; v < 4; ++v) s4 += *(const f32x4*)(rp + v * 1024);
    return s4;
  };
  u32 o[2];
  auto finish = [&](f32x4 s4) INL {
#pragma unroll
    for (int e = 0; e < 2; ++e) {
      auto sw = __builtin_amdgcn_permlane32_swap(__float_as_uint(s4[e]), __float_as_uint(s4[e + 2]), false, false);
      const float y = (__uint_as_float(sw[0]) + __uint_as_float(sw[1])) * scale;
      const float yc = __builtin_fminf(__builtin_fmaxf(y, -60.f), 60.f);
      o[e] = __float_as_uint(y * __builtin_amdgcn_rcpf(1.f + __expf(-yc)));
    }
  };
  auto put = [&](bool last = false) INL {
    auto b0 = __builtin_amdgcn_permlane32_swap(o[0], o[0], false, false);
    auto b1 = __builtin_amdgcn_permlane32_swap(o[1], o[1], false, false);
    const u32 c0 = (b0[0] < 0xfffffffeu ? b0[0] : 0xfffffffeu) ^ omark;
    auto r = __builtin_amdgcn_make_buffer_rsrc((void*)obase, 0, NR * MS * NS * 4, 0x00020000);
    if (OAUX != AUX_SYS && last && !isr) __builtin_amdgcn_raw_buffer_store_b128(u32x4{c0, b1[0], b0[1], b1[1]}, r, voff_o | __builtin_amdgcn_readlane(jo, it), 0, OAUX);
    else __builtin_amdgcn_raw_buffer_store_b128(u32x4{c0, b1[0], b0[1], b1[1]}, r, voff_o | __builtin_amdgcn_readlane(jo, it), 0, AUX_SYS);
    __builtin_amdgcn_sched_barrier(0);
    STAMPJ(3 + 2 * it, since(tstart));
  };
  auto wipe = [&]() INL {
    const u32 k = it * 2 * NRS + set * 2 + half;
    const u32 z = !isr ? __builtin_amdgcn_readlane(jt, it) : k >= 64 + NST ? OOB : dl * SHARD + (k < 64 ? NPRE * TILE_B : (NUC - 8) * TILE_B) + k * 8192;
    __builtin_amdgcn_raw_buffer_store_b128(u32x4{0, 0, 0, 0}, rzr, voff_z + z, 0, AUX_SYS);
  };
  auto step = [&](auto first_wave, u32x4 (&a)[8], u32x4 (&an)[8], u32& tv) INL {
    wipe();
    const int gate = __builtin_amdgcn_readlane(jg, it);
    sure = gate == 0x7fffffff;
    for (bool first = true; !sure; first = false) {
      if (__builtin_expect(!hole(a, 0, 8), 1)) break;
      const u32 off = __builtin_amdgcn_readlane(jt, it);
      gu32* f = flags + (off / SHARD * 8 + ((wv - off / TILE_B) & 7)) * 32;
      if (!first) slow(f, f, false);
      else if (__builtin_amdgcn_readlane(pk, it) == 0) {
        while (since(tstart) < gate) __builtin_amdgcn_s_sleep(2);
        if (since(tstart) < gate + LONG_DUE)
          while (!sure) {
            if (__builtin_amdgcn_ballot_w64(probe(off, 7) == 0) == 0) break;
            slow(f, f, true);
          }
      }
      refill(a, 0, off);
      __builtin_amdgcn_s_waitcnt(0x0f70);
    }
    now = since(tstart);
    STAMPJ(2 + 2 * it, now);
#pragma unroll
    for (int ks = 0; ks < 4; ++ks) mul(a, ks);
    share();
    known |= (u32)__builtin_amdgcn_ballot_w64(pk != 0) | 1u << it;
    if (__builtin_expect((known & ~got) >> (it + 1) & 1, 0)) fetch(an, it + 1);
    __builtin_amdgcn_sched_barrier(0);
    auto ahead = [&]() INL {
      fetch(a, it + 2);
      __builtin_amdgcn_sched_barrier(0);
      asm volatile("" ::"v"(tv));
      tv = touch(it + 3);
      __builtin_amdgcn_sched_barrier(0);
    };
    if (first_wave()) ahead();
    pk = peek(now + PEEK_LA);
    __builtin_amdgcn_sched_barrier(0);
    const f32x4 s4 = gather();
    if (!first_wave()) ahead();
    finish(s4);
    put();
    ++it;
  };
  auto solo = [&](u32x4 (&a)[8]) INL {
    const u32 off = __builtin_amdgcn_readlane(jt, it);
    gu32* f = flags + (off / SHARD * 8 + ((wv - off / TILE_B) & 7)) * 32;
    sure = off == OOB;
    known |= (u32)__builtin_amdgcn_ballot_w64(pk != 0);
    if (WIPE0) wipe();
    const u32 pace = __builtin_amdgcn_raw_buffer_load_b32(__builtin_amdgcn_make_buffer_rsrc((void*)(myws + PACE_OFF + dl * 128), 0, it ? 32 : 0, 0x00020000), voff_g >> 2, 0, AUX_AGENT);
    pk = peek(it ? -1 : since(tstart));
    u32 v = 0;
    if (!sure && hole(a, 0, 8)) {
      bool follow = false;
      if (!(known >> it & 1)) {
        __builtin_amdgcn_s_waitcnt(0x0f70);
        const bool tail = (off / TILE_B & 31) >= (isr ? TAIL_R : TAIL_L);
        const int gate = __builtin_amdgcn_readlane(jg, it) - (tail ? TAIL_T : 0);
        while ((now = since(tstart)) < gate) __builtin_amdgcn_s_sleep(1);
        if (now < gate + LONG_DUE) {
          const u32 po = off + (tail ? PROBE_C * 1024 : 7168);
          follow = tail;
          while (!sure) {
            asm volatile("buffer_load_dword %0, off, %1, %2 offset:1008 sc0 sc1\n\ts_waitcnt vmcnt(%3)" : "+v"(v) : "s"(rxg), "s"(po), "n"(PROBE_D - 1) : "memory");
            if (!tail) asm volatile("s_waitcnt vmcnt(0)" : "+v"(v));
            if (__builtin_amdgcn_readfirstlane(v)) break;
            slow(f, f, true);
          }
        }
      }
#pragma unroll
      for (int i = 0; i < 8; ++i) {
        if (i == 4 && FOL_4 && follow) __builtin_amdgcn_s_sleep(FOL_4);
        if (i == 6 && FOL_6 && follow) __builtin_amdgcn_s_sleep(FOL_6);
        a[i] = __builtin_amdgcn_raw_buffer_load_b128(rxg, voff_g + (i & 3) * 1024, off + (i >> 2) * 4096, AUX_SYS);
        __builtin_amdgcn_sched_barrier(0);
      }
    }
#pragma unroll
    for (int ks = 0; ks < 4; ++ks) {
      while (!sure && hole(a, 2 * ks, 2 * ks + 2)) {
        slow(f, f, false);
        refill(a, 2 * ks, off);
        __builtin_amdgcn_s_waitcnt(0x0f70);
      }
      if (ks == 3) STAMPJ(2 + 2 * it, since(tstart));
      mul(a, ks);
      if (ks == 0) asm volatile("" ::"v"(v));
    }
    share();
    known |= 1u;
    drop(a);
    int due = 0;
    if (it == 0) {
#pragma unroll
      for (int i = 0; i < 8; ++i) a[i] = *park(8 + i);
    } else if (isr) {
      due = O_T0 + (set * 2 + half) * O_DT / 16 + origin(pace, tstart, 50);
      obase = (char*)outp;
      omark = 0;
      voff_o = voff_out(NS * 4, g * 256);
    }
    finish(gather());
    if (due) while (since(tstart) < due) __builtin_amdgcn_s_sleep(1);
    put(true);
    if (!WIPE0) wipe();
    __builtin_amdgcn_sched_barrier(0);
    __builtin_amdgcn_s_waitcnt(0x0f72 - WIPE0);
    ++it;
  };
  auto run = [&](auto first_wave) INL {
#pragma unroll
    for (int i = 0; i < 8; ++i) a0[i] = *park(i);
    pk = 0;
    __builtin_amdgcn_s_waitcnt(0x0f70);
    fetch(a1, 2);
    for (;;) {
      solo(a0);
      if (it == len) break;
      do { step(first_wave, a0, a1, tv[0]); step(first_wave, a1, a0, tv[1]); } while (it < len - 1);
    }
  };
  int second = wv >> 2;
  if (second) run(std::false_type{});
  asm volatile("" : "+s"(second));
  if (!second) run(std::true_type{});
  if (!SENT && isr) {
    asm volatile("s_waitcnt vmcnt(0)" ::: "memory");
    if (lane == 0 && __hip_atomic_fetch_add(&turn[0], 1u, __ATOMIC_RELAXED, __HIP_MEMORY_SCOPE_WORKGROUP) == 7)
      __hip_atomic_store((gu32*)(lws + OFL_OFF) + ((dl - 4) * 32 + set * 2 + half) * 32, call, __ATOMIC_RELAXED, __HIP_MEMORY_SCOPE_SYSTEM);
    STAMPJ(60, since(tstart));
  }
}

void build_jobs(u64 jobs[NSETS][2]) {
  unsigned char lst[2][256];
  int n[2] = {0, LPAD};
  for (int i = 0; i < 256; ++i) lst[0][i] = lst[1][i] = 0xff;
  auto add = [&](int k, int d, int u) { lst[k][n[k]++] = (unsigned char)(d << 5 | u); if (k == 0 && n[0] == NST) n[0] += RPAD; };
  for (int u = 0, v = 0; u < NH; ++u) {
    for (; v < NH && SELF_T0 + (v + 1) * SELF_DT <= PUSH_T1 + (u + 1) * PUSH_DT; ++v) add(0, 0, v);
    for (int d = 1; d < 4; ++d) add(0, d, u);
    if (u < NUC && !is_w(u) && cross_tile(u) >= NUI) for (int d = 5; d < 8; ++d) add(0, d, u);
  }
  unsigned char ex[64];
  int ne = 0, ie = 0;
  for (int u = 0; u < 2 * NUC; ++u) ex[ne++] = (unsigned char)u;
  for (int u = 0; u < NUC; ++u) if (!is_w(u)) for (int d = 4; d < 8; ++d) ex[ne++] = (unsigned char)(d << 5 | u);
  for (int u = 2 * NUC; u < NT; ++u) ex[ne++] = (unsigned char)u;
  for (int u = 0; u < NUI; ++u) {
    for (int d = 1; d < 4; ++d) add(1, d, u);
    for (int k = 0; k < (u < NUC ? 2 : 4) && ie < ne; ++k) lst[1][n[1]++] = ex[ie++];
  }
  for (int s = 0; s < NSETS; ++s) {
    const int k = s >= NRS, ns = k ? NLS : NRS, s0 = k ? s - NRS : s;
    jobs[s][0] = jobs[s][1] = ~0ull;
    for (int j = 0; j < 16 && s0 + j * ns < n[k]; ++j) ((unsigned char*)jobs[s])[j] = lst[k][s0 + j * ns];
  }
}
static_assert(RLEN <= 16 && LLEN <= 16 && NRS <= 16 && 2 * NUC <= NT && 64 + NST <= 2 * NRS * RLEN, "job lists");

}

extern "C" void kernel_launch(int rank, int nranks, ncclComm_t comm, hipStream_t stream,
                              void* const* d_in, const int* in_sizes, int n_in,
                              void* d_out, int out_size, const int* in_dtypes, int out_dtype,
                              void* const* d_out_all, void* const* d_ws_all, size_t ws_bytes) {
  (void)comm; (void)in_dtypes; (void)out_dtype;
  if (nranks != NR || (unsigned)rank >= (unsigned)NR || n_in != 4 || ws_bytes < WS_NEED ||
      in_sizes[0] != (int)SHARD || in_sizes[1] != (int)SHARD || out_size != NR * MS * NS) {
    fprintf(stderr, "kernel_launch: unexpected problem shape\n");
    return;
  }
  Params p;
  p.x = (const unsigned char*)d_in[0];
  p.w = (const unsigned char*)d_in[1];
  p.sx = (const float*)d_in[2];
  p.sw = (const float*)d_in[3];
  p.out = (float*)d_out;
  for (int d = 0; d < NR; ++d) p.ws[d] = (char*)d_ws_all[rank ^ d];
  for (int i = 0; i < 4; ++i) p.outp[i] = (float*)d_out_all[rank ^ (4 + i)];
  p.rank = rank;
  p.pad = 0;
  build_jobs(p.jobs);
  hipLaunchKernelGGL(ag_gemm_kernel, dim3(256), dim3(512), 0, stream, p);
}
```

```cpp
#include <hip/hip_runtime.h>
#include <rccl/rccl.h>
#include <cstdio>
#include <type_traits>

typedef unsigned int u32;
typedef unsigned long long u64;
typedef u32 u32x4 __attribute__((ext_vector_type(4)));
typedef u32 u32x2 __attribute__((ext_vector_type(2)));
typedef int i32x8 __attribute__((ext_vector_type(8)));
typedef float f32x4 __attribute__((ext_vector_type(4)));
typedef __attribute__((address_space(1))) u32 gu32;

namespace {

constexpr int NR = 8;
constexpr int KD = 4096;
constexpr int MS = 512;
constexpr int NS = 512;
constexpr u32 SHARD = (u32)MS * KD;
constexpr u32 TILE_B = 16 * KD;
constexpr int NT = MS / 16;
#define NH 26
#define ND 2
#define NA 4
static_assert(NH + ND + NA == NT, "tile kinds");
constexpr int NUI = NH + ND;
constexpr int NUC = 8 + NA + ND;
#define NPRE 1
#define NRS 12
constexpr int NSETS = 24, NLS = NSETS - NRS;
constexpr int NRJ = 4 * NH + 3 * NA, NLJ = NT + 3 * NUI + 4 * (NA + ND);
constexpr int RLEN = (NRJ + 2 * NRS - 1) / (2 * NRS) * 2, LLEN = (NLJ + 2 * NLS - 1) / (2 * NLS) * 2;
constexpr int RPAD = RLEN * NRS - NRJ, LPAD = LLEN * NLS - NLJ;
constexpr int NST = NRJ - NRS;
static_assert(NUC * 8 + NST <= NT * 8, "staging");
constexpr u32 BUF_B = (u32)NR * SHARD;
constexpr int NBUF = 3;
constexpr size_t FLAG_OFF = (size_t)NBUF * BUF_B;
constexpr size_t FLAG_B = (size_t)NR * 8 * 128;
constexpr size_t OFL_OFF = FLAG_OFF + FLAG_B;
constexpr size_t OFL_B = 4 * 4096;
constexpr size_t CNT_OFF = OFL_OFF + OFL_B;
constexpr size_t CNT_B = 256 * 8 * 4;
constexpr size_t TMO_OFF = CNT_OFF + CNT_B;
constexpr size_t PACE_OFF = TMO_OFF + 4096;
constexpr size_t DBG_OFF = PACE_OFF + 4096;
constexpr size_t DBG_B = 256 * 128 * 8;
constexpr size_t WS_NEED = DBG_OFF + DBG_B;
constexpr u64 DEADLINE = 100000000ull;
#define PUSH_DT 110
#define PUSH_T1 45
#define I_DT PUSH_DT
#define I_T1 PUSH_T1
#define BONUS_U 2
#define BONUS_SH 5
#define BONUS_MAX 1
#define PACE_U 20
#define O_M 5
#define PUSH_G 2
#define PUSH_PRE 500
#define ST_PRE 100
#define ST_PROBE 0
#define ST_SLEEP 4
#define PUSH_LEADT 8
#define NLEAD 2
#define PUSH_DT0 55
#define W_DELAY 150
#define W_DT 100
#define W_LAG 0
#define W_SLOT 50
#define GATE 80
#define R_GATE 140
#ifndef LONG_DUE
#define LONG_DUE 150
#endif
#define O_T0 (PUSH_T1 + NUC * PUSH_DT + NST * PUSH_DT / 8 + O_M)
#define O_DT 113
#define SELF_T0 200
#define SELF_DT 50
#define TAIL_R (NH - 1)
#define TAIL_L (NUI - 1)
#define TAIL_T 140
#define PROBE_C 2
#define PROBE_D 3
#define PEEK_LA 25
#define POLL_SLEEP 4
#define FAUX 0
#define WAIT_D 6
#define WAIT_S 3
#define SENT 1
#define SN_G 60
#define SN_L 130
#define OAUX 17
#define WIPE0 1
#define FAST0 1
#define W1_SLEEP 0
#define BONUS0 0
#define WSPLIT 1
#define FOL_4 0
#define FOL_6 4
#define PWARM 1
#define AD_T 95
#define ST_TURN 14
#define SN_D 3
#define SN_S 4
#define SN_W 0
#define PUT1 1
#define PEEK0 1
#define AUX_SYS 17
#define AUX_AGENT 16
constexpr int TR_ROW = 528;
constexpr int TR_B = 16 * TR_ROW;

struct Params {
  char* ws[NR];
  float* outp[4];
  const unsigned char* x;
  const unsigned char* w;
  const float* sx;
  const float* sw;
  float* out;
  int rank, pad;
  u64 jobs[NSETS][2];
};

__host__ __device__ constexpr bool is_w(int u) { return u >= NPRE && u < NPRE + 8; }
__host__ __device__ constexpr int cross_tile(int u) { u = u < NPRE ? u : u - 8; return u < NA ? NUI + u : NH + u - NA; }
__device__ __forceinline__ void job_of(u32 ent, bool isr, int dl, int& d, int& u) {
  u = ent & 31;
  d = isr && ent >= 128 ? dl ^ ((ent >> 5) - 4) : ent >> 5;
}
__device__ __forceinline__ u32 row_of(int rank, int d, int u) { return (u32)((rank ^ d) * MS + (d < 4 ? u : cross_tile(u)) * 16); }
constexpr u32 OOB = 0x80000000u;

#define STAMP(i) do {} while (0)
#define STAMPV(i, v) do {} while (0)
#define STAMPL(i) do {} while (0)
#define STAMPJ(i, v) do {} while (0)

#define INL __attribute__((always_inline))
__device__ __forceinline__ int since(u64 t) { return (int)(__builtin_amdgcn_s_memrealtime() - t); }

__device__ __forceinline__ int origin(u32 v, u64 tstart, int dflt) {
  int sum = 0;
  bool ok = true;
#pragma unroll
  for (int k = 0; k < 8; ++k) {
    const int r = (int)(__builtin_amdgcn_readlane(v, k) - (u32)tstart);
    ok = ok && r > -100 && r < 1000;
    sum += r;
  }
  return ok ? sum >> 3 : dflt;
}

__device__ __forceinline__ void transpose4(u32 d0, u32 d1, u32 d2, u32 d3, u32& c0, u32& c1, u32& c2, u32& c3) {
  const u32 t0 = __builtin_amdgcn_perm(d1, d0, 0x05010400u), t1 = __builtin_amdgcn_perm(d1, d0, 0x07030602u);
  const u32 t2 = __builtin_amdgcn_perm(d3, d2, 0x05010400u), t3 = __builtin_amdgcn_perm(d3, d2, 0x07030602u);
  c0 = __builtin_amdgcn_perm(t2, t0, 0x05040100u);
  c1 = __builtin_amdgcn_perm(t2, t0, 0x07060302u);
  c2 = __builtin_amdgcn_perm(t3, t1, 0x05040100u);
  c3 = __builtin_amdgcn_perm(t3, t1, 0x07060302u);
}

__global__ void __launch_bounds__(512, 2) ag_gemm_kernel(Params p) {
  __shared__ __attribute__((aligned(16))) char smem[2 * TILE_B];
  __shared__ u32 turn[4];
  const int lane = threadIdx.x & 63;
  const int wv = __builtin_amdgcn_readfirstlane(threadIdx.x >> 6);
  const int xg = blockIdx.x & 7;
  const int ci = blockIdx.x >> 3;
  int rank = p.rank;
  char* myws = p.ws[0];
  char* pws = p.ws[(ci + 1) & 7];
  const unsigned char* xp = p.x;
  const unsigned char* wp = p.w;
  const int d = (ci + 1) & 7;
  const int kb = (xg + wv) & 7;
  u32 so, sst, vo;
  auto source = [&](int u) INL {
    const bool w = d >= 4 && is_w(u);
    u32 sw_ = (u32)(kb * 512 + (u - NPRE) * 64) * NS + (d & 3) * 128, sx_ = (d < 4 ? u : cross_tile(u)) * TILE_B + kb * 512;
    asm("" : "+s"(sw_), "+s"(sx_));
    so = w ? sw_ : sx_;
    sst = w ? 8 * NS : 2 * KD;
    const int sh = w ? 3 : 5;
    vo = (u32)(lane >> sh) * (sst >> (6 - sh)) + (lane & ((1 << sh) - 1)) * 16;
  };
  __builtin_assume(wv < 8);
  source(wv);
  static_assert(NPRE > 0, "the first unit is a tile of x");
  sst = wv || !d ? sst : 64;
  vo = wv || !d ? vo : (lane & 15) * KD + (lane >> 4) * 16;
  asm volatile("" : "+s"(so), "+s"(sst));
  asm volatile("" : "+v"(vo));
  asm volatile("" : "+s"(rank), "+s"(myws), "+s"(pws), "+s"(xp), "+s"(wp));
  const int m = lane & 15, q = lane >> 4;
  const u64 tstart = __builtin_amdgcn_s_memrealtime();

  gu32* cntp = (gu32*)(myws + CNT_OFF) + blockIdx.x * 8 + wv;
  auto rw = __builtin_amdgcn_make_buffer_rsrc((void*)wp, 0, (int)SHARD, 0x00020000);

  if (__builtin_expect(ci < 8, 1)) {
    auto due_of = [&](int u8) INL {
      return d == 0 ? SELF_T0 + u8 * SELF_DT / 8 : d < 4 ? I_T1 + u8 * I_DT / 8 : PUSH_T1 + u8 * PUSH_DT / 8;
    };
    auto until = [&](int t) INL { while (since(tstart) < t) __builtin_amdgcn_s_sleep(2); };
    u32x4 e[8];
    auto load = [&]() INL {
      auto r = __builtin_amdgcn_make_buffer_rsrc((void*)(sst == 8 * NS ? wp : xp), 0, (int)SHARD, 0x00020000);
#pragma unroll
      for (int j = 0; j < 8; ++j) e[j] = __builtin_amdgcn_raw_buffer_load_b128(r, vo, so + j * sst, 0);
    };
    auto rout = [&]() INL { return __builtin_amdgcn_make_buffer_rsrc((void*)p.out, 0, NR * MS * NS * 4, 0x00020000); };
    auto direct = [&](int s) INL {
      int jd, ju;
      job_of(((const u32*)p.jobs[s])[(RLEN - 1) >> 2] >> ((RLEN - 1) & 3) * 8 & 0xff, true, d, jd, ju);
      return (row_of(rank ^ d, jd, ju) + xg * 2) * (NS * 4) + (d & 3) * 512;
    };
    const u32 ovo = (lane >> 5) * (NS * 4) + (lane & 31) * 16;
    if (__builtin_expect(wv >= NLEAD, 0)) {
      __hip_atomic_store(&turn[lane & 3], 0u, __ATOMIC_RELAXED, __HIP_MEMORY_SCOPE_WORKGROUP);
      asm volatile("s_waitcnt lgkmcnt(0)" ::: "memory");
      __builtin_amdgcn_s_barrier();
      if (wv == 2) {
        u32 t = __builtin_amdgcn_raw_buffer_load_b32(__builtin_amdgcn_make_buffer_rsrc((void*)(pws + d * SHARD + (SHARD - 128)), 0, NBUF * BUF_B, 0x00020000), lane < NBUF ? lane * BUF_B : OOB, 0, AUX_SYS);
        asm volatile("" :: "v"(t));
      }
      static_assert(is_w(7) && NLEAD <= 7, "the marks are in place before the peer has all of w");
      if (wv == 7 && d >= 4)
#pragma unroll 1
        for (int s = 0; s < NRS; ++s) __builtin_amdgcn_raw_buffer_store_b128(u32x4{~0u, ~0u, ~0u, ~0u}, rout(), ovo + direct(s), 0, AUX_SYS);
      until(wv < PUSH_LEADT ? wv * PUSH_DT0 : due_of(wv * 8) - PUSH_PRE);
    }
    u32 cs;
    asm volatile("s_load_dword %0, %1, 0x0" : "=s"(cs) : "s"(cntp) : "memory");
    if (W1_SLEEP && wv == 1) __builtin_amdgcn_s_sleep(W1_SLEEP);
    load();
    __builtin_amdgcn_sched_barrier(0);
    if (wv < NLEAD) __builtin_amdgcn_s_barrier();
    const int nx = d == 0 ? NT : d < 4 ? NUI : NUC;
    const int nu = d < 4 ? nx : NUC + (NST + 7 - kb) / 8;
    char* tw = smem + wv * TR_B + (lane >> 5) * TR_ROW + (lane & 31) * 16;
    const char* tr = smem + wv * TR_B + m * TR_ROW + q * 16;
    const u32 soff = lane * 16;
    int u = wv, tz = 0;
    auto acks = [&](int bonus) INL {
#pragma unroll
      for (int k = 7; k >= 0; --k) {
        asm volatile("s_waitcnt vmcnt(%0)" ::"n"(k) : "memory");
        __hip_atomic_fetch_max(&turn[0], (u32)(u * 8 + 8 - k + bonus), __ATOMIC_RELAXED, __HIP_MEMORY_SCOPE_WORKGROUP);
      }
    };
    auto next = [&]() INL {
      source(u);
      until(due_of(u * 8) - PUSH_PRE);
      load();
    };
    asm volatile("s_waitcnt lgkmcnt(0)" : "+s"(cs) :: "memory");
    const u32 call = cs + 1;
    const u32 boff = call % NBUF * BUF_B + d * SHARD + kb * 8192;
    if (__builtin_expect(sst == 64, 1)) {
      auto rd = __builtin_amdgcn_make_buffer_rsrc((void*)(pws + boff), 0, 8192, 0x00020000);
#pragma unroll
      for (int i = 0; i < 8; ++i) {
        e[i][0] = ~e[i][0];
        __builtin_amdgcn_raw_buffer_store_b128(e[i], rd, soff + i * 1024, 0, AUX_SYS);
        __builtin_amdgcn_sched_barrier(0);
        if (i == 0) STAMPL(0);
        if (i == 7) STAMPL(5);
        __builtin_amdgcn_sched_barrier(0);
      }
      const int t = since(tstart) - AD_T;
      __hip_atomic_store(&turn[3], (u32)(t > 0 ? t : 0), __ATOMIC_RELAXED, __HIP_MEMORY_SCOPE_WORKGROUP);
      acks(BONUS0);
      u = 8;
      next();
    }
    char* to = nullptr;
    u32 dvo = soff, dst = 1024;
#pragma unroll 1
    for (;;) {
      int bonus = 0;
      u32 kind = __builtin_amdgcn_readfirstlane(sst);
      asm volatile("" : "+s"(kind));
      if (kind == 2 * KD) {
#pragma unroll
        for (int j = 0; j < 8; ++j) *(u32x4*)(tw + j * 2 * TR_ROW) = e[j];
#pragma unroll
        for (int i = 0; i < 8; ++i) e[i] = *(const u32x4*)(tr + i * 64);
      }
      asm volatile("" : "+s"(kind));
      if (kind == 8 * NS) {
#pragma unroll
        for (int j = 0; j < 8; ++j) *(u32x4*)(smem + wv * TR_B + lane * 16 + j * 1024) = e[j];
        const u32* wr = (const u32*)(smem + wv * TR_B + q * 2048 + m * 4);
#pragma unroll
        for (int j = 0; j < 8; ++j) {
          const u32* w4 = wr + (j >> 2) * 16 + (j & 3) * 128;
          u32 c0, c1, c2, c3;
          transpose4(w4[0], w4[32], w4[64], w4[96], c0, c1, c2, c3);
          e[(j & 4) + 0][j & 3] = c0; e[(j & 4) + 1][j & 3] = c1; e[(j & 4) + 2][j & 3] = c2; e[(j & 4) + 3][j & 3] = c3;
        }
      }
#pragma unroll
      for (int i = 0; i < 8; ++i) e[i][0] = ~e[i][0];
      if (u < nx) to = pws + (boff + u * TILE_B);
      auto rd = __builtin_amdgcn_make_buffer_rsrc((void*)to, 0, u < nx ? 8192 : 16 * NS * 4, 0x00020000);
#pragma unroll
      for (int i = 0; i < 8; ++i) {
        if (i == 7) { if (!bonus) asm volatile("s_waitcnt vmcnt(6)" ::: "memory"); }
        else if (u > 0 && d) {
          while ((int)__hip_atomic_load(&turn[0], __ATOMIC_RELAXED, __HIP_MEMORY_SCOPE_WORKGROUP) < u * 8 + i - 6) __builtin_amdgcn_s_sleep(1);
        }
        if (i % PUSH_G == 0) {
          if (AD_T && i == 0) tz = __builtin_amdgcn_readfirstlane(__hip_atomic_load(&turn[3], __ATOMIC_RELAXED, __HIP_MEMORY_SCOPE_WORKGROUP));
          const int due = due_of(u * 8 + i) + tz;
          int t;
          while ((t = since(tstart)) < due) __builtin_amdgcn_s_sleep(1);
          if (i == 6 && u >= BONUS_U) { bonus = (t - due) >> BONUS_SH; bonus = bonus > BONUS_MAX ? BONUS_MAX : bonus; }
          if (i == 0 && u == PACE_U)
            __builtin_amdgcn_raw_buffer_store_b32((u32)tstart + (u32)(t - due + tz), __builtin_amdgcn_make_buffer_rsrc((void*)(myws + PACE_OFF + d * 128 + xg * 4), 0, 4, 0x00020000), soff, 0, AUX_AGENT);
        }
        if (i == 0) STAMPL(u >> 3);
        __builtin_amdgcn_raw_buffer_store_b128(e[i], rd, dvo + i * dst, 0, AUX_SYS);
        __builtin_amdgcn_sched_barrier(0);
      }
      acks(bonus);
      if (u < nx && u + 8 >= nx) {
        u32 done = 0;
        if (lane == 0) done = __hip_atomic_fetch_add(&turn[1], 1u, __ATOMIC_RELAXED, __HIP_MEMORY_SCOPE_WORKGROUP);
        if (__builtin_amdgcn_readfirstlane(done) == 7)
          __builtin_amdgcn_raw_buffer_store_b32(call, __builtin_amdgcn_make_buffer_rsrc((void*)(pws + FLAG_OFF + (d * 8 + xg) * 128), 0, 4, 0x00020000), soff, 0, AUX_SYS);
      }
      u += 8;
      if (u >= nu) break;
      if (u < nx) {
        next();
      } else {
        sst = 0;
        dvo = ovo;
        dst = 2 * NS * 4;
        const int n = (u - NUC) * 8 + kb, j = n / NRS;
        int jd, ju;
        job_of(((const u32*)p.jobs[n % NRS])[j >> 2] >> (j & 3) * 8 & 0xff, true, d, jd, ju);
        to = (char*)p.outp[d - 4] + (row_of(rank, jd, ju) * (NS * 4) + (d & 3) * 512);
        auto rs = __builtin_amdgcn_make_buffer_rsrc((void*)(myws + (boff + u * TILE_B)), 0, 8192, 0x00020000);
        until(due_of(u * 8) + tz - ST_PRE);
        if (ST_TURN) while ((int)__hip_atomic_load(&turn[0], __ATOMIC_RELAXED, __HIP_MEMORY_SCOPE_WORKGROUP) < u * 8 - ST_TURN) __builtin_amdgcn_s_sleep(2);
        for (;;) {
#pragma unroll
          for (int j = 0; j < 8; ++j) e[j] = __builtin_amdgcn_raw_buffer_load_b128(rs, soff + j * 1024, 0, AUX_SYS);
          u32 mn = e[0][0];
#pragma unroll
          for (int j = 1; j < 8; ++j) mn = e[j][0] < mn ? e[j][0] : mn;
          if (__builtin_amdgcn_ballot_w64(mn == 0) == 0 || __builtin_amdgcn_s_memrealtime() - tstart > DEADLINE) break;
          __builtin_amdgcn_s_sleep(ST_SLEEP);
        }
      }
    }
    STAMPL(4);
    if (lane == 0) {
      if (d >= 4 && __hip_atomic_fetch_add(&turn[2], 1u, __ATOMIC_RELAXED, __HIP_MEMORY_SCOPE_WORKGROUP) == 7)
        __hip_atomic_store((gu32*)(pws + OFL_OFF) + ((d - 4) * 32 + 2 * NRS + xg) * 32, call, __ATOMIC_RELAXED, __HIP_MEMORY_SCOPE_SYSTEM);
      __hip_atomic_store(cntp, call, __ATOMIC_RELAXED, __HIP_MEMORY_SCOPE_AGENT);
    }
    if (d >= 4) {
      const bool two = wv + NRS - 8 >= 8;
      auto ro = rout();
      const u32 oa = direct(wv), ob = direct(two ? wv + NRS - 8 : wv);
      until(O_T0 + tz + (two ? SN_G : SN_L) + wv * O_DT / 8);
      u32 v = two ? ~0u : 0u, n = 0;
      auto expired = [&]() INL { return (++n & 63u) == 0 && __builtin_amdgcn_s_memrealtime() - tstart > DEADLINE; };
      while (__builtin_amdgcn_readfirstlane(v) == ~0u && !expired()) {
        asm volatile("buffer_load_dword %0, off, %1, %2 offset:256 sc0 sc1\n\ts_waitcnt vmcnt(%3)" : "+v"(v) : "s"(ro), "s"(oa), "n"(SN_D - 1) : "memory");
        if (__builtin_amdgcn_readfirstlane(v) != ~0u) break;
        asm volatile("s_sleep %1" : "+v"(v) : "n"(SN_S));
      }
      if (SN_W) __builtin_amdgcn_s_sleep(SN_W);
      for (;;) {
        const u32 b = __builtin_amdgcn_raw_buffer_load_b32(ro, ovo, ob, AUX_SYS), a = __builtin_amdgcn_raw_buffer_load_b32(ro, ovo, oa, AUX_SYS);
        if (__builtin_amdgcn_ballot_w64(a == ~0u || b == ~0u) == 0 || expired()) break;
      }
      asm volatile("" :: "v"(v));
    }
    {
      if (d >= 4 && xg == 0 && wv == 0) {
        const char* fp = myws + OFL_OFF + (d - 4) * 4096 + (SENT ? 2 * NRS + (lane & 7) : lane < 2 * NRS + 8 ? lane : 0) * 128;
        while (since(tstart) < O_T0) __builtin_amdgcn_s_sleep(4);
        u32 v = 0, n = 0;
        for (;;) {
          asm volatile("global_load_dword %0, %1, off sc0 sc1\n\ts_waitcnt vmcnt(%2)" : "+v"(v) : "v"(fp), "n"(WAIT_D - 1) : "memory");
          if (__builtin_amdgcn_ballot_w64(v < call) == 0) break;
          asm volatile("s_sleep %1" : "+v"(v) : "n"(WAIT_S));
          if (__builtin_amdgcn_ballot_w64(v < call) == 0) break;
          if ((++n & 255u) == 0 && __builtin_amdgcn_s_memrealtime() - tstart > DEADLINE) {
            if (lane == 0) p.out[0] = __builtin_nanf("");
            break;
          }
        }
        asm volatile("s_waitcnt vmcnt(0)" : "+v"(v) :: "memory");
        STAMP(1);
      }
    }
  }
  int role = ci;
  asm volatile("" : "+s"(role));
  if (role < 8) return;

  STAMP(0);
  u32 call = __hip_atomic_load(cntp, __ATOMIC_RELAXED, __HIP_MEMORY_SCOPE_AGENT) + 1;
  gu32* tmo = (gu32*)(myws + TMO_OFF);
  const int g = ci & 7;
  const int sl = (ci - 8) >> 3;
  const int set = sl == 0 ? xg : sl == 1 ? 16 + xg : 8 + (xg >> 1) + ((xg ^ g) & 1) * 4;
  const bool isr = set < NRS;
  const int dl = 4 + (g >> 1), half = g & 1;
  const int len = isr ? RLEN : LLEN;
  u64 jlo = p.jobs[set][0], jhi = p.jobs[set][1];
  float* outp = isr ? p.outp[dl - 4] : p.out;
  char* lws = p.ws[dl];
  float scale = p.sx[0] * p.sw[0];
  asm volatile("" : "+s"(jlo), "+s"(jhi), "+s"(outp), "+s"(lws));
  asm volatile("" : "+v"(scale));
  if (!FAST0 && threadIdx.x == 0) __hip_atomic_store(&turn[0], 0u, __ATOMIC_RELAXED, __HIP_MEMORY_SCOPE_WORKGROUP);
  gu32* flags = (gu32*)(myws + FLAG_OFF);
  const u32 voff_g = lane * 16;

  bool sure;
  u32 n = 0;
  auto word = [&](gu32* a) INL { return __builtin_amdgcn_readfirstlane(__hip_atomic_load(a, __ATOMIC_RELAXED, __HIP_MEMORY_SCOPE_SYSTEM)); };
  auto slow = [&](gu32* f, gu32* f2, bool nap) INL {
    if ((++n & 31u) == 0) {
      if (word(f) >= call && word(f2) >= call) { sure = true; return; }
      if (word(tmo) == call || __builtin_amdgcn_s_memrealtime() - tstart > DEADLINE) {
        __hip_atomic_store(tmo, call, __ATOMIC_RELAXED, __HIP_MEMORY_SCOPE_SYSTEM);
        sure = true;
        return;
      }
    }
    if (nap) __builtin_amdgcn_s_sleep(POLL_SLEEP); else __builtin_amdgcn_s_sleep(1);
  };

  call = __builtin_amdgcn_readfirstlane(call);
  if (lane == 0) __hip_atomic_store(cntp, call, __ATOMIC_RELAXED, __HIP_MEMORY_SCOPE_AGENT);
  char* gbuf = myws + (size_t)(call % NBUF) * BUF_B + wv * 8192;
  auto rxg = __builtin_amdgcn_make_buffer_rsrc((void*)gbuf, 0, (int)BUF_B, 0x00020000);
  auto rzr = __builtin_amdgcn_make_buffer_rsrc((void*)(myws + (size_t)((call + NBUF - 1) % NBUF) * BUF_B), 0, (int)BUF_B, 0x00020000);
  char* obase = isr ? myws + (size_t)(call % NBUF) * BUF_B + (size_t)dl * SHARD + NUC * TILE_B : (char*)outp;
  u32 omark = isr ? ~0u : 0u;
  auto voff_out = [&](u32 pitch, u32 col) INL { return (wv * 2 + ((voff_g >> 8) & 1)) * pitch + col + (voff_g & 255) + (voff_g < 512 ? 0 : OOB); };
  u32 voff_o = isr ? voff_out(512, half * 256) : voff_out(NS * 4, g * 256);
  const u32 voff_z = (isr ? 0u : (u32)g * 8192) + threadIdx.x * 16;
  const u32 voff_t = lane < (sl == 2 ? 16 : 8) ? (sl == 2 ? g & 6 : g) * 1024 + lane * 128 : OOB;

  u32 jt, jo;
  int jg;
  {
    const u32 ent = lane < 16 ? (u32)((lane < 8 ? jlo : jhi) >> (lane & 7) * 8) & 0xff : 0xff;
    int d, u;
    job_of(ent, isr, dl, d, u);
    jt = ent == 0xff ? OOB : d * SHARD + u * TILE_B;
    jg = ent == 0xff ? 0x7fffffff : (d >= 4 ? PUSH_T1 + (u + 1) * PUSH_DT : d ? I_T1 + (u + 1) * I_DT : SELF_T0 + (u + 1) * SELF_DT) + (isr && u < TAIL_R ? R_GATE : GATE);
    jo = ent == 0xff ? OOB : isr && lane < RLEN - 1 ? (u32)(lane * NRS + set) * 8192 : row_of(rank, d, u) * (NS * 4);
  }
  u32 known = 0;
  u32 got = 0;
  u32 warm = 2;
  int now;

  auto probe = [&](u32 off, int chunk) INL { return __builtin_amdgcn_raw_buffer_load_b32(rxg, 1008, off + chunk * 1024, AUX_SYS); };
  auto rsrc_if = [&](bool on) INL { return __builtin_amdgcn_make_buffer_rsrc((void*)gbuf, 0, on ? (int)BUF_B : 0, 0x00020000); };
  auto fetch = [&](u32x4 (&a)[8], int j) INL {
    auto r = rsrc_if(known >> j & 1);
    got |= known & 1u << j;
    const u32 off = __builtin_amdgcn_readlane(jt, j);
#pragma unroll
    for (int i = 0; i < 8; ++i) a[i] = __builtin_amdgcn_raw_buffer_load_b128(r, voff_g + (i & 3) * 1024, off + (i >> 2) * 4096, FAUX);
  };
  auto touch = [&](int j) INL {
    j += __builtin_ctz(((known & ~warm) | 1u << 31) >> j);
    warm |= 1u << (j & 15);
    return __builtin_amdgcn_raw_buffer_load_b32(rsrc_if(j < 16), voff_t, __builtin_amdgcn_readlane(jt, j), AUX_SYS);
  };
  auto peek = [&](int t) INL {
    const bool on = jg <= t && !(known >> (lane & 31) & 1);
    return __builtin_amdgcn_raw_buffer_load_b32(rxg, on ? jt : OOB, 8176, AUX_SYS);
  };
  u32 pk, tv[4];
  u32x4 a0[8], a1[8];
  auto park = [&](int i) INL { return (u32x4*)(smem + (i >> 2) * 32768 + wv * 4096 + (i & 3) * 1024 + voff_g); };
  auto early = [&](int phase) INL {
    if (phase) known |= (u32)__builtin_amdgcn_ballot_w64(pk != 0);
    if (phase < 3) pk = peek(since(tstart));
    if (phase == 1 || phase == 2) fetch(a1, phase == 2 ? 1 : isr ? 0 : 31);
#pragma unroll
    for (int k = 0; k < 4; ++k) {
      if (phase == 2) tv[k] = touch(isr ? 2 : 0);
      if (phase == 3) asm volatile("" ::"v"(tv[k]));
    }
    if (phase == 1 || phase == 2)
#pragma unroll
      for (int i = 0; i < 8; ++i) *park(phase * 8 - 8 + i) = a1[i];
  };

  i32x8 bf[4][4];
  if (!isr) {
    const u32 voff = (u32)(wv * 512 + q * 16) * NS + g * 64 + m * 4;
#pragma unroll
    for (int ks = 0; ks < 4; ++ks)
#pragma unroll
      for (int h = 0; h < 2; ++h) {
        const int due = W_DELAY + sl * W_SLOT + (ks * 2 + h) * W_DT;
        while (since(tstart) < due) __builtin_amdgcn_s_sleep(4);
        if (ks * 2 + h >= 5) early(ks * 2 + h - 5);
#pragma unroll
        for (int r = 0; r < 4; ++r) {
          u32 d0 = __builtin_amdgcn_raw_buffer_load_b32(rw, voff + (r * 4 + 0) * NS, (ks * 128 + h * 64) * NS, 0);
          u32 d1 = __builtin_amdgcn_raw_buffer_load_b32(rw, voff + (r * 4 + 1) * NS, (ks * 128 + h * 64) * NS, 0);
          u32 d2 = __builtin_amdgcn_raw_buffer_load_b32(rw, voff + (r * 4 + 2) * NS, (ks * 128 + h * 64) * NS, 0);
          u32 d3 = __builtin_amdgcn_raw_buffer_load_b32(rw, voff + (r * 4 + 3) * NS, (ks * 128 + h * 64) * NS, 0);
          u32 c0, c1, c2, c3;
          transpose4(d0, d1, d2, d3, c0, c1, c2, c3);
          bf[0][ks][h * 4 + r] = (int)c0; bf[1][ks][h * 4 + r] = (int)c1; bf[2][ks][h * 4 + r] = (int)c2; bf[3][ks][h * 4 + r] = (int)c3;
        }
      }
  } else {
#pragma unroll
    for (int ks = 0; ks < 4; ++ks) {
      const int u = NPRE + ks * 2;
      const u32 off = dl * SHARD + u * TILE_B + half * 4096;
      gu32* f = flags + (dl * 8 + ((wv - u) & 7)) * 32;
      gu32* f2 = flags + (dl * 8 + ((wv - u - 1) & 7)) * 32;
      if (ks >= 1) early(ks - 1);
      while (since(tstart) < PUSH_T1 + (u + 2) * PUSH_DT - (1 - half) * PUSH_DT / 2 + (ks == 3 && W_LAG ? W_LAG : GATE)) __builtin_amdgcn_s_sleep(4);
      sure = false;
#pragma unroll
      for (int i = 0; i < 4; ++i) a1[i] = __builtin_amdgcn_raw_buffer_load_b128(rxg, voff_g + i * 1024, off, AUX_SYS);
      while (!sure && !(ks == 3 && W_LAG)) {
        if (__builtin_amdgcn_ballot_w64(probe(off + TILE_B, 3) == 0) == 0) break;
        slow(f, f2, true);
      }
      for (bool all = !WSPLIT || sure;; all = true) {
#pragma unroll
        for (int i = 0; i < 8; ++i)
          if (i >= 4 || all) a1[i] = __builtin_amdgcn_raw_buffer_load_b128(rxg, voff_g + (i & 3) * 1024, off + (i >> 2) * TILE_B, AUX_SYS);
        if (sure) break;
        u32 mn = a1[0][0];
#pragma unroll
        for (int i = 1; i < 8; ++i) mn = a1[i][0] < mn ? a1[i][0] : mn;
        if (__builtin_amdgcn_ballot_w64(mn == 0) == 0) break;
        slow(f, f2, true);
      }
#pragma unroll
      for (int i = 0; i < 8; ++i)
#pragma unroll
        for (int e = 0; e < 4; ++e) bf[i & 3][ks][(i >> 2) * 4 + e] = (int)(e ? a1[i][e] : ~a1[i][e]);
    }
  }
  early(3);

  int it = 0;
  auto hole = [&](u32x4 (&a)[8], int i0, int i1) INL {
    u32 mn = a[i0][0];
#pragma unroll
    for (int i = i0 + 1; i < i1; ++i) mn = a[i][0] < mn ? a[i][0] : mn;
    return __builtin_amdgcn_ballot_w64(mn == 0) != 0;
  };
  auto refill = [&](u32x4 (&a)[8], int i0, u32 off) INL {
#pragma unroll
    for (int i = i0; i < 8; ++i)
      if (hole(a, i, i + 1)) a[i] = __builtin_amdgcn_raw_buffer_load_b128(rxg, voff_g + (i & 3) * 1024, off + (i >> 2) * 4096, AUX_SYS);
  };
  auto drop = [&](u32x4 (&a)[8]) INL {
#pragma unroll
    for (int i = 0; i < 8; ++i) asm volatile("" : "=v"(a[i]));
  };
  f32x4 acc[4];
  auto mul = [&](u32x4 (&a)[8], int ks) INL {
    i32x8 af;
#pragma unroll
    for (int e = 0; e < 4; ++e) { af[e] = (int)a[2 * ks][e]; af[4 + e] = (int)a[2 * ks + 1][e]; }
    af[0] = ~af[0]; af[4] = ~af[4];
#pragma unroll
    for (int c = 0; c < 4; ++c)
      acc[c] = __builtin_amdgcn_mfma_scale_f32_16x16x128_f8f6f4(af, bf[c][ks], ks ? acc[c] : f32x4{0.f, 0.f, 0.f, 0.f}, 0, 1, 0, 0, 0, 0);
  };
  auto share = [&]() INL {
    float* rb = (float*)smem + (it & 1) * 8192;
#pragma unroll
    for (int j = 0; j < 4; ++j)
      *(f32x4*)(rb + wv * 1024 + (q * 4 + j) * 64 + m * 4) = f32x4{acc[0][j], acc[1][j], acc[2][j], acc[3][j]};
    __builtin_amdgcn_sched_barrier(0);
  };
  auto gather = [&]() INL {
    asm volatile("s_waitcnt lgkmcnt(0)" ::: "memory");
    __builtin_amdgcn_s_barrier();
    const float* rp = (float*)smem + (it & 1) * 8192 + (lane >> 5) * 4096 + wv * 128 + (lane & 31) * 4;
    f32x4 s4 = *(const f32x4*)rp;
#pragma unroll
    for (int v = 1; v < 4; ++v) s4 += *(const f32x4*)(rp + v * 1024);
    return s4;
  };
  u32 o[2];
  auto finish = [&](f32x4 s4) INL {
#pragma unroll
    for (int e = 0; e < 2; ++e) {
      auto sw = __builtin_amdgcn_permlane32_swap(__float_as_uint(s4[e]), __float_as_uint(s4[e + 2]), false, false);
      const float y = (__uint_as_float(sw[0]) + __uint_as_float(sw[1])) * scale;
      const float yc = __builtin_fminf(__builtin_fmaxf(y, -60.f), 60.f);
      o[e] = __float_as_uint(y * __builtin_amdgcn_rcpf(1.f + __expf(-yc)));
    }
  };
  auto put = [&](bool last = false) INL {
    auto b0 = __builtin_amdgcn_permlane32_swap(o[0], o[0], false, false);
    auto b1 = __builtin_amdgcn_permlane32_swap(o[1], o[1], false, false);
    const u32 c0 = (b0[0] < 0xfffffffeu ? b0[0] : 0xfffffffeu) ^ omark;
    auto r = __builtin_amdgcn_make_buffer_rsrc((void*)obase, 0, NR * MS * NS * 4, 0x00020000);
    if (OAUX != AUX_SYS && last && !isr) __builtin_amdgcn_raw_buffer_store_b128(u32x4{c0, b1[0], b0[1], b1[1]}, r, voff_o | __builtin_amdgcn_readlane(jo, it), 0, OAUX);
    else __builtin_amdgcn_raw_buffer_store_b128(u32x4{c0, b1[0], b0[1], b1[1]}, r, voff_o | __builtin_amdgcn_readlane(jo, it), 0, AUX_SYS);
    __builtin_amdgcn_sched_barrier(0);
    STAMPJ(3 + 2 * it, since(tstart));
  };
  auto wipe = [&]() INL {
    const u32 k = it * 2 * NRS + set * 2 + half;
    const u32 z = !isr ? __builtin_amdgcn_readlane(jt, it) : k >= 64 + NST ? OOB : dl * SHARD + (k < 64 ? NPRE * TILE_B : (NUC - 8) * TILE_B) + k * 8192;
    __builtin_amdgcn_raw_buffer_store_b128(u32x4{0, 0, 0, 0}, rzr, voff_z + z, 0, AUX_SYS);
  };
  auto step = [&](auto first_wave, u32x4 (&a)[8], u32x4 (&an)[8], u32& tv) INL {
    wipe();
    const int gate = __builtin_amdgcn_readlane(jg, it);
    sure = gate == 0x7fffffff;
    for (bool first = true; !sure; first = false) {
      if (__builtin_expect(!hole(a, 0, 8), 1)) break;
      const u32 off = __builtin_amdgcn_readlane(jt, it);
      gu32* f = flags + (off / SHARD * 8 + ((wv - off / TILE_B) & 7)) * 32;
      if (!first) slow(f, f, false);
      else if (__builtin_amdgcn_readlane(pk, it) == 0) {
        while (since(tstart) < gate) __builtin_amdgcn_s_sleep(2);
        if (since(tstart) < gate + LONG_DUE)
          while (!sure) {
            if (__builtin_amdgcn_ballot_w64(probe(off, 7) == 0) == 0) break;
            slow(f, f, true);
          }
      }
      refill(a, 0, off);
      __builtin_amdgcn_s_waitcnt(0x0f70);
    }
    now = since(tstart);
    STAMPJ(2 + 2 * it, now);
#pragma unroll
    for (int ks = 0; ks < 4; ++ks) mul(a, ks);
    share();
    known |= (u32)__builtin_amdgcn_ballot_w64(pk != 0) | 1u << it;
    if (__builtin_expect((known & ~got) >> (it + 1) & 1, 0)) fetch(an, it + 1);
    __builtin_amdgcn_sched_barrier(0);
    auto ahead = [&]() INL {
      fetch(a, it + 2);
      __builtin_amdgcn_sched_barrier(0);
      asm volatile("" ::"v"(tv));
      tv = touch(it + 3);
      __builtin_amdgcn_sched_barrier(0);
    };
    if (first_wave()) ahead();
    pk = peek(now + PEEK_LA);
    __builtin_amdgcn_sched_barrier(0);
    const f32x4 s4 = gather();
    if (!first_wave()) ahead();
    finish(s4);
    put();
    ++it;
  };
  auto solo = [&](u32x4 (&a)[8]) INL {
    const u32 off = __builtin_amdgcn_readlane(jt, it);
    gu32* f = flags + (off / SHARD * 8 + ((wv - off / TILE_B) & 7)) * 32;
    sure = off == OOB;
    known |= (u32)__builtin_amdgcn_ballot_w64(pk != 0);
    if (WIPE0) wipe();
    const u32 pace = __builtin_amdgcn_raw_buffer_load_b32(__builtin_amdgcn_make_buffer_rsrc((void*)(myws + PACE_OFF + dl * 128), 0, it ? 32 : 0, 0x00020000), voff_g >> 2, 0, AUX_AGENT);
    pk = peek(it ? -1 : since(tstart));
    u32 v = 0;
    if (!sure && hole(a, 0, 8)) {
      bool follow = false;
      if (!(known >> it & 1)) {
        __builtin_amdgcn_s_waitcnt(0x0f70);
        const bool tail = (off / TILE_B & 31) >= (isr ? TAIL_R : TAIL_L);
        const int gate = __builtin_amdgcn_readlane(jg, it) - (tail ? TAIL_T : 0);
        while ((now = since(tstart)) < gate) __builtin_amdgcn_s_sleep(1);
        if (now < gate + LONG_DUE) {
          const u32 po = off + (tail ? PROBE_C * 1024 : 7168);
          follow = tail;
          while (!sure) {
            asm volatile("buffer_load_dword %0, off, %1, %2 offset:1008 sc0 sc1\n\ts_waitcnt vmcnt(%3)" : "+v"(v) : "s"(rxg), "s"(po), "n"(PROBE_D - 1) : "memory");
            if (!tail) asm volatile("s_waitcnt vmcnt(0)" : "+v"(v));
            if (__builtin_amdgcn_readfirstlane(v)) break;
            slow(f, f, true);
          }
        }
      }
#pragma unroll
      for (int i = 0; i < 8; ++i) {
        if (i == 4 && FOL_4 && follow) __builtin_amdgcn_s_sleep(FOL_4);
        if (i == 6 && FOL_6 && follow) __builtin_amdgcn_s_sleep(FOL_6);
        a[i] = __builtin_amdgcn_raw_buffer_load_b128(rxg, voff_g + (i & 3) * 1024, off + (i >> 2) * 4096, AUX_SYS);
        __builtin_amdgcn_sched_barrier(0);
      }
    }
#pragma unroll
    for (int ks = 0; ks < 4; ++ks) {
      while (!sure && hole(a, 2 * ks, 2 * ks + 2)) {
        slow(f, f, false);
        refill(a, 2 * ks, off);
        __builtin_amdgcn_s_waitcnt(0x0f70);
      }
      if (ks == 3) STAMPJ(2 + 2 * it, since(tstart));
      mul(a, ks);
      if (ks == 0) asm volatile("" ::"v"(v));
    }
    share();
    known |= 1u;
    drop(a);
    int due = 0;
    if (it == 0) {
#pragma unroll
      for (int i = 0; i < 8; ++i) a[i] = *park(8 + i);
    } else if (isr) {
      due = O_T0 + (set * 2 + half) * O_DT / 16 + origin(pace, tstart, 50);
      obase = (char*)outp;
      omark = 0;
      voff_o = voff_out(NS * 4, g * 256);
    }
    finish(gather());
    if (due) while (since(tstart) < due) __builtin_amdgcn_s_sleep(1);
    put(true);
    if (!WIPE0) wipe();
    __builtin_amdgcn_sched_barrier(0);
    __builtin_amdgcn_s_waitcnt(0x0f72 - WIPE0);
    ++it;
  };
  auto run = [&](auto first_wave) INL {
#pragma unroll
    for (int i = 0; i < 8; ++i) a0[i] = *park(i);
    pk = 0;
    __builtin_amdgcn_s_waitcnt(0x0f70);
    fetch(a1, 2);
    for (;;) {
      solo(a0);
      if (it == len) break;
      do { step(first_wave, a0, a1, tv[0]); step(first_wave, a1, a0, tv[1]); } while (it < len - 1);
    }
  };
  int second = wv >> 2;
  if (second) run(std::false_type{});
  asm volatile("" : "+s"(second));
  if (!second) run(std::true_type{});
  if (!SENT && isr) {
    asm volatile("s_waitcnt vmcnt(0)" ::: "memory");
    if (lane == 0 && __hip_atomic_fetch_add(&turn[0], 1u, __ATOMIC_RELAXED, __HIP_MEMORY_SCOPE_WORKGROUP) == 7)
      __hip_atomic_store((gu32*)(lws + OFL_OFF) + ((dl - 4) * 32 + set * 2 + half) * 32, call, __ATOMIC_RELAXED, __HIP_MEMORY_SCOPE_SYSTEM);
    STAMPJ(60, since(tstart));
  }
}

void build_jobs(u64 jobs[NSETS][2]) {
  unsigned char lst[2][256];
  int n[2] = {0, LPAD};
  for (int i = 0; i < 256; ++i) lst[0][i] = lst[1][i] = 0xff;
  auto add = [&](int k, int d, int u) { lst[k][n[k]++] = (unsigned char)(d << 5 | u); if (k == 0 && n[0] == NST) n[0] += RPAD; };
  for (int u = 0, v = 0; u < NH; ++u) {
    for (; v < NH && SELF_T0 + (v + 1) * SELF_DT <= PUSH_T1 + (u + 1) * PUSH_DT; ++v) add(0, 0, v);
    for (int d = 1; d < 4; ++d) add(0, d, u);
    if (u < NUC && !is_w(u) && cross_tile(u) >= NUI) for (int d = 5; d < 8; ++d) add(0, d, u);
  }
  unsigned char ex[64];
  int ne = 0, ie = 0;
  for (int u = 0; u < 2 * NUC; ++u) ex[ne++] = (unsigned char)u;
  for (int u = 0; u < NUC; ++u) if (!is_w(u)) for (int d = 4; d < 8; ++d) ex[ne++] = (unsigned char)(d << 5 | u);
  for (int u = 2 * NUC; u < NT; ++u) ex[ne++] = (unsigned char)u;
  for (int u = 0; u < NUI; ++u) {
    for (int d = 1; d < 4; ++d) add(1, d, u);
    for (int k = 0; k < (u < NUC ? 2 : 4) && ie < ne; ++k) lst[1][n[1]++] = ex[ie++];
  }
  for (int s = 0; s < NSETS; ++s) {
    const int k = s >= NRS, ns = k ? NLS : NRS, s0 = k ? s - NRS : s;
    jobs[s][0] = jobs[s][1] = ~0ull;
    for (int j = 0; j < 16 && s0 + j * ns < n[k]; ++j) ((unsigned char*)jobs[s])[j] = lst[k][s0 + j * ns];
  }
}
static_assert(RLEN <= 16 && LLEN <= 16 && NRS <= 16 && 2 * NUC <= NT && 64 + NST <= 2 * NRS * RLEN, "job lists");

}

extern "C" void kernel_launch(int rank, int nranks, ncclComm_t comm, hipStream_t stream,
                              void* const* d_in, const int* in_sizes, int n_in,
                              void* d_out, int out_size, const int* in_dtypes, int out_dtype,
                              void* const* d_out_all, void* const* d_ws_all, size_t ws_bytes) {
  (void)comm; (void)in_dtypes; (void)out_dtype;
  if (nranks != NR || (unsigned)rank >= (unsigned)NR || n_in != 4 || ws_bytes < WS_NEED ||
      in_sizes[0] != (int)SHARD || in_sizes[1] != (int)SHARD || out_size != NR * MS * NS) {
    fprintf(stderr, "kernel_launch: unexpected problem shape\n");
    return;
  }
  Params p;
  p.x = (const unsigned char*)d_in[0];
  p.w = (const unsigned char*)d_in[1];
  p.sx = (const float*)d_in[2];
  p.sw = (const float*)d_in[3];
  p.out = (float*)d_out;
  for (int d = 0; d < NR; ++d) p.ws[d] = (char*)d_ws_all[rank ^ d];
  for (int i = 0; i < 4; ++i) p.outp[i] = (float*)d_out_all[rank ^ (4 + i)];
  p.rank = rank;
  p.pad = 0;
  build_jobs(p.jobs);
  hipLaunchKernelGGL(ag_gemm_kernel, dim3(256), dim3(512), 0, stream, p);
}
```

```cpp
#include <hip/hip_runtime.h>
#include <rccl/rccl.h>
#include <cstdio>
#include <type_traits>

typedef unsigned int u32;
typedef unsigned long long u64;
typedef u32 u32x4 __attribute__((ext_vector_type(4)));
typedef u32 u32x2 __attribute__((ext_vector_type(2)));
typedef int i32x8 __attribute__((ext_vector_type(8)));
typedef float f32x4 __attribute__((ext_vector_type(4)));
typedef __attribute__((address_space(1))) u32 gu32;

namespace {

constexpr int NR = 8;
constexpr int KD = 4096;
constexpr int MS = 512;
constexpr int NS = 512;
constexpr u32 SHARD = (u32)MS * KD;
constexpr u32 TILE_B = 16 * KD;
constexpr int NT = MS / 16;
constexpr int NH = 26, ND = 2, NA = 4;
static_assert(NH + ND + NA == NT, "tile kinds");
constexpr int NUI = NH + ND;
constexpr int NUC = 8 + NA + ND;
constexpr int NPRE = 1;
constexpr int NRS = 12;
constexpr int NSETS = 24, NLS = NSETS - NRS;
constexpr int NRJ = 4 * NH + 3 * NA, NLJ = NT + 3 * NUI + 4 * (NA + ND);
constexpr int RLEN = (NRJ + 2 * NRS - 1) / (2 * NRS) * 2, LLEN = (NLJ + 2 * NLS - 1) / (2 * NLS) * 2;
constexpr int RPAD = RLEN * NRS - NRJ, LPAD = LLEN * NLS - NLJ;
constexpr int NST = NRJ - NRS;
static_assert(NUC * 8 + NST <= NT * 8, "staging");
constexpr u32 BUF_B = (u32)NR * SHARD;
constexpr int NBUF = 3;
constexpr size_t FLAG_OFF = (size_t)NBUF * BUF_B;
constexpr size_t FLAG_B = (size_t)NR * 8 * 128;
constexpr size_t OFL_OFF = FLAG_OFF + FLAG_B;
constexpr size_t OFL_B = 4 * 4096;
constexpr size_t CNT_OFF = OFL_OFF + OFL_B;
constexpr size_t CNT_B = 256 * 8 * 4;
constexpr size_t TMO_OFF = CNT_OFF + CNT_B;
constexpr size_t PACE_OFF = TMO_OFF + 4096;
constexpr size_t WS_NEED = PACE_OFF + 4096;
constexpr u64 DEADLINE = 100000000ull;
#define PUSH_DT 110
#define PUSH_T1 45
#define AD_T 95
#define PUSH_G 2
#define BONUS_U 2
#define BONUS_SH 5
#define BONUS_MAX 1
#define PACE_U 20
#define PUSH_PRE 500
#define PUSH_DT0 55
#define NLEAD 2
#define ST_PRE 100
#define ST_TURN 14
#define ST_SLEEP 4
#define W_DELAY 300
#define W_DT 100
#define W_SLOT 50
#define GATE 80
#define R_GATE 140
#define LONG_DUE 150
#define O_T0 (PUSH_T1 + NUC * PUSH_DT + NST * PUSH_DT / 8 + 5)
#define O_DT 113
#define SELF_T0 200
#define SELF_DT 50
#define TAIL_R (NH - 1)
#define TAIL_L (NUI - 1)
#define TAIL_T 140
#define PROBE_C 2
#define PROBE_D 3
#define FOLLOW_S 4
#define PEEK_LA 25
#define POLL_SLEEP 4
#define WAIT_D 6
#define WAIT_S 3
#define SN_G 60
#define SN_L 130
#define SN_D 3
#define SN_S 4
#define FAUX 0
#define AUX_SYS 17
#define AUX_AGENT 16
constexpr int TR_ROW = 528;
constexpr int TR_B = 16 * TR_ROW;

struct Params {
  char* ws[NR];
  float* outp[4];
  const unsigned char* x;
  const unsigned char* w;
  const float* sx;
  const float* sw;
  float* out;
  int rank, pad;
  u64 jobs[NSETS][2];
};

__host__ __device__ constexpr bool is_w(int u) { return u >= NPRE && u < NPRE + 8; }
__host__ __device__ constexpr int cross_tile(int u) { u = u < NPRE ? u : u - 8; return u < NA ? NUI + u : NH + u - NA; }
__device__ __forceinline__ void job_of(u32 ent, bool isr, int dl, int& d, int& u) {
  u = ent & 31;
  d = isr && ent >= 128 ? dl ^ ((ent >> 5) - 4) : ent >> 5;
}
__device__ __forceinline__ u32 row_of(int rank, int d, int u) { return (u32)((rank ^ d) * MS + (d < 4 ? u : cross_tile(u)) * 16); }
constexpr u32 OOB = 0x80000000u;

#define INL __attribute__((always_inline))
__device__ __forceinline__ int since(u64 t) { return (int)(__builtin_amdgcn_s_memrealtime() - t); }

__device__ __forceinline__ int origin(u32 v, u64 tstart, int dflt) {
  int sum = 0;
  bool ok = true;
#pragma unroll
  for (int k = 0; k < 8; ++k) {
    const int r = (int)(__builtin_amdgcn_readlane(v, k) - (u32)tstart);
    ok = ok && r > -100 && r < 1000;
    sum += r;
  }
  return ok ? sum >> 3 : dflt;
}

__device__ __forceinline__ void transpose4(u32 d0, u32 d1, u32 d2, u32 d3, u32& c0, u32& c1, u32& c2, u32& c3) {
  const u32 t0 = __builtin_amdgcn_perm(d1, d0, 0x05010400u), t1 = __builtin_amdgcn_perm(d1, d0, 0x07030602u);
  const u32 t2 = __builtin_amdgcn_perm(d3, d2, 0x05010400u), t3 = __builtin_amdgcn_perm(d3, d2, 0x07030602u);
  c0 = __builtin_amdgcn_perm(t2, t0, 0x05040100u);
  c1 = __builtin_amdgcn_perm(t2, t0, 0x07060302u);
  c2 = __builtin_amdgcn_perm(t3, t1, 0x05040100u);
  c3 = __builtin_amdgcn_perm(t3, t1, 0x07060302u);
}

__global__ void __launch_bounds__(512, 2) ag_gemm_kernel(Params p) {
  __shared__ __attribute__((aligned(16))) char smem[2 * TILE_B];
  __shared__ u32 turn[4];
  const int lane = threadIdx.x & 63;
  const int wv = __builtin_amdgcn_readfirstlane(threadIdx.x >> 6);
  const int xg = blockIdx.x & 7;
  const int ci = blockIdx.x >> 3;
  int rank = p.rank;
  char* myws = p.ws[0];
  char* pws = p.ws[(ci + 1) & 7];
  const unsigned char* xp = p.x;
  const unsigned char* wp = p.w;
  const int d = (ci + 1) & 7;
  const int kb = (xg + wv) & 7;
  u32 so, sst, vo;
  auto source = [&](int u) INL {
    const bool w = d >= 4 && is_w(u);
    u32 sw_ = (u32)(kb * 512 + (u - NPRE) * 64) * NS + (d & 3) * 128, sx_ = (d < 4 ? u : cross_tile(u)) * TILE_B + kb * 512;
    asm("" : "+s"(sw_), "+s"(sx_));
    so = w ? sw_ : sx_;
    sst = w ? 8 * NS : 2 * KD;
    const int sh = w ? 3 : 5;
    vo = (u32)(lane >> sh) * (sst >> (6 - sh)) + (lane & ((1 << sh) - 1)) * 16;
  };
  __builtin_assume(wv < 8);
  source(wv);
  static_assert(NPRE > 0, "the first unit is a tile of x");
  sst = wv || !d ? sst : 64;
  vo = wv || !d ? vo : (lane & 15) * KD + (lane >> 4) * 16;
  asm volatile("" : "+s"(so), "+s"(sst));
  asm volatile("" : "+v"(vo));
  asm volatile("" : "+s"(rank), "+s"(myws), "+s"(pws), "+s"(xp), "+s"(wp));
  const int m = lane & 15, q = lane >> 4;
  const u64 tstart = __builtin_amdgcn_s_memrealtime();

  gu32* cntp = (gu32*)(myws + CNT_OFF) + blockIdx.x * 8 + wv;
  auto rw = __builtin_amdgcn_make_buffer_rsrc((void*)wp, 0, (int)SHARD, 0x00020000);

  if (__builtin_expect(ci < 8, 1)) {
    auto due_of = [&](int u8) INL {
      return d == 0 ? SELF_T0 + u8 * SELF_DT / 8 : PUSH_T1 + u8 * PUSH_DT / 8;
    };
    auto until = [&](int t) INL { while (since(tstart) < t) __builtin_amdgcn_s_sleep(2); };
    u32x4 e[8];
    auto load = [&]() INL {
      auto r = __builtin_amdgcn_make_buffer_rsrc((void*)(sst == 8 * NS ? wp : xp), 0, (int)SHARD, 0x00020000);
#pragma unroll
      for (int j = 0; j < 8; ++j) e[j] = __builtin_amdgcn_raw_buffer_load_b128(r, vo, so + j * sst, 0);
    };
    auto rout = [&]() INL { return __builtin_amdgcn_make_buffer_rsrc((void*)p.out, 0, NR * MS * NS * 4, 0x00020000); };
    auto direct = [&](int s) INL {
      int jd, ju;
      job_of(((const u32*)p.jobs[s])[(RLEN - 1) >> 2] >> ((RLEN - 1) & 3) * 8 & 0xff, true, d, jd, ju);
      return (row_of(rank ^ d, jd, ju) + xg * 2) * (NS * 4) + (d & 3) * 512;
    };
    const u32 ovo = (lane >> 5) * (NS * 4) + (lane & 31) * 16;
    if (__builtin_expect(wv >= NLEAD, 0)) {
      __hip_atomic_store(&turn[lane & 3], 0u, __ATOMIC_RELAXED, __HIP_MEMORY_SCOPE_WORKGROUP);
      asm volatile("s_waitcnt lgkmcnt(0)" ::: "memory");
      __builtin_amdgcn_s_barrier();
      if (wv == (d ? 6 : 3)) {
        u32 t = __builtin_amdgcn_raw_buffer_load_b32(__builtin_amdgcn_make_buffer_rsrc((void*)(pws + d * SHARD + (SHARD - 128)), 0, NBUF * BUF_B, 0x00020000), lane < NBUF ? lane * BUF_B : OOB, 0, AUX_SYS);
        asm volatile("" :: "v"(t));
      }
      if (wv >= 4 && d == 0) {
        u32 t = __builtin_amdgcn_raw_buffer_load_b32(__builtin_amdgcn_make_buffer_rsrc((void*)p.outp[wv - 4], 0, NR * MS * NS * 4, 0x00020000),
                                                     lane < 4 ? (lane ^ (rank >> 2) * 2) * SHARD + SHARD / 2 : OOB, 0, AUX_SYS);
        asm volatile("" :: "v"(t));
      }
      static_assert(is_w(7) && NLEAD <= 7, "the marks are in place before the peer has all of w");
      if (wv == 7 && d >= 4)
#pragma unroll 1
        for (int s = 0; s < NRS; ++s) __builtin_amdgcn_raw_buffer_store_b128(u32x4{~0u, ~0u, ~0u, ~0u}, rout(), ovo + direct(s), 0, AUX_SYS);
      until(wv * PUSH_DT0);
    }
    u32 cs;
    asm volatile("s_load_dword %0, %1, 0x0" : "=s"(cs) : "s"(cntp) : "memory");
    load();
    __builtin_amdgcn_sched_barrier(0);
    if (wv < NLEAD) __builtin_amdgcn_s_barrier();
    const int nx = d == 0 ? NT : d < 4 ? NUI : NUC;
    const int nu = d < 4 ? nx : NUC + (NST + 7 - kb) / 8;
    char* tw = smem + wv * TR_B + (lane >> 5) * TR_ROW + (lane & 31) * 16;
    const char* tr = smem + wv * TR_B + m * TR_ROW + q * 16;
    const u32 soff = lane * 16;
    int u = wv, tz = 0;
    auto acks = [&](int bonus) INL {
#pragma unroll
      for (int k = 7; k >= 0; --k) {
        asm volatile("s_waitcnt vmcnt(%0)" ::"n"(k) : "memory");
        __hip_atomic_fetch_max(&turn[0], (u32)(u * 8 + 8 - k + bonus), __ATOMIC_RELAXED, __HIP_MEMORY_SCOPE_WORKGROUP);
      }
    };
    auto next = [&]() INL {
      source(u);
      until(due_of(u * 8) - PUSH_PRE);
      load();
    };
    asm volatile("s_waitcnt lgkmcnt(0)" : "+s"(cs) :: "memory");
    const u32 call = cs + 1;
    const u32 boff = call % NBUF * BUF_B + d * SHARD + kb * 8192;
    if (__builtin_expect(sst == 64, 1)) {
      auto rd = __builtin_amdgcn_make_buffer_rsrc((void*)(pws + boff), 0, 8192, 0x00020000);
#pragma unroll
      for (int i = 0; i < 8; ++i) {
        e[i][0] = ~e[i][0];
        __builtin_amdgcn_raw_buffer_store_b128(e[i], rd, soff + i * 1024, 0, AUX_SYS);
        __builtin_amdgcn_sched_barrier(0);
      }
      const int t = since(tstart) - AD_T;
      __hip_atomic_store(&turn[3], (u32)(t > 0 ? t : 0), __ATOMIC_RELAXED, __HIP_MEMORY_SCOPE_WORKGROUP);
      acks(0);
      u = 8;
      next();
    }
    char* to = nullptr;
    u32 dvo = soff, dst = 1024;
#pragma unroll 1
    for (;;) {
      int bonus = 0;
      u32 kind = __builtin_amdgcn_readfirstlane(sst);
      asm volatile("" : "+s"(kind));
      if (kind == 2 * KD) {
#pragma unroll
        for (int j = 0; j < 8; ++j) *(u32x4*)(tw + j * 2 * TR_ROW) = e[j];
#pragma unroll
        for (int i = 0; i < 8; ++i) e[i] = *(const u32x4*)(tr + i * 64);
      }
      asm volatile("" : "+s"(kind));
      if (kind == 8 * NS) {
#pragma unroll
        for (int j = 0; j < 8; ++j) *(u32x4*)(smem + wv * TR_B + lane * 16 + j * 1024) = e[j];
        const u32* wr = (const u32*)(smem + wv * TR_B + q * 2048 + m * 4);
#pragma unroll
        for (int j = 0; j < 8; ++j) {
          const u32* w4 = wr + (j >> 2) * 16 + (j & 3) * 128;
          u32 c0, c1, c2, c3;
          transpose4(w4[0], w4[32], w4[64], w4[96], c0, c1, c2, c3);
          e[(j & 4) + 0][j & 3] = c0; e[(j & 4) + 1][j & 3] = c1; e[(j & 4) + 2][j & 3] = c2; e[(j & 4) + 3][j & 3] = c3;
        }
      }
#pragma unroll
      for (int i = 0; i < 8; ++i) e[i][0] = ~e[i][0];
      if (u < nx) to = pws + (boff + u * TILE_B);
      auto rd = __builtin_amdgcn_make_buffer_rsrc((void*)to, 0, u < nx ? 8192 : 16 * NS * 4, 0x00020000);
#pragma unroll
      for (int i = 0; i < 8; ++i) {
        if (i == 7) { if (!bonus) asm volatile("s_waitcnt vmcnt(6)" ::: "memory"); }
        else if (u > 0 && d) {
          while ((int)__hip_atomic_load(&turn[0], __ATOMIC_RELAXED, __HIP_MEMORY_SCOPE_WORKGROUP) < u * 8 + i - 6) __builtin_amdgcn_s_sleep(1);
        }
        if (i % PUSH_G == 0) {
          if (i == 0) tz = __builtin_amdgcn_readfirstlane(__hip_atomic_load(&turn[3], __ATOMIC_RELAXED, __HIP_MEMORY_SCOPE_WORKGROUP));
          const int due = due_of(u * 8 + i) + tz;
          int t;
          while ((t = since(tstart)) < due) __builtin_amdgcn_s_sleep(1);
          if (i == 6 && u >= BONUS_U) { bonus = (t - due) >> BONUS_SH; bonus = bonus > BONUS_MAX ? BONUS_MAX : bonus; }
          if (i == 0 && u == PACE_U)
            __builtin_amdgcn_raw_buffer_store_b32((u32)tstart + (u32)(t - due + tz), __builtin_amdgcn_make_buffer_rsrc((void*)(myws + PACE_OFF + d * 128 + xg * 4), 0, 4, 0x00020000), soff, 0, AUX_AGENT);
        }
        __builtin_amdgcn_raw_buffer_store_b128(e[i], rd, dvo + i * dst, 0, AUX_SYS);
        __builtin_amdgcn_sched_barrier(0);
      }
      acks(bonus);
      if (u < nx && u + 8 >= nx) {
        u32 done = 0;
        if (lane == 0) done = __hip_atomic_fetch_add(&turn[1], 1u, __ATOMIC_RELAXED, __HIP_MEMORY_SCOPE_WORKGROUP);
        if (__builtin_amdgcn_readfirstlane(done) == 7)
          __builtin_amdgcn_raw_buffer_store_b32(call, __builtin_amdgcn_make_buffer_rsrc((void*)(pws + FLAG_OFF + (d * 8 + xg) * 128), 0, 4, 0x00020000), soff, 0, AUX_SYS);
      }
      u += 8;
      if (u >= nu) break;
      if (u < nx) {
        next();
      } else {
        sst = 0;
        dvo = ovo;
        dst = 2 * NS * 4;
        const int n = (u - NUC) * 8 + kb, j = n / NRS;
        int jd, ju;
        job_of(((const u32*)p.jobs[n % NRS])[j >> 2] >> (j & 3) * 8 & 0xff, true, d, jd, ju);
        to = (char*)p.outp[d - 4] + (row_of(rank, jd, ju) * (NS * 4) + (d & 3) * 512);
        auto rs = __builtin_amdgcn_make_buffer_rsrc((void*)(myws + (boff + u * TILE_B)), 0, 8192, 0x00020000);
        until(due_of(u * 8) + tz - ST_PRE);
        while ((int)__hip_atomic_load(&turn[0], __ATOMIC_RELAXED, __HIP_MEMORY_SCOPE_WORKGROUP) < u * 8 - ST_TURN) __builtin_amdgcn_s_sleep(2);
        for (;;) {
#pragma unroll
          for (int j = 0; j < 8; ++j) e[j] = __builtin_amdgcn_raw_buffer_load_b128(rs, soff + j * 1024, 0, AUX_SYS);
          u32 mn = e[0][0];
#pragma unroll
          for (int j = 1; j < 8; ++j) mn = e[j][0] < mn ? e[j][0] : mn;
          if (__builtin_amdgcn_ballot_w64(mn == 0) == 0 || __builtin_amdgcn_s_memrealtime() - tstart > DEADLINE) break;
          __builtin_amdgcn_s_sleep(ST_SLEEP);
        }
      }
    }
    if (lane == 0) {
      if (d >= 4 && __hip_atomic_fetch_add(&turn[2], 1u, __ATOMIC_RELAXED, __HIP_MEMORY_SCOPE_WORKGROUP) == 7)
        __hip_atomic_store((gu32*)(pws + OFL_OFF) + ((d - 4) * 32 + xg) * 32, call, __ATOMIC_RELAXED, __HIP_MEMORY_SCOPE_SYSTEM);
      __hip_atomic_store(cntp, call, __ATOMIC_RELAXED, __HIP_MEMORY_SCOPE_AGENT);
    }
    if (d >= 4) {
      const bool two = wv + NRS - 8 >= 8;
      auto ro = rout();
      const u32 oa = direct(wv), ob = direct(two ? wv + NRS - 8 : wv);
      until(O_T0 + tz + (two ? SN_G : SN_L) + wv * O_DT / 8);
      u32 v = two ? ~0u : 0u, n = 0;
      auto expired = [&]() INL { return (++n & 63u) == 0 && __builtin_amdgcn_s_memrealtime() - tstart > DEADLINE; };
      while (__builtin_amdgcn_readfirstlane(v) == ~0u && !expired()) {
        asm volatile("buffer_load_dword %0, off, %1, %2 offset:256 sc0 sc1\n\ts_waitcnt vmcnt(%3)" : "+v"(v) : "s"(ro), "s"(oa), "n"(SN_D - 1) : "memory");
        if (__builtin_amdgcn_readfirstlane(v) != ~0u) break;
        asm volatile("s_sleep %1" : "+v"(v) : "n"(SN_S));
      }
      for (;;) {
        const u32 b = __builtin_amdgcn_raw_buffer_load_b32(ro, ovo, ob, AUX_SYS), a = __builtin_amdgcn_raw_buffer_load_b32(ro, ovo, oa, AUX_SYS);
        if (__builtin_amdgcn_ballot_w64(a == ~0u || b == ~0u) == 0 || expired()) break;
      }
      asm volatile("" :: "v"(v));
    }
    if (d >= 4 && xg == 0 && wv == 0) {
      const char* fp = myws + OFL_OFF + (d - 4) * 4096 + (lane & 7) * 128;
      while (since(tstart) < O_T0) __builtin_amdgcn_s_sleep(4);
      u32 v = 0, n = 0;
      for (;;) {
        asm volatile("global_load_dword %0, %1, off sc0 sc1\n\ts_waitcnt vmcnt(%2)" : "+v"(v) : "v"(fp), "n"(WAIT_D - 1) : "memory");
        if (__builtin_amdgcn_ballot_w64(v < call) == 0) break;
        asm volatile("s_sleep %1" : "+v"(v) : "n"(WAIT_S));
        if (__builtin_amdgcn_ballot_w64(v < call) == 0) break;
        if ((++n & 255u) == 0 && __builtin_amdgcn_s_memrealtime() - tstart > DEADLINE) {
          if (lane == 0) p.out[0] = __builtin_nanf("");
          break;
        }
      }
      asm volatile("s_waitcnt vmcnt(0)" : "+v"(v) :: "memory");
    }
  }
  int role = ci;
  asm volatile("" : "+s"(role));
  if (role < 8) return;

  u32 call = __hip_atomic_load(cntp, __ATOMIC_RELAXED, __HIP_MEMORY_SCOPE_AGENT) + 1;
  gu32* tmo = (gu32*)(myws + TMO_OFF);
  const int g = ci & 7;
  const int sl = (ci - 8) >> 3;
  const int set = sl == 0 ? xg : sl == 1 ? 16 + xg : 8 + (xg >> 1) + ((xg ^ g) & 1) * 4;
  const bool isr = set < NRS;
  const int dl = 4 + (g >> 1), half = g & 1;
  const int len = isr ? RLEN : LLEN;
  u64 jlo = p.jobs[set][0], jhi = p.jobs[set][1];
  float* outp = isr ? p.outp[dl - 4] : p.out;
  float scale = p.sx[0] * p.sw[0];
  asm volatile("" : "+s"(jlo), "+s"(jhi), "+s"(outp));
  asm volatile("" : "+v"(scale));
  gu32* flags = (gu32*)(myws + FLAG_OFF);
  const u32 voff_g = lane * 16;

  bool sure;
  u32 n = 0;
  auto word = [&](gu32* a) INL { return __builtin_amdgcn_readfirstlane(__hip_atomic_load(a, __ATOMIC_RELAXED, __HIP_MEMORY_SCOPE_SYSTEM)); };
  auto slow = [&](gu32* f, gu32* f2, bool nap) INL {
    if ((++n & 31u) == 0) {
      if (word(f) >= call && word(f2) >= call) { sure = true; return; }
      if (word(tmo) == call || __builtin_amdgcn_s_memrealtime() - tstart > DEADLINE) {
        __hip_atomic_store(tmo, call, __ATOMIC_RELAXED, __HIP_MEMORY_SCOPE_SYSTEM);
        sure = true;
        return;
      }
    }
    if (nap) __builtin_amdgcn_s_sleep(POLL_SLEEP); else __builtin_amdgcn_s_sleep(1);
  };

  call = __builtin_amdgcn_readfirstlane(call);
  if (lane == 0) __hip_atomic_store(cntp, call, __ATOMIC_RELAXED, __HIP_MEMORY_SCOPE_AGENT);
  char* gbuf = myws + (size_t)(call % NBUF) * BUF_B + wv * 8192;
  auto rxg = __builtin_amdgcn_make_buffer_rsrc((void*)gbuf, 0, (int)BUF_B, 0x00020000);
  auto rzr = __builtin_amdgcn_make_buffer_rsrc((void*)(myws + (size_t)((call + NBUF - 1) % NBUF) * BUF_B), 0, (int)BUF_B, 0x00020000);
  char* obase = isr ? myws + (size_t)(call % NBUF) * BUF_B + (size_t)dl * SHARD + NUC * TILE_B : (char*)outp;
  u32 omark = isr ? ~0u : 0u;
  auto voff_out = [&](u32 pitch, u32 col) INL { return (wv * 2 + ((voff_g >> 8) & 1)) * pitch + col + (voff_g & 255) + (voff_g < 512 ? 0 : OOB); };
  u32 voff_o = isr ? voff_out(512, half * 256) : voff_out(NS * 4, g * 256);
  const u32 voff_z = (isr ? 0u : (u32)g * 8192) + threadIdx.x * 16;
  const u32 voff_t = lane < (sl == 2 ? 16 : 8) ? (sl == 2 ? g & 6 : g) * 1024 + lane * 128 : OOB;

  u32 jt, jo;
  int jg;
  {
    const u32 ent = lane < 16 ? (u32)((lane < 8 ? jlo : jhi) >> (lane & 7) * 8) & 0xff : 0xff;
    int d, u;
    job_of(ent, isr, dl, d, u);
    jt = ent == 0xff ? OOB : d * SHARD + u * TILE_B;
    jg = ent == 0xff ? 0x7fffffff : (d ? PUSH_T1 + (u + 1) * PUSH_DT : SELF_T0 + (u + 1) * SELF_DT) + (isr && u < TAIL_R ? R_GATE : GATE);
    jo = ent == 0xff ? OOB : isr && lane < RLEN - 1 ? (u32)(lane * NRS + set) * 8192 : row_of(rank, d, u) * (NS * 4);
  }
  u32 known = 0;
  u32 got = 0;
  u32 warm = 2;
  int now;

  auto probe = [&](u32 off, int chunk) INL { return __builtin_amdgcn_raw_buffer_load_b32(rxg, 1008, off + chunk * 1024, AUX_SYS); };
  auto rsrc_if = [&](bool on) INL { return __builtin_amdgcn_make_buffer_rsrc((void*)gbuf, 0, on ? (int)BUF_B : 0, 0x00020000); };
  auto fetch = [&](u32x4 (&a)[8], int j) INL {
    auto r = rsrc_if(known >> j & 1);
    got |= known & 1u << j;
    const u32 off = __builtin_amdgcn_readlane(jt, j);
#pragma unroll
    for (int i = 0; i < 8; ++i) a[i] = __builtin_amdgcn_raw_buffer_load_b128(r, voff_g + (i & 3) * 1024, off + (i >> 2) * 4096, FAUX);
  };
  auto touch = [&](int j) INL {
    j += __builtin_ctz(((known & ~warm) | 1u << 31) >> j);
    warm |= 1u << (j & 15);
    return __builtin_amdgcn_raw_buffer_load_b32(rsrc_if(j < 16), voff_t, __builtin_amdgcn_readlane(jt, j), AUX_SYS);
  };
  auto peek = [&](int t) INL {
    const bool on = jg <= t && !(known >> (lane & 31) & 1);
    return __builtin_amdgcn_raw_buffer_load_b32(rxg, on ? jt : OOB, 8176, AUX_SYS);
  };
  u32 pk, tv[4];
  u32x4 a0[8], a1[8];
  auto park = [&](int i) INL { return (u32x4*)(smem + (i >> 2) * 32768 + wv * 4096 + (i & 3) * 1024 + voff_g); };
  auto early = [&](int phase) INL {
    if (phase) known |= (u32)__builtin_amdgcn_ballot_w64(pk != 0);
    if (phase < 3) pk = peek(since(tstart));
    if (phase == 1 || phase == 2) fetch(a1, phase == 2 ? 1 : isr ? 0 : 31);
#pragma unroll
    for (int k = 0; k < 4; ++k) {
      if (phase == 2) tv[k] = touch(isr ? 2 : 0);
      if (phase == 3) asm volatile("" ::"v"(tv[k]));
    }
    if (phase == 1 || phase == 2)
#pragma unroll
      for (int i = 0; i < 8; ++i) *park(phase * 8 - 8 + i) = a1[i];
  };

  i32x8 bf[4][4];
  if (!isr) {
    const u32 voff = (u32)(wv * 512 + q * 16) * NS + g * 64 + m * 4;
#pragma unroll
    for (int ks = 0; ks < 4; ++ks)
#pragma unroll
      for (int h = 0; h < 2; ++h) {
        const int due = W_DELAY + sl * W_SLOT + (ks * 2 + h) * W_DT;
        while (since(tstart) < due) __builtin_amdgcn_s_sleep(4);
        if (ks * 2 + h >= 5) early(ks * 2 + h - 5);
#pragma unroll
        for (int r = 0; r < 4; ++r) {
          u32 d0 = __builtin_amdgcn_raw_buffer_load_b32(rw, voff + (r * 4 + 0) * NS, (ks * 128 + h * 64) * NS, 0);
          u32 d1 = __builtin_amdgcn_raw_buffer_load_b32(rw, voff + (r * 4 + 1) * NS, (ks * 128 + h * 64) * NS, 0);
          u32 d2 = __builtin_amdgcn_raw_buffer_load_b32(rw, voff + (r * 4 + 2) * NS, (ks * 128 + h * 64) * NS, 0);
          u32 d3 = __builtin_amdgcn_raw_buffer_load_b32(rw, voff + (r * 4 + 3) * NS, (ks * 128 + h * 64) * NS, 0);
          u32 c0, c1, c2, c3;
          transpose4(d0, d1, d2, d3, c0, c1, c2, c3);
          bf[0][ks][h * 4 + r] = (int)c0; bf[1][ks][h * 4 + r] = (int)c1; bf[2][ks][h * 4 + r] = (int)c2; bf[3][ks][h * 4 + r] = (int)c3;
        }
      }
  } else {
#pragma unroll
    for (int ks = 0; ks < 4; ++ks) {
      const int u = NPRE + ks * 2;
      const u32 off = dl * SHARD + u * TILE_B + half * 4096;
      gu32* f = flags + (dl * 8 + ((wv - u) & 7)) * 32;
      gu32* f2 = flags + (dl * 8 + ((wv - u - 1) & 7)) * 32;
      if (ks >= 1) early(ks - 1);
      while (since(tstart) < PUSH_T1 + (u + 2) * PUSH_DT - (1 - half) * PUSH_DT / 2 + GATE) __builtin_amdgcn_s_sleep(4);
      sure = false;
#pragma unroll
      for (int i = 0; i < 4; ++i) a1[i] = __builtin_amdgcn_raw_buffer_load_b128(rxg, voff_g + i * 1024, off, AUX_SYS);
      while (!sure) {
        if (__builtin_amdgcn_ballot_w64(probe(off + TILE_B, 3) == 0) == 0) break;
        slow(f, f2, true);
      }
      for (bool all = sure;; all = true) {
#pragma unroll
        for (int i = 0; i < 8; ++i)
          if (i >= 4 || all) a1[i] = __builtin_amdgcn_raw_buffer_load_b128(rxg, voff_g + (i & 3) * 1024, off + (i >> 2) * TILE_B, AUX_SYS);
        if (sure) break;
        u32 mn = a1[0][0];
#pragma unroll
        for (int i = 1; i < 8; ++i) mn = a1[i][0] < mn ? a1[i][0] : mn;
        if (__builtin_amdgcn_ballot_w64(mn == 0) == 0) break;
        slow(f, f2, true);
      }
#pragma unroll
      for (int i = 0; i < 8; ++i)
#pragma unroll
        for (int e = 0; e < 4; ++e) bf[i & 3][ks][(i >> 2) * 4 + e] = (int)(e ? a1[i][e] : ~a1[i][e]);
    }
  }
  early(3);

  int it = 0;
  auto hole = [&](u32x4 (&a)[8], int i0, int i1) INL {
    u32 mn = a[i0][0];
#pragma unroll
    for (int i = i0 + 1; i < i1; ++i) mn = a[i][0] < mn ? a[i][0] : mn;
    return __builtin_amdgcn_ballot_w64(mn == 0) != 0;
  };
  auto refill = [&](u32x4 (&a)[8], int i0, u32 off) INL {
#pragma unroll
    for (int i = i0; i < 8; ++i)
      if (hole(a, i, i + 1)) a[i] = __builtin_amdgcn_raw_buffer_load_b128(rxg, voff_g + (i & 3) * 1024, off + (i >> 2) * 4096, AUX_SYS);
  };
  auto drop = [&](u32x4 (&a)[8]) INL {
#pragma unroll
    for (int i = 0; i < 8; ++i) asm volatile("" : "=v"(a[i]));
  };
  f32x4 acc[4];
  auto mul = [&](u32x4 (&a)[8], int ks) INL {
    i32x8 af;
#pragma unroll
    for (int e = 0; e < 4; ++e) { af[e] = (int)a[2 * ks][e]; af[4 + e] = (int)a[2 * ks + 1][e]; }
    af[0] = ~af[0]; af[4] = ~af[4];
#pragma unroll
    for (int c = 0; c < 4; ++c)
      acc[c] = __builtin_amdgcn_mfma_scale_f32_16x16x128_f8f6f4(af, bf[c][ks], ks ? acc[c] : f32x4{0.f, 0.f, 0.f, 0.f}, 0, 1, 0, 0, 0, 0);
  };
  auto share = [&]() INL {
    float* rb = (float*)smem + (it & 1) * 8192;
#pragma unroll
    for (int j = 0; j < 4; ++j)
      *(f32x4*)(rb + wv * 1024 + (q * 4 + j) * 64 + m * 4) = f32x4{acc[0][j], acc[1][j], acc[2][j], acc[3][j]};
    __builtin_amdgcn_sched_barrier(0);
  };
  auto gather = [&]() INL {
    asm volatile("s_waitcnt lgkmcnt(0)" ::: "memory");
    __builtin_amdgcn_s_barrier();
    const float* rp = (float*)smem + (it & 1) * 8192 + (lane >> 5) * 4096 + wv * 128 + (lane & 31) * 4;
    f32x4 s4 = *(const f32x4*)rp;
#pragma unroll
    for (int v = 1; v < 4; ++v) s4 += *(const f32x4*)(rp + v * 1024);
    return s4;
  };
  u32 o[2];
  auto finish = [&](f32x4 s4) INL {
#pragma unroll
    for (int e = 0; e < 2; ++e) {
      auto sw = __builtin_amdgcn_permlane32_swap(__float_as_uint(s4[e]), __float_as_uint(s4[e + 2]), false, false);
      const float y = (__uint_as_float(sw[0]) + __uint_as_float(sw[1])) * scale;
      const float yc = __builtin_fminf(__builtin_fmaxf(y, -60.f), 60.f);
      o[e] = __float_as_uint(y * __builtin_amdgcn_rcpf(1.f + __expf(-yc)));
    }
  };
  auto put = [&]() INL {
    auto b0 = __builtin_amdgcn_permlane32_swap(o[0], o[0], false, false);
    auto b1 = __builtin_amdgcn_permlane32_swap(o[1], o[1], false, false);
    const u32 c0 = (b0[0] < 0xfffffffeu ? b0[0] : 0xfffffffeu) ^ omark;
    auto r = __builtin_amdgcn_make_buffer_rsrc((void*)obase, 0, NR * MS * NS * 4, 0x00020000);
    __builtin_amdgcn_raw_buffer_store_b128(u32x4{c0, b1[0], b0[1], b1[1]}, r, voff_o | __builtin_amdgcn_readlane(jo, it), 0, AUX_SYS);
    __builtin_amdgcn_sched_barrier(0);
  };
  auto wipe = [&]() INL {
    const u32 k = it * 2 * NRS + set * 2 + half;
    const u32 z = !isr ? __builtin_amdgcn_readlane(jt, it) : k >= 64 + NST ? OOB : dl * SHARD + (k < 64 ? NPRE * TILE_B : (NUC - 8) * TILE_B) + k * 8192;
    __builtin_amdgcn_raw_buffer_store_b128(u32x4{0, 0, 0, 0}, rzr, voff_z + z, 0, AUX_SYS);
  };
  auto step = [&](auto first_wave, u32x4 (&a)[8], u32x4 (&an)[8], u32& tv) INL {
    wipe();
    const int gate = __builtin_amdgcn_readlane(jg, it);
    sure = gate == 0x7fffffff;
    for (bool first = true; !sure; first = false) {
      if (__builtin_expect(!hole(a, 0, 8), 1)) break;
      const u32 off = __builtin_amdgcn_readlane(jt, it);
      gu32* f = flags + (off / SHARD * 8 + ((wv - off / TILE_B) & 7)) * 32;
      if (!first) slow(f, f, false);
      else if (__builtin_amdgcn_readlane(pk, it) == 0) {
        while (since(tstart) < gate) __builtin_amdgcn_s_sleep(2);
        if (since(tstart) < gate + LONG_DUE)
          while (!sure) {
            if (__builtin_amdgcn_ballot_w64(probe(off, 7) == 0) == 0) break;
            slow(f, f, true);
          }
      }
      refill(a, 0, off);
      __builtin_amdgcn_s_waitcnt(0x0f70);
    }
    now = since(tstart);
#pragma unroll
    for (int ks = 0; ks < 4; ++ks) mul(a, ks);
    share();
    known |= (u32)__builtin_amdgcn_ballot_w64(pk != 0) | 1u << it;
    if (__builtin_expect((known & ~got) >> (it + 1) & 1, 0)) fetch(an, it + 1);
    __builtin_amdgcn_sched_barrier(0);
    auto ahead = [&]() INL {
      fetch(a, it + 2);
      __builtin_amdgcn_sched_barrier(0);
      asm volatile("" ::"v"(tv));
      tv = touch(it + 3);
      __builtin_amdgcn_sched_barrier(0);
    };
    if (first_wave()) ahead();
    pk = peek(now + PEEK_LA);
    __builtin_amdgcn_sched_barrier(0);
    const f32x4 s4 = gather();
    if (!first_wave()) ahead();
    finish(s4);
    put();
    ++it;
  };
  auto solo = [&](u32x4 (&a)[8]) INL {
    const u32 off = __builtin_amdgcn_readlane(jt, it);
    gu32* f = flags + (off / SHARD * 8 + ((wv - off / TILE_B) & 7)) * 32;
    sure = off == OOB;
    known |= (u32)__builtin_amdgcn_ballot_w64(pk != 0);
    wipe();
    const u32 pace = __builtin_amdgcn_raw_buffer_load_b32(__builtin_amdgcn_make_buffer_rsrc((void*)(myws + PACE_OFF + dl * 128), 0, it ? 32 : 0, 0x00020000), voff_g >> 2, 0, AUX_AGENT);
    pk = peek(it ? -1 : since(tstart));
    u32 v = 0;
    if (!sure && hole(a, 0, 8)) {
      bool follow = false;
      if (!(known >> it & 1)) {
        __builtin_amdgcn_s_waitcnt(0x0f70);
        const bool tail = (off / TILE_B & 31) >= (isr ? TAIL_R : TAIL_L);
        const int gate = __builtin_amdgcn_readlane(jg, it) - (tail ? TAIL_T : 0);
        while ((now = since(tstart)) < gate) __builtin_amdgcn_s_sleep(1);
        if (now < gate + LONG_DUE) {
          const u32 po = off + (tail ? PROBE_C * 1024 : 7168);
          follow = tail;
          while (!sure) {
            asm volatile("buffer_load_dword %0, off, %1, %2 offset:1008 sc0 sc1\n\ts_waitcnt vmcnt(%3)" : "+v"(v) : "s"(rxg), "s"(po), "n"(PROBE_D - 1) : "memory");
            if (!tail) asm volatile("s_waitcnt vmcnt(0)" : "+v"(v));
            if (__builtin_amdgcn_readfirstlane(v)) break;
            slow(f, f, true);
          }
        }
      }
#pragma unroll
      for (int i = 0; i < 8; ++i) {
        if (i == 6 && follow) __builtin_amdgcn_s_sleep(FOLLOW_S);
        a[i] = __builtin_amdgcn_raw_buffer_load_b128(rxg, voff_g + (i & 3) * 1024, off + (i >> 2) * 4096, AUX_SYS);
        __builtin_amdgcn_sched_barrier(0);
      }
    }
#pragma unroll
    for (int ks = 0; ks < 4; ++ks) {
      while (!sure && hole(a, 2 * ks, 2 * ks + 2)) {
        slow(f, f, false);
        refill(a, 2 * ks, off);
        __builtin_amdgcn_s_waitcnt(0x0f70);
      }
      mul(a, ks);
      if (ks == 0) asm volatile("" ::"v"(v));
    }
    share();
    known |= 1u;
    drop(a);
    int due = 0;
    if (it == 0) {
#pragma unroll
      for (int i = 0; i < 8; ++i) a[i] = *park(8 + i);
    } else if (isr) {
      due = O_T0 + (set * 2 + half) * O_DT / 16 + origin(pace, tstart, 50);
      obase = (char*)outp;
      omark = 0;
      voff_o = voff_out(NS * 4, g * 256);
    }
    finish(gather());
    if (due) while (since(tstart) < due) __builtin_amdgcn_s_sleep(1);
    put();
    __builtin_amdgcn_sched_barrier(0);
    __builtin_amdgcn_s_waitcnt(0x0f71);
    ++it;
  };
  auto run = [&](auto first_wave) INL {
#pragma unroll
    for (int i = 0; i < 8; ++i) a0[i] = *park(i);
    pk = 0;
    __builtin_amdgcn_s_waitcnt(0x0f70);
    fetch(a1, 2);
    for (;;) {
      solo(a0);
      if (it == len) break;
      do { step(first_wave, a0, a1, tv[0]); step(first_wave, a1, a0, tv[1]); } while (it < len - 1);
    }
  };
  int second = wv >> 2;
  if (second) run(std::false_type{});
  asm volatile("" : "+s"(second));
  if (!second) run(std::true_type{});
}

void build_jobs(u64 jobs[NSETS][2]) {
  unsigned char lst[2][256];
  int n[2] = {0, LPAD};
  for (int i = 0; i < 256; ++i) lst[0][i] = lst[1][i] = 0xff;
  auto add = [&](int k, int d, int u) { lst[k][n[k]++] = (unsigned char)(d << 5 | u); if (k == 0 && n[0] == NST) n[0] += RPAD; };
  for (int u = 0, v = 0; u < NH; ++u) {
    for (; v < NH && SELF_T0 + (v + 1) * SELF_DT <= PUSH_T1 + (u + 1) * PUSH_DT; ++v) add(0, 0, v);
    for (int d = 1; d < 4; ++d) add(0, d, u);
    if (u < NUC && !is_w(u) && cross_tile(u) >= NUI) for (int d = 5; d < 8; ++d) add(0, d, u);
  }
  unsigned char ex[64];
  int ne = 0, ie = 0;
  for (int u = 0; u < 2 * NUC; ++u) ex[ne++] = (unsigned char)u;
  for (int u = 0; u < NUC; ++u) if (!is_w(u)) for (int d = 4; d < 8; ++d) ex[ne++] = (unsigned char)(d << 5 | u);
  for (int u = 2 * NUC; u < NT; ++u) ex[ne++] = (unsigned char)u;
  for (int u = 0; u < NUI; ++u) {
    for (int d = 1; d < 4; ++d) add(1, d, u);
    for (int k = 0; k < (u < NUC ? 2 : 4) && ie < ne; ++k) lst[1][n[1]++] = ex[ie++];
  }
  for (int s = 0; s < NSETS; ++s) {
    const int k = s >= NRS, ns = k ? NLS : NRS, s0 = k ? s - NRS : s;
    jobs[s][0] = jobs[s][1] = ~0ull;
    for (int j = 0; j < 16 && s0 + j * ns < n[k]; ++j) ((unsigned char*)jobs[s])[j] = lst[k][s0 + j * ns];
  }
}
static_assert(RLEN <= 16 && LLEN <= 16 && NRS <= 16 && 2 * NUC <= NT && 64 + NST <= 2 * NRS * RLEN, "job lists");

}

extern "C" void kernel_launch(int rank, int nranks, ncclComm_t comm, hipStream_t stream,
                              void* const* d_in, const int* in_sizes, int n_in,
                              void* d_out, int out_size, const int* in_dtypes, int out_dtype,
                              void* const* d_out_all, void* const* d_ws_all, size_t ws_bytes) {
  (void)comm; (void)in_dtypes; (void)out_dtype;
  if (nranks != NR || (unsigned)rank >= (unsigned)NR || n_in != 4 || ws_bytes < WS_NEED ||
      in_sizes[0] != (int)SHARD || in_sizes[1] != (int)SHARD || out_size != NR * MS * NS) {
    fprintf(stderr, "kernel_launch: unexpected problem shape\n");
    return;
  }
  Params p;
  p.x = (const unsigned char*)d_in[0];
  p.w = (const unsigned char*)d_in[1];
  p.sx = (const float*)d_in[2];
  p.sw = (const float*)d_in[3];
  p.out = (float*)d_out;
  for (int d = 0; d < NR; ++d) p.ws[d] = (char*)d_ws_all[rank ^ d];
  for (int i = 0; i < 4; ++i) p.outp[i] = (float*)d_out_all[rank ^ (4 + i)];
  p.rank = rank;
  p.pad = 0;
  build_jobs(p.jobs);
  hipLaunchKernelGGL(ag_gemm_kernel, dim3(256), dim3(512), 0, stream, p);
}
```

```cpp
#include <hip/hip_runtime.h>
#include <rccl/rccl.h>
#include <cstdio>
#include <type_traits>

typedef unsigned int u32;
typedef unsigned long long u64;
typedef u32 u32x4 __attribute__((ext_vector_type(4)));
typedef u32 u32x2 __attribute__((ext_vector_type(2)));
typedef int i32x8 __attribute__((ext_vector_type(8)));
typedef float f32x4 __attribute__((ext_vector_type(4)));
typedef __attribute__((address_space(1))) u32 gu32;

namespace {

constexpr int NR = 8;
constexpr int KD = 4096;
constexpr int MS = 512;
constexpr int NS = 512;
constexpr u32 SHARD = (u32)MS * KD;
constexpr u32 TILE_B = 16 * KD;
constexpr int NT = MS / 16;
constexpr int NH = 26, ND = 2, NA = 4;
static_assert(NH + ND + NA == NT, "tile kinds");
constexpr int NUI = NH + ND;
constexpr int NUC = 8 + NA + ND;
constexpr int NPRE = 1;
constexpr int NRS = 12;
constexpr int NSETS = 24, NLS = NSETS - NRS;
constexpr int NRJ = 4 * NH + 3 * NA, NLJ = NT + 3 * NUI + 4 * (NA + ND);
constexpr int RLEN = (NRJ + 2 * NRS - 1) / (2 * NRS) * 2, LLEN = (NLJ + 2 * NLS - 1) / (2 * NLS) * 2;
constexpr int RPAD = RLEN * NRS - NRJ, LPAD = LLEN * NLS - NLJ;
constexpr int NST = NRJ - NRS;
static_assert(NUC * 8 + NST <= NT * 8, "staging");
constexpr u32 BUF_B = (u32)NR * SHARD;
constexpr int NBUF = 3;
constexpr size_t FLAG_OFF = (size_t)NBUF * BUF_B;
constexpr size_t FLAG_B = (size_t)NR * 8 * 128;
constexpr size_t OFL_OFF = FLAG_OFF + FLAG_B;
constexpr size_t OFL_B = 4 * 4096;
constexpr size_t CNT_OFF = OFL_OFF + OFL_B;
constexpr size_t CNT_B = 256 * 8 * 4;
constexpr size_t TMO_OFF = CNT_OFF + CNT_B;
constexpr size_t PACE_OFF = TMO_OFF + 4096;
constexpr size_t WS_NEED = PACE_OFF + 4096;
constexpr u64 DEADLINE = 100000000ull;
#define PUSH_DT 110
#define PUSH_T1 45
#define AD_T 115
#define PUSH_G 2
#define BONUS_U 2
#define BONUS_SH 5
#define BONUS_MAX 1
#define PACE_U 20
#define PUSH_PRE 500
#define PUSH_DT0 55
#define NLEAD 2
#define ST_PRE 100
#define ST_TURN 14
#define ST_SLEEP 4
#define W_DELAY 300
#define W_DT 100
#define W_SLOT 50
#define GATE 80
#define R_GATE 140
#define LONG_DUE 150
#define O_T0 (PUSH_T1 + NUC * PUSH_DT + NST * PUSH_DT / 8 + 5)
#define O_DT 113
#define SELF_T0 200
#define SELF_DT 50
#define TAIL_R (NH - 1)
#define TAIL_L (NUI - 1)
#define TAIL_T 140
#define PROBE_C 2
#define PROBE_D 3
#define FOLLOW_S 4
#define PEEK_LA 25
#define POLL_SLEEP 4
#define WAIT_D 6
#define WAIT_S 3
#define MARK_T 250
#define SN_G 60
#define SN_L 130
#define SN_D 3
#define SN_S 4
#define FAUX 0
#define AUX_SYS 17
#define AUX_AGENT 16
constexpr int TR_ROW = 528;
constexpr int TR_B = 16 * TR_ROW;

struct Params {
  char* ws[NR];
  float* outp[4];
  const unsigned char* x;
  const unsigned char* w;
  const float* sx;
  const float* sw;
  float* out;
  int rank, pad;
  u64 jobs[NSETS][2];
};

__host__ __device__ constexpr bool is_w(int u) { return u >= NPRE && u < NPRE + 8; }
__host__ __device__ constexpr int cross_tile(int u) { u = u < NPRE ? u : u - 8; return u < NA ? NUI + u : NH + u - NA; }
__device__ __forceinline__ void job_of(u32 ent, bool isr, int dl, int& d, int& u) {
  u = ent & 31;
  d = isr && ent >= 128 ? dl ^ ((ent >> 5) - 4) : ent >> 5;
}
__device__ __forceinline__ u32 row_of(int rank, int d, int u) { return (u32)((rank ^ d) * MS + (d < 4 ? u : cross_tile(u)) * 16); }
constexpr u32 OOB = 0x80000000u;

#define INL __attribute__((always_inline))
__device__ __forceinline__ int since(u64 t) { return (int)(__builtin_amdgcn_s_memrealtime() - t); }

__device__ __forceinline__ int origin(u32 v, u64 tstart, int dflt) {
  int sum = 0;
  bool ok = true;
#pragma unroll
  for (int k = 0; k < 8; ++k) {
    const int r = (int)(__builtin_amdgcn_readlane(v, k) - (u32)tstart);
    ok = ok && r > -100 && r < 1000;
    sum += r;
  }
  return ok ? sum >> 3 : dflt;
}

__device__ __forceinline__ void transpose4(u32 d0, u32 d1, u32 d2, u32 d3, u32& c0, u32& c1, u32& c2, u32& c3) {
  const u32 t0 = __builtin_amdgcn_perm(d1, d0, 0x05010400u), t1 = __builtin_amdgcn_perm(d1, d0, 0x07030602u);
  const u32 t2 = __builtin_amdgcn_perm(d3, d2, 0x05010400u), t3 = __builtin_amdgcn_perm(d3, d2, 0x07030602u);
  c0 = __builtin_amdgcn_perm(t2, t0, 0x05040100u);
  c1 = __builtin_amdgcn_perm(t2, t0, 0x07060302u);
  c2 = __builtin_amdgcn_perm(t3, t1, 0x05040100u);
  c3 = __builtin_amdgcn_perm(t3, t1, 0x07060302u);
}

__global__ void __launch_bounds__(512, 2) ag_gemm_kernel(Params p) {
  __shared__ __attribute__((aligned(16))) char smem[2 * TILE_B];
  __shared__ u32 turn[4];
  const int lane = threadIdx.x & 63;
  const int wv = __builtin_amdgcn_readfirstlane(threadIdx.x >> 6);
  const int xg = blockIdx.x & 7;
  const int ci = blockIdx.x >> 3;
  int rank = p.rank;
  char* myws = p.ws[0];
  char* pws = p.ws[(ci + 1) & 7];
  const unsigned char* xp = p.x;
  const unsigned char* wp = p.w;
  const int d = (ci + 1) & 7;
  const int kb = (xg + wv) & 7;
  u32 so, sst, vo;
  auto source = [&](int u) INL {
    const bool w = d >= 4 && is_w(u);
    u32 sw_ = (u32)(kb * 512 + (u - NPRE) * 64) * NS + (d & 3) * 128, sx_ = (d < 4 ? u : cross_tile(u)) * TILE_B + kb * 512;
    asm("" : "+s"(sw_), "+s"(sx_));
    so = w ? sw_ : sx_;
    sst = w ? 8 * NS : 2 * KD;
    const int sh = w ? 3 : 5;
    vo = (u32)(lane >> sh) * (sst >> (6 - sh)) + (lane & ((1 << sh) - 1)) * 16;
  };
  __builtin_assume(wv < 8);
  source(wv);
  static_assert(NPRE > 0, "the first unit is a tile of x");
  sst = wv || !d ? sst : 64;
  vo = wv || !d ? vo : (lane & 15) * KD + (lane >> 4) * 16;
  asm volatile("" : "+s"(so), "+s"(sst));
  asm volatile("" : "+v"(vo));
  asm volatile("" : "+s"(rank), "+s"(myws), "+s"(pws), "+s"(xp), "+s"(wp));
  const int m = lane & 15, q = lane >> 4;
  const u64 tstart = __builtin_amdgcn_s_memrealtime();

  gu32* cntp = (gu32*)(myws + CNT_OFF) + blockIdx.x * 8 + wv;
  auto rw = __builtin_amdgcn_make_buffer_rsrc((void*)wp, 0, (int)SHARD, 0x00020000);

  if (__builtin_expect(ci < 8, 1)) {
    auto due_of = [&](int u8) INL {
      return d == 0 ? SELF_T0 + u8 * SELF_DT / 8 : PUSH_T1 + u8 * PUSH_DT / 8;
    };
    auto until = [&](int t) INL { while (since(tstart) < t) __builtin_amdgcn_s_sleep(2); };
    u32x4 e[8];
    auto load = [&]() INL {
      auto r = __builtin_amdgcn_make_buffer_rsrc((void*)(sst == 8 * NS ? wp : xp), 0, (int)SHARD, 0x00020000);
#pragma unroll
      for (int j = 0; j < 8; ++j) e[j] = __builtin_amdgcn_raw_buffer_load_b128(r, vo, so + j * sst, 0);
    };
    auto rout = [&]() INL { return __builtin_amdgcn_make_buffer_rsrc((void*)p.out, 0, NR * MS * NS * 4, 0x00020000); };
    auto direct = [&](int s) INL {
      int jd, ju;
      job_of(((const u32*)p.jobs[s])[(RLEN - 1) >> 2] >> ((RLEN - 1) & 3) * 8 & 0xff, true, d, jd, ju);
      return (row_of(rank ^ d, jd, ju) + xg * 2) * (NS * 4) + (d & 3) * 512;
    };
    const u32 ovo = (lane >> 5) * (NS * 4) + (lane & 31) * 16;
    if (__builtin_expect(wv >= NLEAD, 0)) {
      __hip_atomic_store(&turn[lane & 3], 0u, __ATOMIC_RELAXED, __HIP_MEMORY_SCOPE_WORKGROUP);
      asm volatile("s_waitcnt lgkmcnt(0)" ::: "memory");
      __builtin_amdgcn_s_barrier();
      static_assert(is_w(7) && NLEAD <= 7, "the marks are in place before the peer has all of w");
      if (wv == 7) until(MARK_T);
      if (wv == 7 && d >= 4)
#pragma unroll 1
        for (int s = 0; s < NRS; ++s) __builtin_amdgcn_raw_buffer_store_b128(u32x4{~0u, ~0u, ~0u, ~0u}, rout(), ovo + direct(s), 0, AUX_SYS);
      until(wv * PUSH_DT0);
    }
    u32 cs;
    asm volatile("s_load_dword %0, %1, 0x0" : "=s"(cs) : "s"(cntp) : "memory");
    load();
    __builtin_amdgcn_sched_barrier(0);
    if (wv < NLEAD) __builtin_amdgcn_s_barrier();
    const int nx = d == 0 ? NT : d < 4 ? NUI : NUC;
    const int nu = d < 4 ? nx : NUC + (NST + 7 - kb) / 8;
    char* tw = smem + wv * TR_B + (lane >> 5) * TR_ROW + (lane & 31) * 16;
    const char* tr = smem + wv * TR_B + m * TR_ROW + q * 16;
    const u32 soff = lane * 16;
    int u = wv, tz = 0;
    auto acks = [&](int bonus) INL {
#pragma unroll
      for (int k = 7; k >= 0; --k) {
        asm volatile("s_waitcnt vmcnt(%0)" ::"n"(k) : "memory");
        __hip_atomic_fetch_max(&turn[0], (u32)(u * 8 + 8 - k + bonus), __ATOMIC_RELAXED, __HIP_MEMORY_SCOPE_WORKGROUP);
      }
    };
    auto next = [&]() INL {
      source(u);
      until(due_of(u * 8) - PUSH_PRE);
      load();
    };
    asm volatile("s_waitcnt lgkmcnt(0)" : "+s"(cs) :: "memory");
    const u32 call = cs + 1;
    const u32 boff = call % NBUF * BUF_B + d * SHARD + kb * 8192;
    if (__builtin_expect(sst == 64, 1)) {
      auto rd = __builtin_amdgcn_make_buffer_rsrc((void*)(pws + boff), 0, 8192, 0x00020000);
#pragma unroll
      for (int i = 0; i < 8; ++i) {
        e[i][0] = ~e[i][0];
        __builtin_amdgcn_raw_buffer_store_b128(e[i], rd, soff + i * 1024, 0, AUX_SYS);
        __builtin_amdgcn_sched_barrier(0);
      }
      const int t = since(tstart) - AD_T;
      __hip_atomic_store(&turn[3], (u32)(t > 0 ? t : 0), __ATOMIC_RELAXED, __HIP_MEMORY_SCOPE_WORKGROUP);
      acks(0);
      u = 8;
      next();
    }
    char* to = nullptr;
    u32 dvo = soff, dst = 1024;
#pragma unroll 1
    for (;;) {
      int bonus = 0;
      u32 kind = __builtin_amdgcn_readfirstlane(sst);
      asm volatile("" : "+s"(kind));
      if (kind == 2 * KD) {
#pragma unroll
        for (int j = 0; j < 8; ++j) *(u32x4*)(tw + j * 2 * TR_ROW) = e[j];
#pragma unroll
        for (int i = 0; i < 8; ++i) e[i] = *(const u32x4*)(tr + i * 64);
      }
      asm volatile("" : "+s"(kind));
      if (kind == 8 * NS) {
#pragma unroll
        for (int j = 0; j < 8; ++j) *(u32x4*)(smem + wv * TR_B + lane * 16 + j * 1024) = e[j];
        const u32* wr = (const u32*)(smem + wv * TR_B + q * 2048 + m * 4);
#pragma unroll
        for (int j = 0; j < 8; ++j) {
          const u32* w4 = wr + (j >> 2) * 16 + (j & 3) * 128;
          u32 c0, c1, c2, c3;
          transpose4(w4[0], w4[32], w4[64], w4[96], c0, c1, c2, c3);
          e[(j & 4) + 0][j & 3] = c0; e[(j & 4) + 1][j & 3] = c1; e[(j & 4) + 2][j & 3] = c2; e[(j & 4) + 3][j & 3] = c3;
        }
      }
#pragma unroll
      for (int i = 0; i < 8; ++i) e[i][0] = ~e[i][0];
      if (u < nx) to = pws + (boff + u * TILE_B);
      auto rd = __builtin_amdgcn_make_buffer_rsrc((void*)to, 0, u < nx ? 8192 : 16 * NS * 4, 0x00020000);
#pragma unroll
      for (int i = 0; i < 8; ++i) {
        if (i == 7) { if (!bonus) asm volatile("s_waitcnt vmcnt(6)" ::: "memory"); }
        else if (u > 0 && d) {
          while ((int)__hip_atomic_load(&turn[0], __ATOMIC_RELAXED, __HIP_MEMORY_SCOPE_WORKGROUP) < u * 8 + i - 6) __builtin_amdgcn_s_sleep(1);
        }
        if (i % PUSH_G == 0) {
          if (i == 0) tz = __builtin_amdgcn_readfirstlane(__hip_atomic_load(&turn[3], __ATOMIC_RELAXED, __HIP_MEMORY_SCOPE_WORKGROUP));
          const int due = due_of(u * 8 + i) + tz;
          int t;
          while ((t = since(tstart)) < due) __builtin_amdgcn_s_sleep(1);
          if (i == 6 && u >= BONUS_U) { bonus = (t - due) >> BONUS_SH; bonus = bonus > BONUS_MAX ? BONUS_MAX : bonus; }
          if (i == 0 && u == PACE_U)
            __builtin_amdgcn_raw_buffer_store_b32((u32)tstart + (u32)(t - due + tz), __builtin_amdgcn_make_buffer_rsrc((void*)(myws + PACE_OFF + d * 128 + xg * 4), 0, 4, 0x00020000), soff, 0, AUX_AGENT);
        }
        __builtin_amdgcn_raw_buffer_store_b128(e[i], rd, dvo + i * dst, 0, AUX_SYS);
        __builtin_amdgcn_sched_barrier(0);
      }
      acks(bonus);
      if (u < nx && u + 8 >= nx) {
        u32 done = 0;
        if (lane == 0) done = __hip_atomic_fetch_add(&turn[1], 1u, __ATOMIC_RELAXED, __HIP_MEMORY_SCOPE_WORKGROUP);
        if (__builtin_amdgcn_readfirstlane(done) == 7)
          __builtin_amdgcn_raw_buffer_store_b32(call, __builtin_amdgcn_make_buffer_rsrc((void*)(pws + FLAG_OFF + (d * 8 + xg) * 128), 0, 4, 0x00020000), soff, 0, AUX_SYS);
      }
      u += 8;
      if (u >= nu) break;
      if (u < nx) {
        next();
      } else {
        sst = 0;
        dvo = ovo;
        dst = 2 * NS * 4;
        const int n = (u - NUC) * 8 + kb, j = n / NRS;
        int jd, ju;
        job_of(((const u32*)p.jobs[n % NRS])[j >> 2] >> (j & 3) * 8 & 0xff, true, d, jd, ju);
        to = (char*)p.outp[d - 4] + (row_of(rank, jd, ju) * (NS * 4) + (d & 3) * 512);
        auto rs = __builtin_amdgcn_make_buffer_rsrc((void*)(myws + (boff + u * TILE_B)), 0, 8192, 0x00020000);
        until(due_of(u * 8) + tz - ST_PRE);
        while ((int)__hip_atomic_load(&turn[0], __ATOMIC_RELAXED, __HIP_MEMORY_SCOPE_WORKGROUP) < u * 8 - ST_TURN) __builtin_amdgcn_s_sleep(2);
        for (;;) {
#pragma unroll
          for (int j = 0; j < 8; ++j) e[j] = __builtin_amdgcn_raw_buffer_load_b128(rs, soff + j * 1024, 0, AUX_SYS);
          u32 mn = e[0][0];
#pragma unroll
          for (int j = 1; j < 8; ++j) mn = e[j][0] < mn ? e[j][0] : mn;
          if (__builtin_amdgcn_ballot_w64(mn == 0) == 0 || __builtin_amdgcn_s_memrealtime() - tstart > DEADLINE) break;
          __builtin_amdgcn_s_sleep(ST_SLEEP);
        }
      }
    }
    if (lane == 0) {
      if (d >= 4 && __hip_atomic_fetch_add(&turn[2], 1u, __ATOMIC_RELAXED, __HIP_MEMORY_SCOPE_WORKGROUP) == 7)
        __hip_atomic_store((gu32*)(pws + OFL_OFF) + ((d - 4) * 32 + xg) * 32, call, __ATOMIC_RELAXED, __HIP_MEMORY_SCOPE_SYSTEM);
      __hip_atomic_store(cntp, call, __ATOMIC_RELAXED, __HIP_MEMORY_SCOPE_AGENT);
    }
    if (d >= 4) {
      const bool two = wv + NRS - 8 >= 8;
      auto ro = rout();
      const u32 oa = direct(wv), ob = direct(two ? wv + NRS - 8 : wv);
      until(O_T0 + tz + (two ? SN_G : SN_L) + wv * O_DT / 8);
      u32 v = two ? ~0u : 0u, n = 0;
      auto expired = [&]() INL { return (++n & 63u) == 0 && __builtin_amdgcn_s_memrealtime() - tstart > DEADLINE; };
      while (__builtin_amdgcn_readfirstlane(v) == ~0u && !expired()) {
        asm volatile("buffer_load_dword %0, off, %1, %2 offset:256 sc0 sc1\n\ts_waitcnt vmcnt(%3)" : "+v"(v) : "s"(ro), "s"(oa), "n"(SN_D - 1) : "memory");
        if (__builtin_amdgcn_readfirstlane(v) != ~0u) break;
        asm volatile("s_sleep %1" : "+v"(v) : "n"(SN_S));
      }
      for (;;) {
        const u32 b = __builtin_amdgcn_raw_buffer_load_b32(ro, ovo, ob, AUX_SYS), a = __builtin_amdgcn_raw_buffer_load_b32(ro, ovo, oa, AUX_SYS);
        if (__builtin_amdgcn_ballot_w64(a == ~0u || b == ~0u) == 0 || expired()) break;
      }
      asm volatile("" :: "v"(v));
    }
    if (d >= 4 && xg == 0 && wv == 0) {
      const char* fp = myws + OFL_OFF + (d - 4) * 4096 + (lane & 7) * 128;
      while (since(tstart) < O_T0) __builtin_amdgcn_s_sleep(4);
      u32 v = 0, n = 0;
      for (;;) {
        asm volatile("global_load_dword %0, %1, off sc0 sc1\n\ts_waitcnt vmcnt(%2)" : "+v"(v) : "v"(fp), "n"(WAIT_D - 1) : "memory");
        if (__builtin_amdgcn_ballot_w64(v < call) == 0) break;
        asm volatile("s_sleep %1" : "+v"(v) : "n"(WAIT_S));
        if (__builtin_amdgcn_ballot_w64(v < call) == 0) break;
        if ((++n & 255u) == 0 && __builtin_amdgcn_s_memrealtime() - tstart > DEADLINE) {
          if (lane == 0) p.out[0] = __builtin_nanf("");
          break;
        }
      }
      asm volatile("s_waitcnt vmcnt(0)" : "+v"(v) :: "memory");
    }
  }
  int role = ci;
  asm volatile("" : "+s"(role));
  if (role < 8) return;

  if (wv == 0) {
    const u32 b = (role - 8) >> 3, nb = b == 2 ? 4 : 1;
    auto ld = [&](const void* base, u32 off) INL { return __builtin_amdgcn_raw_buffer_load_b32(__builtin_amdgcn_make_buffer_rsrc((void*)base, 0, (int)WS_NEED, 0x00020000), off, 0, AUX_SYS); };
    const u32 ow = lane ? OOB : b * BUF_B + d * SHARD + SHARD - 128;
    const u32 t0 = ld(d ? (const void*)pws : b == 0 ? (const void*)wp : b == 1 ? (const void*)xp : (const void*)p.out,
                      d ? ow : lane > nb ? OOB : lane == nb ? nb * SHARD - 128 : lane * SHARD);
    __builtin_amdgcn_sched_barrier(0);
    const u32 t1 = ld(d >= 4 ? (const void*)p.outp[d & 3] : (const void*)pws, d == 0 ? ow : d >= 4 && lane * 3 + b < 4 ? (lane * 3 + b) * SHARD + SHARD / 2 : OOB);
    const u32 t2 = ld(pws, b == 2 && lane == 0 ? (u32)FLAG_OFF : OOB);
    asm volatile("" :: "v"(t0), "v"(t1), "v"(t2));
  } else __builtin_amdgcn_s_sleep(4);
  u32 call = __hip_atomic_load(cntp, __ATOMIC_RELAXED, __HIP_MEMORY_SCOPE_AGENT) + 1;
  gu32* tmo = (gu32*)(myws + TMO_OFF);
  const int g = ci & 7;
  const int sl = (ci - 8) >> 3;
  const int set = sl == 0 ? xg : sl == 1 ? 16 + xg : 8 + (xg >> 1) + ((xg ^ g) & 1) * 4;
  const bool isr = set < NRS;
  const int dl = 4 + (g >> 1), half = g & 1;
  const int len = isr ? RLEN : LLEN;
  u64 jlo = p.jobs[set][0], jhi = p.jobs[set][1];
  float* outp = isr ? p.outp[dl - 4] : p.out;
  float scale = p.sx[0] * p.sw[0];
  asm volatile("" : "+s"(jlo), "+s"(jhi), "+s"(outp));
  asm volatile("" : "+v"(scale));
  gu32* flags = (gu32*)(myws + FLAG_OFF);
  const u32 voff_g = lane * 16;

  bool sure;
  u32 n = 0;
  auto word = [&](gu32* a) INL { return __builtin_amdgcn_readfirstlane(__hip_atomic_load(a, __ATOMIC_RELAXED, __HIP_MEMORY_SCOPE_SYSTEM)); };
  auto slow = [&](gu32* f, gu32* f2, bool nap) INL {
    if ((++n & 31u) == 0) {
      if (word(f) >= call && word(f2) >= call) { sure = true; return; }
      if (word(tmo) == call || __builtin_amdgcn_s_memrealtime() - tstart > DEADLINE) {
        __hip_atomic_store(tmo, call, __ATOMIC_RELAXED, __HIP_MEMORY_SCOPE_SYSTEM);
        sure = true;
        return;
      }
    }
    if (nap) __builtin_amdgcn_s_sleep(POLL_SLEEP); else __builtin_amdgcn_s_sleep(1);
  };

  call = __builtin_amdgcn_readfirstlane(call);
  if (lane == 0) __hip_atomic_store(cntp, call, __ATOMIC_RELAXED, __HIP_MEMORY_SCOPE_AGENT);
  char* gbuf = myws + (size_t)(call % NBUF) * BUF_B + wv * 8192;
  auto rxg = __builtin_amdgcn_make_buffer_rsrc((void*)gbuf, 0, (int)BUF_B, 0x00020000);
  auto rzr = __builtin_amdgcn_make_buffer_rsrc((void*)(myws + (size_t)((call + NBUF - 1) % NBUF) * BUF_B), 0, (int)BUF_B, 0x00020000);
  char* obase = isr ? myws + (size_t)(call % NBUF) * BUF_B + (size_t)dl * SHARD + NUC * TILE_B : (char*)outp;
  u32 omark = isr ? ~0u : 0u;
  auto voff_out = [&](u32 pitch, u32 col) INL { return (wv * 2 + ((voff_g >> 8) & 1)) * pitch + col + (voff_g & 255) + (voff_g < 512 ? 0 : OOB); };
  u32 voff_o = isr ? voff_out(512, half * 256) : voff_out(NS * 4, g * 256);
  const u32 voff_z = (isr ? 0u : (u32)g * 8192) + threadIdx.x * 16;
  const u32 voff_t = lane < (sl == 2 ? 16 : 8) ? (sl == 2 ? g & 6 : g) * 1024 + lane * 128 : OOB;

  u32 jt, jo;
  int jg;
  {
    const u32 ent = lane < 16 ? (u32)((lane < 8 ? jlo : jhi) >> (lane & 7) * 8) & 0xff : 0xff;
    int d, u;
    job_of(ent, isr, dl, d, u);
    jt = ent == 0xff ? OOB : d * SHARD + u * TILE_B;
    jg = ent == 0xff ? 0x7fffffff : (d ? PUSH_T1 + (u + 1) * PUSH_DT : SELF_T0 + (u + 1) * SELF_DT) + (isr && u < TAIL_R ? R_GATE : GATE);
    jo = ent == 0xff ? OOB : isr && lane < RLEN - 1 ? (u32)(lane * NRS + set) * 8192 : row_of(rank, d, u) * (NS * 4);
  }
  u32 known = 0;
  u32 got = 0;
  u32 warm = 2;
  int now;

  auto probe = [&](u32 off, int chunk) INL { return __builtin_amdgcn_raw_buffer_load_b32(rxg, 1008, off + chunk * 1024, AUX_SYS); };
  auto rsrc_if = [&](bool on) INL { return __builtin_amdgcn_make_buffer_rsrc((void*)gbuf, 0, on ? (int)BUF_B : 0, 0x00020000); };
  auto fetch = [&](u32x4 (&a)[8], int j) INL {
    auto r = rsrc_if(known >> j & 1);
    got |= known & 1u << j;
    const u32 off = __builtin_amdgcn_readlane(jt, j);
#pragma unroll
    for (int i = 0; i < 8; ++i) a[i] = __builtin_amdgcn_raw_buffer_load_b128(r, voff_g + (i & 3) * 1024, off + (i >> 2) * 4096, FAUX);
  };
  auto touch = [&](int j) INL {
    j += __builtin_ctz(((known & ~warm) | 1u << 31) >> j);
    warm |= 1u << (j & 15);
    return __builtin_amdgcn_raw_buffer_load_b32(rsrc_if(j < 16), voff_t, __builtin_amdgcn_readlane(jt, j), AUX_SYS);
  };
  auto peek = [&](int t) INL {
    const bool on = jg <= t && !(known >> (lane & 31) & 1);
    return __builtin_amdgcn_raw_buffer_load_b32(rxg, on ? jt : OOB, 8176, AUX_SYS);
  };
  u32 pk, tv[4];
  u32x4 a0[8], a1[8];
  auto park = [&](int i) INL { return (u32x4*)(smem + (i >> 2) * 32768 + wv * 4096 + (i & 3) * 1024 + voff_g); };
  auto early = [&](int phase) INL {
    if (phase) known |= (u32)__builtin_amdgcn_ballot_w64(pk != 0);
    if (phase < 3) pk = peek(since(tstart));
    if (phase == 1 || phase == 2) fetch(a1, phase == 2 ? 1 : isr ? 0 : 31);
#pragma unroll
    for (int k = 0; k < 4; ++k) {
      if (phase == 2) tv[k] = touch(isr ? 2 : 0);
      if (phase == 3) asm volatile("" ::"v"(tv[k]));
    }
    if (phase == 1 || phase == 2)
#pragma unroll
      for (int i = 0; i < 8; ++i) *park(phase * 8 - 8 + i) = a1[i];
  };

  i32x8 bf[4][4];
  if (!isr) {
    const u32 voff = (u32)(wv * 512 + q * 16) * NS + g * 64 + m * 4;
#pragma unroll
    for (int ks = 0; ks < 4; ++ks)
#pragma unroll
      for (int h = 0; h < 2; ++h) {
        const int due = W_DELAY + sl * W_SLOT + (ks * 2 + h) * W_DT;
        while (since(tstart) < due) __builtin_amdgcn_s_sleep(4);
        if (ks * 2 + h >= 5) early(ks * 2 + h - 5);
#pragma unroll
        for (int r = 0; r < 4; ++r) {
          u32 d0 = __builtin_amdgcn_raw_buffer_load_b32(rw, voff + (r * 4 + 0) * NS, (ks * 128 + h * 64) * NS, 0);
          u32 d1 = __builtin_amdgcn_raw_buffer_load_b32(rw, voff + (r * 4 + 1) * NS, (ks * 128 + h * 64) * NS, 0);
          u32 d2 = __builtin_amdgcn_raw_buffer_load_b32(rw, voff + (r * 4 + 2) * NS, (ks * 128 + h * 64) * NS, 0);
          u32 d3 = __builtin_amdgcn_raw_buffer_load_b32(rw, voff + (r * 4 + 3) * NS, (ks * 128 + h * 64) * NS, 0);
          u32 c0, c1, c2, c3;
          transpose4(d0, d1, d2, d3, c0, c1, c2, c3);
          bf[0][ks][h * 4 + r] = (int)c0; bf[1][ks][h * 4 + r] = (int)c1; bf[2][ks][h * 4 + r] = (int)c2; bf[3][ks][h * 4 + r] = (int)c3;
        }
      }
  } else {
#pragma unroll
    for (int ks = 0; ks < 4; ++ks) {
      const int u = NPRE + ks * 2;
      const u32 off = dl * SHARD + u * TILE_B + half * 4096;
      gu32* f = flags + (dl * 8 + ((wv - u) & 7)) * 32;
      gu32* f2 = flags + (dl * 8 + ((wv - u - 1) & 7)) * 32;
      if (ks >= 1) early(ks - 1);
      while (since(tstart) < PUSH_T1 + (u + 2) * PUSH_DT - (1 - half) * PUSH_DT / 2 + GATE) __builtin_amdgcn_s_sleep(4);
      sure = false;
#pragma unroll
      for (int i = 0; i < 4; ++i) a1[i] = __builtin_amdgcn_raw_buffer_load_b128(rxg, voff_g + i * 1024, off, AUX_SYS);
      while (!sure) {
        if (__builtin_amdgcn_ballot_w64(probe(off + TILE_B, 3) == 0) == 0) break;
        slow(f, f2, true);
      }
      for (bool all = sure;; all = true) {
#pragma unroll
        for (int i = 0; i < 8; ++i)
          if (i >= 4 || all) a1[i] = __builtin_amdgcn_raw_buffer_load_b128(rxg, voff_g + (i & 3) * 1024, off + (i >> 2) * TILE_B, AUX_SYS);
        if (sure) break;
        u32 mn = a1[0][0];
#pragma unroll
        for (int i = 1; i < 8; ++i) mn = a1[i][0] < mn ? a1[i][0] : mn;
        if (__builtin_amdgcn_ballot_w64(mn == 0) == 0) break;
        slow(f, f2, true);
      }
#pragma unroll
      for (int i = 0; i < 8; ++i)
#pragma unroll
        for (int e = 0; e < 4; ++e) bf[i & 3][ks][(i >> 2) * 4 + e] = (int)(e ? a1[i][e] : ~a1[i][e]);
    }
  }
  early(3);

  int it = 0;
  auto hole = [&](u32x4 (&a)[8], int i0, int i1) INL {
    u32 mn = a[i0][0];
#pragma unroll
    for (int i = i0 + 1; i < i1; ++i) mn = a[i][0] < mn ? a[i][0] : mn;
    return __builtin_amdgcn_ballot_w64(mn == 0) != 0;
  };
  auto refill = [&](u32x4 (&a)[8], int i0, u32 off) INL {
#pragma unroll
    for (int i = i0; i < 8; ++i)
      if (hole(a, i, i + 1)) a[i] = __builtin_amdgcn_raw_buffer_load_b128(rxg, voff_g + (i & 3) * 1024, off + (i >> 2) * 4096, AUX_SYS);
  };
  auto drop = [&](u32x4 (&a)[8]) INL {
#pragma unroll
    for (int i = 0; i < 8; ++i) asm volatile("" : "=v"(a[i]));
  };
  f32x4 acc[4];
  auto mul = [&](u32x4 (&a)[8], int ks) INL {
    i32x8 af;
#pragma unroll
    for (int e = 0; e < 4; ++e) { af[e] = (int)a[2 * ks][e]; af[4 + e] = (int)a[2 * ks + 1][e]; }
    af[0] = ~af[0]; af[4] = ~af[4];
#pragma unroll
    for (int c = 0; c < 4; ++c)
      acc[c] = __builtin_amdgcn_mfma_scale_f32_16x16x128_f8f6f4(af, bf[c][ks], ks ? acc[c] : f32x4{0.f, 0.f, 0.f, 0.f}, 0, 1, 0, 0, 0, 0);
  };
  auto share = [&]() INL {
    float* rb = (float*)smem + (it & 1) * 8192;
#pragma unroll
    for (int j = 0; j < 4; ++j)
      *(f32x4*)(rb + wv * 1024 + (q * 4 + j) * 64 + m * 4) = f32x4{acc[0][j], acc[1][j], acc[2][j], acc[3][j]};
    __builtin_amdgcn_sched_barrier(0);
  };
  auto gather = [&]() INL {
    asm volatile("s_waitcnt lgkmcnt(0)" ::: "memory");
    __builtin_amdgcn_s_barrier();
    const float* rp = (float*)smem + (it & 1) * 8192 + (lane >> 5) * 4096 + wv * 128 + (lane & 31) * 4;
    f32x4 s4 = *(const f32x4*)rp;
#pragma unroll
    for (int v = 1; v < 4; ++v) s4 += *(const f32x4*)(rp + v * 1024);
    return s4;
  };
  u32 o[2];
  auto finish = [&](f32x4 s4) INL {
#pragma unroll
    for (int e = 0; e < 2; ++e) {
      auto sw = __builtin_amdgcn_permlane32_swap(__float_as_uint(s4[e]), __float_as_uint(s4[e + 2]), false, false);
      const float y = (__uint_as_float(sw[0]) + __uint_as_float(sw[1])) * scale;
      const float yc = __builtin_fminf(__builtin_fmaxf(y, -60.f), 60.f);
      o[e] = __float_as_uint(y * __builtin_amdgcn_rcpf(1.f + __expf(-yc)));
    }
  };
  auto put = [&]() INL {
    auto b0 = __builtin_amdgcn_permlane32_swap(o[0], o[0], false, false);
    auto b1 = __builtin_amdgcn_permlane32_swap(o[1], o[1], false, false);
    const u32 c0 = (b0[0] < 0xfffffffeu ? b0[0] : 0xfffffffeu) ^ omark;
    auto r = __builtin_amdgcn_make_buffer_rsrc((void*)obase, 0, NR * MS * NS * 4, 0x00020000);
    __builtin_amdgcn_raw_buffer_store_b128(u32x4{c0, b1[0], b0[1], b1[1]}, r, voff_o | __builtin_amdgcn_readlane(jo, it), 0, AUX_SYS);
    __builtin_amdgcn_sched_barrier(0);
  };
  auto wipe = [&]() INL {
    const u32 k = it * 2 * NRS + set * 2 + half;
    const u32 z = !isr ? __builtin_amdgcn_readlane(jt, it) : k >= 64 + NST ? OOB : dl * SHARD + (k < 64 ? NPRE * TILE_B : (NUC - 8) * TILE_B) + k * 8192;
    __builtin_amdgcn_raw_buffer_store_b128(u32x4{0, 0, 0, 0}, rzr, voff_z + z, 0, AUX_SYS);
  };
  auto step = [&](auto first_wave, u32x4 (&a)[8], u32x4 (&an)[8], u32& tv) INL {
    wipe();
    const int gate = __builtin_amdgcn_readlane(jg, it);
    sure = gate == 0x7fffffff;
    for (bool first = true; !sure; first = false) {
      if (__builtin_expect(!hole(a, 0, 8), 1)) break;
      const u32 off = __builtin_amdgcn_readlane(jt, it);
      gu32* f = flags + (off / SHARD * 8 + ((wv - off / TILE_B) & 7)) * 32;
      if (!first) slow(f, f, false);
      else if (__builtin_amdgcn_readlane(pk, it) == 0) {
        while (since(tstart) < gate) __builtin_amdgcn_s_sleep(2);
        if (since(tstart) < gate + LONG_DUE)
          while (!sure) {
            if (__builtin_amdgcn_ballot_w64(probe(off, 7) == 0) == 0) break;
            slow(f, f, true);
          }
      }
      refill(a, 0, off);
      __builtin_amdgcn_s_waitcnt(0x0f70);
    }
    now = since(tstart);
#pragma unroll
    for (int ks = 0; ks < 4; ++ks) mul(a, ks);
    share();
    known |= (u32)__builtin_amdgcn_ballot_w64(pk != 0) | 1u << it;
    if (__builtin_expect((known & ~got) >> (it + 1) & 1, 0)) fetch(an, it + 1);
    __builtin_amdgcn_sched_barrier(0);
    auto ahead = [&]() INL {
      fetch(a, it + 2);
      __builtin_amdgcn_sched_barrier(0);
      asm volatile("" ::"v"(tv));
      tv = touch(it + 3);
      __builtin_amdgcn_sched_barrier(0);
    };
    if (first_wave()) ahead();
    pk = peek(now + PEEK_LA);
    __builtin_amdgcn_sched_barrier(0);
    const f32x4 s4 = gather();
    if (!first_wave()) ahead();
    finish(s4);
    put();
    ++it;
  };
  auto solo = [&](u32x4 (&a)[8]) INL {
    const u32 off = __builtin_amdgcn_readlane(jt, it);
    gu32* f = flags + (off / SHARD * 8 + ((wv - off / TILE_B) & 7)) * 32;
    sure = off == OOB;
    known |= (u32)__builtin_amdgcn_ballot_w64(pk != 0);
    wipe();
    const u32 pace = __builtin_amdgcn_raw_buffer_load_b32(__builtin_amdgcn_make_buffer_rsrc((void*)(myws + PACE_OFF + dl * 128), 0, it ? 32 : 0, 0x00020000), voff_g >> 2, 0, AUX_AGENT);
    pk = peek(it ? -1 : since(tstart));
    u32 v = 0;
    if (!sure && hole(a, 0, 8)) {
      bool follow = false;
      if (!(known >> it & 1)) {
        __builtin_amdgcn_s_waitcnt(0x0f70);
        const bool tail = (off / TILE_B & 31) >= (isr ? TAIL_R : TAIL_L);
        const int gate = __builtin_amdgcn_readlane(jg, it) - (tail ? TAIL_T : 0);
        while ((now = since(tstart)) < gate) __builtin_amdgcn_s_sleep(1);
        if (now < gate + LONG_DUE) {
          const u32 po = off + (tail ? PROBE_C * 1024 : 7168);
          follow = tail;
          while (!sure) {
            asm volatile("buffer_load_dword %0, off, %1, %2 offset:1008 sc0 sc1\n\ts_waitcnt vmcnt(%3)" : "+v"(v) : "s"(rxg), "s"(po), "n"(PROBE_D - 1) : "memory");
            if (!tail) asm volatile("s_waitcnt vmcnt(0)" : "+v"(v));
            if (__builtin_amdgcn_readfirstlane(v)) break;
            slow(f, f, true);
          }
        }
      }
#pragma unroll
      for (int i = 0; i < 8; ++i) {
        if (i == 6 && follow) __builtin_amdgcn_s_sleep(FOLLOW_S);
        a[i] = __builtin_amdgcn_raw_buffer_load_b128(rxg, voff_g + (i & 3) * 1024, off + (i >> 2) * 4096, AUX_SYS);
        __builtin_amdgcn_sched_barrier(0);
      }
    }
#pragma unroll
    for (int ks = 0; ks < 4; ++ks) {
      while (!sure && hole(a, 2 * ks, 2 * ks + 2)) {
        slow(f, f, false);
        refill(a, 2 * ks, off);
        __builtin_amdgcn_s_waitcnt(0x0f70);
      }
      mul(a, ks);
      if (ks == 0) asm volatile("" ::"v"(v));
    }
    share();
    known |= 1u;
    drop(a);
    int due = 0;
    if (it == 0) {
#pragma unroll
      for (int i = 0; i < 8; ++i) a[i] = *park(8 + i);
    } else if (isr) {
      due = O_T0 + (set * 2 + half) * O_DT / 16 + origin(pace, tstart, 50);
      obase = (char*)outp;
      omark = 0;
      voff_o = voff_out(NS * 4, g * 256);
    }
    finish(gather());
    if (due) while (since(tstart) < due) __builtin_amdgcn_s_sleep(1);
    put();
    __builtin_amdgcn_sched_barrier(0);
    __builtin_amdgcn_s_waitcnt(0x0f71);
    ++it;
  };
  auto run = [&](auto first_wave) INL {
#pragma unroll
    for (int i = 0; i < 8; ++i) a0[i] = *park(i);
    pk = 0;
    __builtin_amdgcn_s_waitcnt(0x0f70);
    fetch(a1, 2);
    for (;;) {
      solo(a0);
      if (it == len) break;
      do { step(first_wave, a0, a1, tv[0]); step(first_wave, a1, a0, tv[1]); } while (it < len - 1);
    }
  };
  int second = wv >> 2;
  if (second) run(std::false_type{});
  asm volatile("" : "+s"(second));
  if (!second) run(std::true_type{});
}

void build_jobs(u64 jobs[NSETS][2]) {
  unsigned char lst[2][256];
  int n[2] = {0, LPAD};
  for (int i = 0; i < 256; ++i) lst[0][i] = lst[1][i] = 0xff;
  auto add = [&](int k, int d, int u) { lst[k][n[k]++] = (unsigned char)(d << 5 | u); if (k == 0 && n[0] == NST) n[0] += RPAD; };
  for (int u = 0, v = 0; u < NH; ++u) {
    for (; v < NH && SELF_T0 + (v + 1) * SELF_DT <= PUSH_T1 + (u + 1) * PUSH_DT; ++v) add(0, 0, v);
    for (int d = 1; d < 4; ++d) add(0, d, u);
    if (u < NUC && !is_w(u) && cross_tile(u) >= NUI) for (int d = 5; d < 8; ++d) add(0, d, u);
  }
  unsigned char ex[64];
  int ne = 0, ie = 0;
  for (int u = 0; u < 2 * NUC; ++u) ex[ne++] = (unsigned char)u;
  for (int u = 0; u < NUC; ++u) if (!is_w(u)) for (int d = 4; d < 8; ++d) ex[ne++] = (unsigned char)(d << 5 | u);
  for (int u = 2 * NUC; u < NT; ++u) ex[ne++] = (unsigned char)u;
  for (int u = 0; u < NUI; ++u) {
    for (int d = 1; d < 4; ++d) add(1, d, u);
    for (int k = 0; k < (u < NUC ? 2 : 4) && ie < ne; ++k) lst[1][n[1]++] = ex[ie++];
  }
  for (int s = 0; s < NSETS; ++s) {
    const int k = s >= NRS, ns = k ? NLS : NRS, s0 = k ? s - NRS : s;
    jobs[s][0] = jobs[s][1] = ~0ull;
    for (int j = 0; j < 16 && s0 + j * ns < n[k]; ++j) ((unsigned char*)jobs[s])[j] = lst[k][s0 + j * ns];
  }
}
static_assert(NBUF == 3 && NSETS == 8 * NBUF && NR * MS * NS * 4 == 4 * SHARD, "translation warm-up");
static_assert(RLEN <= 16 && LLEN <= 16 && NRS <= 16 && 2 * NUC <= NT && 64 + NST <= 2 * NRS * RLEN, "job lists");

}

extern "C" void kernel_launch(int rank, int nranks, ncclComm_t comm, hipStream_t stream,
                              void* const* d_in, const int* in_sizes, int n_in,
                              void* d_out, int out_size, const int* in_dtypes, int out_dtype,
                              void* const* d_out_all, void* const* d_ws_all, size_t ws_bytes) {
  (void)comm; (void)in_dtypes; (void)out_dtype;
  if (nranks != NR || (unsigned)rank >= (unsigned)NR || n_in != 4 || ws_bytes < WS_NEED ||
      in_sizes[0] != (int)SHARD || in_sizes[1] != (int)SHARD || out_size != NR * MS * NS) {
    fprintf(stderr, "kernel_launch: unexpected problem shape\n");
    return;
  }
  Params p;
  p.x = (const unsigned char*)d_in[0];
  p.w = (const unsigned char*)d_in[1];
  p.sx = (const float*)d_in[2];
  p.sw = (const float*)d_in[3];
  p.out = (float*)d_out;
  for (int d = 0; d < NR; ++d) p.ws[d] = (char*)d_ws_all[rank ^ d];
  for (int i = 0; i < 4; ++i) p.outp[i] = (float*)d_out_all[rank ^ (4 + i)];
  p.rank = rank;
  p.pad = 0;
  build_jobs(p.jobs);
  hipLaunchKernelGGL(ag_gemm_kernel, dim3(256), dim3(512), 0, stream, p);
}
```
